```python
import numpy as np

kernel_name = "gpu_gemm_rs_m4096_k4096_n32768_f32_none"


ABSMAX_RTOL = 0.001

COMPUTE_DTYPE = "float32"
EPILOGUE = "none"
IN_QUANT = None
OUT_QUANT = None
IN_DTYPES = {"x": "float32", "w_mat": "float32"}
WORLD_SIZE = 8

def setup_inputs():

    x = np.random.randn(4096, 4096).astype(np.float32)
    w_mat = (np.random.randn(4096, 32768) / np.sqrt(4096)).astype(np.float32)
    return {"x": x, "w_mat": w_mat}

def _epilogue(y):
    return y

def reference(x, w_mat):
    y = (x @ w_mat).astype(np.float32)
    return _epilogue(y)
```

```cpp
#include <hip/hip_runtime.h>
#include <rccl/rccl.h>
#include <cstdio>

#define HIP_CHECK(expr) do { hipError_t _e = (expr); if (_e != hipSuccess) { \
    fprintf(stderr, "HIP error %s at %s:%d\n", hipGetErrorString(_e), __FILE__, __LINE__); \
    return; } } while (0)

#define LAS __attribute__((address_space(3)))
#define GAS __attribute__((address_space(1)))
typedef _Float16 half8 __attribute__((ext_vector_type(8)));
typedef _Float16 half2v __attribute__((ext_vector_type(2)));
typedef float f32x2 __attribute__((ext_vector_type(2)));
typedef float f32x4 __attribute__((ext_vector_type(4)));
typedef unsigned u32x4 __attribute__((ext_vector_type(4)));
typedef GAS unsigned gu32;
typedef GAS char gchar;

constexpr int NRANK = 8;
constexpr int KSH = 512;
constexpr int NCOLS = 32768, NLOC = 4096;
constexpr int GRID = 256;
constexpr size_t CTRL_BYTES = 1u << 20;
constexpr size_t PANEL = 32768;
constexpr size_t KT_STRIDE = 16 * PANEL;
constexpr size_t MAT_BYTES = 64 * KT_STRIDE;
constexpr size_t BUF_BYTES = 2 * MAT_BYTES;
constexpr size_t WS_NEED = CTRL_BYTES + 2 * BUF_BYTES;
constexpr int OFF_TICKET = 0, OFF_TMO = 128, OFF_INFLAG = 4096, OFF_OUTFLAG = 8192, OFF_STAMP = 65536;
constexpr unsigned long long DEADLINE_TICKS = 50000000ull;
#define LAG 3
#define PBAR 1
#define EW 1
#define PD 2
constexpr int NUNIT = 64;

struct Params {
    const float* x; const float* w;
    float* out[NRANK];
    char* ws[NRANK];
    int rank; int pad;
};

#define STAMP(k) do {} while (0)

__device__ __forceinline__ void poll_ge(gu32* addr, unsigned need, gu32* tmo, unsigned call) {
    const unsigned long long t0 = __builtin_amdgcn_s_memrealtime();
    for (;;) {
        const unsigned v = __hip_atomic_load(addr, __ATOMIC_RELAXED, __HIP_MEMORY_SCOPE_SYSTEM);
        if (__all(v >= need)) break;
        const unsigned tv = __builtin_amdgcn_readfirstlane(__hip_atomic_load(tmo, __ATOMIC_RELAXED, __HIP_MEMORY_SCOPE_AGENT));
        if (tv == call || __builtin_amdgcn_s_memrealtime() - t0 > DEADLINE_TICKS) {
            __hip_atomic_store(tmo, call, __ATOMIC_RELAXED, __HIP_MEMORY_SCOPE_AGENT);
            break;
        }
        __builtin_amdgcn_s_sleep(8);
    }
}

__device__ __forceinline__ unsigned pk16(float a, float b) {
    f32x2 v = {a, b};
    return __builtin_bit_cast(unsigned, __builtin_convertvector(v, half2v));
}

__device__ __forceinline__ void unit_decode(int u, int& q, int& r) {
    constexpr int n1 = 6 * LAG, n2 = n1 + 8 * (8 - LAG);
    if (u < n1) { q = u / 6; r = u - 6 * q; }
    else if (u < n2) { q = LAG + ((u - n1) >> 3); r = (u - n1) & 7; }
    else { q = 8 + ((u - n2) >> 1); r = 6 + ((u - n2) & 1); }
}

__global__ __launch_bounds__(512, 2) void gemm_rs_kernel(const Params p) {
    __shared__ __attribute__((aligned(16))) unsigned char shm[131072 + 16];
    LAS unsigned char* lds = (LAS unsigned char*)shm;
    const int tid = threadIdx.x, wid = __builtin_amdgcn_readfirstlane(tid >> 6), lane = tid & 63;
    const int rank = p.rank & (NRANK - 1), bid = blockIdx.x;
    gchar* myws = (gchar*)p.ws[rank];
    gu32* tmo = (gu32*)(myws + OFF_TMO);
    STAMP(0);

    if (tid == 0) {
        const unsigned ticket = __hip_atomic_fetch_add((gu32*)(myws + OFF_TICKET), 1u, __ATOMIC_RELAXED, __HIP_MEMORY_SCOPE_AGENT);
        *(LAS unsigned*)(lds + 131072) = ticket / GRID + 1u;
    }
    __syncthreads();
    const unsigned call = __builtin_amdgcn_readfirstlane(*(LAS unsigned*)(lds + 131072));
    const size_t bufoff = CTRL_BYTES + (size_t)(call & 1u) * BUF_BYTES;

    const int xcd = bid & 7, idx = bid >> 3;
    const int dest = (rank + (idx >> 2)) & (NRANK - 1);
    const int mt = 2 * dest + (xcd & 1), nt = 4 * (xcd >> 1) + (idx & 3);

    if (bid < 32) {
        const int d = (rank + wid) & (NRANK - 1);
        gchar* dws = (gchar*)p.ws[d];
        gchar* dX = dws + bufoff + (size_t)rank * KT_STRIDE + (size_t)bid * 16384;
        gchar* dW = dws + bufoff + MAT_BYTES + (size_t)rank * KT_STRIDE + (size_t)(bid >> 2) * PANEL + (size_t)(bid & 3) * 4096;
        gu32* dflag = (gu32*)(dws + OFF_INFLAG + rank * 128 + bid * 4);
        const gchar* sX = (const gchar*)p.x + (size_t)bid * 128 * (KSH * 4);
        const gchar* sW = (const gchar*)p.w + ((size_t)d * NLOC + (size_t)bid * 64) * 4;
        const unsigned xoff = (unsigned)(lane & 15) * (KSH * 4) + (unsigned)(lane >> 4) * 32u;
        const unsigned woff = (unsigned)(lane >> 4) * 8u * (NCOLS * 4) + (unsigned)(lane & 15) * 16u;
        const unsigned soff = (unsigned)lane * 16u;
        u32x4 buf[PD][8];
#pragma unroll
        for (int a = 0; a < PD; ++a)
#pragma unroll
            for (int b = 0; b < 8; ++b) buf[a][b] = (u32x4){0u, 0u, 0u, 0u};

#define P_LOAD(B, U) do { int q_, r_; unit_decode((U), q_, r_); const bool isx_ = r_ < 4; \
        const int s_ = r_ >= 6 ? q_ - LAG : q_; \
        const gchar* b_ = isx_ ? sX + (size_t)r_ * 32 * (KSH * 4) + (size_t)s_ * 256 \
                               : sW + (size_t)(s_ * 64 + (r_ & 1) * 32) * (NCOLS * 4) + (r_ >= 6 ? 2048 * 4 : 0); \
        const unsigned vo_ = isx_ ? xoff : woff; \
        const gchar* b1_ = b_ + (isx_ ? 16 : 1 * NCOLS * 4);          const gchar* b2_ = b_ + (isx_ ? 128 : 2 * NCOLS * 4); \
        const gchar* b3_ = b_ + (isx_ ? 144 : 3 * NCOLS * 4);         const gchar* b4_ = b_ + (isx_ ? 32768 : 4 * NCOLS * 4); \
        const gchar* b5_ = b_ + (isx_ ? 32768 + 16 : 5 * NCOLS * 4);  const gchar* b6_ = b_ + (isx_ ? 32768 + 128 : 6 * NCOLS * 4); \
        const gchar* b7_ = b_ + (isx_ ? 32768 + 144 : 7 * NCOLS * 4); \
        asm volatile("global_load_dwordx4 %0, %8, %9\n\tglobal_load_dwordx4 %1, %8, %10\n\t" \
                     "global_load_dwordx4 %2, %8, %11\n\tglobal_load_dwordx4 %3, %8, %12\n\t" \
                     "global_load_dwordx4 %4, %8, %13\n\tglobal_load_dwordx4 %5, %8, %14\n\t" \
                     "global_load_dwordx4 %6, %8, %15\n\tglobal_load_dwordx4 %7, %8, %16" \
                     : "+v"(B[0]), "+v"(B[1]), "+v"(B[2]), "+v"(B[3]), "+v"(B[4]), "+v"(B[5]), "+v"(B[6]), "+v"(B[7]) \
                     : "v"(vo_), "s"(b_), "s"(b1_), "s"(b2_), "s"(b3_), "s"(b4_), "s"(b5_), "s"(b6_), "s"(b7_) : "memory"); } while (0)

#pragma unroll
        for (int a = 0; a < PD; ++a) P_LOAD(buf[a], a);
        unsigned done = 0;
        for (int u0 = 0; u0 < NUNIT; u0 += PD) {
#pragma unroll
            for (int a = 0; a < PD; ++a) {
                const int u = u0 + a;
                if (u0 == 0) asm volatile("s_waitcnt vmcnt(%0)" :: "i"(8 * (PD - 1)) : "memory");
                asm volatile("s_waitcnt vmcnt(%8)" : "+v"(buf[a][0]), "+v"(buf[a][1]), "+v"(buf[a][2]), "+v"(buf[a][3]),
                             "+v"(buf[a][4]), "+v"(buf[a][5]), "+v"(buf[a][6]), "+v"(buf[a][7]) : "i"(4 + 12 * (PD - 1)) : "memory");
                const int e = u - PD - 1;
                if (e >= 0) {
                    int qe, re; unit_decode(e, qe, re);
                    if (re == 7 || (re == 5 && qe < LAG)) {
                        done = (unsigned)qe + 1u;
                        if (lane == 0) __hip_atomic_store(dflag, call * 16u + done, __ATOMIC_RELAXED, __HIP_MEMORY_SCOPE_SYSTEM);
                    }
                }
                int q, r; unit_decode(u, q, r);
                const bool isx = r < 4;
                const int s = r >= 6 ? q - LAG : q;
                u32x4 o[4];
                if (isx) {
#pragma unroll
                    for (int b = 0; b < 4; ++b) {
                        const f32x4 lo = __builtin_bit_cast(f32x4, buf[a][2 * b]), hi = __builtin_bit_cast(f32x4, buf[a][2 * b + 1]);
                        o[b] = (u32x4){pk16(lo[0], lo[1]), pk16(lo[2], lo[3]), pk16(hi[0], hi[1]), pk16(hi[2], hi[3])};
                    }
                } else {
#pragma unroll
                    for (int j = 0; j < 4; ++j) {
#define F(k) __builtin_bit_cast(f32x4, buf[a][k])[j]
                        o[j] = (u32x4){pk16(F(0), F(1)), pk16(F(2), F(3)), pk16(F(4), F(5)), pk16(F(6), F(7))};
#undef F
                    }
                }
                asm volatile("" : "+v"(o[0]), "+v"(o[1]), "+v"(o[2]), "+v"(o[3]));
                const int un = u + PD < NUNIT ? u + PD : NUNIT - 1;
                P_LOAD(buf[a], un);
                gchar* t_ = isx ? dX + (size_t)s * 8 * KT_STRIDE + (size_t)r * 4096
                                : dW + (size_t)s * 8 * KT_STRIDE + (r >= 6 ? 8 * PANEL : 0) + (size_t)(r & 1) * 1024;
                gchar* t1 = t_ + (isx ? 1024 : 2048); gchar* t2 = t_ + (isx ? 2048 : 16384); gchar* t3 = t_ + (isx ? 3072 : 16384 + 2048);
                asm volatile("global_store_dwordx4 %4, %0, %5 sc0 sc1\n\tglobal_store_dwordx4 %4, %1, %6 sc0 sc1\n\t"
                             "global_store_dwordx4 %4, %2, %7 sc0 sc1\n\tglobal_store_dwordx4 %4, %3, %8 sc0 sc1\n\ts_nop 1"
                             :: "v"(o[0]), "v"(o[1]), "v"(o[2]), "v"(o[3]), "v"(soff), "s"(t_), "s"(t1), "s"(t2), "s"(t3) : "memory");
                if (PBAR) __builtin_amdgcn_s_barrier();
            }
        }
        asm volatile("s_waitcnt vmcnt(0)" ::: "memory");
        if (lane == 0) __hip_atomic_store(dflag, call * 16u + 8u + LAG, __ATOMIC_RELAXED, __HIP_MEMORY_SCOPE_SYSTEM);
#undef P_LOAD
    }

    const int wr = wid >> 2, wc = wid & 3;
    const gchar* cA = myws + bufoff + (size_t)mt * PANEL;
    const gchar* cB = myws + bufoff + MAT_BYTES + (size_t)nt * PANEL;
    const unsigned voff = (unsigned)tid * 16u;
    const unsigned ldsw = (unsigned)wid * 1024u;
    const int aoff = wr * 8192 + lane * 16, boff = wc * 4096 + lane * 16;
    const unsigned need0 = call * 16u + 1u + (nt >= 8 ? LAG : 0);
    constexpr int HTB = 16384;
#define SA(b, h) (((b) * 2 + (h)) * HTB)
#define SB(b, h) ((4 + (b) * 2 + (h)) * HTB)
#define STAGE(bufo, gbase) do { _Pragma("unroll") for (int _i = 0; _i < 2; ++_i) \
        __builtin_amdgcn_global_load_lds((const GAS unsigned*)((gbase) + voff + _i * 8192), (LAS unsigned*)(lds + (bufo) + ldsw + _i * 8192), 16, 0, 17); } while (0)
#define LDA(dst, b, h) do { _Pragma("unroll") for (int m = 0; m < 4; ++m) _Pragma("unroll") for (int k = 0; k < 2; ++k) \
        dst[m][k] = *(const LAS half8*)(lds + SA(b, h) + aoff + m * 2048 + k * 1024); } while (0)
#define LDB(dst, b, h) do { _Pragma("unroll") for (int n = 0; n < 2; ++n) _Pragma("unroll") for (int k = 0; k < 2; ++k) \
        dst[n][k] = *(const LAS half8*)(lds + SB(b, h) + boff + n * 2048 + k * 1024); } while (0)
#define MMA(ai, bj, At_, Bt_) do { __builtin_amdgcn_s_setprio(1); \
        _Pragma("unroll") for (int m = 0; m < 4; ++m) _Pragma("unroll") for (int n = 0; n < 2; ++n) _Pragma("unroll") for (int k = 0; k < 2; ++k) \
            acc[ai][bj][m][n] = __builtin_amdgcn_mfma_f32_16x16x32_f16(At_[m][k], Bt_[n][k], acc[ai][bj][m][n], 0, 0, 0); \
        __builtin_amdgcn_s_setprio(0); } while (0)
#define WAIT_V(n) asm volatile("s_waitcnt vmcnt(" #n ")" ::: "memory")
#define WAIT_L(n) asm volatile("s_waitcnt lgkmcnt(" #n ")" ::: "memory")
#define BAR __builtin_amdgcn_s_barrier()
#define SCHED __builtin_amdgcn_sched_barrier(0)
#define WAIT_STEP(s) do { if (wid == 0) { int lz_ = lane; asm volatile("" : "+v"(lz_)); const int wh_ = (lz_ >> 3) > 5 ? 5 : (lz_ >> 3); \
        const int pr_ = wh_ < 2 ? 2 * mt + wh_ : 4 * (nt & 7) + wh_ - 2; \
        poll_ge((gu32*)(myws + OFF_INFLAG + (lz_ & 7) * 128 + pr_ * 4), need0 + (unsigned)(s), tmo, call); } } while (0)

    f32x4 acc[2][2][4][2];
#pragma unroll
    for (int a = 0; a < 2; ++a)
#pragma unroll
        for (int b = 0; b < 2; ++b)
#pragma unroll
            for (int m = 0; m < 4; ++m)
#pragma unroll
                for (int n = 0; n < 2; ++n) acc[a][b][m][n] = (f32x4){0.f, 0.f, 0.f, 0.f};
    half8 At[4][2], B0[2][2], B1[2][2];

    WAIT_STEP(0);
    asm volatile("" ::: "memory"); BAR; asm volatile("" ::: "memory");
    STAMP(1);
    STAGE(SB(0, 0), cB); STAGE(SA(0, 0), cA); STAGE(SB(0, 1), cB + HTB); STAGE(SA(0, 1), cA + HTB);
    if (wr == 1) BAR;
    WAIT_V(4); BAR;
    STAGE(SB(1, 0), cB + KT_STRIDE); STAGE(SA(1, 0), cA + KT_STRIDE); STAGE(SB(1, 1), cB + KT_STRIDE + HTB);
    WAIT_V(6); BAR;
    for (int t = 0; t < 64; t += 2) {
        const int t2 = (t + 2) & 63;
        const gchar* a1 = cA + (size_t)(t + 1) * KT_STRIDE;
        const gchar* a2 = cA + (size_t)t2 * KT_STRIDE; const gchar* b2 = cB + (size_t)t2 * KT_STRIDE;
        const gchar* a3 = a2 + KT_STRIDE; const gchar* b3 = b2 + KT_STRIDE;
        LDB(B0, 0, 0); SCHED; LDA(At, 0, 0); STAGE(SA(1, 1), a1 + HTB);
        WAIT_L(8); BAR; WAIT_L(0); MMA(0, 0, At, B0);
        if ((t2 & 7) == 0 && t2 != 0) { WAIT_STEP(t2 >> 3);
        }
        asm volatile("" ::: "memory"); BAR; SCHED;
        LDB(B1, 0, 1); STAGE(SB(0, 0), b2);
        BAR; WAIT_L(0); MMA(0, 1, At, B1); BAR;
        LDA(At, 0, 1); STAGE(SA(0, 0), a2);
        BAR; WAIT_L(0); MMA(1, 0, At, B0); BAR; SCHED;
        STAGE(SB(0, 1), b2 + HTB);
        WAIT_V(6); BAR; MMA(1, 1, At, B1); BAR;
        LDB(B0, 1, 0); SCHED; LDA(At, 1, 0); STAGE(SA(0, 1), a2 + HTB);
        WAIT_L(8); BAR; WAIT_L(0); MMA(0, 0, At, B0); BAR; SCHED;
        LDB(B1, 1, 1); STAGE(SB(1, 0), b3);
        BAR; WAIT_L(0); MMA(0, 1, At, B1); BAR;
        LDA(At, 1, 1); STAGE(SA(1, 0), a3);
        BAR; WAIT_L(0); MMA(1, 0, At, B0); BAR; SCHED;
        STAGE(SB(1, 1), b3 + HTB);
        WAIT_V(6); BAR; MMA(1, 1, At, B1); BAR;
    }
    if (wr == 0) BAR;
    STAMP(9);

    {
        int tz = threadIdx.x; asm volatile("" : "+v"(tz));
        const int fr = tz & 15, fq = (tz >> 4) & 3;
        const bool bad = __builtin_amdgcn_readfirstlane(__hip_atomic_load(tmo, __ATOMIC_RELAXED, __HIP_MEMORY_SCOPE_AGENT)) == call;
        gchar* obase = (gchar*)p.out[dest] + ((size_t)((mt & 1) * 256 + wr * 64) * NCOLS + (size_t)rank * NLOC + nt * 256 + wc * 64) * 4;
        const unsigned lo = (unsigned)(fq * 4) * (NCOLS * 4) + (unsigned)fr * 16u;
        const float qnan = __builtin_nanf("");
#pragma unroll
        for (int ai = 0; ai < 2; ++ai)
#pragma unroll
            for (int m = 0; m < 4; ++m)
#pragma unroll
                for (int j = 0; j < 4; ++j) {
                    f32x4 v = {acc[ai][0][m][0][j], acc[ai][0][m][1][j], acc[ai][1][m][0][j], acc[ai][1][m][1][j]};
                    if (bad) v = (f32x4){qnan, qnan, qnan, qnan};
                    gchar* rowp = obase + (size_t)(ai * 128 + m * 16 + j) * (NCOLS * 4);
                    asm volatile("global_store_dwordx4 %0, %1, %2 sc0 sc1\n\ts_nop 1\n\ts_waitcnt vmcnt(%3)" :: "v"(lo), "v"(v), "s"(rowp), "i"(EW - 1) : "memory");
                }
        asm volatile("s_waitcnt vmcnt(0)" ::: "memory");
        BAR; asm volatile("" ::: "memory");
        STAMP(10);
        const int slot = (mt & 1) * 16 + nt;
        if (dest != rank) {
            if (tz == 0) __hip_atomic_store((gu32*)((gchar*)p.ws[dest] + OFF_OUTFLAG + slot * 128 + rank * 4), call, __ATOMIC_RELAXED, __HIP_MEMORY_SCOPE_SYSTEM);
        } else if (wid == 0) {
            const int src = (rank + 1 + (tz & 63) % 7) & (NRANK - 1);
            poll_ge((gu32*)(myws + OFF_OUTFLAG + slot * 128 + src * 4), call, tmo, call);
            if (tz == 0 && __hip_atomic_load(tmo, __ATOMIC_RELAXED, __HIP_MEMORY_SCOPE_AGENT) == call)
                *(GAS float*)obase = qnan;
            STAMP(11);
        }
    }
}

extern "C" void kernel_launch(int rank, int nranks, ncclComm_t comm,
                              hipStream_t stream,
                              void* const* d_in, const int* in_sizes, int n_in,
                              void* d_out, int out_size,
                              const int* in_dtypes, int out_dtype,
                              void* const* d_out_all,
                              void* const* d_ws_all, size_t ws_bytes) {
    if (nranks != NRANK || (unsigned)rank >= (unsigned)NRANK || ws_bytes < WS_NEED || n_in < 2 ||
        in_sizes[0] != 4096 * KSH || in_sizes[1] != KSH * NCOLS || out_size != 512 * NCOLS) {
        fprintf(stderr, "unexpected problem shape\n");
        return;
    }
    Params p;
    p.x = (const float*)d_in[0]; p.w = (const float*)d_in[1];
    for (int r = 0; r < NRANK; ++r) { p.out[r] = (float*)d_out_all[r]; p.ws[r] = (char*)d_ws_all[r]; }
    p.rank = rank; p.pad = 0;
    gemm_rs_kernel<<<GRID, 512, 0, stream>>>(p);
    HIP_CHECK(hipGetLastError());
}
```

```cpp
#include <hip/hip_runtime.h>
#include <rccl/rccl.h>
#include <cstdio>

#define HIP_CHECK(expr) do { hipError_t _e = (expr); if (_e != hipSuccess) { \
    fprintf(stderr, "HIP error %s at %s:%d\n", hipGetErrorString(_e), __FILE__, __LINE__); \
    return; } } while (0)

#define LAS __attribute__((address_space(3)))
#define GAS __attribute__((address_space(1)))
typedef _Float16 half8 __attribute__((ext_vector_type(8)));
typedef _Float16 half2v __attribute__((ext_vector_type(2)));
typedef float f32x2 __attribute__((ext_vector_type(2)));
typedef float f32x4 __attribute__((ext_vector_type(4)));
typedef unsigned u32x4 __attribute__((ext_vector_type(4)));
typedef GAS unsigned gu32;
typedef GAS char gchar;

constexpr int NRANK = 8;
constexpr int KSH = 512;
constexpr int NCOLS = 32768, NLOC = 4096;
constexpr int GRID = 256, NPROD = 32;
constexpr size_t CTRL_BYTES = 1u << 20;
constexpr size_t PANEL = 32768;
constexpr size_t KT_STRIDE = 16 * PANEL;
constexpr size_t MAT_BYTES = 64 * KT_STRIDE;
constexpr size_t BUF_BYTES = 2 * MAT_BYTES;
constexpr size_t PK_WAVE = 28 * 1024;
constexpr size_t OFF_PK = CTRL_BYTES + 2 * BUF_BYTES;
constexpr size_t WS_NEED = OFF_PK + (size_t)NRANK * 32 * 8 * PK_WAVE;
constexpr int OFF_TICKET = 0, OFF_TMO = 128, OFF_QUEUE = 256, OFF_PDONE = 384;
constexpr int OFF_GATE = 4096;
constexpr int OFF_INFLAG = 32768;
constexpr int OFF_OUTFLAG = 65536;
constexpr int OFF_STAMP = 262144;
constexpr unsigned long long DEADLINE_TICKS = 50000000ull;
#define LAG 5
#define EW 3
#define PW 6
constexpr int NITEM = 1024;
constexpr unsigned QUEUE_PER_CALL = NITEM - 2 * NPROD + NPROD;

struct Params {
    const float* x; const float* w;
    float* out[NRANK];
    char* ws[NRANK];
    int rank; int pad;
};

#define STAMPW(k) do {} while (0)
#define STAMP(k) do {} while (0)

__device__ __forceinline__ unsigned poll_ge(gu32* addr, unsigned need, gu32* tmo, unsigned call) {
    const unsigned long long t0 = __builtin_amdgcn_s_memrealtime();
    unsigned v;
    for (;;) {
        v = __hip_atomic_load(addr, __ATOMIC_RELAXED, __HIP_MEMORY_SCOPE_SYSTEM);
        if (__all(v >= need)) break;
        const unsigned tv = __builtin_amdgcn_readfirstlane(__hip_atomic_load(tmo, __ATOMIC_RELAXED, __HIP_MEMORY_SCOPE_AGENT));
        if (tv == call || __builtin_amdgcn_s_memrealtime() - t0 > DEADLINE_TICKS) {
            __hip_atomic_store(tmo, call, __ATOMIC_RELAXED, __HIP_MEMORY_SCOPE_AGENT);
            break;
        }
        __builtin_amdgcn_s_sleep(8);
    }
    asm volatile("" ::: "memory");
    return v;
}

__device__ __forceinline__ unsigned pk16(float a, float b) {
    f32x2 v = {a, b};
    return __builtin_bit_cast(unsigned, __builtin_convertvector(v, half2v));
}

__device__ __forceinline__ void item_decode(int it, int& s, int& r) {
    constexpr int n1 = 96 * LAG, n2 = n1 + 128 * (8 - LAG);
    int q;
    if (it < n1) { q = it / 96; r = it - 96 * q; }
    else if (it < n2) { q = LAG + ((it - n1) >> 7); r = (it - n1) & 127; }
    else { q = 8 + ((it - n2) >> 5); r = 96 + ((it - n2) & 31); }
    s = r >= 96 ? q - LAG : q;
}

#define UNIT_LOAD(B, S_, R_, H_) do { const bool isx_ = (R_) < 64; \
        const gchar* b_ = isx_ ? sX + (size_t)((R_) * 64 + (H_) * 32) * (KSH * 4) + (size_t)(S_) * 256 \
                               : sW + (size_t)((S_) * 64 + (H_) * 32) * (NCOLS * 4) + (size_t)((R_) - 64) * 256; \
        const unsigned vo_ = isx_ ? xoff : woff; \
        const gchar* b1_ = b_ + (isx_ ? 16 : 1 * NCOLS * 4);          const gchar* b2_ = b_ + (isx_ ? 128 : 2 * NCOLS * 4); \
        const gchar* b3_ = b_ + (isx_ ? 144 : 3 * NCOLS * 4);         const gchar* b4_ = b_ + (isx_ ? 32768 : 4 * NCOLS * 4); \
        const gchar* b5_ = b_ + (isx_ ? 32768 + 16 : 5 * NCOLS * 4);  const gchar* b6_ = b_ + (isx_ ? 32768 + 128 : 6 * NCOLS * 4); \
        const gchar* b7_ = b_ + (isx_ ? 32768 + 144 : 7 * NCOLS * 4); \
        asm volatile("global_load_dwordx4 %0, %8, %9\n\tglobal_load_dwordx4 %1, %8, %10\n\t" \
                     "global_load_dwordx4 %2, %8, %11\n\tglobal_load_dwordx4 %3, %8, %12\n\t" \
                     "global_load_dwordx4 %4, %8, %13\n\tglobal_load_dwordx4 %5, %8, %14\n\t" \
                     "global_load_dwordx4 %6, %8, %15\n\tglobal_load_dwordx4 %7, %8, %16" \
                     : "+v"(B[0]), "+v"(B[1]), "+v"(B[2]), "+v"(B[3]), "+v"(B[4]), "+v"(B[5]), "+v"(B[6]), "+v"(B[7]) \
                     : "v"(vo_), "s"(b_), "s"(b1_), "s"(b2_), "s"(b3_), "s"(b4_), "s"(b5_), "s"(b6_), "s"(b7_) : "memory"); } while (0)
#define UNIT_WAIT(B, N) asm volatile("s_waitcnt vmcnt(%8)" : "+v"(B[0]), "+v"(B[1]), "+v"(B[2]), "+v"(B[3]), \
                                     "+v"(B[4]), "+v"(B[5]), "+v"(B[6]), "+v"(B[7]) : "i"(N) : "memory")
#define UNIT_CVT(O, B, R_) do { if ((R_) < 64) { \
            _Pragma("unroll") for (int b = 0; b < 4; ++b) { \
                const f32x4 lo_ = __builtin_bit_cast(f32x4, B[2 * b]), hi_ = __builtin_bit_cast(f32x4, B[2 * b + 1]); \
                O[b] = (u32x4){pk16(lo_[0], lo_[1]), pk16(lo_[2], lo_[3]), pk16(hi_[0], hi_[1]), pk16(hi_[2], hi_[3])}; } \
        } else { \
            _Pragma("unroll") for (int j = 0; j < 4; ++j) \
                O[j] = (u32x4){pk16(__builtin_bit_cast(f32x4, B[0])[j], __builtin_bit_cast(f32x4, B[1])[j]), \
                               pk16(__builtin_bit_cast(f32x4, B[2])[j], __builtin_bit_cast(f32x4, B[3])[j]), \
                               pk16(__builtin_bit_cast(f32x4, B[4])[j], __builtin_bit_cast(f32x4, B[5])[j]), \
                               pk16(__builtin_bit_cast(f32x4, B[6])[j], __builtin_bit_cast(f32x4, B[7])[j])}; } \
        asm volatile("" : "+v"(O[0]), "+v"(O[1]), "+v"(O[2]), "+v"(O[3])); } while (0)
#define UNIT_DST(S_, R_, H_) const bool isxd_ = (R_) < 64; \
        gchar* t0_ = isxd_ ? dimg + (size_t)(S_) * 8 * KT_STRIDE + (size_t)(R_) * 8192 + (H_) * 4096 \
                           : dimg + MAT_BYTES + (size_t)(S_) * 8 * KT_STRIDE + (size_t)(((R_) - 64) >> 2) * PANEL + (size_t)((R_) & 3) * 4096 + (H_) * 1024; \
        gchar* t1_ = t0_ + (isxd_ ? 1024 : 2048); gchar* t2_ = t0_ + (isxd_ ? 2048 : 16384); gchar* t3_ = t0_ + (isxd_ ? 3072 : 16384 + 2048)
#define STORE16(VOFF, DATA, BASE, N) asm volatile("s_waitcnt vmcnt(%3)\n\tglobal_store_dwordx4 %0, %1, %2 sc0 sc1\n\ts_nop 1" \
                                                  :: "v"(VOFF), "v"(DATA), "s"(BASE), "i"(N) : "memory")

__global__ __launch_bounds__(512, 2) void gemm_rs_kernel(const Params p) {
    __shared__ __attribute__((aligned(16))) unsigned char shm[131072 + 32];
    LAS unsigned char* lds = (LAS unsigned char*)shm;
    volatile LAS unsigned* lctl = (volatile LAS unsigned*)(lds + 131072);
    const int tid = threadIdx.x, wid = __builtin_amdgcn_readfirstlane(tid >> 6), lane = tid & 63;
    const int rank = p.rank & (NRANK - 1), bid = blockIdx.x;
    gchar* myws = (gchar*)p.ws[rank];
    gu32* tmo = (gu32*)(myws + OFF_TMO);

    if (tid == 0) {
        const unsigned ticket = __hip_atomic_fetch_add((gu32*)(myws + OFF_TICKET), 1u, __ATOMIC_RELAXED, __HIP_MEMORY_SCOPE_AGENT);
        lctl[0] = ticket / GRID + 1u; lctl[1] = 0u;
    }
    __syncthreads();
    const unsigned call = __builtin_amdgcn_readfirstlane(lctl[0]);
    const size_t bufoff = CTRL_BYTES + (size_t)(call & 1u) * BUF_BYTES;

    const int xcd = bid & 7, idx = bid >> 3, dd = idx >> 2;
    const int dest = (rank + dd) & (NRANK - 1);
    const int mt = 2 * dest + (xcd & 1), nt = 2 * (xcd >> 1) + (idx & 1) + 8 * ((idx >> 1) & 1);

    if (bid < NPROD + NITEM / 8) {
        const int d = bid < NPROD ? (rank + wid) & (NRANK - 1) : rank;
        gchar* dws = (gchar*)p.ws[d];
        gchar* dimg = dws + bufoff + (size_t)rank * KT_STRIDE;
        gu32* dflag = (gu32*)(dws + OFF_INFLAG + rank * 4096);
        const gchar* sX = (const gchar*)p.x;
        const gchar* sW = (const gchar*)p.w + (size_t)d * NLOC * 4;
        const unsigned xoff = (unsigned)(lane & 15) * (KSH * 4) + (unsigned)(lane >> 4) * 32u;
        const unsigned woff = (unsigned)(lane >> 4) * 8u * (NCOLS * 4) + (unsigned)(lane & 15) * 16u;
        const unsigned soff = (unsigned)lane * 16u;
        u32x4 buf0[8], buf1[8], o[4];
#pragma unroll
        for (int b = 0; b < 8; ++b) { buf0[b] = (u32x4){0u, 0u, 0u, 0u}; buf1[b] = (u32x4){0u, 0u, 0u, 0u}; }
        if (bid >= NPROD) {
            int s, r; item_decode((bid - NPROD) * 8 + wid, s, r);
            UNIT_LOAD(buf0, s, r, 0); UNIT_LOAD(buf1, s, r, 1);
            UNIT_WAIT(buf0, 8); UNIT_CVT(o, buf0, r);
            { UNIT_DST(s, r, 0); STORE16(soff, o[0], t0_, 63); STORE16(soff, o[1], t1_, 63); STORE16(soff, o[2], t2_, 63); STORE16(soff, o[3], t3_, 63); }
            UNIT_WAIT(buf1, 4); UNIT_CVT(o, buf1, r);
            { UNIT_DST(s, r, 1); STORE16(soff, o[0], t0_, 63); STORE16(soff, o[1], t1_, 63); STORE16(soff, o[2], t2_, 63); STORE16(soff, o[3], t3_, 63); }
            asm volatile("s_waitcnt vmcnt(0)" ::: "memory");
            if (lane == 0) __hip_atomic_store(dflag + s * 128 + r, call, __ATOMIC_RELAXED, __HIP_MEMORY_SCOPE_SYSTEM);
        } else if (wid == 0) {
            int cur = bid, nxt = bid + NPROD; bool more = true;
            for (int i = 0; cur < NITEM; ++i) {
                int nn = NITEM;
                if (more) {
                    unsigned t = 0;
                    if (lane == 0) t = __hip_atomic_fetch_add((gu32*)(myws + OFF_QUEUE), 1u, __ATOMIC_RELAXED, __HIP_MEMORY_SCOPE_AGENT);
                    t = __builtin_amdgcn_readfirstlane(t) - (call - 1u) * QUEUE_PER_CALL + 2u * NPROD;
                    nn = t < (unsigned)NITEM ? (int)t : NITEM;
                    more = nn < NITEM;
                }
                lctl[4 + (i & 3)] = (unsigned)nn;
                asm volatile("s_waitcnt lgkmcnt(0)" ::: "memory");
                __builtin_amdgcn_s_barrier(); __builtin_amdgcn_s_barrier();
                cur = nxt; nxt = nn;
            }
            if (lane == 0) __hip_atomic_fetch_add((gu32*)(myws + OFF_PDONE), 1u, __ATOMIC_RELAXED, __HIP_MEMORY_SCOPE_AGENT);
        } else {
            int cur = bid, nxt = bid + NPROD, s, r, sn, rn, pflag = -1;
            item_decode(cur, s, r);
            UNIT_LOAD(buf0, s, r, 0); UNIT_LOAD(buf1, s, r, 1);
            asm volatile("s_waitcnt vmcnt(8)" ::: "memory");
            constexpr int JF = (PW + 3) & 3, EB = (PW + 3) / 4;
            static_assert(EB == 1 || EB == 2, "window");
#define WN(j) (PW - 1 + 8 * ((PW - (j) + 3) / 4))
#define P_FLAG() do { if (pflag >= 0 && lane == 0) __hip_atomic_store(dflag + pflag, call, __ATOMIC_RELAXED, __HIP_MEMORY_SCOPE_SYSTEM); } while (0)
#define P_STORES(H) do { UNIT_DST(s, r, H); \
                STORE16(soff, o[0], t0_, WN(0)); if (JF == 0 && EB == H + 1) P_FLAG(); \
                STORE16(soff, o[1], t1_, WN(1)); if (JF == 1 && EB == H + 1) P_FLAG(); \
                STORE16(soff, o[2], t2_, WN(2)); if (JF == 2 && EB == H + 1) P_FLAG(); \
                STORE16(soff, o[3], t3_, WN(3)); if (JF == 3 && EB == H + 1) P_FLAG(); } while (0)
            for (int i = 0; cur < NITEM; ++i) {
                item_decode(nxt < NITEM ? nxt : cur, sn, rn);
                UNIT_WAIT(buf0, 16); UNIT_CVT(o, buf0, r);
                UNIT_LOAD(buf0, sn, rn, 0);
                P_STORES(0);
                __builtin_amdgcn_s_barrier();
                if (i == 0) asm volatile("s_waitcnt vmcnt(12)" ::: "memory");
                UNIT_WAIT(buf1, 16); UNIT_CVT(o, buf1, r);
                UNIT_LOAD(buf1, sn, rn, 1);
                P_STORES(1);
                const int nn = (int)__builtin_amdgcn_readfirstlane(lctl[4 + (i & 3)]);
                __builtin_amdgcn_s_barrier();
                pflag = s * 128 + r; cur = nxt; nxt = nn; s = sn; r = rn;
            }
            asm volatile("s_waitcnt vmcnt(0)" : "+v"(buf0[0]), "+v"(buf1[0]) :: "memory");
            P_FLAG();
            STAMPW(16 + wid);
        }
    }

    const int wr = wid >> 2, wc = wid & 3;
    const gchar* cA = myws + bufoff + (size_t)mt * PANEL;
    const gchar* cB = myws + bufoff + MAT_BYTES + (size_t)nt * PANEL;
    const unsigned voff = (unsigned)tid * 16u;
    const unsigned ldsw = (unsigned)wid * 1024u;
    const int aoff = wr * 8192 + lane * 16, boff = wc * 4096 + lane * 16;
    constexpr int HTB = 16384;
#define SA(b, h) (((b) * 2 + (h)) * HTB)
#define SB(b, h) ((4 + (b) * 2 + (h)) * HTB)
#define STAGE(bufo, gbase) do { _Pragma("unroll") for (int _i = 0; _i < 2; ++_i) \
        __builtin_amdgcn_global_load_lds((const GAS unsigned*)((gbase) + voff + _i * 8192), (LAS unsigned*)(lds + (bufo) + ldsw + _i * 8192), 16, 0, 17); } while (0)
#define LDA(dst, b, h) do { _Pragma("unroll") for (int m = 0; m < 4; ++m) _Pragma("unroll") for (int k = 0; k < 2; ++k) \
        dst[m][k] = *(const LAS half8*)(lds + SA(b, h) + aoff + m * 2048 + k * 1024); } while (0)
#define LDB(dst, b, h) do { _Pragma("unroll") for (int n = 0; n < 2; ++n) _Pragma("unroll") for (int k = 0; k < 2; ++k) \
        dst[n][k] = *(const LAS half8*)(lds + SB(b, h) + boff + n * 2048 + k * 1024); } while (0)
#define MMA(ai, bj, At_, Bt_) do { __builtin_amdgcn_s_setprio(1); \
        _Pragma("unroll") for (int m = 0; m < 4; ++m) _Pragma("unroll") for (int n = 0; n < 2; ++n) _Pragma("unroll") for (int k = 0; k < 2; ++k) \
            acc[ai][bj][m][n] = __builtin_amdgcn_mfma_f32_16x16x32_f16(At_[m][k], Bt_[n][k], acc[ai][bj][m][n], 0, 0, 0); \
        __builtin_amdgcn_s_setprio(0); } while (0)
#define WAIT_V(n) asm volatile("s_waitcnt vmcnt(" #n ")" ::: "memory")
#define WAIT_L(n) asm volatile("s_waitcnt lgkmcnt(" #n ")" ::: "memory")
#define BAR __builtin_amdgcn_s_barrier()
#define SCHED __builtin_amdgcn_sched_barrier(0)
#define WAIT_STEP(s) do { if (wid == 0) { int lz_ = lane; asm volatile("" : "+v"(lz_)); const int wh_ = lz_ >> 3; \
        const int it_ = wh_ < 4 ? 4 * mt + wh_ : 60 + 4 * nt + wh_; \
        poll_ge((gu32*)(myws + OFF_INFLAG + (lz_ & 7) * 4096 + (s) * 512 + it_ * 4), call, tmo, call); } } while (0)

    f32x4 acc[2][2][4][2];
#pragma unroll
    for (int a = 0; a < 2; ++a)
#pragma unroll
        for (int b = 0; b < 2; ++b)
#pragma unroll
            for (int m = 0; m < 4; ++m)
#pragma unroll
                for (int n = 0; n < 2; ++n) acc[a][b][m][n] = (f32x4){0.f, 0.f, 0.f, 0.f};
    half8 At[4][2], B0[2][2], B1[2][2];

    WAIT_STEP(0);
    asm volatile("" ::: "memory"); BAR; asm volatile("" ::: "memory");
    STAMP(1);
    STAGE(SB(0, 0), cB); STAGE(SA(0, 0), cA); STAGE(SB(0, 1), cB + HTB); STAGE(SA(0, 1), cA + HTB);
    if (wr == 1) BAR;
    WAIT_V(4); BAR;
    STAGE(SB(1, 0), cB + KT_STRIDE); STAGE(SA(1, 0), cA + KT_STRIDE); STAGE(SB(1, 1), cB + KT_STRIDE + HTB);
    WAIT_V(6); BAR;
    for (int t = 0; t < 64; t += 2) {
        const int t2 = (t + 2) & 63;
        const gchar* a1 = cA + (size_t)(t + 1) * KT_STRIDE;
        const gchar* a2 = cA + (size_t)t2 * KT_STRIDE; const gchar* b2 = cB + (size_t)t2 * KT_STRIDE;
        const gchar* a3 = a2 + KT_STRIDE; const gchar* b3 = b2 + KT_STRIDE;
        LDB(B0, 0, 0); SCHED; LDA(At, 0, 0); STAGE(SA(1, 1), a1 + HTB);
        WAIT_L(8); BAR; WAIT_L(0); MMA(0, 0, At, B0);
        if ((t2 & 7) == 0 && t2 != 0) { WAIT_STEP(t2 >> 3); STAMP(1 + (t2 >> 3)); }
        asm volatile("" ::: "memory"); BAR; SCHED;
        LDB(B1, 0, 1); STAGE(SB(0, 0), b2);
        BAR; WAIT_L(0); MMA(0, 1, At, B1); BAR;
        LDA(At, 0, 1); STAGE(SA(0, 0), a2);
        BAR; WAIT_L(0); MMA(1, 0, At, B0); BAR; SCHED;
        STAGE(SB(0, 1), b2 + HTB);
        WAIT_V(6); BAR; MMA(1, 1, At, B1); BAR;
        LDB(B0, 1, 0); SCHED; LDA(At, 1, 0); STAGE(SA(0, 1), a2 + HTB);
        WAIT_L(8); BAR; WAIT_L(0); MMA(0, 0, At, B0); BAR; SCHED;
        LDB(B1, 1, 1); STAGE(SB(1, 0), b3);
        BAR; WAIT_L(0); MMA(0, 1, At, B1); BAR;
        LDA(At, 1, 1); STAGE(SA(1, 0), a3);
        BAR; WAIT_L(0); MMA(1, 0, At, B0); BAR; SCHED;
        STAGE(SB(1, 1), b3 + HTB);
        WAIT_V(6); BAR; MMA(1, 1, At, B1); BAR;
    }
    if (wr == 0) BAR;
    STAMP(9);

    int tz = threadIdx.x; asm volatile("" : "+v"(tz));
    const int fr = tz & 15, fq = (tz >> 4) & 3;
    const unsigned lo = (unsigned)(fq * 4) * (NCOLS * 4) + (unsigned)fr * 16u;
    const unsigned soff = (unsigned)(tz & 63) * 16u;
    const size_t torg = ((size_t)((mt & 1) * 256 + wr * 64) * NCOLS + nt * 256 + wc * 64) * 4;
    const int slot = (mt & 1) * 16 + nt;
    const unsigned qnan = 0x7fc00000u;
#define ROWOFF(c, g, e) ((size_t)(((c) >> 1) * 128 + (((c) & 1) * 2 + ((g) >> 1)) * 16 + ((g) & 1) * 2 + ((e) >> 2)) * (NCOLS * 4))
    unsigned u[4][4][8];
    {
        const bool bad = __builtin_amdgcn_readfirstlane(__hip_atomic_load(tmo, __ATOMIC_RELAXED, __HIP_MEMORY_SCOPE_AGENT)) == call;
#pragma unroll
        for (int c = 0; c < 4; ++c)
#pragma unroll
            for (int g = 0; g < 4; ++g)
#pragma unroll
                for (int e = 0; e < 8; ++e) {
                    const float v = acc[c >> 1][(e >> 1) & 1][(c & 1) * 2 + (g >> 1)][e & 1][(g & 1) * 2 + (e >> 2)];
                    u[c][g][e] = bad ? qnan : __builtin_bit_cast(unsigned, v);
                }
    }
    if (dd == 0) {
        gchar* obase = (gchar*)p.out[rank] + torg + (size_t)rank * NLOC * 4;
#pragma unroll
        for (int c = 0; c < 4; ++c)
#pragma unroll
            for (int g = 0; g < 4; ++g)
#pragma unroll
                for (int h = 0; h < 2; ++h) {
                    const u32x4 v = {u[c][g][4 * h], u[c][g][4 * h + 1], u[c][g][4 * h + 2], u[c][g][4 * h + 3]};
                    STORE16(lo, v, obase + ROWOFF(c, g, 4 * h), 63);
                }
        STAMP(10);
    }
    { int dz = dd; asm volatile("" : "+s"(dz)); if (dz == 0) return; }
    unsigned ebase;
    {
        unsigned mx = 0;
#pragma unroll
        for (int c = 0; c < 4; ++c)
#pragma unroll
            for (int g = 0; g < 4; ++g) {
#pragma unroll
                for (int e = 0; e < 8; ++e) { u[c][g][e] = __builtin_amdgcn_alignbit(u[c][g][e], u[c][g][e], 31); mx = max(mx, u[c][g][e]); }
                asm volatile("" : "+v"(u[c][g][0]), "+v"(u[c][g][1]), "+v"(u[c][g][2]), "+v"(u[c][g][3]), "+v"(u[c][g][4]), "+v"(u[c][g][5]), "+v"(u[c][g][6]), "+v"(u[c][g][7]));
            }
#pragma unroll
        for (int sft = 1; sft < 64; sft <<= 1) mx = max(mx, (unsigned)__shfl_xor((int)mx, sft));
        const unsigned emax = (unsigned)__builtin_amdgcn_readfirstlane(mx) >> 24;
        ebase = emax > 15u ? emax - 15u : 0u;
        const unsigned B = ebase << 24;
        gchar* rbase = (gchar*)p.out[dest] + torg + (size_t)rank * NLOC * 4;
#pragma unroll
        for (int c = 0; c < 4; ++c)
#pragma unroll
            for (int g = 0; g < 4; ++g) {
                unsigned t[8], mn = 0xffffffffu;
#pragma unroll
                for (int e = 0; e < 8; ++e) { t[e] = __builtin_elementwise_sub_sat(u[c][g][e], B); mn = min(mn, t[e]); }
                if (__any(mn < (1u << 24))) {
#pragma unroll
                    for (int e = 0; e < 8; ++e)
                        if (t[e] < (1u << 24)) {
                            t[e] = min(u[c][g][e], 2u);
                            if (u[c][g][e] > 1u) {
                                const unsigned raw = __builtin_amdgcn_alignbit(u[c][g][e], u[c][g][e], 1);
                                asm volatile("global_store_dword %0, %1, %2 offset:%3 sc0 sc1" :: "v"(lo), "v"(raw), "s"(rbase + ROWOFF(c, g, e)), "i"((e & 3) * 4) : "memory");
                            }
                        }
                }
#pragma unroll
                for (int e = 0; e < 7; ++e) u[c][g][e] = ((t[7] << (4 + 4 * e)) & 0xf0000000u) | t[e];
                asm volatile("" : "+v"(u[c][g][0]), "+v"(u[c][g][1]), "+v"(u[c][g][2]), "+v"(u[c][g][3]), "+v"(u[c][g][4]), "+v"(u[c][g][5]), "+v"(u[c][g][6]));
            }
    }
    STAMP(12);
    if (wid == 0) {
        if (nt < 8) poll_ge((gu32*)(myws + OFF_PDONE), (call - 1u) * NPROD + 2u * (unsigned)((mt & 1) * 8 + nt) + 1u, tmo, call);
        else poll_ge((gu32*)(myws + OFF_GATE + (mt * 8 + nt - 8) * 128), call, tmo, call);
        STAMP(13);
    } else {
        while (__builtin_amdgcn_readfirstlane(lctl[1]) < (unsigned)wid) __builtin_amdgcn_s_sleep(4);
    }
    {
        gchar* pk = (gchar*)p.ws[dest] + OFF_PK + (size_t)((rank * 32 + slot) * 8 + wid) * PK_WAVE;
#pragma unroll
        for (int i = 0; i < 28; ++i) {
#define PKW(n) u[(n) / 28][((n) % 28) / 7][(n) % 7]
            const u32x4 v = {PKW(4 * i), PKW(4 * i + 1), PKW(4 * i + 2), PKW(4 * i + 3)};
            STORE16(soff, v, pk + i * 1024, EW - 1);
        }
        lctl[1] = (unsigned)wid + 1u;
        if (nt < 8 && tz == 448) __hip_atomic_store((gu32*)(myws + OFF_GATE + (mt * 8 + nt) * 128), call, __ATOMIC_RELAXED, __HIP_MEMORY_SCOPE_AGENT);
        asm volatile("s_waitcnt vmcnt(0)" ::: "memory");
        if ((tz & 63) == 0)
            __hip_atomic_store((gu32*)((gchar*)p.ws[dest] + OFF_OUTFLAG + ((rank * 32 + slot) * 8 + wid) * 64), (call << 8) | ebase, __ATOMIC_RELAXED, __HIP_MEMORY_SCOPE_SYSTEM);
        if (wid == 7) STAMPW(10);
    }
    {
        const int src = (rank - dd) & (NRANK - 1);
        const unsigned f = __builtin_amdgcn_readfirstlane(poll_ge((gu32*)(myws + OFF_OUTFLAG + ((src * 32 + slot) * 8 + wid) * 64), call << 8, tmo, call));
        if (wid == 7) STAMPW(14);
        const bool bad = __builtin_amdgcn_readfirstlane(__hip_atomic_load(tmo, __ATOMIC_RELAXED, __HIP_MEMORY_SCOPE_AGENT)) == call;
        const unsigned B = (f & 255u) << 24;
        gchar* pk = myws + OFF_PK + (size_t)((src * 32 + slot) * 8 + wid) * PK_WAVE;
        gchar* obase = (gchar*)p.out[rank] + torg + (size_t)src * NLOC * 4;
        const __amdgpu_buffer_rsrc_t rs = __builtin_amdgcn_make_buffer_rsrc((void*)pk, 0, bad ? 0 : (int)PK_WAVE, 0x00020000);
        const __amdgpu_buffer_rsrc_t ro = __builtin_amdgcn_make_buffer_rsrc((void*)obase, 0, 192 * NCOLS * 4, 0x00020000);
        u32x4 in[28];
#pragma unroll
        for (int i = 0; i < 28; ++i) in[i] = __builtin_amdgcn_raw_buffer_load_b128(rs, soff, i * 1024, 17);
#pragma unroll
        for (int c = 0; c < 4; ++c)
#pragma unroll
            for (int g = 0; g < 4; ++g) {
                unsigned t[8], v[8], mn;
                t[7] = 0;
#pragma unroll
                for (int e = 0; e < 7; ++e) {
                    const int n = c * 28 + g * 7 + e;
                    const unsigned dw = in[n >> 2][n & 3];
                    t[7] = __builtin_amdgcn_alignbit(t[7], dw, 28); t[e] = dw & 0x0fffffffu;
                }
                mn = t[0];
#pragma unroll
                for (int e = 0; e < 8; ++e) { mn = min(mn, t[e]); const unsigned x = t[e] + B; v[e] = __builtin_amdgcn_alignbit(x, x, 1); }
                if (__any(mn < (1u << 24))) {
#pragma unroll
                    for (int e = 0; e < 8; ++e) {
                        const unsigned x = bad ? qnan : (t[e] < (1u << 24) ? t[e] << 31 : v[e]);
                        if (t[e] != 2u) __builtin_amdgcn_raw_buffer_store_b32(x, ro, lo + (e & 3) * 4, (int)ROWOFF(c, g, e), 17);
                    }
                } else {
                    __builtin_amdgcn_raw_buffer_store_b128((u32x4){v[0], v[1], v[2], v[3]}, ro, lo, (int)ROWOFF(c, g, 0), 17);
                    __builtin_amdgcn_raw_buffer_store_b128((u32x4){v[4], v[5], v[6], v[7]}, ro, lo, (int)ROWOFF(c, g, 4), 17);
                }
            }
    }
}

extern "C" void kernel_launch(int rank, int nranks, ncclComm_t comm,
                              hipStream_t stream,
                              void* const* d_in, const int* in_sizes, int n_in,
                              void* d_out, int out_size,
                              const int* in_dtypes, int out_dtype,
                              void* const* d_out_all,
                              void* const* d_ws_all, size_t ws_bytes) {
    if (nranks != NRANK || (unsigned)rank >= (unsigned)NRANK || ws_bytes < WS_NEED || n_in < 2 ||
        in_sizes[0] != 4096 * KSH || in_sizes[1] != KSH * NCOLS || out_size != 512 * NCOLS) {
        fprintf(stderr, "unexpected problem shape\n");
        return;
    }
    Params p;
    p.x = (const float*)d_in[0]; p.w = (const float*)d_in[1];
    for (int r = 0; r < NRANK; ++r) { p.out[r] = (float*)d_out_all[r]; p.ws[r] = (char*)d_ws_all[r]; }
    p.rank = rank; p.pad = 0;
    gemm_rs_kernel<<<GRID, 512, 0, stream>>>(p);
    HIP_CHECK(hipGetLastError());
}
```

```cpp
#include <hip/hip_runtime.h>
#include <rccl/rccl.h>
#include <cstdio>

#define HIP_CHECK(expr) do { hipError_t _e = (expr); if (_e != hipSuccess) { \
    fprintf(stderr, "HIP error %s at %s:%d\n", hipGetErrorString(_e), __FILE__, __LINE__); \
    return; } } while (0)

#define LAS __attribute__((address_space(3)))
#define GAS __attribute__((address_space(1)))
typedef _Float16 half8 __attribute__((ext_vector_type(8)));
typedef _Float16 half2v __attribute__((ext_vector_type(2)));
typedef float f32x2 __attribute__((ext_vector_type(2)));
typedef float f32x4 __attribute__((ext_vector_type(4)));
typedef unsigned u32x4 __attribute__((ext_vector_type(4)));
typedef GAS unsigned gu32;
typedef GAS char gchar;

constexpr int NRANK = 8;
constexpr int KSH = 512;
constexpr int NCOLS = 32768, NLOC = 4096;
constexpr int GRID = 256, NPROD = 32;
constexpr size_t CTRL_BYTES = 1u << 20;
constexpr size_t PANEL = 32768;
constexpr size_t KT_STRIDE = 16 * PANEL;
constexpr size_t MAT_BYTES = 64 * KT_STRIDE;
constexpr size_t PI_ITEM = 7 * 1024 + 256;
constexpr size_t OFF_PI = 3 * MAT_BYTES;
constexpr size_t BUF_BYTES = 3 * MAT_BYTES + (33u << 20);
constexpr size_t PK_WAVE = 28 * 1024;
constexpr size_t RAW_WAVE = 128 * 256;
constexpr size_t OFF_PK = CTRL_BYTES + 2 * BUF_BYTES;
constexpr size_t OFF_RAW = OFF_PK + (size_t)GRID * 8 * PK_WAVE;
constexpr size_t OFF_PARK = OFF_RAW + (size_t)GRID * 8 * RAW_WAVE;
constexpr size_t WS_NEED = OFF_PARK + (size_t)GRID * 8 * 32768;
constexpr int OFF_TMO = 0, OFF_QUEUE = 20480 + 256, OFF_PDONE = 24576 + 512, OFF_CDONE = 28672 + 768;
constexpr int OFF_GATE = 4096;
constexpr int OFF_INFLAG = 32768;
constexpr int OFF_CFLAG = 65536;
constexpr int OFF_OUTFLAG = 131072;
constexpr int OFF_STAMP = 262144;
constexpr int OFF_TICKET = 393216;
constexpr unsigned long long DEADLINE_TICKS = 50000000ull;
#define LAG 7
#define EW 3
#define PW 3
#define NPUSH 36
#define PX 4
#define DSTAG 800
#define PSL 16
#define ISL 16
#define GAUX 17
#define NTIN " nt"
#define NTOUT " nt"
#define EARLY 1
#define UW 16
#define CW 3
constexpr int NCONV = 8 * (64 + 512);
constexpr int NITEM = 960;
constexpr unsigned QUEUE_PER_CALL = NITEM - NPUSH - 2 * NPROD + NPROD;

struct Params {
    const float* x; const float* w;
    float* out[NRANK];
    char* ws[NRANK];
    int rank; int pad;
};

#define STAMPW(k) do {} while (0)
#define STAMP(k) do {} while (0)

template <int SLEEP = 8>
__device__ __forceinline__ unsigned poll_ge(gu32* addr, unsigned need, gu32* tmo, unsigned call) {
    const unsigned long long t0 = __builtin_amdgcn_s_memrealtime();
    unsigned v;
    for (unsigned n = 1;; ++n) {
        v = __hip_atomic_load(addr, __ATOMIC_RELAXED, __HIP_MEMORY_SCOPE_SYSTEM);
        if (__all(v >= need)) break;
        if ((n & 15u) == 0u) {
            const unsigned tv = __builtin_amdgcn_readfirstlane(__hip_atomic_load(tmo, __ATOMIC_RELAXED, __HIP_MEMORY_SCOPE_AGENT));
            if (tv == call || __builtin_amdgcn_s_memrealtime() - t0 > DEADLINE_TICKS) {
                __hip_atomic_store(tmo, call, __ATOMIC_RELAXED, __HIP_MEMORY_SCOPE_AGENT);
                break;
            }
        }
        __builtin_amdgcn_s_sleep(SLEEP);
    }
    asm volatile("" ::: "memory");
    return v;
}

__device__ __forceinline__ unsigned pk16(float a, float b) {
    f32x2 v = {a, b};
    return __builtin_bit_cast(unsigned, __builtin_convertvector(v, half2v));
}

typedef unsigned short u16x2 __attribute__((ext_vector_type(2)));
__device__ __forceinline__ unsigned pk_max(unsigned a, unsigned b) { return __builtin_bit_cast(unsigned, __builtin_elementwise_max(__builtin_bit_cast(u16x2, a), __builtin_bit_cast(u16x2, b))); }
__device__ __forceinline__ unsigned pk_subs(unsigned a, unsigned b) { return __builtin_bit_cast(unsigned, __builtin_elementwise_sub_sat(__builtin_bit_cast(u16x2, a), __builtin_bit_cast(u16x2, b))); }

__device__ __forceinline__ unsigned pack_item(unsigned (&D)[32], unsigned (&P)[28]) {
    unsigned mx = 0;
#pragma unroll
    for (int i = 0; i < 32; ++i) mx = pk_max(mx, D[i] & 0x7c007c00u);
    mx = max(mx & 0xffffu, mx >> 16);
    const unsigned b10 = mx > (7u << 10) ? mx - (7u << 10) : 0u, BB = b10 * 0x10001u;
    unsigned st = 0xffffffffu, sumk = 0, sumz = 0;
#pragma unroll
    for (int i = 31; i >= 0; --i) {
        const unsigned E = D[i] & 0x7c007c00u, C = pk_subs(E, BB), K = pk_subs(BB, E) >> 10, Z = pk_subs(0x00010001u, C);
        sumk += K; sumz += Z;
        st = ((st << (Z >> 16)) | (Z >> 16)) << (K >> 16);
        st = ((st << (Z & 1u)) | (Z & 1u)) << (K & 31u);
        D[i] = (D[i] & 0x83ff83ffu) | C;
    }
    const unsigned nk = (sumk & 0xffffu) + (sumk >> 16), nz = (sumz & 0xffffu) + (sumz >> 16);
#pragma unroll
    for (int g = 0; g < 4; ++g) {
        const unsigned t7 = (D[8 * g + 7] & 0x1fff1fffu) | ((D[8 * g + 7] >> 2) & 0x20002000u);
#pragma unroll
        for (int i = 0; i < 7; ++i) P[7 * g + i] = D[8 * g + i] | ((t7 << (13 - 2 * i)) & 0x60006000u);
    }
    return (st & 0x03ffffffu) | (b10 << 16) | (nk != 0u && nk + nz > 26u ? 0x80000000u : 0u);
}
__device__ __forceinline__ void unpack_group(const unsigned (&P)[7], unsigned BB, int& st, unsigned (&H)[8]) {
    unsigned t7 = 0;
#pragma unroll
    for (int i = 0; i < 7; ++i) { t7 |= (P[i] >> (13 - 2 * i)) & (0x00030003u << (2 * i)); H[i] = P[i] & 0x9fff9fffu; }
    H[7] = (t7 & 0x1fff1fffu) | ((t7 << 2) & 0x80008000u);
#pragma unroll
    for (int i = 0; i < 8; ++i) {
        const unsigned T = H[i];
        unsigned h = T + BB;
        { const bool z = (T & 0x00001c00u) == 0u; const unsigned k = (unsigned)__builtin_ctz((unsigned)st); h -= z ? k << 10 : 0u; st >>= z ? k + 1u : 0u; }
        { const bool z = (T & 0x1c000000u) == 0u; const unsigned k = (unsigned)__builtin_ctz((unsigned)st); h -= z ? k << 26 : 0u; st >>= z ? k + 1u : 0u; }
        H[i] = h;
    }
}
constexpr int pull_younger(int k) {
    int n = 0;
    for (int j = k - PW; j < k; ++j) {
        const int jj = (j + 7) % 7;
        if (j > k - PW) n += (j + PW) % 7 == 0 ? 2 : 1;
        if (jj == 1 || jj == 3 || jj == 5 || jj == 6) n += 2;
    }
    return n;
}

__device__ __forceinline__ void item_decode(int it, int src, int& s, int& r) {
    constexpr int n1 = 88 * LAG, n2 = n1 + 120 * (8 - LAG);
    int q, e;
    if (it < n1) { q = it / 88; e = it - 88 * q; }
    else if (it < n2) { q = (it - n1) / 120; e = it - n1 - 120 * q; q += LAG; }
    else { q = 8 + ((it - n2) >> 5); e = 88 + ((it - n2) & 31); }
    s = e >= 88 ? q - LAG : q;
    r = e < 56 ? (8 * src + 8 + e) & 63 : e + 8;
}

#define UNIT_LOAD(B, S_, R_, H_) do { const bool isx_ = (R_) < 64; \
        const gchar* b_ = isx_ ? sX + (size_t)((R_) * 64 + (H_) * 32) * (KSH * 4) + (size_t)(S_) * 256 \
                               : sW + (size_t)((S_) * 64 + (H_) * 32) * (NCOLS * 4) + (size_t)((R_) - 64) * 256; \
        const unsigned vo_ = isx_ ? xoff : woff; \
        const gchar* b1_ = b_ + (isx_ ? 16 : 1 * NCOLS * 4);          const gchar* b2_ = b_ + (isx_ ? 128 : 2 * NCOLS * 4); \
        const gchar* b3_ = b_ + (isx_ ? 144 : 3 * NCOLS * 4);         const gchar* b4_ = b_ + (isx_ ? 32768 : 4 * NCOLS * 4); \
        const gchar* b5_ = b_ + (isx_ ? 32768 + 16 : 5 * NCOLS * 4);  const gchar* b6_ = b_ + (isx_ ? 32768 + 128 : 6 * NCOLS * 4); \
        const gchar* b7_ = b_ + (isx_ ? 32768 + 144 : 7 * NCOLS * 4); \
        asm volatile("global_load_dwordx4 %0, %8, %9" NTIN "\n\tglobal_load_dwordx4 %1, %8, %10" NTIN "\n\t" \
                     "global_load_dwordx4 %2, %8, %11" NTIN "\n\tglobal_load_dwordx4 %3, %8, %12" NTIN "\n\t" \
                     "global_load_dwordx4 %4, %8, %13" NTIN "\n\tglobal_load_dwordx4 %5, %8, %14" NTIN "\n\t" \
                     "global_load_dwordx4 %6, %8, %15" NTIN "\n\tglobal_load_dwordx4 %7, %8, %16" NTIN "" \
                     : "+v"(B[0]), "+v"(B[1]), "+v"(B[2]), "+v"(B[3]), "+v"(B[4]), "+v"(B[5]), "+v"(B[6]), "+v"(B[7]) \
                     : "v"(vo_), "s"(b_), "s"(b1_), "s"(b2_), "s"(b3_), "s"(b4_), "s"(b5_), "s"(b6_), "s"(b7_) : "memory"); } while (0)
#define UNIT_WAIT(B, N) asm volatile("s_waitcnt vmcnt(%8)" : "+v"(B[0]), "+v"(B[1]), "+v"(B[2]), "+v"(B[3]), \
                                     "+v"(B[4]), "+v"(B[5]), "+v"(B[6]), "+v"(B[7]) : "i"(N) : "memory")
#define UNIT_CVT(O, B, R_) do { if ((R_) < 64) { \
            _Pragma("unroll") for (int b = 0; b < 4; ++b) { \
                const f32x4 lo_ = __builtin_bit_cast(f32x4, B[2 * b]), hi_ = __builtin_bit_cast(f32x4, B[2 * b + 1]); \
                O[b] = (u32x4){pk16(lo_[0], lo_[1]), pk16(lo_[2], lo_[3]), pk16(hi_[0], hi_[1]), pk16(hi_[2], hi_[3])}; } \
        } else { \
            _Pragma("unroll") for (int j = 0; j < 4; ++j) \
                O[j] = (u32x4){pk16(__builtin_bit_cast(f32x4, B[0])[j], __builtin_bit_cast(f32x4, B[1])[j]), \
                               pk16(__builtin_bit_cast(f32x4, B[2])[j], __builtin_bit_cast(f32x4, B[3])[j]), \
                               pk16(__builtin_bit_cast(f32x4, B[4])[j], __builtin_bit_cast(f32x4, B[5])[j]), \
                               pk16(__builtin_bit_cast(f32x4, B[6])[j], __builtin_bit_cast(f32x4, B[7])[j])}; } \
        asm volatile("" : "+v"(O[0]), "+v"(O[1]), "+v"(O[2]), "+v"(O[3])); } while (0)
#define ITEM_OFF(R_) ((R_) < 64 ? (size_t)(R_) * 8192 : (size_t)(((R_) - 64) >> 2) * PANEL + (size_t)((R_) & 3) * 4096)
#define UNIT_DST(BASE, R_, H_) const bool isxd_ = (R_) < 64; \
        gchar* t0_ = (BASE) + (isxd_ ? (H_) * 4096 : (H_) * 1024); \
        gchar* t1_ = t0_ + (isxd_ ? 1024 : 2048); gchar* t2_ = t0_ + (isxd_ ? 2048 : 16384); gchar* t3_ = t0_ + (isxd_ ? 3072 : 16384 + 2048)
#define STOREO(VOFF, DATA, BASE) asm volatile("global_store_dwordx4 %0, %1, %2 sc0 sc1" NTOUT "\n\ts_nop 1" :: "v"(VOFF), "v"(DATA), "s"(BASE) : "memory")
#define STORE16(VOFF, DATA, BASE, N) asm volatile("s_waitcnt vmcnt(%3)\n\tglobal_store_dwordx4 %0, %1, %2 sc0 sc1\n\ts_nop 1" \
                                                  :: "v"(VOFF), "v"(DATA), "s"(BASE), "i"(N) : "memory")

__global__ __launch_bounds__(512, 2) void gemm_rs_kernel(const Params p) {
    __shared__ __attribute__((aligned(16))) unsigned char shm[131072 + 32];
    LAS unsigned char* lds = (LAS unsigned char*)shm;
    volatile LAS unsigned* lctl = (volatile LAS unsigned*)(lds + 131072);
    const int tid = threadIdx.x, wid = __builtin_amdgcn_readfirstlane(tid >> 6), lane = tid & 63;
    const int rank = p.rank & (NRANK - 1), bid = blockIdx.x;
    gchar* myws = (gchar*)p.ws[rank];
    gu32* tmo = (gu32*)(myws + OFF_TMO);

    if (tid == 0) {
        lctl[0] = __hip_atomic_fetch_add((gu32*)(myws + OFF_TICKET + bid * 128), 1u, __ATOMIC_RELAXED, __HIP_MEMORY_SCOPE_AGENT) + 1u;
        lctl[1] = 0u;
    }
    unsigned call; size_t bufoff;
#define GET_CALL() do { asm volatile("s_waitcnt lgkmcnt(0)" ::: "memory"); __builtin_amdgcn_s_barrier(); asm volatile("" ::: "memory"); \
        call = __builtin_amdgcn_readfirstlane(lctl[0]); bufoff = CTRL_BYTES + (size_t)(call & 1u) * BUF_BYTES; } while (0)

    const int xcd = bid & 7, idx = bid >> 3, dd = idx >> 2;
    const int dest = (rank + dd) & (NRANK - 1);
    const int mt = 2 * dest + (xcd & 1), nt = 2 * (xcd >> 1) + (idx & 1) + 8 * ((idx >> 1) & 1);
    const int hidx = (dd - 1) * 16 + (bid & 15);

    const unsigned soff0 = (unsigned)lane * 16u;
    const gchar* sX = (const gchar*)p.x;
    const unsigned xoff = (unsigned)(lane & 15) * (KSH * 4) + (unsigned)(lane >> 4) * 32u;
    const unsigned woff = (unsigned)(lane >> 4) * 8u * (NCOLS * 4) + (unsigned)(lane & 15) * 16u;
    if (bid >= NPROD && nt >= 8 && wid >= 1 && wid <= CW) {
        u32x4 buf0[8], buf1[8], o[4], o1[4];
#pragma unroll
        for (int b = 0; b < 8; ++b) { buf0[b] = (u32x4){0u, 0u, 0u, 0u}; buf1[b] = (u32x4){0u, 0u, 0u, 0u}; }
        bool first = true;
        for (int job = NPUSH + (wid - 1) * 112 + hidx; job < NCONV; job += CW * 112) {
            constexpr int n1 = 320 * LAG, n2 = n1 + 576 * (8 - LAG);
            int q, e;
            if (job < n1) { q = job / 320; e = job - 320 * q; }
            else if (job < n2) { q = (job - n1) / 576; e = job - n1 - 576 * q; q += LAG; }
            else { q = 8 + ((job - n2) >> 8); e = 320 + ((job - n2) & 255); }
            const int s = e >= 320 ? q - LAG : q;
            const int d = e < 64 ? rank : (rank + 1 + ((e - 64) & 7)) & (NRANK - 1);
            const int r = e < 64 ? (8 * rank + 8 + e) & 63 : 64 + ((e - 64) >> 3);
            const gchar* sW = (const gchar*)p.w + (size_t)d * NLOC * 4;
            UNIT_LOAD(buf0, s, r, 0); UNIT_LOAD(buf1, s, r, 1);
            if (first) { GET_CALL(); first = false; }
            gchar* tb = myws + bufoff + ITEM_OFF(r) + (e < 64 ? (size_t)(s * 8 + rank) * KT_STRIDE
                        : d == rank ? MAT_BYTES + (size_t)(s * 8 + rank) * KT_STRIDE : 2 * MAT_BYTES + (size_t)(d * 8 + s) * KT_STRIDE);
            UNIT_WAIT(buf0, 8); UNIT_CVT(o, buf0, r);
            { UNIT_DST(tb, r, 0); STORE16(soff0, o[0], t0_, 63); STORE16(soff0, o[1], t1_, 63); STORE16(soff0, o[2], t2_, 63); STORE16(soff0, o[3], t3_, 63); }
            UNIT_WAIT(buf1, 4); UNIT_CVT(o1, buf1, r);
            { UNIT_DST(tb, r, 1); STORE16(soff0, o1[0], t0_, 63); STORE16(soff0, o1[1], t1_, 63); STORE16(soff0, o1[2], t2_, 63); STORE16(soff0, o1[3], t3_, 63); }
            if (e < 56 || d != rank) {
                unsigned D[32], P[28];
#pragma unroll
                for (int i = 0; i < 16; ++i) { D[i] = o[i >> 2][i & 3]; D[16 + i] = o1[i >> 2][i & 3]; }
                const unsigned side = pack_item(D, P);
                gchar* pi = myws + bufoff + OFF_PI + (size_t)(e < 64 ? s * 64 + r : 512 + (d * 8 + s) * 64 + r - 64) * PI_ITEM;
#pragma unroll
                for (int i = 0; i < 7; ++i) { const u32x4 v = {P[4 * i], P[4 * i + 1], P[4 * i + 2], P[4 * i + 3]}; STORE16(soff0, v, pi + i * 1024, 63); }
                asm volatile("global_store_dword %0, %1, %2 sc0 sc1" :: "v"(soff0 >> 2), "v"(side), "s"(pi + 7168) : "memory");
            }
            asm volatile("s_waitcnt vmcnt(0)" ::: "memory");
            const int fo = rank * 4096 + (s * 128 + r) * 4;
#pragma unroll
            for (int k = 0; k < NRANK; ++k)
                if ((e < 56 || k == d) && lane == 0)
                    __hip_atomic_store((gu32*)((gchar*)p.ws[k] + (k == rank ? OFF_INFLAG : OFF_CFLAG) + fo), call, __ATOMIC_RELAXED, __HIP_MEMORY_SCOPE_SYSTEM);
        }
        if (lane == 0) __hip_atomic_fetch_add((gu32*)(myws + OFF_CDONE), 1u, __ATOMIC_RELAXED, __HIP_MEMORY_SCOPE_AGENT);
        if (wid == 1) STAMPW(15);
    } else if (bid >= NPROD && nt < 8 && wid == 1 && hidx < NPUSH) {
        u32x4 buf0[8], buf1[8], o0[4], o1[4];
#pragma unroll
        for (int b = 0; b < 8; ++b) { buf0[b] = (u32x4){0u, 0u, 0u, 0u}; buf1[b] = (u32x4){0u, 0u, 0u, 0u}; }
        const gchar* sW = sX;
        const int r = (8 * rank + 8 + hidx) & 63;
        UNIT_LOAD(buf0, 0, r, 0); UNIT_LOAD(buf1, 0, r, 1);
        GET_CALL();
        UNIT_WAIT(buf0, 8); UNIT_CVT(o0, buf0, 0);
        UNIT_WAIT(buf1, 0); UNIT_CVT(o1, buf1, 0);
        const size_t ioff = bufoff + (size_t)rank * KT_STRIDE + (size_t)r * 8192;
#pragma unroll
        for (int b = 0; b < 8; ++b)
            for (int k = 1; k <= NRANK; ++k) {
                gchar* t = (gchar*)p.ws[(rank + hidx + k) & (NRANK - 1)] + ioff + b * 1024;
                STORE16(soff0, b < 4 ? o0[b & 3] : o1[b & 3], t, UW - 1);
            }
        asm volatile("s_waitcnt vmcnt(0)" ::: "memory");
#pragma unroll
        for (int k = 0; k < NRANK; ++k)
            if (lane == 0) __hip_atomic_store((gu32*)((gchar*)p.ws[k] + OFF_INFLAG + rank * 4096 + r * 4), call, __ATOMIC_RELAXED, __HIP_MEMORY_SCOPE_SYSTEM);
        STAMPW(15);
    } else if (bid >= NPROD) {
        GET_CALL();
    } else if (wid == 0) {
        GET_CALL();
        const int psrc = (rank + 1 + (lane % 7)) & (NRANK - 1);
        bool more = true;
        for (int i = 0;; ++i) {
            int nn = NITEM;
            if (i < 2) nn = NPUSH + bid + i * NPROD;
            else if (more) {
                unsigned t = 0;
                if (lane == 0) t = __hip_atomic_fetch_add((gu32*)(myws + OFF_QUEUE), 1u, __ATOMIC_RELAXED, __HIP_MEMORY_SCOPE_AGENT);
                t = __builtin_amdgcn_readfirstlane(t) - (call - 1u) * QUEUE_PER_CALL + NPUSH + 2u * NPROD;
                nn = t < (unsigned)NITEM ? (int)t : NITEM;
            }
            if (nn < NITEM) { int s, r; item_decode(nn, psrc, s, r); poll_ge((gu32*)(myws + OFF_CFLAG + psrc * 4096 + (s * 128 + r) * 4), call, tmo, call); }
            lctl[4 + (i & 3)] = (unsigned)nn;
            asm volatile("s_waitcnt lgkmcnt(0)" ::: "memory");
            __builtin_amdgcn_s_barrier();
            if (!more) break;
            more = nn < NITEM;
        }
        if (lane == 0) __hip_atomic_fetch_add((gu32*)(myws + OFF_PDONE), 1u, __ATOMIC_RELAXED, __HIP_MEMORY_SCOPE_AGENT);
    } else {
        GET_CALL();
        const int src = (rank + wid) & (NRANK - 1);
        const gchar* rimg = (const gchar*)p.ws[src] + bufoff;
        gchar* limg = myws + bufoff + (size_t)src * KT_STRIDE;
        gu32* lflag = (gu32*)(myws + OFF_INFLAG + src * 4096);
        u32x4 in[7]; unsigned side = 0;
#pragma unroll
        for (int b = 0; b < 7; ++b) in[b] = (u32x4){0u, 0u, 0u, 0u};
        const unsigned soff4 = (unsigned)lane * 4u;
#define PI_OF(S_, R_) (rimg + OFF_PI + 3072 + (size_t)((R_) < 64 ? (S_) * 64 + (R_) : 512 + (rank * 8 + (S_)) * 64 + (R_) - 64) * PI_ITEM)
#define RAW_OF(S_, R_) (rimg + ITEM_OFF(R_) + ((R_) < 64 ? (size_t)((S_) * 8 + src) * KT_STRIDE : 2 * MAT_BYTES + (size_t)(rank * 8 + (S_)) * KT_STRIDE))
#define DST_OF(S_, R_) (limg + ITEM_OFF(R_) + ((R_) < 64 ? 0 : MAT_BYTES) + (size_t)(S_) * 8 * KT_STRIDE)
#define PULL(k, BASE) do { if ((k) == 0) asm volatile("global_load_dword %0, %1, %2 sc0 sc1" : "+v"(side) : "v"(soff4), "s"((BASE) + 4096) : "memory"); \
        asm volatile("global_load_dwordx4 %0, %1, %2 offset:%3 sc0 sc1" : "+v"(in[k]) : "v"(soff0), "s"(BASE), "i"((k) * 1024 - 3072) : "memory"); } while (0)
        __builtin_amdgcn_s_barrier();
        int cur = (int)__builtin_amdgcn_readfirstlane(lctl[4]), s, r, pflag = -1;
        item_decode(cur, src, s, r);
        const gchar* sb = PI_OF(s, r);
#pragma unroll
        for (int k = 0; k < PW; ++k) PULL(k, sb);
        if (wid == 1) STAMPW(24);
        for (int i = 0; cur < NITEM; ++i) {
            __builtin_amdgcn_s_barrier();
            if (wid == 1 && i % 5 == 0 && i <= 25) STAMPW(25 + i / 5);
            const int nxt = (int)__builtin_amdgcn_readfirstlane(lctl[4 + ((i + 1) & 3)]);
            int sn, rn; item_decode(nxt < NITEM ? nxt : cur, src, sn, rn);
            const gchar* sbn = nxt < NITEM ? PI_OF(sn, rn) : (const gchar*)myws + bufoff + OFF_PI + 3072;
            gchar* db = DST_OF(s, r);
            const bool isx = r < 64;
            const gchar* rraw = RAW_OF(s, r);
            asm volatile("" : "+s"(rraw), "+s"(db));
            int st = 0; unsigned BB = 0; bool ovf = false;
#pragma unroll
            for (int k = 0; k < 7; ++k) {
                if (k < PW && i == 0) asm volatile("s_waitcnt vmcnt(%0)" :: "i"(PW - 1) : "memory");
                asm volatile("s_waitcnt vmcnt(%2)" : "+v"(in[k]), "+v"(side) : "i"(pull_younger(k)) : "memory");
                if (k == 0) { st = (int)(side | 0xfc000000u); BB = ((side >> 16) & 0x7c00u) * 0x10001u; ovf = (int)side < 0;
                              asm volatile("" : "+v"(st), "+v"(BB)); }
                if (k == PW && pflag >= 0 && lane == 0) __hip_atomic_store(lflag + pflag, call, __ATOMIC_RELAXED, __HIP_MEMORY_SCOPE_SYSTEM);
                if (k + PW < 7) PULL((k + PW) % 7, sb); else PULL((k + PW) % 7, sbn);
                __builtin_amdgcn_sched_barrier(0);
                if (k == 1 || k == 3 || k == 5 || k == 6) {
                    const int g = k >> 1;
                    unsigned P[7], H[8];
#pragma unroll
                    for (int e = 0; e < 7; ++e) P[e] = in[(7 * g + e) >> 2][(7 * g + e) & 3];
                    unpack_group(P, BB, st, H);
                    u32x4 v0 = {H[0], H[1], H[2], H[3]}, v1 = {H[4], H[5], H[6], H[7]};
                    const unsigned o0 = soff0 + (isx ? g * 2048 : (g & 1) * 16384 + (g >> 1) * 1024), o1 = o0 + (isx ? 1024 : 2048);
                    if (ovf) asm volatile("global_load_dwordx4 %0, %2, %4 sc0 sc1\n\tglobal_load_dwordx4 %1, %3, %4 sc0 sc1\n\ts_waitcnt vmcnt(0)"
                                          : "+v"(v0), "+v"(v1) : "v"(o0), "v"(o1), "s"(rraw) : "memory");
                    STORE16(o0, v0, db, 63); STORE16(o1, v1, db, 63);
                }
            }
            pflag = s * 128 + r; cur = nxt; s = sn; r = rn; sb = sbn;
        }
        __builtin_amdgcn_s_barrier();
        asm volatile("s_waitcnt vmcnt(0)" : "+v"(in[0]), "+v"(in[1]), "+v"(in[2]), "+v"(in[3]), "+v"(side) :: "memory");
        if (lane == 0) __hip_atomic_store(lflag + pflag, call, __ATOMIC_RELAXED, __HIP_MEMORY_SCOPE_SYSTEM);
        STAMPW(16 + wid);
    }

    const int wr = wid >> 2, wc = wid & 3;
    const gchar* cA = myws + bufoff + (size_t)mt * PANEL;
    const gchar* cB = myws + bufoff + MAT_BYTES + (size_t)nt * PANEL;
    const unsigned voff = (unsigned)tid * 16u;
    const unsigned ldsw = (unsigned)wid * 1024u;
    const int aoff = wr * 8192 + lane * 16, boff = wc * 4096 + lane * 16;
    constexpr int HTB = 16384;
#define SA(b, h) (((b) * 2 + (h)) * HTB)
#define SB(b, h) ((4 + (b) * 2 + (h)) * HTB)
#define STAGE(bufo, gbase) do { _Pragma("unroll") for (int _i = 0; _i < 2; ++_i) \
        __builtin_amdgcn_global_load_lds((const GAS unsigned*)((gbase) + voff + _i * 8192), (LAS unsigned*)(lds + (bufo) + ldsw + _i * 8192), 16, 0, GAUX); } while (0)
#define LDA(dst, b, h) do { _Pragma("unroll") for (int m = 0; m < 4; ++m) _Pragma("unroll") for (int k = 0; k < 2; ++k) \
        dst[m][k] = *(const LAS half8*)(lds + SA(b, h) + aoff + m * 2048 + k * 1024); } while (0)
#define LDB(dst, b, h) do { _Pragma("unroll") for (int n = 0; n < 2; ++n) _Pragma("unroll") for (int k = 0; k < 2; ++k) \
        dst[n][k] = *(const LAS half8*)(lds + SB(b, h) + boff + n * 2048 + k * 1024); } while (0)
#define MMA(ai, bj, At_, Bt_) do { __builtin_amdgcn_s_setprio(1); \
        _Pragma("unroll") for (int m = 0; m < 4; ++m) _Pragma("unroll") for (int n = 0; n < 2; ++n) _Pragma("unroll") for (int k = 0; k < 2; ++k) \
            acc[ai][bj][m][n] = __builtin_amdgcn_mfma_f32_16x16x32_f16(At_[m][k], Bt_[n][k], acc[ai][bj][m][n], 0, 0, 0); \
        __builtin_amdgcn_s_setprio(0); } while (0)
#define WAIT_V(n) asm volatile("s_waitcnt vmcnt(" #n ")" ::: "memory")
#define WAIT_L(n) asm volatile("s_waitcnt lgkmcnt(" #n ")" ::: "memory")
#define BAR __builtin_amdgcn_s_barrier()
#define SCHED __builtin_amdgcn_sched_barrier(0)
#define WAIT_STEP(s) do { if (wid == 0) { int lz_ = lane; asm volatile("" : "+v"(lz_)); const int wh_ = lz_ >> 3; \
        const int it_ = wh_ < 4 && (lz_ & 7) != dest ? 4 * mt + wh_ : 64 + 4 * nt + (wh_ & 3); \
        poll_ge<ISL>((gu32*)(myws + OFF_INFLAG + (lz_ & 7) * 4096 + (s) * 512 + it_ * 4), call, tmo, call); } } while (0)

    f32x4 acc[2][2][4][2];
#pragma unroll
    for (int a = 0; a < 2; ++a)
#pragma unroll
        for (int b = 0; b < 2; ++b)
#pragma unroll
            for (int m = 0; m < 4; ++m)
#pragma unroll
                for (int n = 0; n < 2; ++n) acc[a][b][m][n] = (f32x4){0.f, 0.f, 0.f, 0.f};
    half8 At[4][2], B0[2][2], B1[2][2];

#define GEMM_LOOP(NT, AOFF, BOFF, HOOK) do { \
    { const gchar* a0 = AOFF(0); const gchar* b0 = BOFF(0); const gchar* a1 = AOFF(1); const gchar* b1 = BOFF(1); \
      STAGE(SB(0, 0), b0); STAGE(SA(0, 0), a0); STAGE(SB(0, 1), b0 + HTB); STAGE(SA(0, 1), a0 + HTB); \
      if (wr == 1) BAR; \
      WAIT_V(4); BAR; \
      STAGE(SB(1, 0), b1); STAGE(SA(1, 0), a1); STAGE(SB(1, 1), b1 + HTB); \
      WAIT_V(6); BAR; } \
    for (int t = 0; t < (NT); t += 2) { \
        const int t2 = t + 2 < (NT) ? t + 2 : 0;                        \
        const gchar* a1 = AOFF(t + 1); \
        const gchar* a2 = AOFF(t2); const gchar* b2 = BOFF(t2); \
        const gchar* a3 = AOFF(t2 + 1); const gchar* b3 = BOFF(t2 + 1); \
        LDB(B0, 0, 0); SCHED; LDA(At, 0, 0); STAGE(SA(1, 1), a1 + HTB); \
        WAIT_L(8); BAR; WAIT_L(0); MMA(0, 0, At, B0); \
        HOOK; \
        asm volatile("" ::: "memory"); BAR; SCHED; \
        LDB(B1, 0, 1); STAGE(SB(0, 0), b2); \
        BAR; WAIT_L(0); MMA(0, 1, At, B1); BAR; \
        LDA(At, 0, 1); STAGE(SA(0, 0), a2); \
        BAR; WAIT_L(0); MMA(1, 0, At, B0); BAR; SCHED; \
        STAGE(SB(0, 1), b2 + HTB); \
        WAIT_V(6); BAR; MMA(1, 1, At, B1); BAR; \
        LDB(B0, 1, 0); SCHED; LDA(At, 1, 0); STAGE(SA(0, 1), a2 + HTB); \
        WAIT_L(8); BAR; WAIT_L(0); MMA(0, 0, At, B0); BAR; SCHED; \
        LDB(B1, 1, 1); STAGE(SB(1, 0), b3); \
        BAR; WAIT_L(0); MMA(0, 1, At, B1); BAR; \
        LDA(At, 1, 1); STAGE(SA(1, 0), a3); \
        BAR; WAIT_L(0); MMA(1, 0, At, B0); BAR; SCHED; \
        STAGE(SB(1, 1), b3 + HTB); \
        WAIT_V(6); BAR; MMA(1, 1, At, B1); BAR; \
    } \
    if (wr == 0) BAR;                                                   \
    } while (0)

    const int src = (rank - dd) & (NRANK - 1);
    const bool early = dd != 0 && nt >= 8;
#define A_DIAG(t) (dA + (size_t)(t) * 8 * KT_STRIDE)
#define B_DIAG(t) (dB + (size_t)(t) * bstep)
#define DIAG(DELAY) do { \
        const gchar* dA = myws + bufoff + (size_t)rank * KT_STRIDE + (size_t)(2 * rank + (mt & 1)) * PANEL; \
        const gchar* dB = myws + bufoff + (size_t)nt * PANEL + (dd == 0 ? MAT_BYTES + (size_t)rank * KT_STRIDE : 2 * MAT_BYTES + (size_t)src * 8 * KT_STRIDE); \
        const size_t bstep = dd == 0 ? 8 * KT_STRIDE : KT_STRIDE; \
        BAR; \
        if (wid == 0) { poll_ge<64>((gu32*)(myws + OFF_CDONE), call * (112u * CW), tmo, call); \
            const unsigned long long t0_ = __builtin_amdgcn_s_memrealtime(); \
            while (__builtin_amdgcn_s_memrealtime() - t0_ < (unsigned long long)(DELAY)) __builtin_amdgcn_s_sleep(16); } \
        WAIT_V(0); BAR; asm volatile("" ::: "memory"); \
        GEMM_LOOP(8, A_DIAG, B_DIAG, ); \
        STAMP(16); } while (0)
#define ZERO_ACC() do { _Pragma("unroll") for (int a = 0; a < 2; ++a) _Pragma("unroll") for (int b = 0; b < 2; ++b) \
        _Pragma("unroll") for (int m = 0; m < 4; ++m) _Pragma("unroll") for (int n = 0; n < 2; ++n) acc[a][b][m][n] = (f32x4){0.f, 0.f, 0.f, 0.f}; } while (0)
    gchar* park = myws + OFF_PARK + (size_t)(bid * 8 + wid) * 32768;
    if (EARLY && early) {
        DIAG((dd & 3) * DSTAG);
#pragma unroll
        for (int i = 0; i < 32; ++i) STORE16(soff0, acc[i >> 4][(i >> 3) & 1][(i >> 1) & 3][i & 1], park + i * 1024, 63);
        ZERO_ACC();
        WAIT_V(0);
    }

#define KT_OF(t) ((size_t)(((t) / 7) * 8 + ((dest + 1 + (t) % 7) & 7)) * KT_STRIDE)
#define A_MAIN(t) (cA + KT_OF(t))
#define B_MAIN(t) (cB + KT_OF(t))
    WAIT_STEP(0);
    asm volatile("" ::: "memory"); BAR; asm volatile("" ::: "memory");
    STAMP(1);
    int have = 0;
    GEMM_LOOP(56, A_MAIN, B_MAIN, if ((t2 + 1) / 7 > have) { have = (t2 + 1) / 7; WAIT_STEP(have); STAMP(1 + have); });
    STAMP(9);

    int tz = threadIdx.x; asm volatile("" : "+v"(tz));
    const int fr = tz & 15, fq = (tz >> 4) & 3;
    const unsigned lo = (unsigned)(fq * 4) * (NCOLS * 4) + (unsigned)fr * 16u;
    const unsigned soff = (unsigned)(tz & 63) * 16u;
    const size_t torg = ((size_t)((mt & 1) * 256 + wr * 64) * NCOLS + nt * 256 + wc * 64) * 4;
    const unsigned qnan = 0x7fc00000u;
#define ROWOFF(c, g, e) ((size_t)(((c) >> 1) * 128 + (((c) & 1) * 2 + ((g) >> 1)) * 16 + ((g) & 1) * 2 + ((e) >> 2)) * (NCOLS * 4))
#define ACC(c, g, e) acc[(c) >> 1][((e) >> 1) & 1][((c) & 1) * 2 + ((g) >> 1)][(e) & 1][((g) & 1) * 2 + ((e) >> 2)]
    if (dd != 0) {
    unsigned u[4][4][8];
    {
        const bool bad = __builtin_amdgcn_readfirstlane(__hip_atomic_load(tmo, __ATOMIC_RELAXED, __HIP_MEMORY_SCOPE_AGENT)) == call;
#pragma unroll
        for (int c = 0; c < 4; ++c)
#pragma unroll
            for (int g = 0; g < 4; ++g)
#pragma unroll
                for (int e = 0; e < 8; ++e) { const float v = ACC(c, g, e); u[c][g][e] = bad ? qnan : __builtin_bit_cast(unsigned, v); }
    }
    unsigned ebase;
    {
        unsigned mx = 0;
#pragma unroll
        for (int c = 0; c < 4; ++c)
#pragma unroll
            for (int g = 0; g < 4; ++g) {
#pragma unroll
                for (int e = 0; e < 8; ++e) { u[c][g][e] = __builtin_amdgcn_alignbit(u[c][g][e], u[c][g][e], 31); mx = max(mx, u[c][g][e]); }
                asm volatile("" : "+v"(u[c][g][0]), "+v"(u[c][g][1]), "+v"(u[c][g][2]), "+v"(u[c][g][3]), "+v"(u[c][g][4]), "+v"(u[c][g][5]), "+v"(u[c][g][6]), "+v"(u[c][g][7]));
            }
#pragma unroll
        for (int sft = 1; sft < 64; sft <<= 1) mx = max(mx, (unsigned)__shfl_xor((int)mx, sft));
        const unsigned emax = (unsigned)__builtin_amdgcn_readfirstlane(mx) >> 24;
        ebase = emax > 15u ? emax - 15u : 0u;
        const unsigned B = ebase << 24;
        gchar* rbase = myws + OFF_RAW + (size_t)(bid * 8 + wid) * RAW_WAVE;
        const unsigned roff = (unsigned)(tz & 63) * 4u;
        gchar* pk = myws + OFF_PK + (size_t)(bid * 8 + wid) * PK_WAVE;
#pragma unroll
        for (int c = 0; c < 4; ++c) {
#pragma unroll
            for (int g = 0; g < 4; ++g) {
                unsigned t[8], mn = 0xffffffffu;
#pragma unroll
                for (int e = 0; e < 8; ++e) { t[e] = __builtin_elementwise_sub_sat(u[c][g][e], B); mn = min(mn, t[e]); }
                if (__any(mn < (1u << 24))) {
#pragma unroll
                    for (int e = 0; e < 8; ++e)
                        if (t[e] < (1u << 24)) {
                            t[e] = min(u[c][g][e], 2u);
                            if (u[c][g][e] > 1u) {
                                const unsigned raw = __builtin_amdgcn_alignbit(u[c][g][e], u[c][g][e], 1);
                                asm volatile("global_store_dword %0, %1, %2 offset:%3 sc0 sc1" :: "v"(roff), "v"(raw), "s"(rbase + (c * 4 + g) * 2048), "i"(e * 256) : "memory");
                            }
                        }
                }
#pragma unroll
                for (int e = 0; e < 7; ++e) u[c][g][e] = ((t[7] << (4 + 4 * e)) & 0xf0000000u) | t[e];
                asm volatile("" : "+v"(u[c][g][0]), "+v"(u[c][g][1]), "+v"(u[c][g][2]), "+v"(u[c][g][3]), "+v"(u[c][g][4]), "+v"(u[c][g][5]), "+v"(u[c][g][6]));
            }
#pragma unroll
            for (int i = 0; i < 7; ++i) {
#define PKW(n) u[c][(n) / 7][(n) % 7]
                const u32x4 v = {PKW(4 * i), PKW(4 * i + 1), PKW(4 * i + 2), PKW(4 * i + 3)};
                STORE16(soff, v, pk + (c * 7 + i) * 1024, 63);
            }
        }
        asm volatile("s_waitcnt vmcnt(0)" ::: "memory");
        if ((tz & 63) == 0)
            __hip_atomic_store((gu32*)((gchar*)p.ws[dest] + OFF_OUTFLAG + (bid * 8 + wid) * 64), (call << 8) | ebase, __ATOMIC_RELAXED, __HIP_MEMORY_SCOPE_SYSTEM);
        STAMP(12);
    }
    const __amdgpu_buffer_rsrc_t rp = __builtin_amdgcn_make_buffer_rsrc((void*)park, 0, EARLY && early ? 32768 : 0, 0x00020000);
#pragma unroll
    for (int i = 0; i < 32; ++i)
        acc[i >> 4][(i >> 3) & 1][(i >> 1) & 3][i & 1] = __builtin_bit_cast(f32x4, __builtin_amdgcn_raw_buffer_load_b128(rp, soff, i * 1024, 17));
    }
    if (!EARLY && early) DIAG(0);
    gchar* obase = (gchar*)p.out[rank] + torg + (size_t)src * NLOC * 4;
    if (dd != 0) {
        const unsigned f = __builtin_amdgcn_readfirstlane(poll_ge((gu32*)(myws + OFF_OUTFLAG + (bid * 8 + wid) * 64), call << 8, tmo, call));
        if (wid == 0) {
            STAMP(14);
            if (nt < 8) poll_ge<PSL>((gu32*)(myws + OFF_PDONE), (call - 1u) * NPROD + 2u * (unsigned)((mt & 1) * 8 + nt) + 1u, tmo, call);
            else poll_ge((gu32*)(myws + OFF_GATE + (mt * 8 + ((nt - 8) ^ PX)) * 128), call, tmo, call);
            STAMP(13);
        }
        const bool bad = __builtin_amdgcn_readfirstlane(__hip_atomic_load(tmo, __ATOMIC_RELAXED, __HIP_MEMORY_SCOPE_AGENT)) == call;
        const unsigned B = (f & 255u) << 24;
        const gchar* pk = (const gchar*)p.ws[bad ? rank : src] + OFF_PK + (size_t)(bid * 8 + wid) * PK_WAVE;
        const gchar* rbase = (const gchar*)p.ws[src] + OFF_RAW + (size_t)(bid * 8 + wid) * RAW_WAVE + (tz & 63) * 4;
#pragma unroll
        for (int c = 0; c < 4; ++c) {
            while (__builtin_amdgcn_readfirstlane(lctl[1]) < (unsigned)(c * 8 + wid)) __builtin_amdgcn_s_sleep(2);
            u32x4 in[7];
            const gchar* pc = pk + c * 7168 + 3072;
#pragma unroll
            for (int i = 0; i < 7; ++i)
                asm volatile("s_waitcnt vmcnt(%3)\n\tglobal_load_dwordx4 %0, %1, %2 offset:%4 sc0 sc1" : "=v"(in[i]) : "v"(soff), "s"(pc), "i"(EW - 1), "i"(i * 1024 - 3072) : "memory");
            lctl[1] = (unsigned)(c * 8 + wid) + 1u;
            if (c == 3 && nt < 8 && tz == 448) __hip_atomic_store((gu32*)(myws + OFF_GATE + (mt * 8 + nt) * 128), call, __ATOMIC_RELAXED, __HIP_MEMORY_SCOPE_AGENT);
            asm volatile("s_waitcnt vmcnt(0)" : "+v"(in[0]), "+v"(in[1]), "+v"(in[2]), "+v"(in[3]), "+v"(in[4]), "+v"(in[5]), "+v"(in[6]) :: "memory");
            if (c == 3 && wid == 7) STAMPW(10);
#pragma unroll
            for (int g = 0; g < 4; ++g) {
                unsigned t[8], v[8], mn;
                t[7] = 0;
#pragma unroll
                for (int e = 0; e < 7; ++e) {
                    const int n = g * 7 + e;
                    const unsigned dw = in[n >> 2][n & 3];
                    t[7] = __builtin_amdgcn_alignbit(t[7], dw, 28); t[e] = dw & 0x0fffffffu;
                }
                mn = t[0];
#pragma unroll
                for (int e = 0; e < 8; ++e) { mn = min(mn, t[e]); const unsigned x = t[e] + B; v[e] = __builtin_amdgcn_alignbit(x, x, 1); }
                if (__any(mn < (1u << 24)) || bad) {
#pragma unroll
                    for (int e = 0; e < 8; ++e) {
                        if (t[e] < (1u << 24)) v[e] = t[e] << 31;
                        if (t[e] == 2u && !bad) v[e] = __hip_atomic_load((const gu32*)(rbase + (c * 4 + g) * 2048 + e * 256), __ATOMIC_RELAXED, __HIP_MEMORY_SCOPE_SYSTEM);
                        if (bad) v[e] = qnan;
                    }
                }
#pragma unroll
                for (int e = 0; e < 8; ++e) ACC(c, g, e) += __builtin_bit_cast(float, v[e]);
                if (early) {
                    const f32x4 w0 = {ACC(c, g, 0), ACC(c, g, 1), ACC(c, g, 2), ACC(c, g, 3)}, w1 = {ACC(c, g, 4), ACC(c, g, 5), ACC(c, g, 6), ACC(c, g, 7)};
                    STOREO(lo, w0, obase + ROWOFF(c, g, 0)); STOREO(lo, w1, obase + ROWOFF(c, g, 4));
                }
            }
        }
    }
    if (!early) {
        DIAG(0);
        const bool bad = __builtin_amdgcn_readfirstlane(__hip_atomic_load(tmo, __ATOMIC_RELAXED, __HIP_MEMORY_SCOPE_AGENT)) == call;
#pragma unroll
        for (int c = 0; c < 4; ++c)
#pragma unroll
            for (int g = 0; g < 4; ++g)
#pragma unroll
                for (int h = 0; h < 2; ++h) {
                    f32x4 v = {ACC(c, g, 4 * h), ACC(c, g, 4 * h + 1), ACC(c, g, 4 * h + 2), ACC(c, g, 4 * h + 3)};
                    if (bad) v = __builtin_bit_cast(f32x4, (u32x4){qnan, qnan, qnan, qnan});
                    STOREO(lo, v, obase + ROWOFF(c, g, 4 * h));
                }
        STAMP(23);
    }
}

extern "C" void kernel_launch(int rank, int nranks, ncclComm_t comm,
                              hipStream_t stream,
                              void* const* d_in, const int* in_sizes, int n_in,
                              void* d_out, int out_size,
                              const int* in_dtypes, int out_dtype,
                              void* const* d_out_all,
                              void* const* d_ws_all, size_t ws_bytes) {
    if (nranks != NRANK || (unsigned)rank >= (unsigned)NRANK || ws_bytes < WS_NEED || n_in < 2 ||
        in_sizes[0] != 4096 * KSH || in_sizes[1] != KSH * NCOLS || out_size != 512 * NCOLS) {
        fprintf(stderr, "unexpected problem shape\n");
        return;
    }
    Params p;
    p.x = (const float*)d_in[0]; p.w = (const float*)d_in[1];
    for (int r = 0; r < NRANK; ++r) { p.out[r] = (float*)d_out_all[r]; p.ws[r] = (char*)d_ws_all[r]; }
    p.rank = rank; p.pad = 0;
    gemm_rs_kernel<<<GRID, 512, 0, stream>>>(p);
    HIP_CHECK(hipGetLastError());
}
```

```cpp
#include <hip/hip_runtime.h>
#include <rccl/rccl.h>
#include <cstdio>

#define HIP_CHECK(expr) do { hipError_t _e = (expr); if (_e != hipSuccess) { \
    fprintf(stderr, "HIP error %s at %s:%d\n", hipGetErrorString(_e), __FILE__, __LINE__); \
    return; } } while (0)

#define LAS __attribute__((address_space(3)))
#define GAS __attribute__((address_space(1)))
typedef _Float16 half8 __attribute__((ext_vector_type(8)));
typedef _Float16 half2v __attribute__((ext_vector_type(2)));
typedef float f32x2 __attribute__((ext_vector_type(2)));
typedef float f32x4 __attribute__((ext_vector_type(4)));
typedef unsigned u32x4 __attribute__((ext_vector_type(4)));
typedef GAS unsigned gu32;
typedef GAS char gchar;

constexpr int NRANK = 8;
constexpr int KSH = 512;
constexpr int NCOLS = 32768, NLOC = 4096;
constexpr int GRID = 256, NPROD = 32;
constexpr size_t CTRL_BYTES = 1u << 20;
constexpr size_t PANEL = 32768;
constexpr size_t KT_STRIDE = 16 * PANEL;
constexpr size_t MAT_BYTES = 64 * KT_STRIDE;
constexpr size_t PI_ITEM = 7 * 1024 + 256;
constexpr size_t OFF_PI = 3 * MAT_BYTES;
constexpr size_t BUF_BYTES = 3 * MAT_BYTES + (33u << 20);
constexpr size_t PK_WAVE = 28 * 1024;
constexpr size_t RAW_WAVE = 128 * 256;
constexpr size_t OFF_PK = CTRL_BYTES + 2 * BUF_BYTES;
constexpr size_t OFF_RAW = OFF_PK + (size_t)GRID * 8 * PK_WAVE;
constexpr size_t OFF_PARK = OFF_RAW + (size_t)GRID * 8 * RAW_WAVE;
constexpr size_t WS_NEED = OFF_PARK + (size_t)GRID * 8 * 32768;
constexpr int OFF_TMO = 0, OFF_QUEUE = 20480, OFF_PDONE = 24576 + 256, OFF_CDONE = 28672 + 768, OFF_BDONE = 24576 + 128;
constexpr int OFF_GATE = 4096;
constexpr int OFF_INFLAG = 32768;
constexpr int OFF_CFLAG = 65536;
constexpr int OFF_OUTFLAG = 131072;
constexpr int OFF_STAMP = 262144;
constexpr int OFF_TICKET = 393216;
constexpr unsigned long long DEADLINE_TICKS = 50000000ull;
#define OWNA 18
#define TGATE 500
#define XSTAG 20
#define EW 3
#define PW 2
#define NPUSH 28
#define PX 4
#define DSTAG 500
#define PSL 16
#define ISL 16
#define GAUX 17
#define NTIN " nt"
#define NTOUT " nt"
#define EARLY 0
#define PMAP 0
#define WIDE 4
#define WTOK 3
#define WBE 1
#define UW 16
#define CW 3
constexpr int NCONV = 8 * (64 + 512);
constexpr int NITEM = 960;
constexpr unsigned NSTR = PMAP ? 28 : 32;
constexpr unsigned QUEUE_PER_CALL = NITEM - NPUSH - 2 * NSTR + NSTR;

struct Params {
    const float* x; const float* w;
    float* out[NRANK];
    char* ws[NRANK];
    int rank; int pad;
};

#define STAMPW(k) do {} while (0)
#define STAMP(k) do {} while (0)

template <int SLEEP = 8>
__device__ __forceinline__ unsigned poll_ge(gu32* addr, unsigned need, gu32* tmo, unsigned call) {
    const unsigned long long t0 = __builtin_amdgcn_s_memrealtime();
    unsigned v;
    for (unsigned n = 1;; ++n) {
        v = __hip_atomic_load(addr, __ATOMIC_RELAXED, __HIP_MEMORY_SCOPE_SYSTEM);
        if (__all(v >= need)) break;
        if ((n & 15u) == 0u) {
            const unsigned tv = __builtin_amdgcn_readfirstlane(__hip_atomic_load(tmo, __ATOMIC_RELAXED, __HIP_MEMORY_SCOPE_AGENT));
            if (tv == call || __builtin_amdgcn_s_memrealtime() - t0 > DEADLINE_TICKS) {
                __hip_atomic_store(tmo, call, __ATOMIC_RELAXED, __HIP_MEMORY_SCOPE_AGENT);
                break;
            }
        }
        __builtin_amdgcn_s_sleep(SLEEP);
    }
    asm volatile("" ::: "memory");
    return v;
}

__device__ __forceinline__ unsigned pk16(float a, float b) {
    f32x2 v = {a, b};
    return __builtin_bit_cast(unsigned, __builtin_convertvector(v, half2v));
}

typedef unsigned short u16x2 __attribute__((ext_vector_type(2)));
__device__ __forceinline__ unsigned pk_max(unsigned a, unsigned b) { return __builtin_bit_cast(unsigned, __builtin_elementwise_max(__builtin_bit_cast(u16x2, a), __builtin_bit_cast(u16x2, b))); }
__device__ __forceinline__ unsigned pk_subs(unsigned a, unsigned b) { return __builtin_bit_cast(unsigned, __builtin_elementwise_sub_sat(__builtin_bit_cast(u16x2, a), __builtin_bit_cast(u16x2, b))); }

__device__ __forceinline__ unsigned pack_item(unsigned (&D)[32], unsigned (&P)[28]) {
    unsigned mx = 0;
#pragma unroll
    for (int i = 0; i < 32; ++i) mx = pk_max(mx, D[i] & 0x7c007c00u);
    mx = max(mx & 0xffffu, mx >> 16);
    const unsigned b10 = mx > (7u << 10) ? mx - (7u << 10) : 0u, BB = b10 * 0x10001u;
    unsigned st = 0xffffffffu, sumk = 0, sumz = 0;
#pragma unroll
    for (int i = 31; i >= 0; --i) {
        const unsigned E = D[i] & 0x7c007c00u, C = pk_subs(E, BB), K = pk_subs(BB, E) >> 10, Z = pk_subs(0x00010001u, C);
        sumk += K; sumz += Z;
        st = ((st << (Z >> 16)) | (Z >> 16)) << (K >> 16);
        st = ((st << (Z & 1u)) | (Z & 1u)) << (K & 31u);
        D[i] = (D[i] & 0x83ff83ffu) | C;
    }
    const unsigned nk = (sumk & 0xffffu) + (sumk >> 16), nz = (sumz & 0xffffu) + (sumz >> 16);
#pragma unroll
    for (int g = 0; g < 4; ++g) {
        const unsigned t7 = (D[8 * g + 7] & 0x1fff1fffu) | ((D[8 * g + 7] >> 2) & 0x20002000u);
#pragma unroll
        for (int i = 0; i < 7; ++i) P[7 * g + i] = D[8 * g + i] | ((t7 << (13 - 2 * i)) & 0x60006000u);
    }
    return (st & 0x03ffffffu) | (b10 << 16) | (nk != 0u && nk + nz > 26u ? 0x80000000u : 0u);
}
__device__ __forceinline__ void unpack_group(const unsigned (&P)[7], unsigned BB, int& st, unsigned (&H)[8]) {
    unsigned t7 = 0;
#pragma unroll
    for (int i = 0; i < 7; ++i) { t7 |= (P[i] >> (13 - 2 * i)) & (0x00030003u << (2 * i)); H[i] = P[i] & 0x9fff9fffu; }
    H[7] = (t7 & 0x1fff1fffu) | ((t7 << 2) & 0x80008000u);
#pragma unroll
    for (int i = 0; i < 8; ++i) {
        const unsigned T = H[i];
        unsigned h = T + BB;
        { const bool z = (T & 0x00001c00u) == 0u; const unsigned k = (unsigned)__builtin_ctz((unsigned)st); h -= z ? k << 10 : 0u; st >>= z ? k + 1u : 0u; }
        { const bool z = (T & 0x1c000000u) == 0u; const unsigned k = (unsigned)__builtin_ctz((unsigned)st); h -= z ? k << 26 : 0u; st >>= z ? k + 1u : 0u; }
        H[i] = h;
    }
}
constexpr int pull_younger(int k) {
    int n = 0;
    for (int j = k - PW; j < k; ++j) {
        const int jj = (j + 7) % 7;
        if (j > k - PW) n += (j + PW) % 7 == 0 ? 2 : 1;
        if (jj == 1 || jj == 3 || jj == 5 || jj == 6) n += 2;
    }
    return n;
}

__device__ __forceinline__ void item_decode(int it, int src, int& s, int& r) {
    int e;
    if (it >= 616 && it < 704 + 32 * WBE) it = it < 616 + 32 * WBE ? it + 88 : it - 32 * WBE;
    if (it < 704) { s = it / 88; e = it - 88 * s; }
    else { s = (it - 704) >> 5; e = 88 + (it & 31); }
    r = e < 56 ? (8 * src + 8 + e) & 63 : e + 8;
}

#define UNIT_LOAD(B, S_, R_, H_) do { const bool isx_ = (R_) < 64; \
        const gchar* b_ = isx_ ? sX + (size_t)((R_) * 64 + (H_) * 32) * (KSH * 4) + (size_t)(S_) * 256 \
                               : sW + (size_t)((S_) * 64 + (H_) * 32) * (NCOLS * 4) + (size_t)((R_) - 64) * 256; \
        const unsigned vo_ = isx_ ? xoff : woff; \
        const gchar* b1_ = b_ + (isx_ ? 16 : 1 * NCOLS * 4);          const gchar* b2_ = b_ + (isx_ ? 128 : 2 * NCOLS * 4); \
        const gchar* b3_ = b_ + (isx_ ? 144 : 3 * NCOLS * 4);         const gchar* b4_ = b_ + (isx_ ? 32768 : 4 * NCOLS * 4); \
        const gchar* b5_ = b_ + (isx_ ? 32768 + 16 : 5 * NCOLS * 4);  const gchar* b6_ = b_ + (isx_ ? 32768 + 128 : 6 * NCOLS * 4); \
        const gchar* b7_ = b_ + (isx_ ? 32768 + 144 : 7 * NCOLS * 4); \
        asm volatile("global_load_dwordx4 %0, %8, %9" NTIN "\n\tglobal_load_dwordx4 %1, %8, %10" NTIN "\n\t" \
                     "global_load_dwordx4 %2, %8, %11" NTIN "\n\tglobal_load_dwordx4 %3, %8, %12" NTIN "\n\t" \
                     "global_load_dwordx4 %4, %8, %13" NTIN "\n\tglobal_load_dwordx4 %5, %8, %14" NTIN "\n\t" \
                     "global_load_dwordx4 %6, %8, %15" NTIN "\n\tglobal_load_dwordx4 %7, %8, %16" NTIN "" \
                     : "+v"(B[0]), "+v"(B[1]), "+v"(B[2]), "+v"(B[3]), "+v"(B[4]), "+v"(B[5]), "+v"(B[6]), "+v"(B[7]) \
                     : "v"(vo_), "s"(b_), "s"(b1_), "s"(b2_), "s"(b3_), "s"(b4_), "s"(b5_), "s"(b6_), "s"(b7_) : "memory"); } while (0)
#define UNIT_WAIT(B, N) asm volatile("s_waitcnt vmcnt(%8)" : "+v"(B[0]), "+v"(B[1]), "+v"(B[2]), "+v"(B[3]), \
                                     "+v"(B[4]), "+v"(B[5]), "+v"(B[6]), "+v"(B[7]) : "i"(N) : "memory")
#define UNIT_CVT(O, B, R_) do { if ((R_) < 64) { \
            _Pragma("unroll") for (int b = 0; b < 4; ++b) { \
                const f32x4 lo_ = __builtin_bit_cast(f32x4, B[2 * b]), hi_ = __builtin_bit_cast(f32x4, B[2 * b + 1]); \
                O[b] = (u32x4){pk16(lo_[0], lo_[1]), pk16(lo_[2], lo_[3]), pk16(hi_[0], hi_[1]), pk16(hi_[2], hi_[3])}; } \
        } else { \
            _Pragma("unroll") for (int j = 0; j < 4; ++j) \
                O[j] = (u32x4){pk16(__builtin_bit_cast(f32x4, B[0])[j], __builtin_bit_cast(f32x4, B[1])[j]), \
                               pk16(__builtin_bit_cast(f32x4, B[2])[j], __builtin_bit_cast(f32x4, B[3])[j]), \
                               pk16(__builtin_bit_cast(f32x4, B[4])[j], __builtin_bit_cast(f32x4, B[5])[j]), \
                               pk16(__builtin_bit_cast(f32x4, B[6])[j], __builtin_bit_cast(f32x4, B[7])[j])}; } \
        asm volatile("" : "+v"(O[0]), "+v"(O[1]), "+v"(O[2]), "+v"(O[3])); } while (0)
#define ITEM_OFF(R_) ((R_) < 64 ? (size_t)(R_) * 8192 : (size_t)(((R_) - 64) >> 2) * PANEL + (size_t)((R_) & 3) * 4096)
#define UNIT_DST(BASE, R_, H_) const bool isxd_ = (R_) < 64; \
        gchar* t0_ = (BASE) + (isxd_ ? (H_) * 4096 : (H_) * 1024); \
        gchar* t1_ = t0_ + (isxd_ ? 1024 : 2048); gchar* t2_ = t0_ + (isxd_ ? 2048 : 16384); gchar* t3_ = t0_ + (isxd_ ? 3072 : 16384 + 2048)
#define STOREO(VOFF, DATA, BASE) asm volatile("global_store_dwordx4 %0, %1, %2 sc0 sc1" NTOUT "\n\ts_nop 1" :: "v"(VOFF), "v"(DATA), "s"(BASE) : "memory")
#define STORE16(VOFF, DATA, BASE, N) asm volatile("s_waitcnt vmcnt(%3)\n\tglobal_store_dwordx4 %0, %1, %2 sc0 sc1\n\ts_nop 1" \
                                                  :: "v"(VOFF), "v"(DATA), "s"(BASE), "i"(N) : "memory")

__global__ __launch_bounds__(512, 2) void gemm_rs_kernel(const Params p) {
    __shared__ __attribute__((aligned(16))) unsigned char shm[131072 + 256];
    LAS unsigned char* lds = (LAS unsigned char*)shm;
    volatile LAS unsigned* lctl = (volatile LAS unsigned*)(lds + 131072);
    const int tid = threadIdx.x, wid = __builtin_amdgcn_readfirstlane(tid >> 6), lane = tid & 63;
    const int rank = p.rank & (NRANK - 1), bid = blockIdx.x;
    gchar* myws = (gchar*)p.ws[rank];
    gu32* tmo = (gu32*)(myws + OFF_TMO);
    const unsigned long long tstart = __builtin_amdgcn_s_memrealtime();

    if (tid == 0) {
        lctl[0] = __hip_atomic_fetch_add((gu32*)(myws + OFF_TICKET + bid * 128), 1u, __ATOMIC_RELAXED, __HIP_MEMORY_SCOPE_AGENT) + 1u;
    } else if (tid < 64) lctl[tid] = 0u;
    unsigned call; size_t bufoff;
#define GET_CALL() do { asm volatile("s_waitcnt lgkmcnt(0)" ::: "memory"); __builtin_amdgcn_s_barrier(); asm volatile("" ::: "memory"); \
        call = __builtin_amdgcn_readfirstlane(lctl[0]); bufoff = CTRL_BYTES + (size_t)(call & 1u) * BUF_BYTES; } while (0)

    const int xcd = bid & 7, idx = bid >> 3, dd = idx >> 2;
    const int dest = (rank + dd) & (NRANK - 1);
    const int mt = 2 * dest + (xcd & 1), nt = 2 * (xcd >> 1) + (idx & 1) + 8 * ((idx >> 1) & 1);
    const int hidx = (dd - 1) * 16 + (bid & 15);

    const unsigned soff0 = (unsigned)lane * 16u;
    const gchar* sX = (const gchar*)p.x;
    const unsigned xoff = (unsigned)(lane & 15) * (KSH * 4) + (unsigned)(lane >> 4) * 32u;
    const unsigned woff = (unsigned)(lane >> 4) * 8u * (NCOLS * 4) + (unsigned)(lane & 15) * 16u;
    if (bid >= NPROD && nt >= 8 && wid >= 1 && wid <= CW) {
        u32x4 buf0[8], buf1[8], o[4], o1[4];
#pragma unroll
        for (int b = 0; b < 8; ++b) { buf0[b] = (u32x4){0u, 0u, 0u, 0u}; buf1[b] = (u32x4){0u, 0u, 0u, 0u}; }
        bool first = true;
        for (int job = NPUSH + (wid - 1) * 112 + hidx; job < NCONV; job += CW * 112) {
            int s, e;
            if (job < 2560) { s = job / 320; e = job - 320 * s; }
            else { s = (job - 2560) >> 8; e = 320 + (job & 255); }
            const int d = e < 64 ? rank : (rank + 1 + ((e - 64) & 7)) & (NRANK - 1);
            const int r = e < 64 ? (8 * rank + 8 + e) & 63 : 64 + ((e - 64) >> 3);
            const gchar* sW = (const gchar*)p.w + (size_t)d * NLOC * 4;
            UNIT_LOAD(buf0, s, r, 0); UNIT_LOAD(buf1, s, r, 1);
            if (first) { GET_CALL(); first = false; }
            gchar* tb = myws + bufoff + ITEM_OFF(r) + (e < 64 ? (size_t)(s * 8 + rank) * KT_STRIDE
                        : d == rank ? MAT_BYTES + (size_t)(s * 8 + rank) * KT_STRIDE : 2 * MAT_BYTES + (size_t)(d * 8 + s) * KT_STRIDE);
            UNIT_WAIT(buf0, 8); UNIT_CVT(o, buf0, r);
            { UNIT_DST(tb, r, 0); STORE16(soff0, o[0], t0_, 63); STORE16(soff0, o[1], t1_, 63); STORE16(soff0, o[2], t2_, 63); STORE16(soff0, o[3], t3_, 63); }
            UNIT_WAIT(buf1, 4); UNIT_CVT(o1, buf1, r);
            { UNIT_DST(tb, r, 1); STORE16(soff0, o1[0], t0_, 63); STORE16(soff0, o1[1], t1_, 63); STORE16(soff0, o1[2], t2_, 63); STORE16(soff0, o1[3], t3_, 63); }
            if (e < 56 || d != rank) {
                unsigned D[32], P[28];
#pragma unroll
                for (int i = 0; i < 16; ++i) { D[i] = o[i >> 2][i & 3]; D[16 + i] = o1[i >> 2][i & 3]; }
                const unsigned side = pack_item(D, P);
                gchar* pi = myws + bufoff + OFF_PI + (size_t)(e < 64 ? s * 64 + r : 512 + (d * 8 + s) * 64 + r - 64) * PI_ITEM;
#pragma unroll
                for (int i = 0; i < 7; ++i) { const u32x4 v = {P[4 * i], P[4 * i + 1], P[4 * i + 2], P[4 * i + 3]}; STORE16(soff0, v, pi + i * 1024, 63); }
                asm volatile("global_store_dword %0, %1, %2 sc0 sc1" :: "v"(soff0 >> 2), "v"(side), "s"(pi + 7168) : "memory");
            }
            asm volatile("s_waitcnt vmcnt(0)" ::: "memory");
            if (TGATE) while (__builtin_amdgcn_s_memrealtime() - tstart < TGATE) __builtin_amdgcn_s_sleep(2);
            const int fo = rank * 4096 + (s * 128 + r) * 4;
#pragma unroll
            for (int k = 0; k < NRANK; ++k)
                if ((e < 56 || k == d) && lane == 0)
                    __hip_atomic_store((gu32*)((gchar*)p.ws[k] + (k == rank ? OFF_INFLAG : OFF_CFLAG) + fo), call, __ATOMIC_RELAXED, __HIP_MEMORY_SCOPE_SYSTEM);
        }
        if (lane == 0) __hip_atomic_fetch_add((gu32*)(myws + OFF_CDONE), 1u, __ATOMIC_RELAXED, __HIP_MEMORY_SCOPE_AGENT);
        if (wid == 1) STAMPW(15);
    } else if (bid >= NPROD && nt < 8 && wid == 1 && hidx < NPUSH) {
        u32x4 buf0[8], buf1[8], o0[4], o1[4];
#pragma unroll
        for (int b = 0; b < 8; ++b) { buf0[b] = (u32x4){0u, 0u, 0u, 0u}; buf1[b] = (u32x4){0u, 0u, 0u, 0u}; }
        const gchar* sW = sX;
        const int r = (8 * rank + 8 + hidx) & 63;
        UNIT_LOAD(buf0, 0, r, 0); UNIT_LOAD(buf1, 0, r, 1);
        GET_CALL();
        UNIT_WAIT(buf0, 8); UNIT_CVT(o0, buf0, 0);
        UNIT_WAIT(buf1, 0); UNIT_CVT(o1, buf1, 0);
        const size_t ioff = bufoff + (size_t)rank * KT_STRIDE + (size_t)r * 8192;
#pragma unroll
        for (int b = 0; b < 8; ++b)
            for (int k = 1; k <= NRANK; ++k) {
                gchar* t = (gchar*)p.ws[(rank + hidx + k) & (NRANK - 1)] + ioff + b * 1024;
                STORE16(soff0, b < 4 ? o0[b & 3] : o1[b & 3], t, UW - 1);
            }
        asm volatile("s_waitcnt vmcnt(0)" ::: "memory");
#pragma unroll
        for (int k = 0; k < NRANK; ++k)
            if (lane == 0) __hip_atomic_store((gu32*)((gchar*)p.ws[k] + OFF_INFLAG + rank * 4096 + r * 4), call, __ATOMIC_RELAXED, __HIP_MEMORY_SCOPE_SYSTEM);
        STAMPW(15);
    } else if (bid >= (int)NSTR) {
        GET_CALL();
    } else if (wid == 0) {
        GET_CALL();
        const int j = lane & 7, lk = PMAP ? bid % 7 : (j + bid) % 7, psrc = (rank + 1 + lk) & (NRANK - 1);
        const unsigned long long t0 = __builtin_amdgcn_s_memrealtime();
        unsigned n = 0; int item = 0; bool have = false, fin = lane >= 7, late = false;
        for (unsigned it = 1; !__all(fin); ++it) {
            if (!fin && !have && n <= lctl[8 + j]) {
                unsigned t = NPUSH + (PMAP ? j * 4 + bid / 7 : bid) + n * NSTR;
                if (n >= 2) t = __hip_atomic_fetch_add((gu32*)(myws + OFF_QUEUE + lk * 512), 1u, __ATOMIC_RELAXED, __HIP_MEMORY_SCOPE_AGENT)
                                - (call - 1u) * QUEUE_PER_CALL + NPUSH + 2u * NSTR;
                int s, r; item_decode(t < (unsigned)NITEM ? (int)t : 0, psrc, s, r);
                item = t < (unsigned)NITEM ? s * 128 + r : 1024; have = true;
            }
            if (have) {
                bool ready = item >= 1024 || late;
                if (!ready) ready = __hip_atomic_load((gu32*)(myws + OFF_CFLAG + psrc * 4096 + item * 4), __ATOMIC_RELAXED, __HIP_MEMORY_SCOPE_SYSTEM) >= call;
                if (ready) { ++n; __hip_atomic_store((LAS unsigned*)lctl + 32 + j * 4 + (n & 3u), (n << 16) | (unsigned)item, __ATOMIC_RELEASE, __HIP_MEMORY_SCOPE_WORKGROUP); have = false; fin = item >= 1024; }
            }
            if ((it & 63u) == 0u && !late) {
                late = __builtin_amdgcn_readfirstlane(__hip_atomic_load(tmo, __ATOMIC_RELAXED, __HIP_MEMORY_SCOPE_AGENT)) == call
                       || __builtin_amdgcn_s_memrealtime() - t0 > DEADLINE_TICKS;
                if (late) __hip_atomic_store(tmo, call, __ATOMIC_RELAXED, __HIP_MEMORY_SCOPE_AGENT);
            }
            __builtin_amdgcn_s_sleep(2);
        }
    } else {
        GET_CALL();
        const int lk = PMAP ? bid % 7 : (wid - 1 + bid) % 7, src = (rank + 1 + lk) & (NRANK - 1);
        const gchar* rimg = (const gchar*)p.ws[src] + bufoff;
        gchar* limg = myws + bufoff + (size_t)src * KT_STRIDE;
        gu32* lflag = (gu32*)(myws + OFF_INFLAG + src * 4096);
        u32x4 in[7]; unsigned side = 0;
#pragma unroll
        for (int b = 0; b < 7; ++b) in[b] = (u32x4){0u, 0u, 0u, 0u};
        const unsigned soff4 = (unsigned)lane * 4u;
#define PI_OF(S_, R_) (rimg + OFF_PI + 3072 + (size_t)((R_) < 64 ? (S_) * 64 + (R_) : 512 + (rank * 8 + (S_)) * 64 + (R_) - 64) * PI_ITEM)
#define RAW_OF(S_, R_) (rimg + ITEM_OFF(R_) + ((R_) < 64 ? (size_t)((S_) * 8 + src) * KT_STRIDE : 2 * MAT_BYTES + (size_t)(rank * 8 + (S_)) * KT_STRIDE))
#define DST_OF(S_, R_) (limg + ITEM_OFF(R_) + ((R_) < 64 ? 0 : MAT_BYTES) + (size_t)(S_) * 8 * KT_STRIDE)
#define PULL(k, BASE) do { if ((k) == 0) asm volatile("global_load_dword %0, %1, %2 sc0 sc1" : "+v"(side) : "v"(soff4), "s"((BASE) + 4096) : "memory"); \
        asm volatile("global_load_dwordx4 %0, %1, %2 offset:%3 sc0 sc1" : "+v"(in[k]) : "v"(soff0), "s"(BASE), "i"((k) * 1024 - 3072) : "memory"); } while (0)
#define TAKE(DST, N_) do { unsigned e_; while (((e_ = __builtin_amdgcn_readfirstlane(__hip_atomic_load((LAS unsigned*)lctl + 28 + wid * 4 + ((N_) & 3), __ATOMIC_ACQUIRE, __HIP_MEMORY_SCOPE_WORKGROUP))) >> 16) != (unsigned)(N_)) __builtin_amdgcn_s_sleep(1); \
        lctl[7 + wid] = (unsigned)(N_); asm volatile("" ::: "memory"); DST = (int)(e_ & 0xffffu); } while (0)
        int cur, s, r, pflag = -1;
        TAKE(cur, 1);
        s = cur >> 7; r = cur & 127;
        const gchar* sb = PI_OF(s, r);
#pragma unroll
        for (int k = 0; k < PW; ++k) PULL(k, sb);
        if (wid == 1) STAMPW(24);
        for (int i = 0; cur < 1024; ++i) {
            if (wid == 1 && i % 5 == 0 && i <= 25) STAMPW(25 + i / 5);
            int nxt = 1024, sn = 0, rn = 0; const gchar* sbn = sb;
            gchar* db = DST_OF(s, r);
            const bool isx = r < 64;
            const gchar* rraw = RAW_OF(s, r);
            asm volatile("" : "+s"(rraw), "+s"(db));
            int st = 0; unsigned BB = 0; bool ovf = false;
#pragma unroll
            for (int k = 0; k < 7; ++k) {
                if (k + PW == 7) {
                    TAKE(nxt, i + 2);
                    sn = (nxt >> 7) & 7; rn = nxt & 127;
                    sbn = nxt < 1024 ? PI_OF(sn, rn) : (const gchar*)myws + bufoff + OFF_PI + 3072;
                    if (nxt >= 1024 && lane == 0) __hip_atomic_fetch_add((gu32*)(myws + OFF_PDONE + lk * 512), 1u, __ATOMIC_RELAXED, __HIP_MEMORY_SCOPE_AGENT);
                }
                if (k < PW && i == 0) asm volatile("s_waitcnt vmcnt(%0)" :: "i"(PW - 1) : "memory");
                asm volatile("s_waitcnt vmcnt(%2)" : "+v"(in[k]), "+v"(side) : "i"(pull_younger(k)) : "memory");
                if (k == 0) { st = (int)(side | 0xfc000000u); BB = ((side >> 16) & 0x7c00u) * 0x10001u; ovf = (int)side < 0;
                              asm volatile("" : "+v"(st), "+v"(BB)); }
                if (k == PW && pflag >= 0 && lane == 0) __hip_atomic_store(lflag + pflag, call, __ATOMIC_RELAXED, __HIP_MEMORY_SCOPE_SYSTEM);
                if (k + PW < 7) PULL((k + PW) % 7, sb); else PULL((k + PW) % 7, sbn);
                __builtin_amdgcn_sched_barrier(0);
                if (k == 1 || k == 3 || k == 5 || k == 6) {
                    const int g = k >> 1;
                    unsigned P[7], H[8];
#pragma unroll
                    for (int e = 0; e < 7; ++e) P[e] = in[(7 * g + e) >> 2][(7 * g + e) & 3];
                    unpack_group(P, BB, st, H);
                    u32x4 v0 = {H[0], H[1], H[2], H[3]}, v1 = {H[4], H[5], H[6], H[7]};
                    const unsigned o0 = soff0 + (isx ? g * 2048 : (g & 1) * 16384 + (g >> 1) * 1024), o1 = o0 + (isx ? 1024 : 2048);
                    if (ovf) asm volatile("global_load_dwordx4 %0, %2, %4 sc0 sc1\n\tglobal_load_dwordx4 %1, %3, %4 sc0 sc1\n\ts_waitcnt vmcnt(0)"
                                          : "+v"(v0), "+v"(v1) : "v"(o0), "v"(o1), "s"(rraw) : "memory");
                    STORE16(o0, v0, db, 63); STORE16(o1, v1, db, 63);
                }
            }
            pflag = cur; cur = nxt; s = sn; r = rn; sb = sbn;
        }
        asm volatile("s_waitcnt vmcnt(0)" : "+v"(in[0]), "+v"(in[1]), "+v"(in[2]), "+v"(in[3]), "+v"(side) :: "memory");
        if (lane == 0) __hip_atomic_store(lflag + pflag, call, __ATOMIC_RELAXED, __HIP_MEMORY_SCOPE_SYSTEM);
        STAMPW(16 + wid);
    }

    const int wr = wid >> 2, wc = wid & 3;
    const gchar* cA = myws + bufoff + (size_t)mt * PANEL;
    const gchar* cB = myws + bufoff + MAT_BYTES + (size_t)nt * PANEL;
    const unsigned voff = (unsigned)tid * 16u;
    const unsigned ldsw = (unsigned)wid * 1024u;
    const int aoff = wr * 8192 + lane * 16, boff = wc * 4096 + lane * 16;
    constexpr int HTB = 16384;
#define SA(b, h) (((b) * 2 + (h)) * HTB)
#define SB(b, h) ((4 + (b) * 2 + (h)) * HTB)
#define STAGE(bufo, gbase) do { _Pragma("unroll") for (int _i = 0; _i < 2; ++_i) \
        __builtin_amdgcn_global_load_lds((const GAS unsigned*)((gbase) + voff + _i * 8192), (LAS unsigned*)(lds + (bufo) + ldsw + _i * 8192), 16, 0, GAUX); } while (0)
#define LDA(dst, b, h) do { _Pragma("unroll") for (int m = 0; m < 4; ++m) _Pragma("unroll") for (int k = 0; k < 2; ++k) \
        dst[m][k] = *(const LAS half8*)(lds + SA(b, h) + aoff + m * 2048 + k * 1024); } while (0)
#define LDB(dst, b, h) do { _Pragma("unroll") for (int n = 0; n < 2; ++n) _Pragma("unroll") for (int k = 0; k < 2; ++k) \
        dst[n][k] = *(const LAS half8*)(lds + SB(b, h) + boff + n * 2048 + k * 1024); } while (0)
#define MMA(ai, bj, At_, Bt_) do { __builtin_amdgcn_s_setprio(1); \
        _Pragma("unroll") for (int m = 0; m < 4; ++m) _Pragma("unroll") for (int n = 0; n < 2; ++n) _Pragma("unroll") for (int k = 0; k < 2; ++k) \
            acc[ai][bj][m][n] = __builtin_amdgcn_mfma_f32_16x16x32_f16(At_[m][k], Bt_[n][k], acc[ai][bj][m][n], 0, 0, 0); \
        __builtin_amdgcn_s_setprio(0); } while (0)
#define WAIT_V(n) asm volatile("s_waitcnt vmcnt(" #n ")" ::: "memory")
#define WAIT_L(n) asm volatile("s_waitcnt lgkmcnt(" #n ")" ::: "memory")
#define BAR __builtin_amdgcn_s_barrier()
#define SCHED __builtin_amdgcn_sched_barrier(0)
#define WAIT_SEG(c) do { if (wid == 0) { int lz_ = lane; asm volatile("" : "+v"(lz_)); const int wh_ = lz_ >> 3; \
        const int it_ = wh_ < 4 && ((c) == 8 || (lz_ & 7) != dest) ? 4 * mt + wh_ : 64 + 4 * nt + (wh_ & 3); \
        poll_ge<ISL>((gu32*)(myws + OFF_INFLAG + ((c) == 8 ? rank * 4096 + (lz_ & 7) * 512 : (lz_ & 7) * 4096 + (c) * 512) + it_ * 4), call, tmo, call); } } while (0)

    f32x4 acc[2][2][4][2];
#pragma unroll
    for (int a = 0; a < 2; ++a)
#pragma unroll
        for (int b = 0; b < 2; ++b)
#pragma unroll
            for (int m = 0; m < 4; ++m)
#pragma unroll
                for (int n = 0; n < 2; ++n) acc[a][b][m][n] = (f32x4){0.f, 0.f, 0.f, 0.f};
    half8 At[4][2], B0[2][2], B1[2][2];

#define GEMM_LOOP(NT, AOFF, BOFF, HOOK) do { \
    { const gchar* a0 = AOFF(0); const gchar* b0 = BOFF(0); const gchar* a1 = AOFF(1); const gchar* b1 = BOFF(1); \
      STAGE(SB(0, 0), b0); STAGE(SA(0, 0), a0); STAGE(SB(0, 1), b0 + HTB); STAGE(SA(0, 1), a0 + HTB); \
      if (wr == 1) BAR; \
      WAIT_V(4); BAR; \
      STAGE(SB(1, 0), b1); STAGE(SA(1, 0), a1); STAGE(SB(1, 1), b1 + HTB); \
      WAIT_V(6); BAR; } \
    for (int t = 0; t < (NT); t += 2) { \
        const int t2 = t + 2 < (NT) ? t + 2 : 0;                        \
        const gchar* a1 = AOFF(t + 1); \
        const gchar* a2 = AOFF(t2); const gchar* b2 = BOFF(t2); \
        const gchar* a3 = AOFF(t2 + 1); const gchar* b3 = BOFF(t2 + 1); \
        LDB(B0, 0, 0); SCHED; LDA(At, 0, 0); STAGE(SA(1, 1), a1 + HTB); \
        WAIT_L(8); BAR; WAIT_L(0); MMA(0, 0, At, B0); \
        HOOK; \
        asm volatile("" ::: "memory"); BAR; SCHED; \
        LDB(B1, 0, 1); STAGE(SB(0, 0), b2); \
        BAR; WAIT_L(0); MMA(0, 1, At, B1); BAR; \
        LDA(At, 0, 1); STAGE(SA(0, 0), a2); \
        BAR; WAIT_L(0); MMA(1, 0, At, B0); BAR; SCHED; \
        STAGE(SB(0, 1), b2 + HTB); \
        WAIT_V(6); BAR; MMA(1, 1, At, B1); BAR; \
        LDB(B0, 1, 0); SCHED; LDA(At, 1, 0); STAGE(SA(0, 1), a2 + HTB); \
        WAIT_L(8); BAR; WAIT_L(0); MMA(0, 0, At, B0); BAR; SCHED; \
        LDB(B1, 1, 1); STAGE(SB(1, 0), b3); \
        BAR; WAIT_L(0); MMA(0, 1, At, B1); BAR; \
        LDA(At, 1, 1); STAGE(SA(1, 0), a3); \
        BAR; WAIT_L(0); MMA(1, 0, At, B0); BAR; SCHED; \
        STAGE(SB(1, 1), b3 + HTB); \
        WAIT_V(6); BAR; MMA(1, 1, At, B1); BAR; \
    } \
    if (wr == 0) BAR;                                                   \
    } while (0)

    const int src = (rank - dd) & (NRANK - 1);
    const bool early = dd != 0 && nt >= 8;
#define A_DIAG(t) (dA + (size_t)(t) * 8 * KT_STRIDE)
#define B_DIAG(t) (dB + (size_t)(t) * bstep)
#define DIAG(DELAY) do { \
        const gchar* dA = myws + bufoff + (size_t)rank * KT_STRIDE + (size_t)(2 * rank + (mt & 1)) * PANEL; \
        const gchar* dB = myws + bufoff + (size_t)nt * PANEL + (dd == 0 ? MAT_BYTES + (size_t)rank * KT_STRIDE : 2 * MAT_BYTES + (size_t)src * 8 * KT_STRIDE); \
        const size_t bstep = dd == 0 ? 8 * KT_STRIDE : KT_STRIDE; \
        BAR; \
        if (wid == 0) { poll_ge<64>((gu32*)(myws + OFF_CDONE), call * (112u * CW), tmo, call); \
            const unsigned long long t0_ = __builtin_amdgcn_s_memrealtime(); \
            while (__builtin_amdgcn_s_memrealtime() - t0_ < (unsigned long long)(DELAY)) __builtin_amdgcn_s_sleep(16); } \
        WAIT_V(0); BAR; asm volatile("" ::: "memory"); \
        GEMM_LOOP(8, A_DIAG, B_DIAG, ); \
        STAMP(16); } while (0)
#define ZERO_ACC() do { _Pragma("unroll") for (int a = 0; a < 2; ++a) _Pragma("unroll") for (int b = 0; b < 2; ++b) \
        _Pragma("unroll") for (int m = 0; m < 4; ++m) _Pragma("unroll") for (int n = 0; n < 2; ++n) acc[a][b][m][n] = (f32x4){0.f, 0.f, 0.f, 0.f}; } while (0)
    gchar* park = myws + OFF_PARK + (size_t)(bid * 8 + wid) * 32768;
    if (EARLY && early) {
        DIAG((dd & 3) * DSTAG);
#pragma unroll
        for (int i = 0; i < 32; ++i) STORE16(soff0, acc[i >> 4][(i >> 3) & 1][(i >> 1) & 3][i & 1], park + i * 1024, 63);
        ZERO_ACC();
        WAIT_V(0);
    }

    int tab;
    {
        const int t0 = nt >= 8 ? 0 : OWNA, u = lane < t0 ? lane : lane - 8, oo = 8 - dd;
        const int st = dd == 0 ? lane / 7 : u / 6, jj = dd == 0 ? lane % 7 : u % 6, o = jj + 1 + (jj + 1 >= oo ? 1 : 0);
        const bool own = dd != 0 && lane >= t0 && lane < t0 + 8;
        tab = own ? ((lane - t0) * 8 + rank) | (8 << 8) : ((st & 7) * 8 + ((dest + o) & 7)) | ((st & 7) << 8);
    }
#define KT_OF(t) ((size_t)(__builtin_amdgcn_readlane(tab, (t)) & 63) * KT_STRIDE)
#define A_MAIN(t) (cA + KT_OF(t))
#define B_MAIN(t) (cB + KT_OF(t))
    int have = __builtin_amdgcn_readlane(tab, 0) >> 8;
    WAIT_SEG(have);
    asm volatile("" ::: "memory"); BAR; asm volatile("" ::: "memory");
    STAMP(1);
    GEMM_LOOP(56, A_MAIN, B_MAIN, { const int seg_ = __builtin_amdgcn_readlane(tab, t2 + 1) >> 8; if (seg_ != have && t2 != 0) { have = seg_; WAIT_SEG(have); if (have < 8) STAMP(1 + have); } });
    STAMP(9);

    int tz = threadIdx.x; asm volatile("" : "+v"(tz));
    const int fr = tz & 15, fq = (tz >> 4) & 3;
    const unsigned lo = (unsigned)(fq * 4) * (NCOLS * 4) + (unsigned)fr * 16u;
    const unsigned soff = (unsigned)(tz & 63) * 16u;
    const size_t torg = ((size_t)((mt & 1) * 256 + wr * 64) * NCOLS + nt * 256 + wc * 64) * 4;
    const unsigned qnan = 0x7fc00000u;
#define ROWOFF(c, g, e) ((size_t)(((c) >> 1) * 128 + (((c) & 1) * 2 + ((g) >> 1)) * 16 + ((g) & 1) * 2 + ((e) >> 2)) * (NCOLS * 4))
#define ACC(c, g, e) acc[(c) >> 1][((e) >> 1) & 1][((c) & 1) * 2 + ((g) >> 1)][(e) & 1][((g) & 1) * 2 + ((e) >> 2)]
    if (dd != 0) {
    unsigned u[4][4][8];
    {
        const bool bad = __builtin_amdgcn_readfirstlane(__hip_atomic_load(tmo, __ATOMIC_RELAXED, __HIP_MEMORY_SCOPE_AGENT)) == call;
#pragma unroll
        for (int c = 0; c < 4; ++c)
#pragma unroll
            for (int g = 0; g < 4; ++g)
#pragma unroll
                for (int e = 0; e < 8; ++e) { const float v = ACC(c, g, e); u[c][g][e] = bad ? qnan : __builtin_bit_cast(unsigned, v); }
    }
    for (int i = 0; i < wid * XSTAG; ++i) __builtin_amdgcn_s_sleep(1);
    unsigned ebase;
    {
        unsigned mx = 0;
#pragma unroll
        for (int c = 0; c < 4; ++c)
#pragma unroll
            for (int g = 0; g < 4; ++g) {
#pragma unroll
                for (int e = 0; e < 8; ++e) { u[c][g][e] = __builtin_amdgcn_alignbit(u[c][g][e], u[c][g][e], 31); mx = max(mx, u[c][g][e]); }
                asm volatile("" : "+v"(u[c][g][0]), "+v"(u[c][g][1]), "+v"(u[c][g][2]), "+v"(u[c][g][3]), "+v"(u[c][g][4]), "+v"(u[c][g][5]), "+v"(u[c][g][6]), "+v"(u[c][g][7]));
            }
#pragma unroll
        for (int sft = 1; sft < 64; sft <<= 1) mx = max(mx, (unsigned)__shfl_xor((int)mx, sft));
        const unsigned emax = (unsigned)__builtin_amdgcn_readfirstlane(mx) >> 24;
        ebase = emax > 15u ? emax - 15u : 0u;
        const unsigned B = ebase << 24;
        gchar* rbase = myws + OFF_RAW + (size_t)(bid * 8 + wid) * RAW_WAVE;
        const unsigned roff = (unsigned)(tz & 63) * 4u;
        gchar* pk = myws + OFF_PK + (size_t)(bid * 8 + wid) * PK_WAVE;
#pragma unroll
        for (int c = 0; c < 4; ++c) {
#pragma unroll
            for (int g = 0; g < 4; ++g) {
                unsigned t[8], mn = 0xffffffffu;
#pragma unroll
                for (int e = 0; e < 8; ++e) { t[e] = __builtin_elementwise_sub_sat(u[c][g][e], B); mn = min(mn, t[e]); }
                if (__any(mn < (1u << 24))) {
#pragma unroll
                    for (int e = 0; e < 8; ++e)
                        if (t[e] < (1u << 24)) {
                            t[e] = min(u[c][g][e], 2u);
                            if (u[c][g][e] > 1u) {
                                const unsigned raw = __builtin_amdgcn_alignbit(u[c][g][e], u[c][g][e], 1);
                                asm volatile("global_store_dword %0, %1, %2 offset:%3 sc0 sc1" :: "v"(roff), "v"(raw), "s"(rbase + (c * 4 + g) * 2048), "i"(e * 256) : "memory");
                            }
                        }
                }
#pragma unroll
                for (int e = 0; e < 7; ++e) u[c][g][e] = ((t[7] << (4 + 4 * e)) & 0xf0000000u) | t[e];
                asm volatile("" : "+v"(u[c][g][0]), "+v"(u[c][g][1]), "+v"(u[c][g][2]), "+v"(u[c][g][3]), "+v"(u[c][g][4]), "+v"(u[c][g][5]), "+v"(u[c][g][6]));
            }
#pragma unroll
            for (int i = 0; i < 7; ++i) {
#define PKW(n) u[c][(n) / 7][(n) % 7]
                const u32x4 v = {PKW(4 * i), PKW(4 * i + 1), PKW(4 * i + 2), PKW(4 * i + 3)};
                STORE16(soff, v, pk + (c * 7 + i) * 1024, 63);
            }
        }
        asm volatile("s_waitcnt vmcnt(0)" ::: "memory");
        if ((tz & 63) == 0)
            __hip_atomic_store((gu32*)((gchar*)p.ws[dest] + OFF_OUTFLAG + (bid * 8 + wid) * 64), (call << 8) | ebase, __ATOMIC_RELAXED, __HIP_MEMORY_SCOPE_SYSTEM);
        STAMP(12);
    }
    const __amdgpu_buffer_rsrc_t rp = __builtin_amdgcn_make_buffer_rsrc((void*)park, 0, EARLY && early ? 32768 : 0, 0x00020000);
#pragma unroll
    for (int i = 0; i < 32; ++i)
        acc[i >> 4][(i >> 3) & 1][(i >> 1) & 3][i & 1] = __builtin_bit_cast(f32x4, __builtin_amdgcn_raw_buffer_load_b128(rp, soff, i * 1024, 17));
    }
    if (!EARLY && early) DIAG(0);
    gchar* obase = (gchar*)p.out[rank] + torg + (size_t)src * NLOC * 4;
    if (dd != 0) {
        const unsigned f = __builtin_amdgcn_readfirstlane(poll_ge((gu32*)(myws + OFF_OUTFLAG + (bid * 8 + wid) * 64), call << 8, tmo, call));
        if (wid == 0) {
            STAMP(14);
            if (nt < 8) poll_ge<PSL>((gu32*)(myws + OFF_PDONE + (7 - dd) * 512), (call - 1u) * NSTR + (unsigned)((mt & 1) * 8 + nt) * NSTR / 16u + 1u, tmo, call);
            else poll_ge((gu32*)(myws + OFF_GATE + (mt * 8 + ((nt - 8) ^ PX)) * 128), call, tmo, call);
            STAMP(13);
        }
        const bool bad = __builtin_amdgcn_readfirstlane(__hip_atomic_load(tmo, __ATOMIC_RELAXED, __HIP_MEMORY_SCOPE_AGENT)) == call;
        const unsigned B = (f & 255u) << 24;
        const gchar* pk = (const gchar*)p.ws[bad ? rank : src] + OFF_PK + (size_t)(bid * 8 + wid) * PK_WAVE;
        const gchar* rbase = (const gchar*)p.ws[src] + OFF_RAW + (size_t)(bid * 8 + wid) * RAW_WAVE + (tz & 63) * 4;
#pragma unroll
        for (int c = 0; c < 4; ++c) {
            bool wide = false;
            if (WIDE < 16 && c == 3 && early) {
                while (__builtin_amdgcn_readfirstlane(lctl[1]) + 1u < (unsigned)(c * 8 + wid)) __builtin_amdgcn_s_sleep(2);
                wide = (unsigned)__builtin_amdgcn_readfirstlane(__hip_atomic_load((gu32*)(myws + OFF_BDONE + (7 - dd) * 512), __ATOMIC_RELAXED, __HIP_MEMORY_SCOPE_AGENT)) >= (call - 1u) * 16u + WIDE;
            }
            while (__builtin_amdgcn_readfirstlane(lctl[1]) < (unsigned)(c * 8 + wid)) __builtin_amdgcn_s_sleep(2);
            u32x4 in[7];
            const gchar* pc = pk + c * 7168 + 3072;
#pragma unroll
            for (int i = 0; i < 7; ++i) {
                if (!wide) asm volatile("s_waitcnt vmcnt(%0)" :: "i"(EW - 1) : "memory");
                asm volatile("global_load_dwordx4 %0, %1, %2 offset:%3 sc0 sc1" : "=v"(in[i]) : "v"(soff), "s"(pc), "i"(i * 1024 - 3072) : "memory");
            }
            asm volatile("s_waitcnt vmcnt(%0)" :: "i"(WTOK) : "memory");
            if (WIDE < 16 && c == 3 && early && tz == 448) __hip_atomic_fetch_add((gu32*)(myws + OFF_BDONE + (7 - dd) * 512), 1u, __ATOMIC_RELAXED, __HIP_MEMORY_SCOPE_AGENT);
            lctl[1] = (unsigned)(c * 8 + wid) + 1u;
            if (c == 3 && nt < 8 && tz == 448) __hip_atomic_store((gu32*)(myws + OFF_GATE + (mt * 8 + nt) * 128), call, __ATOMIC_RELAXED, __HIP_MEMORY_SCOPE_AGENT);
            asm volatile("s_waitcnt vmcnt(0)" : "+v"(in[0]), "+v"(in[1]), "+v"(in[2]), "+v"(in[3]), "+v"(in[4]), "+v"(in[5]), "+v"(in[6]) :: "memory");
            if (c == 3 && wid == 7) STAMPW(10);
#pragma unroll
            for (int g = 0; g < 4; ++g) {
                unsigned t[8], v[8], mn;
                t[7] = 0;
#pragma unroll
                for (int e = 0; e < 7; ++e) {
                    const int n = g * 7 + e;
                    const unsigned dw = in[n >> 2][n & 3];
                    t[7] = __builtin_amdgcn_alignbit(t[7], dw, 28); t[e] = dw & 0x0fffffffu;
                }
                mn = t[0];
#pragma unroll
                for (int e = 0; e < 8; ++e) { mn = min(mn, t[e]); const unsigned x = t[e] + B; v[e] = __builtin_amdgcn_alignbit(x, x, 1); }
                if (__any(mn < (1u << 24)) || bad) {
#pragma unroll
                    for (int e = 0; e < 8; ++e) {
                        if (t[e] < (1u << 24)) v[e] = t[e] << 31;
                        if (t[e] == 2u && !bad) v[e] = __hip_atomic_load((const gu32*)(rbase + (c * 4 + g) * 2048 + e * 256), __ATOMIC_RELAXED, __HIP_MEMORY_SCOPE_SYSTEM);
                        if (bad) v[e] = qnan;
                    }
                }
#pragma unroll
                for (int e = 0; e < 8; ++e) ACC(c, g, e) += __builtin_bit_cast(float, v[e]);
                if (early) {
                    const f32x4 w0 = {ACC(c, g, 0), ACC(c, g, 1), ACC(c, g, 2), ACC(c, g, 3)}, w1 = {ACC(c, g, 4), ACC(c, g, 5), ACC(c, g, 6), ACC(c, g, 7)};
                    STOREO(lo, w0, obase + ROWOFF(c, g, 0)); STOREO(lo, w1, obase + ROWOFF(c, g, 4));
                }
            }
        }
    }
    if (!early) {
        DIAG(0);
        const bool bad = __builtin_amdgcn_readfirstlane(__hip_atomic_load(tmo, __ATOMIC_RELAXED, __HIP_MEMORY_SCOPE_AGENT)) == call;
#pragma unroll
        for (int c = 0; c < 4; ++c)
#pragma unroll
            for (int g = 0; g < 4; ++g)
#pragma unroll
                for (int h = 0; h < 2; ++h) {
                    f32x4 v = {ACC(c, g, 4 * h), ACC(c, g, 4 * h + 1), ACC(c, g, 4 * h + 2), ACC(c, g, 4 * h + 3)};
                    if (bad) v = __builtin_bit_cast(f32x4, (u32x4){qnan, qnan, qnan, qnan});
                    STOREO(lo, v, obase + ROWOFF(c, g, 4 * h));
                }
        STAMP(23);
    }
}

extern "C" void kernel_launch(int rank, int nranks, ncclComm_t comm,
                              hipStream_t stream,
                              void* const* d_in, const int* in_sizes, int n_in,
                              void* d_out, int out_size,
                              const int* in_dtypes, int out_dtype,
                              void* const* d_out_all,
                              void* const* d_ws_all, size_t ws_bytes) {
    if (nranks != NRANK || (unsigned)rank >= (unsigned)NRANK || ws_bytes < WS_NEED || n_in < 2 ||
        in_sizes[0] != 4096 * KSH || in_sizes[1] != KSH * NCOLS || out_size != 512 * NCOLS) {
        fprintf(stderr, "unexpected problem shape\n");
        return;
    }
    Params p;
    p.x = (const float*)d_in[0]; p.w = (const float*)d_in[1];
    for (int r = 0; r < NRANK; ++r) { p.out[r] = (float*)d_out_all[r]; p.ws[r] = (char*)d_ws_all[r]; }
    p.rank = rank; p.pad = 0;
    gemm_rs_kernel<<<GRID, 512, 0, stream>>>(p);
    HIP_CHECK(hipGetLastError());
}
```

```cpp
#include <hip/hip_runtime.h>
#include <rccl/rccl.h>
#include <cstdio>

#define HIP_CHECK(expr) do { hipError_t _e = (expr); if (_e != hipSuccess) { \
    fprintf(stderr, "HIP error %s at %s:%d\n", hipGetErrorString(_e), __FILE__, __LINE__); \
    return; } } while (0)

#define LAS __attribute__((address_space(3)))
#define GAS __attribute__((address_space(1)))
typedef _Float16 half8 __attribute__((ext_vector_type(8)));
typedef _Float16 half2v __attribute__((ext_vector_type(2)));
typedef float f32x2 __attribute__((ext_vector_type(2)));
typedef float f32x4 __attribute__((ext_vector_type(4)));
typedef unsigned u32x4 __attribute__((ext_vector_type(4)));
typedef unsigned u32x3 __attribute__((ext_vector_type(3)));
typedef unsigned u32x2 __attribute__((ext_vector_type(2)));
typedef unsigned long long u64;
template <int N> struct IC { static constexpr int value = N; };
typedef GAS unsigned gu32;
typedef GAS char gchar;

constexpr int NRANK = 8;
constexpr int KSH = 512;
constexpr int NCOLS = 32768, NLOC = 4096;
constexpr int GRID = 256, NPROD = 32;
constexpr size_t CTRL_BYTES = 1u << 20;
constexpr size_t PANEL = 32768;
constexpr size_t KT_STRIDE = 16 * PANEL;
constexpr size_t MAT_BYTES = 64 * KT_STRIDE;
constexpr size_t PI_ITEM = 26 * 256 + 512;
constexpr size_t OFF_PI = 3 * MAT_BYTES;
constexpr size_t BUF_BYTES = 3 * MAT_BYTES + (33u << 20);
constexpr size_t PK_CHUNK = 27 * 256;
constexpr size_t PK_WAVE = 4 * PK_CHUNK + 256;
constexpr size_t RAW_WAVE = 64 * 512;
constexpr size_t OFF_PK = CTRL_BYTES + 2 * BUF_BYTES;
constexpr size_t OFF_RAW = OFF_PK + (size_t)GRID * 8 * PK_WAVE;
constexpr size_t WS_NEED = OFF_RAW + (size_t)GRID * 8 * RAW_WAVE;
constexpr int OFF_TMO = 0, OFF_QUEUE = 20480, OFF_PDONE = 24576 + 256, OFF_CDONE = 28672 + 768, OFF_BDONE = 24576 + 128;
constexpr int OFF_GATE = 4096;
constexpr int OFF_INFLAG = 32768;
constexpr int OFF_CFLAG = 65536;
constexpr int OFF_OUTFLAG = 131072;
constexpr int OFF_STAMP = 262144;
constexpr int OFF_TICKET = 393216;
constexpr unsigned long long DEADLINE_TICKS = 50000000ull;
#define OWNA 18
#define TGATE 500
#define XSTAG 20
#define EW 3
#define PW 2
#define NPUSH 28
#define PX 4
#define PSL 16
#define ISL 16
#define GAUX 17
#define NTIN " nt"
#define NTOUT " nt"
#define PMAP 0
#define WTOK 3
#define WBE 1
#define NTPI ""
#define NTPK " nt"
#define NTRAW 0
#define GSL 8
#define XSTAGB 0
#define ISLOW 64
#define SFLAG 0
#define TSL 32
#define TH 12
#define TH2 40
#define UW 16
#define CW 3
constexpr int NCONV = 8 * (64 + 512);
constexpr int NITEM = 960;
constexpr unsigned NSTR = PMAP ? 28 : 32;
constexpr unsigned QUEUE_PER_CALL = NITEM - NPUSH - 2 * NSTR + NSTR;

struct Params {
    const float* x; const float* w;
    float* out[NRANK];
    char* ws[NRANK];
    int rank; int pad;
};

#define STAMPW(k) do {} while (0)
#define STAMP(k) do {} while (0)

template <int SLEEP = 8>
__device__ __forceinline__ unsigned poll_ge(gu32* addr, unsigned need, gu32* tmo, unsigned call) {
    const unsigned long long t0 = __builtin_amdgcn_s_memrealtime();
    unsigned v;
    for (unsigned n = 1;; ++n) {
        v = __hip_atomic_load(addr, __ATOMIC_RELAXED, __HIP_MEMORY_SCOPE_SYSTEM);
        if (__all(v >= need)) break;
        if ((n & 15u) == 0u) {
            const unsigned tv = __builtin_amdgcn_readfirstlane(__hip_atomic_load(tmo, __ATOMIC_RELAXED, __HIP_MEMORY_SCOPE_AGENT));
            if (tv == call || __builtin_amdgcn_s_memrealtime() - t0 > DEADLINE_TICKS) {
                __hip_atomic_store(tmo, call, __ATOMIC_RELAXED, __HIP_MEMORY_SCOPE_AGENT);
                break;
            }
        }
        __builtin_amdgcn_s_sleep(SLEEP);
    }
    asm volatile("" ::: "memory");
    return v;
}

__device__ __forceinline__ unsigned pk16(float a, float b) {
    f32x2 v = {a, b};
    return __builtin_bit_cast(unsigned, __builtin_convertvector(v, half2v));
}

typedef unsigned short u16x2 __attribute__((ext_vector_type(2)));
__device__ __forceinline__ unsigned pk_max(unsigned a, unsigned b) { return __builtin_bit_cast(unsigned, __builtin_elementwise_max(__builtin_bit_cast(u16x2, a), __builtin_bit_cast(u16x2, b))); }
__device__ __forceinline__ unsigned wave_scan(unsigned x) {
    x += (unsigned)__builtin_amdgcn_update_dpp(0, (int)x, 0x111, 0xf, 0xf, false);
    x += (unsigned)__builtin_amdgcn_update_dpp(0, (int)x, 0x112, 0xf, 0xf, false);
    x += (unsigned)__builtin_amdgcn_update_dpp(0, (int)x, 0x114, 0xf, 0xf, false);
    x += (unsigned)__builtin_amdgcn_update_dpp(0, (int)x, 0x118, 0xf, 0xf, false);
    x += (unsigned)__builtin_amdgcn_update_dpp(0, (int)x, 0x142, 0xa, 0xf, false);
    x += (unsigned)__builtin_amdgcn_update_dpp(0, (int)x, 0x143, 0xc, 0xf, false);
    return x;
}

__device__ __forceinline__ unsigned pk_min(unsigned a, unsigned b) { return __builtin_bit_cast(unsigned, __builtin_elementwise_min(__builtin_bit_cast(u16x2, a), __builtin_bit_cast(u16x2, b))); }
__device__ __forceinline__ unsigned pk_subs(unsigned a, unsigned b) { return __builtin_bit_cast(unsigned, __builtin_elementwise_sub_sat(__builtin_bit_cast(u16x2, a), __builtin_bit_cast(u16x2, b))); }

__device__ __forceinline__ unsigned pack_item(unsigned (&D)[32], unsigned (&P)[26], u32x2& sd, LAS unsigned char* sc, int lane) {
    unsigned se = 0;
#pragma unroll
    for (int i = 0; i < 32; ++i) se += (D[i] >> 10) & 0x001f001fu;
    const unsigned S = __builtin_amdgcn_readlane(wave_scan((se & 0xffffu) + (se >> 16)), 63);
    const unsigned b1 = min((S + 246u) >> 12, 29u), B = (b1 + 27u) * 0x10001u;
    *(LAS u32x2*)(sc + lane * 8) = (u32x2){0u, 0u};
    if (lane < 8) *(LAS u32x2*)(sc + 512 + lane * 8) = (u32x2){0u, 0u};
    u64 lo = 1; unsigned hi = 0, len = 0, worst = 0, up = 0;
    bool ovf = false;
#pragma unroll
    for (int q = 15; q >= 0; --q) {
        unsigned W = 1, M[2];
#pragma unroll
        for (int h = 1; h >= 0; --h) {
            const int i = 2 * q + h;
            const unsigned E = ((D[i] >> 10) & 0x001f001fu) | 0x00200020u, dd = pk_subs(E, B), ix = pk_min(dd, 0x00070007u) | 0x0c000c00u;
            const unsigned a3 = pk_subs(dd, 0x00070007u) * 3u;
            up = pk_max(up, a3);
            M[h] = __builtin_amdgcn_perm(0x00000001u, 0x02040507u, ix) + pk_subs(B, E) + a3;
            D[i] = (D[i] & 0x83ff83ffu) | (pk_subs(__builtin_amdgcn_perm(0x03020100u, 0u, ix), a3) << 10);
            W = (W << ((M[h] >> 16) & 31u)) | 1u;
            W = (W << (M[h] & 31u)) | 1u;
        }
        const unsigned ms = M[0] + M[1], sm = (ms & 0xffffu) + (ms >> 16);
        worst = max(worst, sm); len += sm;
        if (q < 8) hi = (unsigned)(((((u64)hi << 32) | (lo >> 32)) << (sm & 31u)) >> 32);
        lo = (lo << (sm & 31u)) | W;
        if (q == 8) ovf = len > 63u;
        asm volatile("" : "+v"(lo), "+v"(hi));
    }
    ovf |= len > 95u || worst > 31u || pk_subs(up, 0x00060006u) != 0u;
    const unsigned end = wave_scan(ovf ? 0u : len), o = end - (ovf ? 0u : len);
    ovf |= end > 3583u;
    sc[lane] = (unsigned char)(ovf ? 255u : len);
    if (!ovf) {
        LAS unsigned* w = (LAS unsigned*)(sc + 64) + (o >> 5);
        const u64 a = lo << (o & 31u), b = (((u64)hi << 32) | (lo >> 32)) << (o & 31u), c = (u64)hi << (o & 31u);
        __hip_atomic_fetch_or(w, (unsigned)a, __ATOMIC_RELAXED, __HIP_MEMORY_SCOPE_WAVEFRONT);
        __hip_atomic_fetch_or(w + 1, (unsigned)(a >> 32), __ATOMIC_RELAXED, __HIP_MEMORY_SCOPE_WAVEFRONT);
        __hip_atomic_fetch_or(w + 2, (unsigned)(b >> 32), __ATOMIC_RELAXED, __HIP_MEMORY_SCOPE_WAVEFRONT);
        __hip_atomic_fetch_or(w + 3, (unsigned)(c >> 32), __ATOMIC_RELAXED, __HIP_MEMORY_SCOPE_WAVEFRONT);
    }
#pragma unroll
    for (int g = 0; g < 2; ++g) {
#pragma unroll
        for (int i = 0; i < 12; ++i) P[13 * g + i] = D[16 * g + i] | ((D[16 * g + 13 + i / 4] << (12 - 3 * (i & 3))) & 0x70007000u);
        P[13 * g + 12] = D[16 * g + 12] | ((D[16 * g + 13] >> 3) & 0x10001000u) | ((D[16 * g + 14] >> 2) & 0x20002000u) | ((D[16 * g + 15] >> 1) & 0x40004000u);
    }
    asm volatile("s_waitcnt lgkmcnt(0)" ::: "memory");
    sd = *(volatile LAS u32x2*)(sc + lane * 8);
    const unsigned tot = min((unsigned)__builtin_amdgcn_readlane(end, 63), 3583u);
    return b1 | (((tot + 512u) >> 9) << 5);
}
template <int Q>
__device__ __forceinline__ void unpack8(unsigned (&H)[8], unsigned BB, unsigned& s0, unsigned& s1, unsigned& s2) {
#pragma unroll
    for (int q = 0; q < 4; ++q) {
        unsigned t = s0, M[2];
#pragma unroll
        for (int h = 0; h < 2; ++h) {
            const unsigned T = H[2 * q + h];
            const half2v f = __builtin_bit_cast(half2v, T);
            const unsigned m0 = (unsigned)__builtin_ctz(t - (__builtin_amdgcn_classh(f[0], 0xf0) ? 1u : 0u));
            t >>= m0 & 31u;
            const unsigned m1 = (unsigned)__builtin_ctz(t - (__builtin_amdgcn_classh(f[1], 0xf0) ? 1u : 0u));
            if (h == 0) t >>= m1 & 31u;
            M[h] = m0 | (m1 << 16);
            H[2 * q + h] = ((__builtin_amdgcn_perm(0x030b0202u, 0x07010100u, pk_min(M[h], 0x00070007u)) - M[h]) << 10) + (T + BB);
        }
        const unsigned ms = M[0] + M[1], sm = ms + (ms >> 16);
        s0 = __builtin_amdgcn_alignbit(s1, s0, sm);
        if (Q < 2) { s1 = __builtin_amdgcn_alignbit(s2, s1, sm); s2 >>= sm & 31u; } else s1 >>= sm & 31u;
        asm volatile("" : "+v"(H[2 * q]), "+v"(H[2 * q + 1]));
    }
}
constexpr int pull_younger(int k) {
    int n = 0;
    for (int j = k - PW; j < k; ++j) {
        const int jj = (j + 7) % 7;
        if (j > k - PW) n += (j + PW) % 7 == 0 ? 2 : 1;
        if (jj == 1 || jj == 3 || jj == 5 || jj == 6) n += 2;
    }
    return n;
}

__device__ __forceinline__ void item_decode(int it, int src, int& s, int& r) {
    int e;
    if (it >= 616 && it < 704 + 32 * WBE) it = it < 616 + 32 * WBE ? it + 88 : it - 32 * WBE;
    if (it < 704) { s = it / 88; e = it - 88 * s; }
    else { s = (it - 704) >> 5; e = 88 + (it & 31); }
    r = e < 56 ? (8 * src + 8 + e) & 63 : e + 8;
}

#define UNIT_LOAD(B, S_, R_, H_) do { const bool isx_ = (R_) < 64; \
        const gchar* b_ = isx_ ? sX + (size_t)((R_) * 64 + (H_) * 32) * (KSH * 4) + (size_t)(S_) * 256 \
                               : sW + (size_t)((S_) * 64 + (H_) * 32) * (NCOLS * 4) + (size_t)((R_) - 64) * 256; \
        const unsigned vo_ = isx_ ? xoff : woff; \
        const gchar* b1_ = b_ + (isx_ ? 16 : 1 * NCOLS * 4);          const gchar* b2_ = b_ + (isx_ ? 128 : 2 * NCOLS * 4); \
        const gchar* b3_ = b_ + (isx_ ? 144 : 3 * NCOLS * 4);         const gchar* b4_ = b_ + (isx_ ? 32768 : 4 * NCOLS * 4); \
        const gchar* b5_ = b_ + (isx_ ? 32768 + 16 : 5 * NCOLS * 4);  const gchar* b6_ = b_ + (isx_ ? 32768 + 128 : 6 * NCOLS * 4); \
        const gchar* b7_ = b_ + (isx_ ? 32768 + 144 : 7 * NCOLS * 4); \
        asm volatile("s_nop 4\n\tglobal_load_dwordx4 %0, %8, %9" NTIN "\n\tglobal_load_dwordx4 %1, %8, %10" NTIN "\n\t" \
                     "global_load_dwordx4 %2, %8, %11" NTIN "\n\tglobal_load_dwordx4 %3, %8, %12" NTIN "\n\t" \
                     "global_load_dwordx4 %4, %8, %13" NTIN "\n\tglobal_load_dwordx4 %5, %8, %14" NTIN "\n\t" \
                     "global_load_dwordx4 %6, %8, %15" NTIN "\n\tglobal_load_dwordx4 %7, %8, %16" NTIN "" \
                     : "+v"(B[0]), "+v"(B[1]), "+v"(B[2]), "+v"(B[3]), "+v"(B[4]), "+v"(B[5]), "+v"(B[6]), "+v"(B[7]) \
                     : "v"(vo_), "s"(b_), "s"(b1_), "s"(b2_), "s"(b3_), "s"(b4_), "s"(b5_), "s"(b6_), "s"(b7_) : "memory"); } while (0)
#define UNIT_WAIT(B, N) asm volatile("s_waitcnt vmcnt(%8)" : "+v"(B[0]), "+v"(B[1]), "+v"(B[2]), "+v"(B[3]), \
                                     "+v"(B[4]), "+v"(B[5]), "+v"(B[6]), "+v"(B[7]) : "i"(N) : "memory")
#define UNIT_CVT(O, B, R_) do { if ((R_) < 64) { \
            _Pragma("unroll") for (int b = 0; b < 4; ++b) { \
                const f32x4 lo_ = __builtin_bit_cast(f32x4, B[2 * b]), hi_ = __builtin_bit_cast(f32x4, B[2 * b + 1]); \
                O[b] = (u32x4){pk16(lo_[0], lo_[1]), pk16(lo_[2], lo_[3]), pk16(hi_[0], hi_[1]), pk16(hi_[2], hi_[3])}; } \
        } else { \
            _Pragma("unroll") for (int j = 0; j < 4; ++j) \
                O[j] = (u32x4){pk16(__builtin_bit_cast(f32x4, B[0])[j], __builtin_bit_cast(f32x4, B[1])[j]), \
                               pk16(__builtin_bit_cast(f32x4, B[2])[j], __builtin_bit_cast(f32x4, B[3])[j]), \
                               pk16(__builtin_bit_cast(f32x4, B[4])[j], __builtin_bit_cast(f32x4, B[5])[j]), \
                               pk16(__builtin_bit_cast(f32x4, B[6])[j], __builtin_bit_cast(f32x4, B[7])[j])}; } \
        asm volatile("" : "+v"(O[0]), "+v"(O[1]), "+v"(O[2]), "+v"(O[3])); } while (0)
#define ITEM_OFF(R_) ((R_) < 64 ? (size_t)(R_) * 8192 : (size_t)(((R_) - 64) >> 2) * PANEL + (size_t)((R_) & 3) * 4096)
#define UNIT_DST(BASE, R_, H_) const bool isxd_ = (R_) < 64; \
        gchar* t0_ = (BASE) + (isxd_ ? (H_) * 4096 : (H_) * 1024); \
        gchar* t1_ = t0_ + (isxd_ ? 1024 : 2048); gchar* t2_ = t0_ + (isxd_ ? 2048 : 16384); gchar* t3_ = t0_ + (isxd_ ? 3072 : 16384 + 2048)
#define STOREO(VOFF, DATA, BASE) asm volatile("s_nop 4\n\tglobal_store_dwordx4 %0, %1, %2 sc0 sc1" NTOUT "\n\ts_nop 1" :: "v"(VOFF), "v"(DATA), "s"(BASE) : "memory")
#define STORE16(VOFF, DATA, BASE, N) asm volatile("s_nop 3\n\ts_waitcnt vmcnt(%3)\n\tglobal_store_dwordx4 %0, %1, %2 sc0 sc1\n\ts_nop 1" \
                                                  :: "v"(VOFF), "v"(DATA), "s"(BASE), "i"(N) : "memory")

__global__ __launch_bounds__(512, 2) void gemm_rs_kernel(const Params p) {
    __shared__ __attribute__((aligned(16))) unsigned char shm[131072 + 256];
    LAS unsigned char* lds = (LAS unsigned char*)shm;
    volatile LAS unsigned* lctl = (volatile LAS unsigned*)(lds + 131072);
    const int tid = threadIdx.x, wid = __builtin_amdgcn_readfirstlane(tid >> 6), lane = tid & 63;
    const int rank = p.rank & (NRANK - 1), bid = blockIdx.x;
    gchar* myws = (gchar*)p.ws[rank];
    gu32* tmo = (gu32*)(myws + OFF_TMO);
    const unsigned long long tstart = __builtin_amdgcn_s_memrealtime();

    if (tid == 0) {
        lctl[0] = __hip_atomic_fetch_add((gu32*)(myws + OFF_TICKET + bid * 128), 1u, __ATOMIC_RELAXED, __HIP_MEMORY_SCOPE_AGENT) + 1u;
    } else if (tid < 64) lctl[tid] = 0u;
    unsigned call; size_t bufoff;
#define GET_CALL() do { asm volatile("s_waitcnt lgkmcnt(0)" ::: "memory"); __builtin_amdgcn_s_barrier(); asm volatile("" ::: "memory"); \
        call = __builtin_amdgcn_readfirstlane(lctl[0]); bufoff = CTRL_BYTES + (size_t)(call & 1u) * BUF_BYTES; } while (0)

    const int xcd = bid & 7, idx = bid >> 3, dd = idx >> 2;
    const int dest = (rank + dd) & (NRANK - 1);
    const int mt = 2 * dest + (xcd & 1), nt = 2 * (xcd >> 1) + (idx & 1) + 8 * ((idx >> 1) & 1);
    const int hidx = (dd - 1) * 16 + (bid & 15);

    const unsigned soff0 = (unsigned)lane * 16u;
    const gchar* sX = (const gchar*)p.x;
    const unsigned xoff = (unsigned)(lane & 15) * (KSH * 4) + (unsigned)(lane >> 4) * 32u;
    const unsigned woff = (unsigned)(lane >> 4) * 8u * (NCOLS * 4) + (unsigned)(lane & 15) * 16u;
    if (bid >= NPROD && nt >= 8 && wid >= 1 && wid <= CW) {
        u32x4 buf0[8], buf1[8], o[4], o1[4];
#pragma unroll
        for (int b = 0; b < 8; ++b) { buf0[b] = (u32x4){0u, 0u, 0u, 0u}; buf1[b] = (u32x4){0u, 0u, 0u, 0u}; }
        bool first = true;
        for (int job = NPUSH + (wid - 1) * 112 + hidx; job < NCONV; job += CW * 112) {
            int s, e;
            if (job < 2560) { s = job / 320; e = job - 320 * s; }
            else { s = (job - 2560) >> 8; e = 320 + (job & 255); }
            const int d = e < 64 ? rank : (rank + 1 + ((e - 64) & 7)) & (NRANK - 1);
            const int r = e < 64 ? (8 * rank + 8 + e) & 63 : 64 + ((e - 64) >> 3);
            const gchar* sW = (const gchar*)p.w + (size_t)d * NLOC * 4;
            UNIT_LOAD(buf0, s, r, 0); UNIT_LOAD(buf1, s, r, 1);
            if (first) { GET_CALL(); first = false; }
            gchar* tb = myws + bufoff + ITEM_OFF(r) + (e < 64 ? (size_t)(s * 8 + rank) * KT_STRIDE
                        : d == rank ? MAT_BYTES + (size_t)(s * 8 + rank) * KT_STRIDE : 2 * MAT_BYTES + (size_t)(d * 8 + s) * KT_STRIDE);
            UNIT_WAIT(buf0, 8); UNIT_CVT(o, buf0, r);
            if (NTRAW && d != rank) { UNIT_DST(tb, r, 0); STOREO(soff0, o[0], t0_); STOREO(soff0, o[1], t1_); STOREO(soff0, o[2], t2_); STOREO(soff0, o[3], t3_); }
            else { UNIT_DST(tb, r, 0); STORE16(soff0, o[0], t0_, 63); STORE16(soff0, o[1], t1_, 63); STORE16(soff0, o[2], t2_, 63); STORE16(soff0, o[3], t3_, 63); }
            UNIT_WAIT(buf1, 4); UNIT_CVT(o1, buf1, r);
            if (NTRAW && d != rank) { UNIT_DST(tb, r, 1); STOREO(soff0, o1[0], t0_); STOREO(soff0, o1[1], t1_); STOREO(soff0, o1[2], t2_); STOREO(soff0, o1[3], t3_); }
            else { UNIT_DST(tb, r, 1); STORE16(soff0, o1[0], t0_, 63); STORE16(soff0, o1[1], t1_, 63); STORE16(soff0, o1[2], t2_, 63); STORE16(soff0, o1[3], t3_, 63); }
            unsigned meta = 0;
            if (e < 56 || d != rank) {
                unsigned D[32], P[26]; u32x2 sd;
#pragma unroll
                for (int i = 0; i < 16; ++i) { D[i] = o[i >> 2][i & 3]; D[16 + i] = o1[i >> 2][i & 3]; }
                meta = pack_item(D, P, sd, lds + wid * 1024, lane);
                gchar* pi = myws + bufoff + OFF_PI + (size_t)(e < 64 ? s * 64 + r : 512 + (d * 8 + s) * 64 + r - 64) * PI_ITEM;
#pragma unroll
                for (int i = 0; i < 6; ++i) { const u32x4 v = {P[4 * i], P[4 * i + 1], P[4 * i + 2], P[4 * i + 3]}; STORE16(soff0, v, pi + i * 1024, 63); }
                const u32x2 v2 = {P[24], P[25]};
                asm volatile("s_nop 4\n\tglobal_store_dwordx2 %0, %1, %3 sc0 sc1\n\tglobal_store_dwordx2 %0, %2, %3 offset:512 sc0 sc1"
                             :: "v"(soff0 >> 1), "v"(v2), "v"(sd), "s"(pi + 6144) : "memory");
            }
            asm volatile("s_waitcnt vmcnt(0)" ::: "memory");
            if (TGATE) while (__builtin_amdgcn_s_memrealtime() - tstart < TGATE) __builtin_amdgcn_s_sleep(2);
            const int fo = rank * 4096 + (s * 128 + r) * 4;
#pragma unroll
            for (int k = 0; k < NRANK; ++k)
                if ((e < 56 || k == d) && lane == 0)
                    __hip_atomic_store((gu32*)((gchar*)p.ws[k] + (k == rank ? OFF_INFLAG : OFF_CFLAG) + fo), k == rank ? call : (call << 8) | meta, __ATOMIC_RELAXED, __HIP_MEMORY_SCOPE_SYSTEM);
        }
        if (lane == 0) __hip_atomic_fetch_add((gu32*)(myws + OFF_CDONE), 1u, __ATOMIC_RELAXED, __HIP_MEMORY_SCOPE_AGENT);
        if (wid == 1) STAMPW(15);
    } else if (bid >= NPROD && nt < 8 && wid == 1 && hidx < NPUSH) {
        u32x4 buf0[8], buf1[8], o0[4], o1[4];
#pragma unroll
        for (int b = 0; b < 8; ++b) { buf0[b] = (u32x4){0u, 0u, 0u, 0u}; buf1[b] = (u32x4){0u, 0u, 0u, 0u}; }
        const gchar* sW = sX;
        const int r = (8 * rank + 8 + hidx) & 63;
        UNIT_LOAD(buf0, 0, r, 0); UNIT_LOAD(buf1, 0, r, 1);
        GET_CALL();
        UNIT_WAIT(buf0, 8); UNIT_CVT(o0, buf0, 0);
        UNIT_WAIT(buf1, 0); UNIT_CVT(o1, buf1, 0);
        const size_t ioff = bufoff + (size_t)rank * KT_STRIDE + (size_t)r * 8192;
#pragma unroll
        for (int b = 0; b < 8; ++b)
            for (int k = 1; k <= NRANK; ++k) {
                gchar* t = (gchar*)p.ws[(rank + hidx + k) & (NRANK - 1)] + ioff + b * 1024;
                STORE16(soff0, b < 4 ? o0[b & 3] : o1[b & 3], t, UW - 1);
            }
        asm volatile("s_waitcnt vmcnt(0)" ::: "memory");
#pragma unroll
        for (int k = 0; k < NRANK; ++k)
            if (lane == 0) __hip_atomic_store((gu32*)((gchar*)p.ws[k] + OFF_INFLAG + rank * 4096 + r * 4), call, __ATOMIC_RELAXED, __HIP_MEMORY_SCOPE_SYSTEM);
        STAMPW(15);
    } else if (bid >= (int)NSTR) {
        GET_CALL();
    } else if (wid == 0) {
        GET_CALL();
        const int j = lane & 7, lk = PMAP ? bid % 7 : (j + bid) % 7, psrc = (rank + 1 + lk) & (NRANK - 1);
        const unsigned long long t0 = __builtin_amdgcn_s_memrealtime();
        unsigned n = 0, nf = 0; int item = 0; bool have = false, fin = lane >= 7, late = false;
        for (unsigned it = 1; !__all(fin && (!SFLAG || lane >= 7 || nf + 1u == n)); ++it) {
            if (SFLAG && lane < 7 && __hip_atomic_load((LAS unsigned*)lctl + 16 + j, __ATOMIC_ACQUIRE, __HIP_MEMORY_SCOPE_WORKGROUP) > nf) {
                ++nf;
                __hip_atomic_store((gu32*)(myws + OFF_INFLAG + psrc * 4096 + (lctl[32 + j * 4 + (nf & 3u)] & 0x7ffu) * 4), call, __ATOMIC_RELAXED, __HIP_MEMORY_SCOPE_SYSTEM);
            }
            if (!fin && !have && n <= lctl[8 + j] && (!SFLAG || n <= nf + 3u)) {
                unsigned t = NPUSH + (PMAP ? j * 4 + bid / 7 : bid) + n * NSTR;
                if (n >= 2) t = __hip_atomic_fetch_add((gu32*)(myws + OFF_QUEUE + lk * 512), 1u, __ATOMIC_RELAXED, __HIP_MEMORY_SCOPE_AGENT)
                                - (call - 1u) * QUEUE_PER_CALL + NPUSH + 2u * NSTR;
                int s, r; item_decode(t < (unsigned)NITEM ? (int)t : 0, psrc, s, r);
                item = t < (unsigned)NITEM ? s * 128 + r : 1024; have = true;
            }
            if (have) {
                bool ready = item >= 1024 || late; unsigned cf = 0;
                if (!ready) { cf = __hip_atomic_load((gu32*)(myws + OFF_CFLAG + psrc * 4096 + item * 4), __ATOMIC_RELAXED, __HIP_MEMORY_SCOPE_SYSTEM); ready = (cf >> 8) >= call; }
                if (ready) { ++n; __hip_atomic_store((LAS unsigned*)lctl + 32 + j * 4 + (n & 3u), (n << 20) | ((cf & 255u) << 12) | (unsigned)item, __ATOMIC_RELEASE, __HIP_MEMORY_SCOPE_WORKGROUP); have = false; fin = item >= 1024; }
            }
            if ((it & 63u) == 0u && !late) {
                late = __builtin_amdgcn_readfirstlane(__hip_atomic_load(tmo, __ATOMIC_RELAXED, __HIP_MEMORY_SCOPE_AGENT)) == call
                       || __builtin_amdgcn_s_memrealtime() - t0 > DEADLINE_TICKS;
                if (late) __hip_atomic_store(tmo, call, __ATOMIC_RELAXED, __HIP_MEMORY_SCOPE_AGENT);
            }
            __builtin_amdgcn_s_sleep(2);
        }
    } else {
        GET_CALL();
        const int lk = PMAP ? bid % 7 : (wid - 1 + bid) % 7, src = (rank + 1 + lk) & (NRANK - 1);
        const gchar* rimg = (const gchar*)p.ws[src] + bufoff;
        gchar* limg = myws + bufoff + (size_t)src * KT_STRIDE;
        gu32* lflag = (gu32*)(myws + OFF_INFLAG + src * 4096);
        u32x4 in[6]; u32x2 in6 = {0u, 0u}, side = {0u, 0u};
#pragma unroll
        for (int b = 0; b < 6; ++b) in[b] = (u32x4){0u, 0u, 0u, 0u};
        const unsigned soff8 = (unsigned)lane * 8u;
        LAS unsigned char* sc = lds + wid * 1024;
#define PI_OF(S_, R_) (rimg + OFF_PI + 3072 + (size_t)((R_) < 64 ? (S_) * 64 + (R_) : 512 + (rank * 8 + (S_)) * 64 + (R_) - 64) * PI_ITEM)
#define RAW_OF(S_, R_) (rimg + ITEM_OFF(R_) + ((R_) < 64 ? (size_t)((S_) * 8 + src) * KT_STRIDE : 2 * MAT_BYTES + (size_t)(rank * 8 + (S_)) * KT_STRIDE))
#define DST_OF(S_, R_) (limg + ITEM_OFF(R_) + ((R_) < 64 ? 0 : MAT_BYTES) + (size_t)(S_) * 8 * KT_STRIDE)
#define PULL(k, BASE, META) do { \
        if ((k) == 0) asm volatile("s_nop 4\n\tglobal_load_dwordx2 %0, %1, %2 offset:3584 sc0 sc1" : "+v"(side) : "v"((soff8 >> 6) <= ((META) >> 5) ? soff8 : 0u), "s"(BASE) : "memory"); \
        if ((k) < 6) asm volatile("s_nop 4\n\tglobal_load_dwordx4 %0, %1, %2 offset:%3 sc0 sc1" : "+v"(in[(k) < 6 ? (k) : 0]) : "v"(soff0), "s"(BASE), "i"((k) * 1024 - 3072) : "memory"); \
        else asm volatile("s_nop 4\n\tglobal_load_dwordx2 %0, %1, %2 offset:3072 sc0 sc1" : "+v"(in6) : "v"(soff8), "s"(BASE) : "memory"); } while (0)
#define TAKE(DST, META, N_) do { unsigned e_; while (((e_ = __builtin_amdgcn_readfirstlane(__hip_atomic_load((LAS unsigned*)lctl + 28 + wid * 4 + ((N_) & 3), __ATOMIC_ACQUIRE, __HIP_MEMORY_SCOPE_WORKGROUP))) >> 20) != (unsigned)(N_)) __builtin_amdgcn_s_sleep(1); \
        lctl[7 + wid] = (unsigned)(N_); asm volatile("" ::: "memory"); DST = (int)(e_ & 0x7ffu); META = (e_ >> 12) & 255u; } while (0)
        int cur, s, r, pflag = -1; unsigned meta;
        TAKE(cur, meta, 1);
        s = cur >> 7; r = cur & 127;
        const gchar* sb = PI_OF(s, r);
#pragma unroll
        for (int k = 0; k < PW; ++k) PULL(k, sb, meta);
        if (wid == 1) STAMPW(24);
        int i = 0;
        for (; cur < 1024; ++i) {
            if (wid == 1 && i % 5 == 0 && i <= 25) STAMPW(25 + i / 5);
            int nxt = 1024, sn = 0, rn = 0; const gchar* sbn = sb; unsigned metan = 0;
            gchar* db = DST_OF(s, r);
            const bool isx = r < 64;
            const gchar* rraw = RAW_OF(s, r);
            asm volatile("" : "+s"(rraw), "+s"(db));
            const unsigned BB = ((meta & 31u) - 1u) * 0x04000400u;
            unsigned s0 = 0, s1 = 0, s2 = 0, dn[3] = {0u, 0u, 0u}; bool ovf = false;
            auto step = [&](auto KC) __attribute__((always_inline)) {
                constexpr int k = decltype(KC)::value;
                (void)&rraw; (void)&db; (void)&sb; (void)&soff0; (void)&soff8; (void)&in; (void)&in6; (void)&side;
                if (k + PW == 7) {
                    TAKE(nxt, metan, i + 2);
                    sn = (nxt >> 7) & 7; rn = nxt & 127;
                    sbn = nxt < 1024 ? PI_OF(sn, rn) : (const gchar*)myws + bufoff + OFF_PI + 3072;
                    if (nxt >= 1024 && lane == 0) __hip_atomic_fetch_add((gu32*)(myws + OFF_PDONE + lk * 512), 1u, __ATOMIC_RELAXED, __HIP_MEMORY_SCOPE_AGENT);
                }
                if (k < PW && i == 0) asm volatile("s_waitcnt vmcnt(%0)" :: "i"(PW - 1) : "memory");
                if (k < 6) asm volatile("s_waitcnt vmcnt(%2)" : "+v"(in[k < 6 ? k : 0]), "+v"(side) : "i"(pull_younger(k)) : "memory");
                else asm volatile("s_waitcnt vmcnt(%1)" : "+v"(in6) : "i"(pull_younger(k)) : "memory");
                if (SFLAG && k == PW) lctl[15 + wid] = (unsigned)i;
                if (!SFLAG && k == PW && pflag >= 0 && lane == 0) __hip_atomic_store(lflag + pflag, call, __ATOMIC_RELAXED, __HIP_MEMORY_SCOPE_SYSTEM);
                if (k + PW < 7) PULL((k + PW) % 7, sb, meta); else PULL((k + PW) % 7, sbn, metan);
                __builtin_amdgcn_sched_barrier(0);
                if (k == 0) {
                    *(volatile LAS u32x2*)(sc + lane * 8) = side;
                    const unsigned lb = *(volatile LAS unsigned char*)(sc + lane);
                    ovf = lb == 255u;
                    const unsigned len = ovf ? 0u : lb, o = min(wave_scan(len) - len, 3583u);
                    const volatile LAS unsigned* w = (const volatile LAS unsigned*)(sc + 64) + (o >> 5);
                    const unsigned w0 = w[0], w1 = w[1], w2 = w[2], w3 = w[3];
                    s0 = __builtin_amdgcn_alignbit(w1, w0, o) | 1u; s1 = __builtin_amdgcn_alignbit(w2, w1, o); s2 = __builtin_amdgcn_alignbit(w3, w2, o);
                    asm volatile("" : "+v"(s0), "+v"(s1), "+v"(s2));
                }
                if (k == 1 || k == 3 || k == 5 || k == 6) {
                    constexpr int g = k >> 1;
#define PIN(n) ((n) < 24 ? in[((n) < 24 ? (n) : 0) >> 2][(n) & 3] : in6[(n) & 1])
                    unsigned H[8];
#pragma unroll
                    for (int e = 0; e < ((g & 1) ? 5 : 8); ++e) {
                        const int ci = (g & 1) * 8 + e; const unsigned pw = PIN(13 * (g >> 1) + ci);
                        H[e] = pw & 0x8fff8fffu;
                        if (ci < 12) { const unsigned pc = (pw >> (12 - 3 * (ci & 3))) & (0x00070007u << (3 * (ci & 3))); dn[ci >> 2] = (ci & 3) ? dn[ci >> 2] | pc : pc; }
                        else { H[5] = dn[0] | ((pw << 3) & 0x80008000u); H[6] = dn[1] | ((pw << 2) & 0x80008000u); H[7] = dn[2] | ((pw << 1) & 0x80008000u); }
                    }
                    unpack8<g>(H, BB, s0, s1, s2);
                    u32x4 v0 = {H[0], H[1], H[2], H[3]}, v1 = {H[4], H[5], H[6], H[7]};
                    const unsigned o0 = soff0 + (isx ? g * 2048 : (g & 1) * 16384 + (g >> 1) * 1024), o1 = o0 + (isx ? 1024 : 2048);
                    if (ovf) asm volatile("s_nop 4\n\tglobal_load_dwordx4 %0, %2, %4 sc0 sc1\n\tglobal_load_dwordx4 %1, %3, %4 sc0 sc1\n\ts_waitcnt vmcnt(0)"
                                          : "+v"(v0), "+v"(v1) : "v"(o0), "v"(o1), "s"(rraw) : "memory");
                    STORE16(o0, v0, db, 63); STORE16(o1, v1, db, 63);
                }
            };
            step(IC<0>{}); step(IC<1>{}); step(IC<2>{}); step(IC<3>{}); step(IC<4>{}); step(IC<5>{}); step(IC<6>{});
            pflag = cur; cur = nxt; s = sn; r = rn; sb = sbn; meta = metan;
        }
        asm volatile("s_waitcnt vmcnt(0)" : "+v"(in[0]), "+v"(in[1]), "+v"(in[2]), "+v"(in[3]), "+v"(side) :: "memory");
        if (SFLAG) lctl[15 + wid] = (unsigned)i;
        else if (lane == 0) __hip_atomic_store(lflag + pflag, call, __ATOMIC_RELAXED, __HIP_MEMORY_SCOPE_SYSTEM);
        STAMPW(16 + wid);
    }

    const int wr = wid >> 2, wc = wid & 3;
    const gchar* img = myws + bufoff;
    const unsigned voff = (unsigned)tid * 16u;
    const unsigned ldsw = (unsigned)wid * 1024u;
    const int aoff = wr * 8192 + lane * 16, boff = wc * 4096 + lane * 16;
    constexpr int HTB = 16384;
#define SA(b, h) (((b) * 2 + (h)) * HTB)
#define SB(b, h) ((4 + (b) * 2 + (h)) * HTB)
#define STAGE(bufo, gbase) do { _Pragma("unroll") for (int _i = 0; _i < 2; ++_i) \
        __builtin_amdgcn_global_load_lds((const GAS unsigned*)((gbase) + voff + _i * 8192), (LAS unsigned*)(lds + (bufo) + ldsw + _i * 8192), 16, 0, GAUX); } while (0)
#define LDA(dst, b, h) do { _Pragma("unroll") for (int m = 0; m < 4; ++m) _Pragma("unroll") for (int k = 0; k < 2; ++k) \
        dst[m][k] = *(const LAS half8*)(lds + SA(b, h) + aoff + m * 2048 + k * 1024); } while (0)
#define LDB(dst, b, h) do { _Pragma("unroll") for (int n = 0; n < 2; ++n) _Pragma("unroll") for (int k = 0; k < 2; ++k) \
        dst[n][k] = *(const LAS half8*)(lds + SB(b, h) + boff + n * 2048 + k * 1024); } while (0)
#define MMA(ai, bj, At_, Bt_) do { __builtin_amdgcn_s_setprio(1); \
        _Pragma("unroll") for (int m = 0; m < 4; ++m) _Pragma("unroll") for (int n = 0; n < 2; ++n) _Pragma("unroll") for (int k = 0; k < 2; ++k) \
            acc[ai][bj][m][n] = __builtin_amdgcn_mfma_f32_16x16x32_f16(At_[m][k], Bt_[n][k], acc[ai][bj][m][n], 0, 0, 0); \
        __builtin_amdgcn_s_setprio(0); } while (0)
#define WAIT_V(n) asm volatile("s_waitcnt vmcnt(" #n ")" ::: "memory")
#define WAIT_L(n) asm volatile("s_waitcnt lgkmcnt(" #n ")" ::: "memory")
#define BAR __builtin_amdgcn_s_barrier()
#define SCHED __builtin_amdgcn_sched_barrier(0)
    bool allin = false;
#define WAIT_SEG(c) do { if (wid == 0 && !allin) { int lz_ = lane; asm volatile("" : "+v"(lz_)); const int wh_ = lz_ >> 3; \
        const int it_ = wh_ < 4 && ((c) == 8 || (lz_ & 7) != dest) ? 4 * mt + wh_ : 64 + 4 * nt + (wh_ & 3); \
        gu32* fa_ = (gu32*)(myws + OFF_INFLAG + ((c) == 8 ? rank * 4096 + (lz_ & 7) * 512 : (lz_ & 7) * 4096 + (c) * 512) + it_ * 4); \
        bool now_ = false; \
        if ((c) < 8 && (nt >= 8 || dd == 0)) { unsigned m_ = __hip_atomic_load((gu32*)(myws + OFF_CDONE), __ATOMIC_RELAXED, __HIP_MEMORY_SCOPE_SYSTEM) >= call * (112u * CW) ? call : 0u; \
            const unsigned f0_ = __hip_atomic_load(fa_, __ATOMIC_RELAXED, __HIP_MEMORY_SCOPE_SYSTEM); \
            _Pragma("unroll") for (int k_ = 1; k_ < 8; ++k_) m_ = min(m_, __hip_atomic_load(fa_ + ((c) + k_ < 8 ? k_ : 0) * 128, __ATOMIC_RELAXED, __HIP_MEMORY_SCOPE_SYSTEM)); \
            now_ = __all(f0_ >= call); allin = now_ && __all(m_ >= call); asm volatile("" ::: "memory"); } \
        if (!now_) { if ((c) == (nt < 8 ? 7 : WBE)) poll_ge<ISL>(fa_, call, tmo, call); else poll_ge<ISLOW>(fa_, call, tmo, call); } \
        if ((c) == 6 && !allin) poll_ge<64>((gu32*)(myws + OFF_CDONE), call * (112u * CW), tmo, call);     \
        } } while (0)

    f32x4 acc[2][2][4][2];
#pragma unroll
    for (int a = 0; a < 2; ++a)
#pragma unroll
        for (int b = 0; b < 2; ++b)
#pragma unroll
            for (int m = 0; m < 4; ++m)
#pragma unroll
                for (int n = 0; n < 2; ++n) acc[a][b][m][n] = (f32x4){0.f, 0.f, 0.f, 0.f};
    half8 At[4][2], B0[2][2], B1[2][2];

#define A_OF(t) (img + ((unsigned)__builtin_amdgcn_readlane(tabA, (t) & 63) & ~15u))
#define B_OF(t) (img + (unsigned)__builtin_amdgcn_readlane(tabB, (t) & 63))
#define GEMM_LOOP(T0, T1, HOOK) do { \
    const gchar* a3 = A_OF((T0) + 1); \
    for (int t = (T0); t < (T1); t += 2) { \
        const gchar* a1 = a3; \
        LDB(B0, 0, 0); SCHED; LDA(At, 0, 0); STAGE(SA(1, 1), a1 + HTB); SCHED; \
        const gchar* a2 = A_OF(t + 2); const gchar* b2 = B_OF(t + 2); const gchar* b3 = B_OF(t + 3); a3 = A_OF(t + 3); \
        WAIT_L(8); BAR; WAIT_L(0); MMA(0, 0, At, B0); \
        HOOK; \
        asm volatile("" ::: "memory"); BAR; SCHED; \
        LDB(B1, 0, 1); STAGE(SB(0, 0), b2); \
        BAR; WAIT_L(0); MMA(0, 1, At, B1); BAR; \
        LDA(At, 0, 1); STAGE(SA(0, 0), a2); \
        BAR; WAIT_L(0); MMA(1, 0, At, B0); BAR; SCHED; \
        STAGE(SB(0, 1), b2 + HTB); \
        WAIT_V(6); BAR; MMA(1, 1, At, B1); BAR; \
        LDB(B0, 1, 0); SCHED; LDA(At, 1, 0); STAGE(SA(0, 1), a2 + HTB); \
        WAIT_L(8); BAR; WAIT_L(0); MMA(0, 0, At, B0); BAR; SCHED; \
        LDB(B1, 1, 1); STAGE(SB(1, 0), b3); \
        BAR; WAIT_L(0); MMA(0, 1, At, B1); BAR; \
        LDA(At, 1, 1); STAGE(SA(1, 0), a3); \
        BAR; WAIT_L(0); MMA(1, 0, At, B0); BAR; SCHED; \
        STAGE(SB(1, 1), b3 + HTB); \
        WAIT_V(6); BAR; MMA(1, 1, At, B1); BAR; \
    } \
    if (wr == 0) BAR;                                                   \
    } while (0)

    const int src = (rank - dd) & (NRANK - 1);
    const bool early = dd != 0 && nt >= 8;
#define DIAG() do { WAIT_V(0); BAR; if (wr == 1) BAR; asm volatile("" ::: "memory"); GEMM_LOOP(56, 64, ); STAMP(16); } while (0)

    int tabA, tabB;
    {
        const int t0 = nt >= 8 ? 0 : OWNA, u = lane < t0 ? lane : lane - 8, oo = 8 - dd;
        const int st = dd == 0 ? lane / 7 : u / 6, jj = dd == 0 ? lane % 7 : u % 6, o = jj + 1 + (jj + 1 >= oo ? 1 : 0);
        const bool own = dd != 0 && lane >= t0 && lane < t0 + 8;
        const int kt = own ? (lane - t0) * 8 + rank : (st & 7) * 8 + ((dest + o) & 7), q = lane & 7;
        tabA = lane < 56 ? (kt * (int)KT_STRIDE + mt * (int)PANEL) | (own ? 8 : st & 7) : ((q * 8 + rank) * (int)KT_STRIDE + (2 * rank + (mt & 1)) * (int)PANEL) | 7;
        tabB = nt * (int)PANEL + (lane < 56 ? (int)MAT_BYTES + kt * (int)KT_STRIDE
                                  : dd == 0 ? (int)MAT_BYTES + (q * 8 + rank) * (int)KT_STRIDE : 2 * (int)MAT_BYTES + (src * 8 + q) * (int)KT_STRIDE);
    }
#define CLK0
    int have = __builtin_amdgcn_readlane(tabA, 0) & 15;
    BAR;
    WAIT_SEG(have);
    asm volatile("" ::: "memory"); BAR; asm volatile("" ::: "memory");
    STAMP(1);
    { const gchar* a0 = A_OF(0); const gchar* b0 = B_OF(0); const gchar* a1 = A_OF(1); const gchar* b1 = B_OF(1);
      STAGE(SB(0, 0), b0); STAGE(SA(0, 0), a0); STAGE(SB(0, 1), b0 + HTB); STAGE(SA(0, 1), a0 + HTB);
      if (wr == 1) BAR;
      WAIT_V(4); BAR;
      STAGE(SB(1, 0), b1); STAGE(SA(1, 0), a1); STAGE(SB(1, 1), b1 + HTB);
      WAIT_V(6); BAR; }
    GEMM_LOOP(0, 56, { const int seg_ = __builtin_amdgcn_readlane(tabA, t + 3) & 15;
        if (seg_ != have) { have = seg_; WAIT_SEG(have); if (have < 8) STAMP(1 + have); CLK0; } });
    STAMP(9);

    int tz = threadIdx.x; asm volatile("" : "+v"(tz));
    const int fr = tz & 15, fq = (tz >> 4) & 3;
    const unsigned lo = (unsigned)(fq * 4) * (NCOLS * 4) + (unsigned)fr * 16u;
    const unsigned soff = (unsigned)(tz & 63) * 16u;
    const size_t torg = ((size_t)((mt & 1) * 256 + wr * 64) * NCOLS + nt * 256 + wc * 64) * 4;
    const unsigned qnan = 0x7fc00000u;
#define ROWOFF(c, g, e) ((size_t)(((c) >> 1) * 128 + (((c) & 1) * 2 + ((g) >> 1)) * 16 + ((g) & 1) * 2 + ((e) >> 2)) * (NCOLS * 4))
#define ACC(c, g, e) acc[(c) >> 1][((e) >> 1) & 1][((c) & 1) * 2 + ((g) >> 1)][(e) & 1][((g) & 1) * 2 + ((e) >> 2)]
    if (dd != 0) {
    unsigned u[4][4][8];
    {
        const bool bad = __builtin_amdgcn_readfirstlane(__hip_atomic_load(tmo, __ATOMIC_RELAXED, __HIP_MEMORY_SCOPE_AGENT)) == call;
#pragma unroll
        for (int c = 0; c < 4; ++c)
#pragma unroll
            for (int g = 0; g < 4; ++g)
#pragma unroll
                for (int e = 0; e < 8; ++e) { const float v = ACC(c, g, e); u[c][g][e] = bad ? qnan : __builtin_bit_cast(unsigned, v); }
    }
    for (int i = 0; i < (early ? XSTAGB : XSTAG) * wid; ++i) __builtin_amdgcn_s_sleep(1);
    unsigned ebase;
    {
        unsigned mx = 0;
#pragma unroll
        for (int c = 0; c < 4; ++c)
#pragma unroll
            for (int g = 0; g < 4; ++g) {
#pragma unroll
                for (int e = 0; e < 8; ++e) { u[c][g][e] = __builtin_amdgcn_alignbit(u[c][g][e], u[c][g][e], 31); mx = max(mx, u[c][g][e]); }
                asm volatile("" : "+v"(u[c][g][0]), "+v"(u[c][g][1]), "+v"(u[c][g][2]), "+v"(u[c][g][3]), "+v"(u[c][g][4]), "+v"(u[c][g][5]), "+v"(u[c][g][6]), "+v"(u[c][g][7]));
            }
        mx >>= 24;
        unsigned wm = mx;
#pragma unroll
        for (int sft = 1; sft < 64; sft <<= 1) wm = max(wm, (unsigned)__shfl_xor((int)wm, sft));
        ebase = __builtin_amdgcn_readfirstlane(wm);
        const unsigned dl = min(ebase - mx, 7u), base = __builtin_elementwise_sub_sat(ebase - dl, 7u);
        gchar* pk = myws + OFF_PK + (size_t)(bid * 8 + wid) * PK_WAVE;
        unsigned st = 0xffffffffu, nk = 0, nz = 0;
#define U32(n) u[c][(n) >> 3][(n) & 7]
#pragma unroll
        for (int c = 3; c >= 0; --c) {
            unsigned cw[4] = {0u, 0u, 0u, 0u};
#pragma unroll
            for (int n = 31; n >= 0; --n) {
                const unsigned ex = U32(n) >> 24, cd = __builtin_elementwise_sub_sat(ex, base), k = __builtin_elementwise_sub_sat(base, ex);
                const unsigned z = __builtin_elementwise_sub_sat(1u, cd);
                st = ((st << z) | z) << (k & 31u); nk += k; nz += z;
                cw[n / 10] = (cw[n / 10] << 3) | cd;
            }
            const u32x3 cv = {cw[0] | (cw[3] << 30), cw[1] | ((cw[3] >> 2) << 30), cw[2] | ((cw[3] >> 4) << 30)};
#pragma unroll
            for (int i = 0; i < 6; ++i) {
#define CARR(n) __builtin_amdgcn_perm(U32(24 + (n) / 3), U32(n), 0x04020100u + (((n) % 3) << 24))
                const u32x4 v = {CARR(4 * i), CARR(4 * i + 1), CARR(4 * i + 2), CARR(4 * i + 3)};
                STORE16(soff, v, pk + c * PK_CHUNK + i * 1024, 63);
            }
            asm volatile("s_nop 4\n\tglobal_store_dwordx3 %0, %1, %2 sc0 sc1\n\ts_nop 1" :: "v"(soff - (soff >> 2)), "v"(cv), "s"(pk + c * PK_CHUNK + 6144) : "memory");
        }
        const bool ovf = nk != 0u && nk + nz > 28u;
        const unsigned side = (st & 0x0fffffffu) | (dl << 28) | (ovf ? 0x80000000u : 0u);
        asm volatile("s_nop 4\n\tglobal_store_dword %0, %1, %2 sc0 sc1" :: "v"(soff >> 2), "v"(side), "s"(pk + 4 * PK_CHUNK) : "memory");
        if (ovf) {
            gchar* rbase = myws + OFF_RAW + (size_t)(bid * 8 + wid) * RAW_WAVE;
#pragma unroll
            for (int c = 0; c < 4; ++c)
#pragma unroll
                for (int i = 0; i < 8; ++i) {
                    const u32x4 v = {U32(4 * i), U32(4 * i + 1), U32(4 * i + 2), U32(4 * i + 3)};
                    STORE16(soff, v, rbase + (c * 8 + i) * 1024, 63);
                }
        }
        asm volatile("s_waitcnt vmcnt(0)" ::: "memory");
        if ((tz & 63) == 0)
            __hip_atomic_store((gu32*)((gchar*)p.ws[dest] + OFF_OUTFLAG + (bid * 8 + wid) * 64), (call << 8) | ebase, __ATOMIC_RELAXED, __HIP_MEMORY_SCOPE_SYSTEM);
        STAMP(12);
    }
#pragma unroll
    for (int i = 0; i < 32; ++i) acc[i >> 4][(i >> 3) & 1][(i >> 1) & 3][i & 1] = (f32x4){0.f, 0.f, 0.f, 0.f};
    }
    if (early) DIAG();
    gchar* obase = (gchar*)p.out[rank] + torg + (size_t)src * NLOC * 4;
    if (dd != 0) {
        const unsigned f = __builtin_amdgcn_readfirstlane(poll_ge((gu32*)(myws + OFF_OUTFLAG + (bid * 8 + wid) * 64), call << 8, tmo, call));
        const bool bad = __builtin_amdgcn_readfirstlane(__hip_atomic_load(tmo, __ATOMIC_RELAXED, __HIP_MEMORY_SCOPE_AGENT)) == call;
        const gchar* pk = (const gchar*)p.ws[bad ? rank : src] + OFF_PK + (size_t)(bid * 8 + wid) * PK_WAVE;
        const gchar* rbase = (const gchar*)p.ws[bad ? rank : src] + OFF_RAW + (size_t)(bid * 8 + wid) * RAW_WAVE;
        unsigned side;
        asm volatile("s_nop 4\n\tglobal_load_dword %0, %1, %2 sc0 sc1\n\ts_waitcnt vmcnt(0)" : "=&v"(side) : "v"(soff >> 2), "s"(pk + 4 * PK_CHUNK) : "memory");
        int st = (int)(side | 0xf0000000u);
        const unsigned base = __builtin_elementwise_sub_sat((f & 255u) - ((side >> 28) & 7u), 7u);
        const bool ovf = (int)side < 0;
        const unsigned poison = bad ? qnan : 0u;
        if (wid == 0) {
            STAMP(14);
            if (nt < 8) poll_ge<PSL>((gu32*)(myws + OFF_PDONE + (7 - dd) * 512), (call - 1u) * NSTR + (unsigned)((mt & 1) * 8 + nt) * NSTR / 16u + 1u, tmo, call);
            else poll_ge<GSL>((gu32*)(myws + OFF_GATE + (mt * 8 + ((nt - 8) ^ PX)) * 128), call, tmo, call);
            STAMP(13);
        }
#pragma unroll
        for (int c = 0; c < 4; ++c) {
            while (__builtin_amdgcn_readfirstlane(lctl[1]) + 1u < (unsigned)(c * 8 + wid)) __builtin_amdgcn_s_sleep(TSL);
            while (__builtin_amdgcn_readfirstlane(lctl[1]) < (unsigned)(c * 8 + wid)) __builtin_amdgcn_s_sleep(2);
            const int lag = (int)__builtin_amdgcn_readfirstlane(lctl[2]);
            u32x4 in[6]; u32x3 cv;
            const gchar* pc = pk + c * PK_CHUNK + 3072;
            asm volatile("s_nop 4\n\tglobal_load_dwordx3 %0, %1, %2 offset:3072 sc0 sc1" NTPK : "=v"(cv) : "v"(soff - (soff >> 2)), "s"(pc) : "memory");
#pragma unroll
            for (int i = 0; i < 6; ++i) {
                asm volatile("s_waitcnt vmcnt(%0)" :: "i"(EW + 1) : "memory");
                if (lag <= TH2) asm volatile("s_waitcnt vmcnt(%0)" :: "i"(EW) : "memory");
                if (lag <= TH) asm volatile("s_waitcnt vmcnt(%0)" :: "i"(EW - 1) : "memory");
                if (lag < -TH) asm volatile("s_waitcnt vmcnt(%0)" :: "i"(EW - 2) : "memory");
                asm volatile("s_nop 4\n\tglobal_load_dwordx4 %0, %1, %2 offset:%3 sc0 sc1" NTPK : "=v"(in[i]) : "v"(soff), "s"(pc), "i"(i * 1024 - 3072) : "memory");
            }
            asm volatile("s_waitcnt vmcnt(%1)" : "+v"(cv) : "i"(WTOK) : "memory");
            lctl[1] = (unsigned)(c * 8 + wid) + 1u;
            if (c == 3 && nt < 8 && tz == 448) __hip_atomic_store((gu32*)(myws + OFF_GATE + (mt * 8 + nt) * 128), call, __ATOMIC_RELAXED, __HIP_MEMORY_SCOPE_AGENT);
            unsigned tot = 0;
            if ((tz & 63) == 0) tot = __hip_atomic_fetch_add((gu32*)(myws + OFF_BDONE + (7 - dd) * 512), 1u, __ATOMIC_RELAXED, __HIP_MEMORY_SCOPE_AGENT);
            __builtin_amdgcn_sched_barrier(0);
            const unsigned cw3 = (cv[0] >> 30) | ((cv[1] >> 30) << 2) | ((cv[2] >> 30) << 4);
            unsigned ex[32];
#pragma unroll
            for (int n = 0; n < 32; ++n) {
                const unsigned cd = ((n < 30 ? cv[n < 30 ? n / 10 : 0] : cw3) >> (3 * (n % 10))) & 7u;
                const bool z = cd == 0u; const unsigned k = (unsigned)__builtin_ctz((unsigned)st);
                ex[n] = base + (z ? 0u - k : cd);
                st >>= z ? k + 1u : 0u;
                if ((n & 3) == 3) asm volatile("" : "+v"(ex[n - 3]), "+v"(ex[n - 2]), "+v"(ex[n - 1]), "+v"(ex[n]), "+v"(st));
            }
            asm volatile("s_waitcnt vmcnt(0)" : "+v"(in[0]), "+v"(in[1]), "+v"(in[2]), "+v"(in[3]), "+v"(in[4]), "+v"(in[5]), "+v"(st), "+v"(ex[31]) :: "memory");
            if ((tz & 63) == 0) lctl[2] = tot - (call - 1u) * 1024u + 8u - 16u * (unsigned)((early ? 32 : 0) + c * 8 + wid + 1);
            if (c == 3 && wid == 7) STAMPW(10);
#pragma unroll
            for (int g = 0; g < 4; ++g) {
                u32x4 q[2];
#pragma unroll
                for (int e = 0; e < 8; ++e) {
                    const int n = g * 8 + e;
#define CIN(n) in[(n) >> 2][(n) & 3]
                    const unsigned lo24 = n < 24 ? CIN(n < 24 ? n : 0)
                        : __builtin_amdgcn_perm(CIN(n < 24 ? 0 : 3 * (n - 24) + 2), __builtin_amdgcn_perm(CIN(n < 24 ? 0 : 3 * (n - 24) + 1), CIN(n < 24 ? 0 : 3 * (n - 24)), 0x00000703u), 0x00070100u);
                    q[e >> 2][e & 3] = __builtin_amdgcn_perm(ex[n], lo24, 0x04020100u);
                }
                if (ovf) asm volatile("s_nop 4\n\tglobal_load_dwordx4 %0, %2, %3 sc0 sc1\n\tglobal_load_dwordx4 %1, %2, %3 offset:1024 sc0 sc1\n\ts_waitcnt vmcnt(0)"
                                      : "+v"(q[0]), "+v"(q[1]) : "v"(soff + (c * 4 + g) * 2048u), "s"(rbase) : "memory");
                unsigned v[8];
#pragma unroll
                for (int e = 0; e < 8; ++e) v[e] = __builtin_amdgcn_alignbit(q[e >> 2][e & 3], q[e >> 2][e & 3], 1) | poison;
#pragma unroll
                for (int e = 0; e < 8; ++e) ACC(c, g, e) += __builtin_bit_cast(float, v[e]);
                if (early) {
                    const f32x4 w0 = {ACC(c, g, 0), ACC(c, g, 1), ACC(c, g, 2), ACC(c, g, 3)}, w1 = {ACC(c, g, 4), ACC(c, g, 5), ACC(c, g, 6), ACC(c, g, 7)};
                    STOREO(lo, w0, obase + ROWOFF(c, g, 0)); STOREO(lo, w1, obase + ROWOFF(c, g, 4));
                }
            }
        }
    }
    if (!early) {
        DIAG();
        const bool bad = __builtin_amdgcn_readfirstlane(__hip_atomic_load(tmo, __ATOMIC_RELAXED, __HIP_MEMORY_SCOPE_AGENT)) == call;
#pragma unroll
        for (int c = 0; c < 4; ++c)
#pragma unroll
            for (int g = 0; g < 4; ++g)
#pragma unroll
                for (int h = 0; h < 2; ++h) {
                    f32x4 v = {ACC(c, g, 4 * h), ACC(c, g, 4 * h + 1), ACC(c, g, 4 * h + 2), ACC(c, g, 4 * h + 3)};
                    if (bad) v = __builtin_bit_cast(f32x4, (u32x4){qnan, qnan, qnan, qnan});
                    STOREO(lo, v, obase + ROWOFF(c, g, 4 * h));
                }
        STAMP(23);
    }
}

extern "C" void kernel_launch(int rank, int nranks, ncclComm_t comm,
                              hipStream_t stream,
                              void* const* d_in, const int* in_sizes, int n_in,
                              void* d_out, int out_size,
                              const int* in_dtypes, int out_dtype,
                              void* const* d_out_all,
                              void* const* d_ws_all, size_t ws_bytes) {
    if (nranks != NRANK || (unsigned)rank >= (unsigned)NRANK || ws_bytes < WS_NEED || n_in < 2 ||
        in_sizes[0] != 4096 * KSH || in_sizes[1] != KSH * NCOLS || out_size != 512 * NCOLS) {
        fprintf(stderr, "unexpected problem shape\n");
        return;
    }
    Params p;
    p.x = (const float*)d_in[0]; p.w = (const float*)d_in[1];
    for (int r = 0; r < NRANK; ++r) { p.out[r] = (float*)d_out_all[r]; p.ws[r] = (char*)d_ws_all[r]; }
    p.rank = rank; p.pad = 0;
    gemm_rs_kernel<<<GRID, 512, 0, stream>>>(p);
    HIP_CHECK(hipGetLastError());
}
```

```cpp
#include <hip/hip_runtime.h>
#include <rccl/rccl.h>
#include <cstdio>

#define HIP_CHECK(expr) do { hipError_t _e = (expr); if (_e != hipSuccess) { \
    fprintf(stderr, "HIP error %s at %s:%d\n", hipGetErrorString(_e), __FILE__, __LINE__); \
    return; } } while (0)

#define LAS __attribute__((address_space(3)))
#define GAS __attribute__((address_space(1)))
typedef _Float16 half8 __attribute__((ext_vector_type(8)));
typedef _Float16 half2v __attribute__((ext_vector_type(2)));
typedef float f32x2 __attribute__((ext_vector_type(2)));
typedef float f32x4 __attribute__((ext_vector_type(4)));
typedef unsigned u32x4 __attribute__((ext_vector_type(4)));
typedef unsigned u32x3 __attribute__((ext_vector_type(3)));
typedef unsigned u32x2 __attribute__((ext_vector_type(2)));
typedef unsigned long long u64;
template <int N> struct IC { static constexpr int value = N; };
typedef GAS unsigned gu32;
typedef GAS char gchar;

constexpr int NRANK = 8;
constexpr int KSH = 512;
constexpr int NCOLS = 32768, NLOC = 4096;
constexpr int GRID = 256, NPROD = 32;
constexpr size_t CTRL_BYTES = 1u << 20;
constexpr size_t PANEL = 32768;
constexpr size_t KT_STRIDE = 16 * PANEL;
constexpr size_t MAT_BYTES = 64 * KT_STRIDE;
constexpr size_t PI_ITEM = 26 * 256 + 512;
constexpr size_t OFF_PI = 3 * MAT_BYTES;
constexpr size_t BUF_BYTES = 3 * MAT_BYTES + (33u << 20);
constexpr size_t PK_CHUNK = 26 * 256;
constexpr size_t PK_WAVE = 4 * PK_CHUNK + 1024;
constexpr size_t RAW_WAVE = 64 * 512;
constexpr size_t OFF_PK = CTRL_BYTES + 2 * BUF_BYTES;
constexpr size_t OFF_RAW = OFF_PK + (size_t)GRID * 8 * PK_WAVE;
constexpr size_t WS_NEED = OFF_RAW + (size_t)GRID * 8 * RAW_WAVE;
constexpr int OFF_TMO = 0, OFF_QUEUE = 20480, OFF_PDONE = 24576 + 256, OFF_CDONE = 28672 + 768, OFF_BDONE = 24576 + 128;
constexpr int OFF_GATE = 425984;
constexpr int OFF_INFLAG = 32768;
constexpr int OFF_CFLAG = 65536;
constexpr int OFF_OUTFLAG = 131072;
constexpr int OFF_STAMP = 262144;
constexpr int OFF_TICKET = 393216;
constexpr unsigned long long DEADLINE_TICKS = 50000000ull;
#define OWNA 18
#define TGATE 500
#define XSTAG 20
#define EW 3
#define PW 2
#define NPUSH 28
#define PSL 16
#define ISL 16
#define GAUX 17
#define NTIN " nt"
#define NTOUT " nt"
#define PMAP 0
#define WTOK 3
#define NTPI ""
#define NTPK " nt"
#define NTRAW 0
#define GSL 8
#define XSTAGB 0
#define ISLOW 64
#define SFLAG 0
#define TSL 32
#define TH 12
#define TH2 40
#define LSTAG 0
#define FW 64
#define LOOK 0
#define LAG 5
#define UW 16
#define CW 3
constexpr int NCONV = 8 * (64 + 512);
constexpr int NITEM = 960;
constexpr unsigned NSTR = PMAP ? 28 : 32;
constexpr unsigned QUEUE_PER_CALL = NITEM - NPUSH - 2 * NSTR + NSTR;

struct Params {
    const float* x; const float* w;
    float* out[NRANK];
    char* ws[NRANK];
    int rank; int pad;
};

#define STAMPW(k) do {} while (0)
#define STAMP(k) do {} while (0)

template <int SLEEP = 8>
__device__ __forceinline__ unsigned poll_ge(gu32* addr, unsigned need, gu32* tmo, unsigned call) {
    const unsigned long long t0 = __builtin_amdgcn_s_memrealtime();
    unsigned v;
    for (unsigned n = 1;; ++n) {
        v = __hip_atomic_load(addr, __ATOMIC_RELAXED, __HIP_MEMORY_SCOPE_SYSTEM);
        if (__all((int)(v - need) >= 0)) break;
        if ((n & 15u) == 0u) {
            const unsigned tv = __builtin_amdgcn_readfirstlane(__hip_atomic_load(tmo, __ATOMIC_RELAXED, __HIP_MEMORY_SCOPE_AGENT));
            if (tv == call || __builtin_amdgcn_s_memrealtime() - t0 > DEADLINE_TICKS) {
                __hip_atomic_store(tmo, call, __ATOMIC_RELAXED, __HIP_MEMORY_SCOPE_AGENT);
                break;
            }
        }
        __builtin_amdgcn_s_sleep(SLEEP);
    }
    asm volatile("" ::: "memory");
    return v;
}

__device__ __forceinline__ unsigned pk16(float a, float b) {
    f32x2 v = {a, b};
    return __builtin_bit_cast(unsigned, __builtin_convertvector(v, half2v));
}

typedef unsigned short u16x2 __attribute__((ext_vector_type(2)));
__device__ __forceinline__ unsigned pk_max(unsigned a, unsigned b) { return __builtin_bit_cast(unsigned, __builtin_elementwise_max(__builtin_bit_cast(u16x2, a), __builtin_bit_cast(u16x2, b))); }
__device__ __forceinline__ unsigned wave_scan(unsigned x) {
    x += (unsigned)__builtin_amdgcn_update_dpp(0, (int)x, 0x111, 0xf, 0xf, false);
    x += (unsigned)__builtin_amdgcn_update_dpp(0, (int)x, 0x112, 0xf, 0xf, false);
    x += (unsigned)__builtin_amdgcn_update_dpp(0, (int)x, 0x114, 0xf, 0xf, false);
    x += (unsigned)__builtin_amdgcn_update_dpp(0, (int)x, 0x118, 0xf, 0xf, false);
    x += (unsigned)__builtin_amdgcn_update_dpp(0, (int)x, 0x142, 0xa, 0xf, false);
    x += (unsigned)__builtin_amdgcn_update_dpp(0, (int)x, 0x143, 0xc, 0xf, false);
    return x;
}

__device__ __forceinline__ unsigned pk_min(unsigned a, unsigned b) { return __builtin_bit_cast(unsigned, __builtin_elementwise_min(__builtin_bit_cast(u16x2, a), __builtin_bit_cast(u16x2, b))); }
__device__ __forceinline__ unsigned pk_subs(unsigned a, unsigned b) { return __builtin_bit_cast(unsigned, __builtin_elementwise_sub_sat(__builtin_bit_cast(u16x2, a), __builtin_bit_cast(u16x2, b))); }

__device__ __forceinline__ unsigned pack_item(unsigned (&D)[32], unsigned (&P)[26], u32x2& sd, LAS unsigned char* sc, int lane) {
    unsigned se = 0;
#pragma unroll
    for (int i = 0; i < 32; ++i) se += (D[i] >> 10) & 0x001f001fu;
    const unsigned S = __builtin_amdgcn_readlane(wave_scan((se & 0xffffu) + (se >> 16)), 63);
    const unsigned b1 = min((S + 246u) >> 12, 29u), B = (b1 + 27u) * 0x10001u;
    *(LAS u32x2*)(sc + lane * 8) = (u32x2){0u, 0u};
    if (lane < 8) *(LAS u32x2*)(sc + 512 + lane * 8) = (u32x2){0u, 0u};
    u64 lo = 1; unsigned hi = 0, len = 0, worst = 0, up = 0;
    bool ovf = false;
#pragma unroll
    for (int q = 15; q >= 0; --q) {
        unsigned W = 1, M[2];
#pragma unroll
        for (int h = 1; h >= 0; --h) {
            const int i = 2 * q + h;
            const unsigned E = ((D[i] >> 10) & 0x001f001fu) | 0x00200020u, dd = pk_subs(E, B), ix = pk_min(dd, 0x00070007u) | 0x0c000c00u;
            const unsigned a3 = pk_subs(dd, 0x00070007u) * 3u;
            up = pk_max(up, a3);
            M[h] = __builtin_amdgcn_perm(0x00000001u, 0x02040507u, ix) + pk_subs(B, E) + a3;
            D[i] = (D[i] & 0x83ff83ffu) | (pk_subs(__builtin_amdgcn_perm(0x03020100u, 0u, ix), a3) << 10);
            W = (W << ((M[h] >> 16) & 31u)) | 1u;
            W = (W << (M[h] & 31u)) | 1u;
        }
        const unsigned ms = M[0] + M[1], sm = (ms & 0xffffu) + (ms >> 16);
        worst = max(worst, sm); len += sm;
        if (q < 8) hi = (unsigned)(((((u64)hi << 32) | (lo >> 32)) << (sm & 31u)) >> 32);
        lo = (lo << (sm & 31u)) | W;
        if (q == 8) ovf = len > 63u;
        asm volatile("" : "+v"(lo), "+v"(hi));
    }
    ovf |= len > 95u || worst > 31u || pk_subs(up, 0x00060006u) != 0u;
    const unsigned end = wave_scan(ovf ? 0u : len), o = end - (ovf ? 0u : len);
    ovf |= end > 3583u;
    sc[lane] = (unsigned char)(ovf ? 255u : len);
    if (!ovf) {
        LAS unsigned* w = (LAS unsigned*)(sc + 64) + (o >> 5);
        const u64 a = lo << (o & 31u), b = (((u64)hi << 32) | (lo >> 32)) << (o & 31u), c = (u64)hi << (o & 31u);
        __hip_atomic_fetch_or(w, (unsigned)a, __ATOMIC_RELAXED, __HIP_MEMORY_SCOPE_WAVEFRONT);
        __hip_atomic_fetch_or(w + 1, (unsigned)(a >> 32), __ATOMIC_RELAXED, __HIP_MEMORY_SCOPE_WAVEFRONT);
        __hip_atomic_fetch_or(w + 2, (unsigned)(b >> 32), __ATOMIC_RELAXED, __HIP_MEMORY_SCOPE_WAVEFRONT);
        __hip_atomic_fetch_or(w + 3, (unsigned)(c >> 32), __ATOMIC_RELAXED, __HIP_MEMORY_SCOPE_WAVEFRONT);
    }
#pragma unroll
    for (int g = 0; g < 2; ++g) {
#pragma unroll
        for (int i = 0; i < 12; ++i) P[13 * g + i] = D[16 * g + i] | ((D[16 * g + 13 + i / 4] << (12 - 3 * (i & 3))) & 0x70007000u);
        P[13 * g + 12] = D[16 * g + 12] | ((D[16 * g + 13] >> 3) & 0x10001000u) | ((D[16 * g + 14] >> 2) & 0x20002000u) | ((D[16 * g + 15] >> 1) & 0x40004000u);
    }
    asm volatile("s_waitcnt lgkmcnt(0)" ::: "memory");
    sd = *(volatile LAS u32x2*)(sc + lane * 8);
    const unsigned tot = min((unsigned)__builtin_amdgcn_readlane(end, 63), 3583u);
    return b1 | (((tot + 512u) >> 9) << 5);
}
template <int Q>
__device__ __forceinline__ void unpack8(unsigned (&H)[8], unsigned BB, unsigned& s0, unsigned& s1, unsigned& s2) {
#pragma unroll
    for (int q = 0; q < 4; ++q) {
        unsigned t = s0, M[2];
#pragma unroll
        for (int h = 0; h < 2; ++h) {
            const unsigned T = H[2 * q + h];
            const half2v f = __builtin_bit_cast(half2v, T);
            const unsigned m0 = (unsigned)__builtin_ctz(t - (__builtin_amdgcn_classh(f[0], 0xf0) ? 1u : 0u));
            t >>= m0 & 31u;
            const unsigned m1 = (unsigned)__builtin_ctz(t - (__builtin_amdgcn_classh(f[1], 0xf0) ? 1u : 0u));
            if (h == 0) t >>= m1 & 31u;
            M[h] = m0 | (m1 << 16);
            H[2 * q + h] = ((__builtin_amdgcn_perm(0x030b0202u, 0x07010100u, pk_min(M[h], 0x00070007u)) - M[h]) << 10) + (T + BB);
        }
        const unsigned ms = M[0] + M[1], sm = ms + (ms >> 16);
        s0 = __builtin_amdgcn_alignbit(s1, s0, sm);
        if (Q < 2) { s1 = __builtin_amdgcn_alignbit(s2, s1, sm); s2 >>= sm & 31u; } else s1 >>= sm & 31u;
        asm volatile("" : "+v"(H[2 * q]), "+v"(H[2 * q + 1]));
    }
}
constexpr int pull_younger(int k) {
    int n = 0;
    for (int j = k - PW; j < k; ++j) {
        const int jj = (j + 7) % 7;
        if (j > k - PW) n += (j + PW) % 7 == 0 ? 2 : 1;
        if (jj == 1 || jj == 3 || jj == 5 || jj == 6) n += 2;
    }
    return n;
}

__device__ __forceinline__ void item_decode(int it, int src, int& s, int& r) {
    int e;
    constexpr int n1 = 72 * LAG, n2 = n1 + 88 * (8 - LAG);
    if (it < n1) { s = it / 72; e = it - 72 * s; }
    else if (it < n2) { s = (it - n1) / 88; e = it - n1 - 88 * s; s += e < 72 ? LAG : 0; }
    else { const int v = it - 576; s = (v >> 4) & 7; e = 72 + (v >> 7) * 16 + (v & 15); }
    r = e < 56 ? (8 * src + 8 + e) & 63 : e + 8;
}

#define UNIT_LOAD(B, S_, R_, H_) do { const bool isx_ = (R_) < 64; \
        const gchar* b_ = isx_ ? sX + (size_t)((R_) * 64 + (H_) * 32) * (KSH * 4) + (size_t)(S_) * 256 \
                               : sW + (size_t)((S_) * 64 + (H_) * 32) * (NCOLS * 4) + (size_t)((R_) - 64) * 256; \
        const unsigned vo_ = isx_ ? xoff : woff; \
        const gchar* b1_ = b_ + (isx_ ? 16 : 1 * NCOLS * 4);          const gchar* b2_ = b_ + (isx_ ? 128 : 2 * NCOLS * 4); \
        const gchar* b3_ = b_ + (isx_ ? 144 : 3 * NCOLS * 4);         const gchar* b4_ = b_ + (isx_ ? 32768 : 4 * NCOLS * 4); \
        const gchar* b5_ = b_ + (isx_ ? 32768 + 16 : 5 * NCOLS * 4);  const gchar* b6_ = b_ + (isx_ ? 32768 + 128 : 6 * NCOLS * 4); \
        const gchar* b7_ = b_ + (isx_ ? 32768 + 144 : 7 * NCOLS * 4); \
        asm volatile("s_nop 4\n\tglobal_load_dwordx4 %0, %8, %9" NTIN "\n\tglobal_load_dwordx4 %1, %8, %10" NTIN "\n\t" \
                     "global_load_dwordx4 %2, %8, %11" NTIN "\n\tglobal_load_dwordx4 %3, %8, %12" NTIN "\n\t" \
                     "global_load_dwordx4 %4, %8, %13" NTIN "\n\tglobal_load_dwordx4 %5, %8, %14" NTIN "\n\t" \
                     "global_load_dwordx4 %6, %8, %15" NTIN "\n\tglobal_load_dwordx4 %7, %8, %16" NTIN "" \
                     : "+v"(B[0]), "+v"(B[1]), "+v"(B[2]), "+v"(B[3]), "+v"(B[4]), "+v"(B[5]), "+v"(B[6]), "+v"(B[7]) \
                     : "v"(vo_), "s"(b_), "s"(b1_), "s"(b2_), "s"(b3_), "s"(b4_), "s"(b5_), "s"(b6_), "s"(b7_) : "memory"); } while (0)
#define UNIT_WAIT(B, N) asm volatile("s_waitcnt vmcnt(%8)" : "+v"(B[0]), "+v"(B[1]), "+v"(B[2]), "+v"(B[3]), \
                                     "+v"(B[4]), "+v"(B[5]), "+v"(B[6]), "+v"(B[7]) : "i"(N) : "memory")
#define UNIT_CVT(O, B, R_) do { if ((R_) < 64) { \
            _Pragma("unroll") for (int b = 0; b < 4; ++b) { \
                const f32x4 lo_ = __builtin_bit_cast(f32x4, B[2 * b]), hi_ = __builtin_bit_cast(f32x4, B[2 * b + 1]); \
                O[b] = (u32x4){pk16(lo_[0], lo_[1]), pk16(lo_[2], lo_[3]), pk16(hi_[0], hi_[1]), pk16(hi_[2], hi_[3])}; } \
        } else { \
            _Pragma("unroll") for (int j = 0; j < 4; ++j) \
                O[j] = (u32x4){pk16(__builtin_bit_cast(f32x4, B[0])[j], __builtin_bit_cast(f32x4, B[1])[j]), \
                               pk16(__builtin_bit_cast(f32x4, B[2])[j], __builtin_bit_cast(f32x4, B[3])[j]), \
                               pk16(__builtin_bit_cast(f32x4, B[4])[j], __builtin_bit_cast(f32x4, B[5])[j]), \
                               pk16(__builtin_bit_cast(f32x4, B[6])[j], __builtin_bit_cast(f32x4, B[7])[j])}; } \
        asm volatile("" : "+v"(O[0]), "+v"(O[1]), "+v"(O[2]), "+v"(O[3])); } while (0)
#define ITEM_OFF(R_) ((R_) < 64 ? (size_t)(R_) * 8192 : (size_t)(((R_) - 64) >> 2) * PANEL + (size_t)((R_) & 3) * 4096)
#define UNIT_DST(BASE, R_, H_) const bool isxd_ = (R_) < 64; \
        gchar* t0_ = (BASE) + (isxd_ ? (H_) * 4096 : (H_) * 1024); \
        gchar* t1_ = t0_ + (isxd_ ? 1024 : 2048); gchar* t2_ = t0_ + (isxd_ ? 2048 : 16384); gchar* t3_ = t0_ + (isxd_ ? 3072 : 16384 + 2048)
#define STOREO(VOFF, DATA, BASE) asm volatile("s_nop 4\n\tglobal_store_dwordx4 %0, %1, %2 sc0 sc1" NTOUT "\n\ts_nop 1" :: "v"(VOFF), "v"(DATA), "s"(BASE) : "memory")
#define STORE16(VOFF, DATA, BASE, N) asm volatile("s_nop 3\n\ts_waitcnt vmcnt(%3)\n\tglobal_store_dwordx4 %0, %1, %2 sc0 sc1\n\ts_nop 1" \
                                                  :: "v"(VOFF), "v"(DATA), "s"(BASE), "i"(N) : "memory")

__global__ __launch_bounds__(512, 2) void gemm_rs_kernel(const Params p) {
    __shared__ __attribute__((aligned(16))) unsigned char shm[131072 + 256 + 8 * 1088];
    LAS unsigned char* lds = (LAS unsigned char*)shm;
    volatile LAS unsigned* lctl = (volatile LAS unsigned*)(lds + 131072);
    const int tid = threadIdx.x, wid = __builtin_amdgcn_readfirstlane(tid >> 6), lane = tid & 63;
    const int rank = p.rank & (NRANK - 1), bid = blockIdx.x;
    gchar* myws = (gchar*)p.ws[rank];
    gu32* tmo = (gu32*)(myws + OFF_TMO);
    const unsigned long long tstart = __builtin_amdgcn_s_memrealtime();

    if (tid == 0) {
        lctl[0] = __hip_atomic_fetch_add((gu32*)(myws + OFF_TICKET + bid * 128), 1u, __ATOMIC_RELAXED, __HIP_MEMORY_SCOPE_AGENT) + 1u;
    } else if (tid < 64) lctl[tid] = 0u;
    unsigned call; size_t bufoff;
#define GET_CALL() do { asm volatile("s_waitcnt lgkmcnt(0)" ::: "memory"); __builtin_amdgcn_s_barrier(); asm volatile("" ::: "memory"); \
        call = __builtin_amdgcn_readfirstlane(lctl[0]); bufoff = CTRL_BYTES + (size_t)(call & 1u) * BUF_BYTES; } while (0)

    const int xcd = bid & 7, idx = bid >> 3, dd = idx >> 2;
    const int dest = (rank + dd) & (NRANK - 1);
    const int mt = 2 * dest + (xcd & 1), grp = idx & 3, nt = 4 * grp + (xcd >> 1);
    const int hidx = (dd - 1) * 16 + (bid & 15);

    const unsigned soff0 = (unsigned)lane * 16u;
    const gchar* sX = (const gchar*)p.x;
    const unsigned xoff = (unsigned)(lane & 15) * (KSH * 4) + (unsigned)(lane >> 4) * 32u;
    const unsigned woff = (unsigned)(lane >> 4) * 8u * (NCOLS * 4) + (unsigned)(lane & 15) * 16u;
    if (bid >= NPROD && nt >= 8 && wid >= 1 && wid <= CW) {
        u32x4 buf0[8], buf1[8], o[4], o1[4];
#pragma unroll
        for (int b = 0; b < 8; ++b) { buf0[b] = (u32x4){0u, 0u, 0u, 0u}; buf1[b] = (u32x4){0u, 0u, 0u, 0u}; }
        bool first = true;
        for (int job = NPUSH + (wid - 1) * 112 + hidx; job < NCONV; job += CW * 112) {
            int s, e;
            if (job < 1536) { s = job / 192; e = job - 192 * s; }
            else { const int v = job - 1536; s = (v >> 7) & 7; e = 192 + (v >> 10) * 128 + (v & 127); }
            const int d = e < 64 ? rank : (rank + 1 + ((e - 64) & 7)) & (NRANK - 1);
            const int r = e < 64 ? (8 * rank + 8 + e) & 63 : 64 + ((e - 64) >> 3);
            const gchar* sW = (const gchar*)p.w + (size_t)d * NLOC * 4;
            UNIT_LOAD(buf0, s, r, 0); UNIT_LOAD(buf1, s, r, 1);
            if (first) { GET_CALL(); first = false; }
            gchar* tb = myws + bufoff + ITEM_OFF(r) + (e < 64 ? (size_t)(s * 8 + rank) * KT_STRIDE
                        : d == rank ? MAT_BYTES + (size_t)(s * 8 + rank) * KT_STRIDE : 2 * MAT_BYTES + (size_t)(d * 8 + s) * KT_STRIDE);
            UNIT_WAIT(buf0, 8); UNIT_CVT(o, buf0, r);
            if (NTRAW && d != rank) { UNIT_DST(tb, r, 0); STOREO(soff0, o[0], t0_); STOREO(soff0, o[1], t1_); STOREO(soff0, o[2], t2_); STOREO(soff0, o[3], t3_); }
            else { UNIT_DST(tb, r, 0); STORE16(soff0, o[0], t0_, 63); STORE16(soff0, o[1], t1_, 63); STORE16(soff0, o[2], t2_, 63); STORE16(soff0, o[3], t3_, 63); }
            UNIT_WAIT(buf1, 4); UNIT_CVT(o1, buf1, r);
            if (NTRAW && d != rank) { UNIT_DST(tb, r, 1); STOREO(soff0, o1[0], t0_); STOREO(soff0, o1[1], t1_); STOREO(soff0, o1[2], t2_); STOREO(soff0, o1[3], t3_); }
            else { UNIT_DST(tb, r, 1); STORE16(soff0, o1[0], t0_, 63); STORE16(soff0, o1[1], t1_, 63); STORE16(soff0, o1[2], t2_, 63); STORE16(soff0, o1[3], t3_, 63); }
            unsigned meta = 0;
            if (e < 56 || d != rank) {
                unsigned D[32], P[26]; u32x2 sd;
#pragma unroll
                for (int i = 0; i < 16; ++i) { D[i] = o[i >> 2][i & 3]; D[16 + i] = o1[i >> 2][i & 3]; }
                meta = pack_item(D, P, sd, lds + wid * 1024, lane);
                gchar* pi = myws + bufoff + OFF_PI + (size_t)(e < 64 ? s * 64 + r : 512 + (d * 8 + s) * 64 + r - 64) * PI_ITEM;
#pragma unroll
                for (int i = 0; i < 6; ++i) { const u32x4 v = {P[4 * i], P[4 * i + 1], P[4 * i + 2], P[4 * i + 3]}; STORE16(soff0, v, pi + i * 1024, 63); }
                const u32x2 v2 = {P[24], P[25]};
                asm volatile("s_nop 4\n\tglobal_store_dwordx2 %0, %1, %3 sc0 sc1\n\tglobal_store_dwordx2 %0, %2, %3 offset:512 sc0 sc1"
                             :: "v"(soff0 >> 1), "v"(v2), "v"(sd), "s"(pi + 6144) : "memory");
            }
            asm volatile("s_waitcnt vmcnt(0)" ::: "memory");
            if (TGATE) while (__builtin_amdgcn_s_memrealtime() - tstart < TGATE) __builtin_amdgcn_s_sleep(2);
            const int fo = rank * 4096 + (s * 128 + r) * 4;
#pragma unroll
            for (int k = 0; k < NRANK; ++k)
                if ((e < 56 || k == d) && lane == 0)
                    __hip_atomic_store((gu32*)((gchar*)p.ws[k] + (k == rank ? OFF_INFLAG : OFF_CFLAG) + fo), k == rank ? call : (call << 8) | meta, __ATOMIC_RELAXED, __HIP_MEMORY_SCOPE_SYSTEM);
        }
        if (lane == 0) __hip_atomic_fetch_add((gu32*)(myws + OFF_CDONE), 1u, __ATOMIC_RELAXED, __HIP_MEMORY_SCOPE_AGENT);
        if (wid == 1) STAMPW(15);
    } else if (bid >= NPROD && nt < 8 && wid == 1 && hidx < NPUSH) {
        u32x4 buf0[8], buf1[8], o0[4], o1[4];
#pragma unroll
        for (int b = 0; b < 8; ++b) { buf0[b] = (u32x4){0u, 0u, 0u, 0u}; buf1[b] = (u32x4){0u, 0u, 0u, 0u}; }
        const gchar* sW = sX;
        const int r = (8 * rank + 8 + hidx) & 63;
        UNIT_LOAD(buf0, 0, r, 0); UNIT_LOAD(buf1, 0, r, 1);
        GET_CALL();
        UNIT_WAIT(buf0, 8); UNIT_CVT(o0, buf0, 0);
        UNIT_WAIT(buf1, 0); UNIT_CVT(o1, buf1, 0);
        const size_t ioff = bufoff + (size_t)rank * KT_STRIDE + (size_t)r * 8192;
#pragma unroll
        for (int b = 0; b < 8; ++b)
            for (int k = 1; k <= NRANK; ++k) {
                gchar* t = (gchar*)p.ws[(rank + hidx + k) & (NRANK - 1)] + ioff + b * 1024;
                STORE16(soff0, b < 4 ? o0[b & 3] : o1[b & 3], t, UW - 1);
            }
        asm volatile("s_waitcnt vmcnt(0)" ::: "memory");
#pragma unroll
        for (int k = 0; k < NRANK; ++k)
            if (lane == 0) __hip_atomic_store((gu32*)((gchar*)p.ws[k] + OFF_INFLAG + rank * 4096 + r * 4), call, __ATOMIC_RELAXED, __HIP_MEMORY_SCOPE_SYSTEM);
        STAMPW(15);
    } else if (bid >= (int)NSTR) {
        GET_CALL();
    } else if (wid == 0) {
        GET_CALL();
        const int j = lane & 7, lk = PMAP ? bid % 7 : (j + bid) % 7, psrc = (rank + 1 + lk) & (NRANK - 1);
        const unsigned long long t0 = __builtin_amdgcn_s_memrealtime();
        unsigned n = 0, nf = 0; int item = 0; bool have = false, fin = lane >= 7, late = false;
        for (unsigned it = 1; !__all(fin && (!SFLAG || lane >= 7 || nf + 1u == n)); ++it) {
            if (SFLAG && lane < 7 && __hip_atomic_load((LAS unsigned*)lctl + 16 + j, __ATOMIC_ACQUIRE, __HIP_MEMORY_SCOPE_WORKGROUP) > nf) {
                ++nf;
                __hip_atomic_store((gu32*)(myws + OFF_INFLAG + psrc * 4096 + (lctl[32 + j * 4 + (nf & 3u)] & 0x7ffu) * 4), call, __ATOMIC_RELAXED, __HIP_MEMORY_SCOPE_SYSTEM);
            }
            if (!fin && !have && n <= lctl[8 + j] && (!SFLAG || n <= nf + 3u)) {
                unsigned t = NPUSH + (PMAP ? j * 4 + bid / 7 : bid) + n * NSTR;
                if (n >= 2) t = __hip_atomic_fetch_add((gu32*)(myws + OFF_QUEUE + lk * 512), 1u, __ATOMIC_RELAXED, __HIP_MEMORY_SCOPE_AGENT)
                                - (call - 1u) * QUEUE_PER_CALL + NPUSH + 2u * NSTR;
                int s, r; item_decode(t < (unsigned)NITEM ? (int)t : 0, psrc, s, r);
                item = t < (unsigned)NITEM ? s * 128 + r : 1024; have = true;
            }
            if (have) {
                bool ready = item >= 1024 || late; unsigned cf = 0;
                if (!ready) { cf = __hip_atomic_load((gu32*)(myws + OFF_CFLAG + psrc * 4096 + item * 4), __ATOMIC_RELAXED, __HIP_MEMORY_SCOPE_SYSTEM); ready = (int)(cf - (call << 8)) >= 0; }
                if (ready) { ++n; __hip_atomic_store((LAS unsigned*)lctl + 32 + j * 4 + (n & 3u), (n << 20) | ((cf & 255u) << 12) | (unsigned)item, __ATOMIC_RELEASE, __HIP_MEMORY_SCOPE_WORKGROUP); have = false; fin = item >= 1024; }
            }
            if ((it & 63u) == 0u && !late) {
                late = __builtin_amdgcn_readfirstlane(__hip_atomic_load(tmo, __ATOMIC_RELAXED, __HIP_MEMORY_SCOPE_AGENT)) == call
                       || __builtin_amdgcn_s_memrealtime() - t0 > DEADLINE_TICKS;
                if (late) __hip_atomic_store(tmo, call, __ATOMIC_RELAXED, __HIP_MEMORY_SCOPE_AGENT);
            }
            __builtin_amdgcn_s_sleep(2);
        }
    } else {
        GET_CALL();
        const int lk = PMAP ? bid % 7 : (wid - 1 + bid) % 7, src = (rank + 1 + lk) & (NRANK - 1);
        const gchar* rimg = (const gchar*)p.ws[src] + bufoff;
        gchar* limg = myws + bufoff + (size_t)src * KT_STRIDE;
        gu32* lflag = (gu32*)(myws + OFF_INFLAG + src * 4096);
        u32x4 in[6]; u32x2 in6 = {0u, 0u}, side = {0u, 0u};
#pragma unroll
        for (int b = 0; b < 6; ++b) in[b] = (u32x4){0u, 0u, 0u, 0u};
        const unsigned soff8 = (unsigned)lane * 8u;
        LAS unsigned char* sc = lds + wid * 1024;
#define PI_OF(S_, R_) (rimg + OFF_PI + 3072 + (size_t)((R_) < 64 ? (S_) * 64 + (R_) : 512 + (rank * 8 + (S_)) * 64 + (R_) - 64) * PI_ITEM)
#define RAW_OF(S_, R_) (rimg + ITEM_OFF(R_) + ((R_) < 64 ? (size_t)((S_) * 8 + src) * KT_STRIDE : 2 * MAT_BYTES + (size_t)(rank * 8 + (S_)) * KT_STRIDE))
#define DST_OF(S_, R_) (limg + ITEM_OFF(R_) + ((R_) < 64 ? 0 : MAT_BYTES) + (size_t)(S_) * 8 * KT_STRIDE)
#define PULL(k, BASE, META) do { \
        if ((k) == 0) asm volatile("s_nop 4\n\tglobal_load_dwordx2 %0, %1, %2 offset:3584 sc0 sc1" : "+v"(side) : "v"((soff8 >> 6) <= ((META) >> 5) ? soff8 : 0u), "s"(BASE) : "memory"); \
        if ((k) < 6) asm volatile("s_nop 4\n\tglobal_load_dwordx4 %0, %1, %2 offset:%3 sc0 sc1" : "+v"(in[(k) < 6 ? (k) : 0]) : "v"(soff0), "s"(BASE), "i"((k) * 1024 - 3072) : "memory"); \
        else asm volatile("s_nop 4\n\tglobal_load_dwordx2 %0, %1, %2 offset:3072 sc0 sc1" : "+v"(in6) : "v"(soff8), "s"(BASE) : "memory"); } while (0)
#define TAKE(DST, META, N_) do { unsigned e_; while (((e_ = __builtin_amdgcn_readfirstlane(__hip_atomic_load((LAS unsigned*)lctl + 28 + wid * 4 + ((N_) & 3), __ATOMIC_ACQUIRE, __HIP_MEMORY_SCOPE_WORKGROUP))) >> 20) != (unsigned)(N_)) __builtin_amdgcn_s_sleep(1); \
        lctl[7 + wid] = (unsigned)(N_); asm volatile("" ::: "memory"); DST = (int)(e_ & 0x7ffu); META = (e_ >> 12) & 255u; } while (0)
        int cur, s, r, pflag = -1; unsigned meta;
        TAKE(cur, meta, 1);
        s = cur >> 7; r = cur & 127;
        const gchar* sb = PI_OF(s, r);
#pragma unroll
        for (int k = 0; k < PW; ++k) PULL(k, sb, meta);
        if (wid == 1) STAMPW(24);
        int i = 0;
        for (; cur < 1024; ++i) {
            if (wid == 1 && i % 5 == 0 && i <= 25) STAMPW(25 + i / 5);
            int nxt = 1024, sn = 0, rn = 0; const gchar* sbn = sb; unsigned metan = 0;
            gchar* db = DST_OF(s, r);
            const bool isx = r < 64;
            const gchar* rraw = RAW_OF(s, r);
            asm volatile("" : "+s"(rraw), "+s"(db));
            const unsigned BB = ((meta & 31u) - 1u) * 0x04000400u;
            unsigned s0 = 0, s1 = 0, s2 = 0, dn[3] = {0u, 0u, 0u}; bool ovf = false;
            auto step = [&](auto KC) __attribute__((always_inline)) {
                constexpr int k = decltype(KC)::value;
                (void)&rraw; (void)&db; (void)&sb; (void)&soff0; (void)&soff8; (void)&in; (void)&in6; (void)&side;
                if (k + PW == 7) {
                    TAKE(nxt, metan, i + 2);
                    sn = (nxt >> 7) & 7; rn = nxt & 127;
                    sbn = nxt < 1024 ? PI_OF(sn, rn) : (const gchar*)myws + bufoff + OFF_PI + 3072;
                    if (nxt >= 1024 && lane == 0) __hip_atomic_fetch_add((gu32*)(myws + OFF_PDONE + lk * 512), 1u, __ATOMIC_RELAXED, __HIP_MEMORY_SCOPE_AGENT);
                }
                if (k < PW && i == 0) asm volatile("s_waitcnt vmcnt(%0)" :: "i"(PW - 1) : "memory");
                if (k < 6) asm volatile("s_waitcnt vmcnt(%2)" : "+v"(in[k < 6 ? k : 0]), "+v"(side) : "i"(pull_younger(k)) : "memory");
                else asm volatile("s_waitcnt vmcnt(%1)" : "+v"(in6) : "i"(pull_younger(k)) : "memory");
                if (SFLAG && k == PW) lctl[15 + wid] = (unsigned)i;
                if (!SFLAG && k == PW && pflag >= 0 && lane == 0) __hip_atomic_store(lflag + pflag, call, __ATOMIC_RELAXED, __HIP_MEMORY_SCOPE_SYSTEM);
                if (k + PW < 7) PULL((k + PW) % 7, sb, meta); else PULL((k + PW) % 7, sbn, metan);
                __builtin_amdgcn_sched_barrier(0);
                if (k == 0) {
                    *(volatile LAS u32x2*)(sc + lane * 8) = side;
                    const unsigned lb = *(volatile LAS unsigned char*)(sc + lane);
                    ovf = lb == 255u;
                    const unsigned len = ovf ? 0u : lb, o = min(wave_scan(len) - len, 3583u);
                    const volatile LAS unsigned* w = (const volatile LAS unsigned*)(sc + 64) + (o >> 5);
                    const unsigned w0 = w[0], w1 = w[1], w2 = w[2], w3 = w[3];
                    s0 = __builtin_amdgcn_alignbit(w1, w0, o) | 1u; s1 = __builtin_amdgcn_alignbit(w2, w1, o); s2 = __builtin_amdgcn_alignbit(w3, w2, o);
                    asm volatile("" : "+v"(s0), "+v"(s1), "+v"(s2));
                }
                if (k == 1 || k == 3 || k == 5 || k == 6) {
                    constexpr int g = k >> 1;
#define PIN(n) ((n) < 24 ? in[((n) < 24 ? (n) : 0) >> 2][(n) & 3] : in6[(n) & 1])
                    unsigned H[8];
#pragma unroll
                    for (int e = 0; e < ((g & 1) ? 5 : 8); ++e) {
                        const int ci = (g & 1) * 8 + e; const unsigned pw = PIN(13 * (g >> 1) + ci);
                        H[e] = pw & 0x8fff8fffu;
                        if (ci < 12) { const unsigned pc = (pw >> (12 - 3 * (ci & 3))) & (0x00070007u << (3 * (ci & 3))); dn[ci >> 2] = (ci & 3) ? dn[ci >> 2] | pc : pc; }
                        else { H[5] = dn[0] | ((pw << 3) & 0x80008000u); H[6] = dn[1] | ((pw << 2) & 0x80008000u); H[7] = dn[2] | ((pw << 1) & 0x80008000u); }
                    }
                    unpack8<g>(H, BB, s0, s1, s2);
                    u32x4 v0 = {H[0], H[1], H[2], H[3]}, v1 = {H[4], H[5], H[6], H[7]};
                    const unsigned o0 = soff0 + (isx ? g * 2048 : (g & 1) * 16384 + (g >> 1) * 1024), o1 = o0 + (isx ? 1024 : 2048);
                    if (ovf) asm volatile("s_nop 4\n\tglobal_load_dwordx4 %0, %2, %4 sc0 sc1\n\tglobal_load_dwordx4 %1, %3, %4 sc0 sc1\n\ts_waitcnt vmcnt(0)"
                                          : "+v"(v0), "+v"(v1) : "v"(o0), "v"(o1), "s"(rraw) : "memory");
                    STORE16(o0, v0, db, 63); STORE16(o1, v1, db, 63);
                }
            };
            step(IC<0>{}); step(IC<1>{}); step(IC<2>{}); step(IC<3>{}); step(IC<4>{}); step(IC<5>{}); step(IC<6>{});
            pflag = cur; cur = nxt; s = sn; r = rn; sb = sbn; meta = metan;
        }
        asm volatile("s_waitcnt vmcnt(0)" : "+v"(in[0]), "+v"(in[1]), "+v"(in[2]), "+v"(in[3]), "+v"(side) :: "memory");
        if (SFLAG) lctl[15 + wid] = (unsigned)i;
        else if (lane == 0) __hip_atomic_store(lflag + pflag, call, __ATOMIC_RELAXED, __HIP_MEMORY_SCOPE_SYSTEM);
        STAMPW(16 + wid);
    }

    const int wr = wid >> 2, wc = wid & 3;
    const gchar* img = myws + bufoff;
    const unsigned voff = (unsigned)tid * 16u;
    const unsigned ldsw = (unsigned)wid * 1024u;
    const int aoff = wr * 8192 + lane * 16, boff = wc * 4096 + lane * 16;
    constexpr int HTB = 16384;
#define SA(b, h) (((b) * 2 + (h)) * HTB)
#define SB(b, h) ((4 + (b) * 2 + (h)) * HTB)
#define STAGE(bufo, gbase) do { _Pragma("unroll") for (int _i = 0; _i < 2; ++_i) \
        __builtin_amdgcn_global_load_lds((const GAS unsigned*)((gbase) + voff + _i * 8192), (LAS unsigned*)(lds + (bufo) + ldsw + _i * 8192), 16, 0, GAUX); } while (0)
#define LDA(dst, b, h) do { _Pragma("unroll") for (int m = 0; m < 4; ++m) _Pragma("unroll") for (int k = 0; k < 2; ++k) \
        dst[m][k] = *(const LAS half8*)(lds + SA(b, h) + aoff + m * 2048 + k * 1024); } while (0)
#define LDB(dst, b, h) do { _Pragma("unroll") for (int n = 0; n < 2; ++n) _Pragma("unroll") for (int k = 0; k < 2; ++k) \
        dst[n][k] = *(const LAS half8*)(lds + SB(b, h) + boff + n * 2048 + k * 1024); } while (0)
#define MMA(ai, bj, At_, Bt_) do { __builtin_amdgcn_s_setprio(1); \
        _Pragma("unroll") for (int m = 0; m < 4; ++m) _Pragma("unroll") for (int n = 0; n < 2; ++n) _Pragma("unroll") for (int k = 0; k < 2; ++k) \
            acc[ai][bj][m][n] = __builtin_amdgcn_mfma_f32_16x16x32_f16(At_[m][k], Bt_[n][k], acc[ai][bj][m][n], 0, 0, 0); \
        __builtin_amdgcn_s_setprio(0); } while (0)
#define WAIT_V(n) asm volatile("s_waitcnt vmcnt(" #n ")" ::: "memory")
#define WAIT_L(n) asm volatile("s_waitcnt lgkmcnt(" #n ")" ::: "memory")
#define BAR __builtin_amdgcn_s_barrier()
#define SCHED __builtin_amdgcn_sched_barrier(0)
    bool allin = false;
#define WAIT_SEG(c) do { if (wid == 0 && !allin) { int lz_ = lane; asm volatile("" : "+v"(lz_)); const int wh_ = lz_ >> 3; \
        const int it_ = wh_ < 4 && ((c) == 8 || (lz_ & 7) != dest) ? 4 * mt + wh_ : 64 + 4 * nt + (wh_ & 3); \
        gu32* fa_ = (gu32*)(myws + OFF_INFLAG + ((c) == 8 ? rank * 4096 + (lz_ & 7) * 512 : (lz_ & 7) * 4096 + (c) * 512) + it_ * 4); \
        bool now_ = false; \
        if ((c) < 8 && (dd == 0 || (c) > (grp == 0 ? 8 : grp == 1 ? 8 - LAG : 0))) { unsigned v_[9]; \
            v_[8] = __hip_atomic_load((gu32*)(myws + OFF_CDONE), __ATOMIC_RELAXED, __HIP_MEMORY_SCOPE_SYSTEM); \
            _Pragma("unroll") for (int k_ = 0; k_ < 8; ++k_) v_[k_] = __hip_atomic_load(fa_ + ((c) + k_ < 8 ? k_ : 0) * 128, __ATOMIC_RELAXED, __HIP_MEMORY_SCOPE_SYSTEM); \
            asm volatile("" : "+v"(v_[0]), "+v"(v_[1]), "+v"(v_[2]), "+v"(v_[3]), "+v"(v_[4]), "+v"(v_[5]), "+v"(v_[6]), "+v"(v_[7]), "+v"(v_[8]) :: "memory"); \
            now_ = __all(v_[0] >= call); \
            allin = now_ && __all(min(min(min(v_[1], v_[2]), min(v_[3], v_[4])), min(min(v_[5], v_[6]), v_[7])) >= call && v_[8] >= call * (112u * CW)); } \
        if (!now_) { if ((c) == (grp == 0 ? 7 : grp == 1 ? 8 - LAG : 0)) poll_ge<ISL>(fa_, call, tmo, call); else poll_ge<ISLOW>(fa_, call, tmo, call); } \
        if ((c) == 6 && !allin) poll_ge<64>((gu32*)(myws + OFF_CDONE), call * (112u * CW), tmo, call);     \
        } } while (0)

    f32x4 acc[2][2][4][2];
#pragma unroll
    for (int a = 0; a < 2; ++a)
#pragma unroll
        for (int b = 0; b < 2; ++b)
#pragma unroll
            for (int m = 0; m < 4; ++m)
#pragma unroll
                for (int n = 0; n < 2; ++n) acc[a][b][m][n] = (f32x4){0.f, 0.f, 0.f, 0.f};
    half8 At[4][2], B0[2][2], B1[2][2];

#define A_OF(t) (img + ((unsigned)__builtin_amdgcn_readlane(tabA, (t) & 63) & ~15u))
#define B_OF(t) (img + (unsigned)__builtin_amdgcn_readlane(tabB, (t) & 63))
#define GEMM_LOOP(T0, T1, HOOK) do { \
    const gchar* a3 = A_OF((T0) + 1); \
    for (int t = (T0); t < (T1); t += 2) { \
        const gchar* a1 = a3; \
        LDB(B0, 0, 0); SCHED; LDA(At, 0, 0); STAGE(SA(1, 1), a1 + HTB); SCHED; \
        const gchar* a2 = A_OF(t + 2); const gchar* b2 = B_OF(t + 2); const gchar* b3 = B_OF(t + 3); a3 = A_OF(t + 3); \
        WAIT_L(8); BAR; WAIT_L(0); MMA(0, 0, At, B0); \
        HOOK; \
        asm volatile("" ::: "memory"); BAR; SCHED; \
        LDB(B1, 0, 1); STAGE(SB(0, 0), b2); \
        BAR; WAIT_L(0); MMA(0, 1, At, B1); BAR; \
        LDA(At, 0, 1); STAGE(SA(0, 0), a2); \
        BAR; WAIT_L(0); MMA(1, 0, At, B0); BAR; SCHED; \
        STAGE(SB(0, 1), b2 + HTB); \
        WAIT_V(6); BAR; MMA(1, 1, At, B1); BAR; \
        LDB(B0, 1, 0); SCHED; LDA(At, 1, 0); STAGE(SA(0, 1), a2 + HTB); \
        WAIT_L(8); BAR; WAIT_L(0); MMA(0, 0, At, B0); BAR; SCHED; \
        LDB(B1, 1, 1); STAGE(SB(1, 0), b3); \
        BAR; WAIT_L(0); MMA(0, 1, At, B1); BAR; \
        LDA(At, 1, 1); STAGE(SA(1, 0), a3); \
        BAR; WAIT_L(0); MMA(1, 0, At, B0); BAR; SCHED; \
        STAGE(SB(1, 1), b3 + HTB); \
        WAIT_V(6); BAR; MMA(1, 1, At, B1); BAR; \
    } \
    if (wr == 0) BAR;                                                   \
    } while (0)

    const int src = (rank - dd) & (NRANK - 1);
    const bool early = dd != 0 && grp == 3;
#define DIAG() do { WAIT_V(0); BAR; if (wr == 1) BAR; asm volatile("" ::: "memory"); GEMM_LOOP(56, 64, ); STAMP(16); } while (0)

    int tabA, tabB;
    {
        const int t0 = grp != 0 ? 0 : OWNA, u = lane < t0 ? lane : lane - 8, oo = 8 - dd;
        const int st = dd == 0 ? lane / 7 : u / 6, jj = dd == 0 ? lane % 7 : u % 6, o = jj + 1 + (jj + 1 >= oo ? 1 : 0);
        const bool own = dd != 0 && lane >= t0 && lane < t0 + 8;
        const int kt = own ? (lane - t0) * 8 + rank : (st & 7) * 8 + ((dest + o) & 7), q = lane & 7;
        tabA = lane < 56 ? (kt * (int)KT_STRIDE + mt * (int)PANEL) | (own ? 8 : st & 7) : ((q * 8 + rank) * (int)KT_STRIDE + (2 * rank + (mt & 1)) * (int)PANEL) | 7;
        tabB = nt * (int)PANEL + (lane < 56 ? (int)MAT_BYTES + kt * (int)KT_STRIDE
                                  : dd == 0 ? (int)MAT_BYTES + (q * 8 + rank) * (int)KT_STRIDE : 2 * (int)MAT_BYTES + (src * 8 + q) * (int)KT_STRIDE);
    }
#define CLK0
    int have = __builtin_amdgcn_readlane(tabA, 0) & 15;
    BAR;
    WAIT_SEG(have);
    asm volatile("" ::: "memory"); BAR; asm volatile("" ::: "memory");
    STAMP(1);
    { const gchar* a0 = A_OF(0); const gchar* b0 = B_OF(0); const gchar* a1 = A_OF(1); const gchar* b1 = B_OF(1);
      STAGE(SB(0, 0), b0); STAGE(SA(0, 0), a0); STAGE(SB(0, 1), b0 + HTB); STAGE(SA(0, 1), a0 + HTB);
      if (wr == 1) BAR;
      WAIT_V(4); BAR;
      STAGE(SB(1, 0), b1); STAGE(SA(1, 0), a1); STAGE(SB(1, 1), b1 + HTB);
      WAIT_V(6); BAR; }
    GEMM_LOOP(0, 56, { const int seg_ = __builtin_amdgcn_readlane(tabA, t + 3) & 15;
        if (seg_ != have) { have = seg_; WAIT_SEG(have); if (have < 8) STAMP(1 + have); CLK0; } });
    STAMP(9);

    int tz = threadIdx.x; asm volatile("" : "+v"(tz));
    const int fr = tz & 15, fq = (tz >> 4) & 3;
    const unsigned lo = (unsigned)(fq * 4) * (NCOLS * 4) + (unsigned)fr * 16u;
    const unsigned soff = (unsigned)(tz & 63) * 16u;
    const size_t torg = ((size_t)((mt & 1) * 256 + wr * 64) * NCOLS + nt * 256 + wc * 64) * 4;
    const unsigned qnan = 0x7fc00000u;
    LAS unsigned char* sc = lds + 131072 + 256 + wid * 1088;
#define ROWOFF(c, g, e) ((size_t)(((c) >> 1) * 128 + (((c) & 1) * 2 + ((g) >> 1)) * 16 + ((g) & 1) * 2 + ((e) >> 2)) * (NCOLS * 4))
#define ACC(c, g, e) acc[(c) >> 1][((e) >> 1) & 1][((c) & 1) * 2 + ((g) >> 1)][(e) & 1][((g) & 1) * 2 + ((e) >> 2)]
    if (dd != 0) {
    if (__builtin_amdgcn_readfirstlane(__hip_atomic_load(tmo, __ATOMIC_RELAXED, __HIP_MEMORY_SCOPE_AGENT)) == call) {
#pragma unroll
        for (int i = 0; i < 32; ++i) {
            acc[i >> 4][(i >> 3) & 1][(i >> 1) & 3][i & 1] = __builtin_bit_cast(f32x4, (u32x4){qnan, qnan, qnan, qnan});
            asm volatile("" : "+v"(acc[i >> 4][(i >> 3) & 1][(i >> 1) & 3][i & 1]));
        }
    }
    unsigned u[4][4][8];
#pragma unroll
    for (int c = 0; c < 4; ++c)
#pragma unroll
        for (int g = 0; g < 4; ++g)
#pragma unroll
            for (int e = 0; e < 8; ++e) { const float v = ACC(c, g, e); u[c][g][e] = __builtin_bit_cast(unsigned, v); }
    for (int i = 0; i < XSTAG * wid; ++i) __builtin_amdgcn_s_sleep(1);
    unsigned meta;
    {
        unsigned se = 0;
#define U32(n) u[c][(n) >> 3][(n) & 7]
#pragma unroll
        for (int c = 0; c < 4; ++c)
#pragma unroll
            for (int g = 0; g < 4; ++g) {
#pragma unroll
                for (int e = 0; e < 8; ++e) u[c][g][e] = __builtin_amdgcn_alignbit(u[c][g][e], u[c][g][e], 31);
#pragma unroll
                for (int e = 0; e < 8; e += 2) se += __builtin_amdgcn_perm(u[c][g][e + 1], u[c][g][e], 0x0c070c03u);
                asm volatile("" : "+v"(u[c][g][0]), "+v"(u[c][g][1]), "+v"(u[c][g][2]), "+v"(u[c][g][3]), "+v"(u[c][g][4]), "+v"(u[c][g][5]), "+v"(u[c][g][6]), "+v"(u[c][g][7]));
            }
        const unsigned S = __builtin_amdgcn_readlane(wave_scan((se & 0xffffu) + (se >> 16)), 63);
        const unsigned b1 = min((S + 492u) >> 13, 253u), B = (b1 + 0xff00u - 5u) * 0x10001u;
        *(LAS u32x4*)(sc + soff) = (u32x4){0u, 0u, 0u, 0u};
        if (soff < 64u) *(LAS u32x4*)(sc + 1024 + soff) = (u32x4){0u, 0u, 0u, 0u};
        gchar* pk = myws + OFF_PK + (size_t)(bid * 8 + wid) * PK_WAVE;
        u64 st[4]; unsigned ln[4], worst = 0, up = 0;
#pragma unroll
        for (int c = 0; c < 4; ++c) {
            u32x2 cw = {0u, 0u};
            st[c] = 1; ln[c] = 0;
#pragma unroll
            for (int q = 7; q >= 0; --q) {
                unsigned W = 1, M[2];
#pragma unroll
                for (int h = 1; h >= 0; --h) {
                    const int n = 4 * q + 2 * h;
                    const unsigned E = __builtin_amdgcn_perm(U32(n + 1), U32(n), 0x0d070d03u), dq = pk_subs(E, B), ix = pk_min(dq, 0x00070007u) | 0x0c000c00u;
                    const unsigned a3 = pk_min(pk_subs(dq, 0x00070007u), 0x00030003u) * 3u;
                    up |= a3;
                    M[h] = __builtin_amdgcn_perm(0x00000001u, 0x02040507u, ix) + pk_subs(B, E) + a3;
                    cw[q >> 2] = (cw[q >> 2] << 2) | pk_subs(__builtin_amdgcn_perm(0x03020100u, 0u, ix), a3);
                    W = (W << ((M[h] >> 16) & 31u)) | 1u;
                    W = (W << (M[h] & 31u)) | 1u;
                }
                const unsigned ms = M[0] + M[1], sm = (ms & 0xffffu) + (ms >> 16);
                worst = max(worst, sm); ln[c] += sm;
                st[c] = (st[c] << (sm & 31u)) | W;
                asm volatile("" : "+v"(st[c]), "+v"(cw));
            }
#pragma unroll
            for (int i = 0; i < 6; ++i) {
#define CARR(n) __builtin_amdgcn_perm(U32(24 + (n) / 3), U32(n), 0x04020100u + (((n) % 3) << 24))
                const u32x4 v = {CARR(4 * i), CARR(4 * i + 1), CARR(4 * i + 2), CARR(4 * i + 3)};
                STORE16(soff, v, pk + c * PK_CHUNK + i * 1024, 63);
            }
            asm volatile("s_nop 4\n\tglobal_store_dwordx2 %0, %1, %2 sc0 sc1" :: "v"(soff >> 1), "v"(cw), "s"(pk + c * PK_CHUNK + 6144) : "memory");
        }
        bool ovf = max(max(ln[0], ln[1]), max(ln[2], ln[3])) > 63u || worst > 31u || (up & 0x00080008u) != 0u;
        const unsigned len = ovf ? 0u : ln[0] + ln[1] + ln[2] + ln[3], end = wave_scan(len);
        ovf |= end > 7679u;
        sc[soff >> 4] = (unsigned char)(ovf ? 255u : len);
        if (!ovf) {
            unsigned o = end - len;
#pragma unroll
            for (int c = 0; c < 4; ++c) {
                LAS unsigned* w = (LAS unsigned*)(sc + 64) + (o >> 5);
                const u64 a = st[c] << (o & 31u), b = (st[c] >> 32) << (o & 31u);
                __hip_atomic_fetch_or(w, (unsigned)a, __ATOMIC_RELAXED, __HIP_MEMORY_SCOPE_WAVEFRONT);
                __hip_atomic_fetch_or(w + 1, (unsigned)(a >> 32), __ATOMIC_RELAXED, __HIP_MEMORY_SCOPE_WAVEFRONT);
                __hip_atomic_fetch_or(w + 2, (unsigned)(b >> 32), __ATOMIC_RELAXED, __HIP_MEMORY_SCOPE_WAVEFRONT);
                o += ln[c];
            }
        }
        asm volatile("s_waitcnt lgkmcnt(0)" ::: "memory");
        const u32x4 sd = *(volatile LAS u32x4*)(sc + soff);
        STORE16(soff, sd, pk + 4 * PK_CHUNK, 63);
        meta = b1 | (((min((unsigned)__builtin_amdgcn_readlane(end, 63), 7679u) + 512u) >> 9) << 8);
        if (ovf) {
            gchar* rbase = myws + OFF_RAW + (size_t)(bid * 8 + wid) * RAW_WAVE;
#pragma unroll
            for (int c = 0; c < 4; ++c)
#pragma unroll
                for (int i = 0; i < 8; ++i) {
                    const u32x4 v = {U32(4 * i), U32(4 * i + 1), U32(4 * i + 2), U32(4 * i + 3)};
                    STORE16(soff, v, rbase + (c * 8 + i) * 1024, 63);
                }
        }
        asm volatile("s_waitcnt vmcnt(0)" ::: "memory");
        if ((tz & 63) == 0)
            __hip_atomic_store((gu32*)((gchar*)p.ws[dest] + OFF_OUTFLAG + (bid * 8 + wid) * 64), (call << 12) | meta, __ATOMIC_RELAXED, __HIP_MEMORY_SCOPE_SYSTEM);
        STAMP(12);
    }
#pragma unroll
    for (int i = 0; i < 32; ++i) acc[i >> 4][(i >> 3) & 1][(i >> 1) & 3][i & 1] = (f32x4){0.f, 0.f, 0.f, 0.f};
    }
    if (early) DIAG();
    gchar* obase = (gchar*)p.out[rank] + torg + (size_t)src * NLOC * 4;
    if (dd != 0) {
        const unsigned f = __builtin_amdgcn_readfirstlane(poll_ge((gu32*)(myws + OFF_OUTFLAG + (bid * 8 + wid) * 64), call << 12, tmo, call));
        const bool bad = __builtin_amdgcn_readfirstlane(__hip_atomic_load(tmo, __ATOMIC_RELAXED, __HIP_MEMORY_SCOPE_AGENT)) == call;
        const gchar* pk = (const gchar*)p.ws[bad ? rank : src] + OFF_PK + (size_t)(bid * 8 + wid) * PK_WAVE;
        const gchar* rbase = (const gchar*)p.ws[bad ? rank : src] + OFF_RAW + (size_t)(bid * 8 + wid) * RAW_WAVE;
        u32x4 side;
        asm volatile("s_nop 4\n\tglobal_load_dwordx4 %0, %1, %2 sc0 sc1\n\ts_waitcnt vmcnt(0)" : "=&v"(side) : "v"((soff >> 6) <= ((f >> 8) & 15u) ? soff : 0u), "s"(pk + 4 * PK_CHUNK) : "memory");
        *(volatile LAS u32x4*)(sc + soff) = side;
        const unsigned lb = *(volatile LAS unsigned char*)(sc + (soff >> 4));
        const bool ovf = lb == 255u;
        unsigned ptr = wave_scan(ovf ? 0u : lb) - (ovf ? 0u : lb);
        const unsigned BB = ((f & 255u) - 1u) * 0x10001u;
        const unsigned poison = bad ? qnan : 0u;
        if (wc == 0) {
            STAMP(14);
            if (grp == 0) poll_ge<PSL>((gu32*)(myws + OFF_PDONE + (7 - dd) * 512), (call - 1u) * NSTR + (unsigned)(xcd * 2 + wr) * NSTR / 16u + 1u, tmo, call);
            else poll_ge<GSL>((gu32*)(myws + OFF_GATE + ((bid - 8) * 2 + wr) * 128), call, tmo, call);
            STAMP(13);
        }
#pragma unroll
        for (int c = 0; c < 4; ++c) {
            while (__builtin_amdgcn_readfirstlane(lctl[1 + wr]) + 1u < (unsigned)(c * 4 + wc)) __builtin_amdgcn_s_sleep(TSL);
            while (__builtin_amdgcn_readfirstlane(lctl[1 + wr]) < (unsigned)(c * 4 + wc)) __builtin_amdgcn_s_sleep(2);
            const int lag = (int)__builtin_amdgcn_readfirstlane(lctl[3 + wr]);
            u32x4 in[6]; u32x2 cv;
            const gchar* pc = pk + c * PK_CHUNK + 3072;
            asm volatile("s_nop 4\n\tglobal_load_dwordx2 %0, %1, %2 offset:3072 sc0 sc1" NTPK : "=v"(cv) : "v"(soff >> 1), "s"(pc) : "memory");
#pragma unroll
            for (int i = 0; i < 6; ++i) {
                asm volatile("s_waitcnt vmcnt(%0)" :: "i"(EW + 1) : "memory");
                if (lag <= TH2) asm volatile("s_waitcnt vmcnt(%0)" :: "i"(EW) : "memory");
                if (lag <= TH) asm volatile("s_waitcnt vmcnt(%0)" :: "i"(EW - 1) : "memory");
                if (lag < -TH) asm volatile("s_waitcnt vmcnt(%0)" :: "i"(EW - 2) : "memory");
                asm volatile("s_nop 4\n\tglobal_load_dwordx4 %0, %1, %2 offset:%3 sc0 sc1" NTPK : "=v"(in[i]) : "v"(soff), "s"(pc), "i"(i * 1024 - 3072) : "memory");
            }
            asm volatile("s_waitcnt vmcnt(%1)" : "+v"(cv) : "i"(WTOK) : "memory");
            lctl[1 + wr] = (unsigned)(c * 4 + wc) + 1u;
            if (c == 3 && (tz & 255) == 192) __hip_atomic_store((gu32*)(myws + OFF_GATE + (bid * 2 + wr) * 128), call, __ATOMIC_RELAXED, __HIP_MEMORY_SCOPE_AGENT);
            unsigned tot = 0;
            if ((tz & 63) == 0) asm volatile("s_nop 4\n\tglobal_atomic_add %0, %1, %2, %3 sc0" : "+v"(tot) : "v"(0u), "v"(1u), "s"(myws + OFF_BDONE + (7 - dd) * 512) : "memory");
            __builtin_amdgcn_sched_barrier(0);
            unsigned ex[16], s0, s1;
            {
                ptr = min(ptr, 7679u);
                const volatile LAS unsigned* w = (const volatile LAS unsigned*)(sc + 64) + (ptr >> 5);
                const unsigned w0 = w[0], w1 = w[1], w2 = w[2];
                s0 = __builtin_amdgcn_alignbit(w1, w0, ptr) | 1u; s1 = __builtin_amdgcn_alignbit(w2, w1, ptr);
            }
#pragma unroll
            for (int q = 0; q < 8; ++q) {
                unsigned t = s0, M[2];
#pragma unroll
                for (int h = 0; h < 2; ++h) {
                    const unsigned C = (cv[q >> 2] >> (2 * ((2 * q + h) & 7))) & 0x00030003u;
                    const unsigned m0 = (unsigned)__builtin_ctz(t - ((C & 0xffffu) == 0u ? 1u : 0u));
                    t >>= m0 & 31u;
                    const unsigned m1 = (unsigned)__builtin_ctz(t - (C < 0x10000u ? 1u : 0u));
                    if (h == 0) t >>= m1 & 31u;
                    M[h] = m0 | (m1 << 16);
                    ex[2 * q + h] = __builtin_amdgcn_perm(0x030b0202u, 0x07010100u, pk_min(M[h], 0x00070007u)) - M[h] + C + BB;
                }
                const unsigned ms = M[0] + M[1], sm = (ms + (ms >> 16)) & 31u;
                s0 = __builtin_amdgcn_alignbit(s1, s0, sm); s1 >>= sm; ptr += sm;
                asm volatile("" : "+v"(ex[2 * q]), "+v"(ex[2 * q + 1]), "+v"(s0));
            }
            asm volatile("s_waitcnt vmcnt(0)" : "+v"(in[0]), "+v"(in[1]), "+v"(in[2]), "+v"(in[3]), "+v"(in[4]), "+v"(in[5]), "+v"(tot), "+v"(ptr) :: "memory");
            if ((tz & 63) == 0) lctl[3 + wr] = tot - (call - 1u) * 1024u + 8u - 16u * (unsigned)(grp * 16 + c * 4 + wc + 1);
            if (c == 3 && wid == 7) STAMPW(10);
#pragma unroll
            for (int g = 0; g < 4; ++g) {
                u32x4 q[2];
#pragma unroll
                for (int e = 0; e < 8; ++e) {
                    const int n = g * 8 + e;
#define CIN(n) in[(n) >> 2][(n) & 3]
                    const unsigned lo24 = n < 24 ? CIN(n < 24 ? n : 0)
                        : __builtin_amdgcn_perm(CIN(n < 24 ? 0 : 3 * (n - 24) + 2), __builtin_amdgcn_perm(CIN(n < 24 ? 0 : 3 * (n - 24) + 1), CIN(n < 24 ? 0 : 3 * (n - 24)), 0x00000703u), 0x00070100u);
                    q[e >> 2][e & 3] = __builtin_amdgcn_perm(ex[n >> 1], lo24, (n & 1) ? 0x06020100u : 0x04020100u);
                }
                if (ovf) asm volatile("s_nop 4\n\tglobal_load_dwordx4 %0, %2, %3 sc0 sc1\n\tglobal_load_dwordx4 %1, %2, %3 offset:1024 sc0 sc1\n\ts_waitcnt vmcnt(0)"
                                      : "+v"(q[0]), "+v"(q[1]) : "v"(soff + (c * 4 + g) * 2048u), "s"(rbase) : "memory");
                unsigned v[8];
#pragma unroll
                for (int e = 0; e < 8; ++e) v[e] = __builtin_amdgcn_alignbit(q[e >> 2][e & 3], q[e >> 2][e & 3], 1) | poison;
#pragma unroll
                for (int e = 0; e < 8; ++e) ACC(c, g, e) += __builtin_bit_cast(float, v[e]);
                if (early) {
                    const f32x4 w0 = {ACC(c, g, 0), ACC(c, g, 1), ACC(c, g, 2), ACC(c, g, 3)}, w1 = {ACC(c, g, 4), ACC(c, g, 5), ACC(c, g, 6), ACC(c, g, 7)};
                    STOREO(lo, w0, obase + ROWOFF(c, g, 0)); STOREO(lo, w1, obase + ROWOFF(c, g, 4));
                }
            }
        }
    }
    if (!early) {
        if (LSTAG && wid == 7) { const unsigned long long t1 = __builtin_amdgcn_s_memrealtime();
            while (__builtin_amdgcn_s_memrealtime() - t1 < (unsigned long long)((dd & 3) * LSTAG)) __builtin_amdgcn_s_sleep(32); }
        DIAG();
        const bool bad = __builtin_amdgcn_readfirstlane(__hip_atomic_load(tmo, __ATOMIC_RELAXED, __HIP_MEMORY_SCOPE_AGENT)) == call;
#pragma unroll
        for (int c = 0; c < 4; ++c)
#pragma unroll
            for (int g = 0; g < 4; ++g)
#pragma unroll
                for (int h = 0; h < 2; ++h) {
                    f32x4 v = {ACC(c, g, 4 * h), ACC(c, g, 4 * h + 1), ACC(c, g, 4 * h + 2), ACC(c, g, 4 * h + 3)};
                    if (bad) v = __builtin_bit_cast(f32x4, (u32x4){qnan, qnan, qnan, qnan});
                    STOREO(lo, v, obase + ROWOFF(c, g, 4 * h));
                    if (FW < 64) asm volatile("s_waitcnt vmcnt(%0)" :: "i"(FW - 1) : "memory");
                }
        STAMP(23);
    }
}

extern "C" void kernel_launch(int rank, int nranks, ncclComm_t comm,
                              hipStream_t stream,
                              void* const* d_in, const int* in_sizes, int n_in,
                              void* d_out, int out_size,
                              const int* in_dtypes, int out_dtype,
                              void* const* d_out_all,
                              void* const* d_ws_all, size_t ws_bytes) {
    if (nranks != NRANK || (unsigned)rank >= (unsigned)NRANK || ws_bytes < WS_NEED || n_in < 2 ||
        in_sizes[0] != 4096 * KSH || in_sizes[1] != KSH * NCOLS || out_size != 512 * NCOLS) {
        fprintf(stderr, "unexpected problem shape\n");
        return;
    }
    Params p;
    p.x = (const float*)d_in[0]; p.w = (const float*)d_in[1];
    for (int r = 0; r < NRANK; ++r) { p.out[r] = (float*)d_out_all[r]; p.ws[r] = (char*)d_ws_all[r]; }
    p.rank = rank; p.pad = 0;
    gemm_rs_kernel<<<GRID, 512, 0, stream>>>(p);
    HIP_CHECK(hipGetLastError());
}
```

```cpp
#include <hip/hip_runtime.h>
#include <rccl/rccl.h>
#include <cstdio>

#define HIP_CHECK(expr) do { hipError_t _e = (expr); if (_e != hipSuccess) { \
    fprintf(stderr, "HIP error %s at %s:%d\n", hipGetErrorString(_e), __FILE__, __LINE__); \
    return; } } while (0)

#define LAS __attribute__((address_space(3)))
#define GAS __attribute__((address_space(1)))
typedef _Float16 half8 __attribute__((ext_vector_type(8)));
typedef _Float16 half2v __attribute__((ext_vector_type(2)));
typedef float f32x2 __attribute__((ext_vector_type(2)));
typedef float f32x4 __attribute__((ext_vector_type(4)));
typedef unsigned u32x4 __attribute__((ext_vector_type(4)));
typedef unsigned u32x2 __attribute__((ext_vector_type(2)));
typedef unsigned long long u64;
template <int N> struct IC { static constexpr int value = N; };
typedef GAS unsigned gu32;
typedef GAS char gchar;

constexpr int NRANK = 8;
constexpr int KSH = 512;
constexpr int NCOLS = 32768, NLOC = 4096;
constexpr int GRID = 256, NPROD = 32;
constexpr size_t CTRL_BYTES = 1u << 20;
constexpr size_t PANEL = 32768;
constexpr size_t KT_STRIDE = 16 * PANEL;
constexpr size_t MAT_BYTES = 64 * KT_STRIDE;
constexpr size_t PI_ITEM = 26 * 256 + 512;
constexpr size_t OFF_PI = 3 * MAT_BYTES;
constexpr size_t BUF_BYTES = 3 * MAT_BYTES + (33u << 20);
constexpr size_t PK_CHUNK = 26 * 256;
constexpr size_t PK_WAVE = 4 * PK_CHUNK + 1024;
constexpr size_t RAW_WAVE = 64 * 512;
constexpr size_t OFF_PK = CTRL_BYTES + 2 * BUF_BYTES;
constexpr size_t OFF_RAW = OFF_PK + (size_t)GRID * 8 * PK_WAVE;
constexpr size_t WS_NEED = OFF_RAW + (size_t)GRID * 8 * RAW_WAVE;
constexpr int OFF_TMO = 0, OFF_CDONE = 28672 + 768;
constexpr int OFF_QUEUE = 20480, OFF_PDONE = 24576 + 256, OFF_PROG = 24576 + 128;
constexpr int OFF_GATE = 425984;
constexpr int OFF_INFLAG = 32768;
constexpr int OFF_CFLAG = 65536;
constexpr int OFF_OUTFLAG = 131072;
constexpr int OFF_TICKET = 393216;
constexpr unsigned long long DEADLINE_TICKS = 50000000ull;
constexpr int NPUSH = 40;
constexpr int UW = 12;
constexpr int CW = 3;
constexpr int PW = 2;
constexpr int LAG = 5;
constexpr int OWNA = 18;
constexpr int XSTAG = 20;
constexpr int EW = 3;
constexpr int TH = 12, TH2 = 40;
constexpr int NCONV = 8 * (64 + 512);
constexpr int NITEM = 960;
constexpr unsigned QUEUE_PER_CALL = NITEM - NPUSH - 2 * NPROD + NPROD;

struct Params {
    const float* x; const float* w;
    float* out;
    char* ws[NRANK];
    int rank; int pad;
};


template <int SLEEP = 8>
__device__ __forceinline__ unsigned poll_ge(gu32* addr, unsigned need, gu32* tmo, unsigned call) {
    const unsigned long long t0 = __builtin_amdgcn_s_memrealtime();
    unsigned v;
    for (unsigned n = 1;; ++n) {
        v = __hip_atomic_load(addr, __ATOMIC_RELAXED, __HIP_MEMORY_SCOPE_SYSTEM);
        if (__all((int)(v - need) >= 0)) break;
        if ((n & 15u) == 0u) {
            const unsigned tv = __builtin_amdgcn_readfirstlane(__hip_atomic_load(tmo, __ATOMIC_RELAXED, __HIP_MEMORY_SCOPE_AGENT));
            if (tv == call || __builtin_amdgcn_s_memrealtime() - t0 > DEADLINE_TICKS) {
                __hip_atomic_store(tmo, call, __ATOMIC_RELAXED, __HIP_MEMORY_SCOPE_AGENT);
                break;
            }
        }
        __builtin_amdgcn_s_sleep(SLEEP);
    }
    asm volatile("" ::: "memory");
    return v;
}

__device__ __forceinline__ unsigned pk16(float a, float b) {
    f32x2 v = {a, b};
    return __builtin_bit_cast(unsigned, __builtin_convertvector(v, half2v));
}

typedef unsigned short u16x2 __attribute__((ext_vector_type(2)));
__device__ __forceinline__ unsigned pk_max(unsigned a, unsigned b) { return __builtin_bit_cast(unsigned, __builtin_elementwise_max(__builtin_bit_cast(u16x2, a), __builtin_bit_cast(u16x2, b))); }
__device__ __forceinline__ unsigned wave_scan(unsigned x) {
    x += (unsigned)__builtin_amdgcn_update_dpp(0, (int)x, 0x111, 0xf, 0xf, false);
    x += (unsigned)__builtin_amdgcn_update_dpp(0, (int)x, 0x112, 0xf, 0xf, false);
    x += (unsigned)__builtin_amdgcn_update_dpp(0, (int)x, 0x114, 0xf, 0xf, false);
    x += (unsigned)__builtin_amdgcn_update_dpp(0, (int)x, 0x118, 0xf, 0xf, false);
    x += (unsigned)__builtin_amdgcn_update_dpp(0, (int)x, 0x142, 0xa, 0xf, false);
    x += (unsigned)__builtin_amdgcn_update_dpp(0, (int)x, 0x143, 0xc, 0xf, false);
    return x;
}

__device__ __forceinline__ unsigned pk_min(unsigned a, unsigned b) { return __builtin_bit_cast(unsigned, __builtin_elementwise_min(__builtin_bit_cast(u16x2, a), __builtin_bit_cast(u16x2, b))); }
__device__ __forceinline__ unsigned pk_subs(unsigned a, unsigned b) { return __builtin_bit_cast(unsigned, __builtin_elementwise_sub_sat(__builtin_bit_cast(u16x2, a), __builtin_bit_cast(u16x2, b))); }

__device__ __forceinline__ unsigned pack_item(unsigned (&D)[32], unsigned (&P)[26], u32x2& sd, LAS unsigned char* sc, int lane) {
    unsigned se = 0;
#pragma unroll
    for (int i = 0; i < 32; ++i) se += (D[i] >> 10) & 0x001f001fu;
    const unsigned S = __builtin_amdgcn_readlane(wave_scan((se & 0xffffu) + (se >> 16)), 63);
    const unsigned b1 = min((S + 246u) >> 12, 29u), B = (b1 + 27u) * 0x10001u;
    *(LAS u32x2*)(sc + lane * 8) = (u32x2){0u, 0u};
    if (lane < 8) *(LAS u32x2*)(sc + 512 + lane * 8) = (u32x2){0u, 0u};
    u64 lo = 1; unsigned hi = 0, len = 0, worst = 0, up = 0;
    bool ovf = false;
#pragma unroll
    for (int q = 15; q >= 0; --q) {
        unsigned W = 1, M[2];
#pragma unroll
        for (int h = 1; h >= 0; --h) {
            const int i = 2 * q + h;
            const unsigned E = ((D[i] >> 10) & 0x001f001fu) | 0x00200020u, dd = pk_subs(E, B), ix = pk_min(dd, 0x00070007u) | 0x0c000c00u;
            const unsigned a3 = pk_subs(dd, 0x00070007u) * 3u;
            up = pk_max(up, a3);
            M[h] = __builtin_amdgcn_perm(0x00000001u, 0x02040507u, ix) + pk_subs(B, E) + a3;
            D[i] = (D[i] & 0x83ff83ffu) | (pk_subs(__builtin_amdgcn_perm(0x03020100u, 0u, ix), a3) << 10);
            W = (W << ((M[h] >> 16) & 31u)) | 1u;
            W = (W << (M[h] & 31u)) | 1u;
        }
        const unsigned ms = M[0] + M[1], sm = (ms & 0xffffu) + (ms >> 16);
        worst = max(worst, sm); len += sm;
        if (q < 8) hi = (unsigned)(((((u64)hi << 32) | (lo >> 32)) << (sm & 31u)) >> 32);
        lo = (lo << (sm & 31u)) | W;
        if (q == 8) ovf = len > 63u;
        asm volatile("" : "+v"(lo), "+v"(hi));
    }
    ovf |= len > 95u || worst > 31u || pk_subs(up, 0x00060006u) != 0u;
    const unsigned end = wave_scan(ovf ? 0u : len), o = end - (ovf ? 0u : len);
    ovf |= end > 3583u;
    asm volatile("" ::: "memory");
    sc[lane] = (unsigned char)(ovf ? 255u : len);
    if (!ovf) {
        LAS unsigned* w = (LAS unsigned*)(sc + 64) + (o >> 5);
        const u64 a = lo << (o & 31u), b = (((u64)hi << 32) | (lo >> 32)) << (o & 31u), c = (u64)hi << (o & 31u);
        __hip_atomic_fetch_or(w, (unsigned)a, __ATOMIC_RELAXED, __HIP_MEMORY_SCOPE_WAVEFRONT);
        __hip_atomic_fetch_or(w + 1, (unsigned)(a >> 32), __ATOMIC_RELAXED, __HIP_MEMORY_SCOPE_WAVEFRONT);
        __hip_atomic_fetch_or(w + 2, (unsigned)(b >> 32), __ATOMIC_RELAXED, __HIP_MEMORY_SCOPE_WAVEFRONT);
        __hip_atomic_fetch_or(w + 3, (unsigned)(c >> 32), __ATOMIC_RELAXED, __HIP_MEMORY_SCOPE_WAVEFRONT);
    }
#pragma unroll
    for (int g = 0; g < 2; ++g) {
#pragma unroll
        for (int i = 0; i < 12; ++i) P[13 * g + i] = D[16 * g + i] | ((D[16 * g + 13 + i / 4] << (12 - 3 * (i & 3))) & 0x70007000u);
        P[13 * g + 12] = D[16 * g + 12] | ((D[16 * g + 13] >> 3) & 0x10001000u) | ((D[16 * g + 14] >> 2) & 0x20002000u) | ((D[16 * g + 15] >> 1) & 0x40004000u);
    }
    asm volatile("s_waitcnt lgkmcnt(0)" ::: "memory");
    sd = *(volatile LAS u32x2*)(sc + lane * 8);
    const unsigned tot = min((unsigned)__builtin_amdgcn_readlane(end, 63), 3583u);
    return b1 | (((tot + 512u) >> 9) << 5);
}
template <int Q>
__device__ __forceinline__ void unpack8(unsigned (&H)[8], unsigned BB, unsigned& s0, unsigned& s1, unsigned& s2) {
#pragma unroll
    for (int q = 0; q < 4; ++q) {
        unsigned t = s0, M[2];
#pragma unroll
        for (int h = 0; h < 2; ++h) {
            const unsigned T = H[2 * q + h];
            const half2v f = __builtin_bit_cast(half2v, T);
            const unsigned m0 = (unsigned)__builtin_ctz(t - (__builtin_amdgcn_classh(f[0], 0xf0) ? 1u : 0u));
            t >>= m0 & 31u;
            const unsigned m1 = (unsigned)__builtin_ctz(t - (__builtin_amdgcn_classh(f[1], 0xf0) ? 1u : 0u));
            if (h == 0) t >>= m1 & 31u;
            M[h] = m0 | (m1 << 16);
            H[2 * q + h] = ((__builtin_amdgcn_perm(0x030b0202u, 0x07010100u, pk_min(M[h], 0x00070007u)) - M[h]) << 10) + (T + BB);
        }
        const unsigned ms = M[0] + M[1], sm = ms + (ms >> 16);
        s0 = __builtin_amdgcn_alignbit(s1, s0, sm);
        if (Q < 2) { s1 = __builtin_amdgcn_alignbit(s2, s1, sm); s2 >>= sm & 31u; } else s1 >>= sm & 31u;
        asm volatile("" : "+v"(H[2 * q]), "+v"(H[2 * q + 1]));
    }
}
constexpr int pull_younger(int k) {
    int n = 0;
    for (int j = k - PW; j < k; ++j) {
        const int jj = (j + 7) % 7;
        if (j > k - PW) n += (j + PW) % 7 == 0 ? 2 : 1;
        if (jj == 1 || jj == 3 || jj == 5 || jj == 6) n += 2;
    }
    return n;
}

__device__ __forceinline__ void item_decode(int it, int src, int& s, int& r) {
    int e;
    constexpr int n1 = 72 * LAG, n2 = n1 + 88 * (8 - LAG);
    if (it < n1) { s = it / 72; e = it - 72 * s; }
    else if (it < n2) { s = (it - n1) / 88; e = it - n1 - 88 * s; s += e < 72 ? LAG : 0; }
    else { const int v = it - 576; s = (v >> 4) & 7; e = 72 + (v >> 7) * 16 + (v & 15); }
    r = e < 56 ? (8 * src + 8 + e) & 63 : e + 8;
}

#define UNIT_LOAD(B, S_, R_, H_) do { const bool isx_ = (R_) < 64; \
        const gchar* b_ = isx_ ? sX + (size_t)((R_) * 64 + (H_) * 32) * (KSH * 4) + (size_t)(S_) * 256 \
                               : sW + (size_t)((S_) * 64 + (H_) * 32) * (NCOLS * 4) + (size_t)((R_) - 64) * 256; \
        const unsigned vo_ = isx_ ? xoff : woff; \
        const gchar* b1_ = b_ + (isx_ ? 16 : 1 * NCOLS * 4);          const gchar* b2_ = b_ + (isx_ ? 128 : 2 * NCOLS * 4); \
        const gchar* b3_ = b_ + (isx_ ? 144 : 3 * NCOLS * 4);         const gchar* b4_ = b_ + (isx_ ? 32768 : 4 * NCOLS * 4); \
        const gchar* b5_ = b_ + (isx_ ? 32768 + 16 : 5 * NCOLS * 4);  const gchar* b6_ = b_ + (isx_ ? 32768 + 128 : 6 * NCOLS * 4); \
        const gchar* b7_ = b_ + (isx_ ? 32768 + 144 : 7 * NCOLS * 4); \
        asm volatile("s_nop 4\n\tglobal_load_dwordx4 %0, %8, %9 nt\n\tglobal_load_dwordx4 %1, %8, %10 nt\n\t" \
                     "global_load_dwordx4 %2, %8, %11 nt\n\tglobal_load_dwordx4 %3, %8, %12 nt\n\t" \
                     "global_load_dwordx4 %4, %8, %13 nt\n\tglobal_load_dwordx4 %5, %8, %14 nt\n\t" \
                     "global_load_dwordx4 %6, %8, %15 nt\n\tglobal_load_dwordx4 %7, %8, %16 nt" \
                     : "+v"(B[0]), "+v"(B[1]), "+v"(B[2]), "+v"(B[3]), "+v"(B[4]), "+v"(B[5]), "+v"(B[6]), "+v"(B[7]) \
                     : "v"(vo_), "s"(b_), "s"(b1_), "s"(b2_), "s"(b3_), "s"(b4_), "s"(b5_), "s"(b6_), "s"(b7_) : "memory"); } while (0)
#define UNIT_WAIT(B, N) asm volatile("s_waitcnt vmcnt(%8)" : "+v"(B[0]), "+v"(B[1]), "+v"(B[2]), "+v"(B[3]), \
                                     "+v"(B[4]), "+v"(B[5]), "+v"(B[6]), "+v"(B[7]) : "i"(N) : "memory")
#define UNIT_CVT(O, B, R_) do { if ((R_) < 64) { \
            _Pragma("unroll") for (int b = 0; b < 4; ++b) { \
                const f32x4 lo_ = __builtin_bit_cast(f32x4, B[2 * b]), hi_ = __builtin_bit_cast(f32x4, B[2 * b + 1]); \
                O[b] = (u32x4){pk16(lo_[0], lo_[1]), pk16(lo_[2], lo_[3]), pk16(hi_[0], hi_[1]), pk16(hi_[2], hi_[3])}; } \
        } else { \
            _Pragma("unroll") for (int j = 0; j < 4; ++j) \
                O[j] = (u32x4){pk16(__builtin_bit_cast(f32x4, B[0])[j], __builtin_bit_cast(f32x4, B[1])[j]), \
                               pk16(__builtin_bit_cast(f32x4, B[2])[j], __builtin_bit_cast(f32x4, B[3])[j]), \
                               pk16(__builtin_bit_cast(f32x4, B[4])[j], __builtin_bit_cast(f32x4, B[5])[j]), \
                               pk16(__builtin_bit_cast(f32x4, B[6])[j], __builtin_bit_cast(f32x4, B[7])[j])}; } \
        asm volatile("" : "+v"(O[0]), "+v"(O[1]), "+v"(O[2]), "+v"(O[3])); } while (0)
#define ITEM_OFF(R_) ((R_) < 64 ? (size_t)(R_) * 8192 : (size_t)(((R_) - 64) >> 2) * PANEL + (size_t)((R_) & 3) * 4096)
#define UNIT_DST(BASE, R_, H_) const bool isxd_ = (R_) < 64; \
        gchar* t0_ = (BASE) + (isxd_ ? (H_) * 4096 : (H_) * 1024); \
        gchar* t1_ = t0_ + (isxd_ ? 1024 : 2048); gchar* t2_ = t0_ + (isxd_ ? 2048 : 16384); gchar* t3_ = t0_ + (isxd_ ? 3072 : 16384 + 2048)
#define STOREO(VOFF, DATA, BASE) asm volatile("s_nop 4\n\tglobal_store_dwordx4 %0, %1, %2 sc0 sc1 nt\n\ts_nop 1" :: "v"(VOFF), "v"(DATA), "s"(BASE) : "memory")
#define STORE16(VOFF, DATA, BASE, N) asm volatile("s_nop 3\n\ts_waitcnt vmcnt(%3)\n\tglobal_store_dwordx4 %0, %1, %2 sc0 sc1\n\ts_nop 1" \
                                                  :: "v"(VOFF), "v"(DATA), "s"(BASE), "i"(N) : "memory")

__global__ __launch_bounds__(512, 2) void gemm_rs_kernel(const Params p) {
    __shared__ __attribute__((aligned(16))) unsigned char shm[131072 + 256 + 8 * 1088];
    LAS unsigned char* lds = (LAS unsigned char*)shm;
    volatile LAS unsigned* lctl = (volatile LAS unsigned*)(lds + 131072);
    const int tid = threadIdx.x, wid = __builtin_amdgcn_readfirstlane(tid >> 6), lane = tid & 63;
    const int rank = p.rank & (NRANK - 1), bid = blockIdx.x;
    gchar* myws = (gchar*)p.ws[rank];
    gu32* tmo = (gu32*)(myws + OFF_TMO);

    if (tid == 0) {
        lctl[0] = __hip_atomic_fetch_add((gu32*)(myws + OFF_TICKET + bid * 128), 1u, __ATOMIC_RELAXED, __HIP_MEMORY_SCOPE_AGENT) + 1u;
    } else if (tid < 64) lctl[tid] = 0u;
    unsigned call; size_t bufoff;
#define GET_CALL() do { asm volatile("s_waitcnt lgkmcnt(0)" ::: "memory"); __builtin_amdgcn_s_barrier(); asm volatile("" ::: "memory"); \
        call = __builtin_amdgcn_readfirstlane(lctl[0]); bufoff = CTRL_BYTES + (size_t)(call & 1u) * BUF_BYTES; } while (0)

    const int xcd = bid & 7, idx = bid >> 3, dd = idx >> 2;
    const int dest = (rank + dd) & (NRANK - 1);
    const int mt = 2 * dest + (xcd & 1), grp = idx & 3, nt = 4 * grp + (xcd >> 1);
    const int hidx = (dd - 1) * 16 + (bid & 15);

    const unsigned soff0 = (unsigned)lane * 16u;
    const gchar* sX = (const gchar*)p.x;
    const unsigned xoff = (unsigned)(lane & 15) * (KSH * 4) + (unsigned)(lane >> 4) * 32u;
    const unsigned woff = (unsigned)(lane >> 4) * 8u * (NCOLS * 4) + (unsigned)(lane & 15) * 16u;
    if (bid >= NPROD && nt >= 8 && wid >= 1 && wid <= CW) {
        u32x4 buf0[8], buf1[8], o[4], o1[4];
#pragma unroll
        for (int b = 0; b < 8; ++b) { buf0[b] = (u32x4){0u, 0u, 0u, 0u}; buf1[b] = (u32x4){0u, 0u, 0u, 0u}; }
        bool first = true;
        for (int job = NPUSH + (wid - 1) * 112 + hidx; job < NCONV; job += CW * 112) {
            int s, e;
            if (job < 1536) { s = job / 192; e = job - 192 * s; }
            else { const int v = job - 1536; s = (v >> 7) & 7; e = 192 + (v >> 10) * 128 + (v & 127); }
            const int d = e < 64 ? rank : (rank + 1 + ((e - 64) & 7)) & (NRANK - 1);
            const int r = e < 64 ? (8 * rank + 8 + e) & 63 : 64 + ((e - 64) >> 3);
            const gchar* sW = (const gchar*)p.w + (size_t)d * NLOC * 4;
            UNIT_LOAD(buf0, s, r, 0); UNIT_LOAD(buf1, s, r, 1);
            if (first) { GET_CALL(); first = false; }
            gchar* tb = myws + bufoff + ITEM_OFF(r) + (e < 64 ? (size_t)(s * 8 + rank) * KT_STRIDE
                        : d == rank ? MAT_BYTES + (size_t)(s * 8 + rank) * KT_STRIDE : 2 * MAT_BYTES + (size_t)(d * 8 + s) * KT_STRIDE);
            UNIT_WAIT(buf0, 8); UNIT_CVT(o, buf0, r);
            { UNIT_DST(tb, r, 0); STORE16(soff0, o[0], t0_, 63); STORE16(soff0, o[1], t1_, 63); STORE16(soff0, o[2], t2_, 63); STORE16(soff0, o[3], t3_, 63); }
            UNIT_WAIT(buf1, 4); UNIT_CVT(o1, buf1, r);
            { UNIT_DST(tb, r, 1); STORE16(soff0, o1[0], t0_, 63); STORE16(soff0, o1[1], t1_, 63); STORE16(soff0, o1[2], t2_, 63); STORE16(soff0, o1[3], t3_, 63); }
            unsigned meta = 0;
            if (e < 56 || d != rank) {
                unsigned D[32], P[26]; u32x2 sd;
#pragma unroll
                for (int i = 0; i < 16; ++i) { D[i] = o[i >> 2][i & 3]; D[16 + i] = o1[i >> 2][i & 3]; }
                meta = pack_item(D, P, sd, lds + wid * 1024, lane);
                gchar* pi = myws + bufoff + OFF_PI + (size_t)(e < 64 ? s * 64 + r : 512 + (d * 8 + s) * 64 + r - 64) * PI_ITEM;
#pragma unroll
                for (int i = 0; i < 6; ++i) { const u32x4 v = {P[4 * i], P[4 * i + 1], P[4 * i + 2], P[4 * i + 3]}; STORE16(soff0, v, pi + i * 1024, 63); }
                const u32x2 v2 = {P[24], P[25]};
                asm volatile("s_nop 4\n\tglobal_store_dwordx2 %0, %1, %3 sc0 sc1\n\tglobal_store_dwordx2 %0, %2, %3 offset:512 sc0 sc1"
                             :: "v"(soff0 >> 1), "v"(v2), "v"(sd), "s"(pi + 6144) : "memory");
            }
            asm volatile("s_waitcnt vmcnt(0)" ::: "memory");
            const int fo = rank * 4096 + (s * 128 + r) * 4;
#pragma unroll
            for (int k = 0; k < NRANK; ++k)
                if ((e < 56 || k == d) && lane == 0)
                    __hip_atomic_store((gu32*)((gchar*)p.ws[k] + (k == rank ? OFF_INFLAG : OFF_CFLAG + (call & 1u) * 32768u) + fo), k == rank ? call : (call << 8) | meta, __ATOMIC_RELAXED, __HIP_MEMORY_SCOPE_SYSTEM);
        }
        if (lane == 0) __hip_atomic_fetch_add((gu32*)(myws + OFF_CDONE), 1u, __ATOMIC_RELAXED, __HIP_MEMORY_SCOPE_AGENT);
    } else if (bid >= NPROD && nt < 8 && wid == 1 && hidx < NPUSH) {
        u32x4 buf0[8], buf1[8], o0[4], o1[4];
#pragma unroll
        for (int b = 0; b < 8; ++b) { buf0[b] = (u32x4){0u, 0u, 0u, 0u}; buf1[b] = (u32x4){0u, 0u, 0u, 0u}; }
        const gchar* sW = sX;
        const int r = (8 * rank + 8 + hidx) & 63;
        UNIT_LOAD(buf0, 0, r, 0); UNIT_LOAD(buf1, 0, r, 1);
        GET_CALL();
        UNIT_WAIT(buf0, 8); UNIT_CVT(o0, buf0, 0);
        UNIT_WAIT(buf1, 0); UNIT_CVT(o1, buf1, 0);
        const size_t ioff = bufoff + (size_t)rank * KT_STRIDE + (size_t)r * 8192;
#pragma unroll
        for (int b = 0; b < 8; ++b)
            for (int k = 1; k <= NRANK; ++k) {
                gchar* t = (gchar*)p.ws[(rank + hidx + k) & (NRANK - 1)] + ioff + b * 1024;
                STORE16(soff0, b < 4 ? o0[b & 3] : o1[b & 3], t, UW - 1);
            }
        asm volatile("s_waitcnt vmcnt(0)" ::: "memory");
#pragma unroll
        for (int k = 0; k < NRANK; ++k)
            if (lane == 0) __hip_atomic_store((gu32*)((gchar*)p.ws[k] + OFF_INFLAG + rank * 4096 + r * 4), call, __ATOMIC_RELAXED, __HIP_MEMORY_SCOPE_SYSTEM);
    } else if (bid >= NPROD) {
        GET_CALL();
    } else if (wid == 0) {
        GET_CALL();
        const int j = lane & 7, lk = (j + bid) % 7, psrc = (rank + 1 + lk) & (NRANK - 1);
        const unsigned long long t0 = __builtin_amdgcn_s_memrealtime();
        unsigned n = 0; int item = 0; bool have = false, fin = lane >= 7, late = false;
        for (unsigned it = 1; !__all(fin); ++it) {
            if (!fin && !have && n <= lctl[8 + j]) {
                unsigned t = NPUSH + bid + n * NPROD;
                if (n >= 2) t = __hip_atomic_fetch_add((gu32*)(myws + OFF_QUEUE + lk * 512), 1u, __ATOMIC_RELAXED, __HIP_MEMORY_SCOPE_AGENT)
                                - (call - 1u) * QUEUE_PER_CALL + NPUSH + 2u * NPROD;
                int s, r; item_decode(t < (unsigned)NITEM ? (int)t : 0, psrc, s, r);
                item = t < (unsigned)NITEM ? s * 128 + r : 1024; have = true;
            }
            if (have) {
                bool ready = item >= 1024 || late; unsigned cf = 0;
                if (!ready) { cf = __hip_atomic_load((gu32*)(myws + OFF_CFLAG + (call & 1u) * 32768u + psrc * 4096 + item * 4), __ATOMIC_RELAXED, __HIP_MEMORY_SCOPE_SYSTEM); ready = (int)(cf - (call << 8)) >= 0; }
                if (ready) { ++n; __hip_atomic_store((LAS unsigned*)lctl + 32 + j * 4 + (n & 3u), (n << 20) | ((cf & 255u) << 12) | (unsigned)item, __ATOMIC_RELEASE, __HIP_MEMORY_SCOPE_WORKGROUP); have = false; fin = item >= 1024; }
            }
            if ((it & 63u) == 0u && !late) {
                late = __builtin_amdgcn_readfirstlane(__hip_atomic_load(tmo, __ATOMIC_RELAXED, __HIP_MEMORY_SCOPE_AGENT)) == call
                       || __builtin_amdgcn_s_memrealtime() - t0 > DEADLINE_TICKS;
                if (late) __hip_atomic_store(tmo, call, __ATOMIC_RELAXED, __HIP_MEMORY_SCOPE_AGENT);
            }
            __builtin_amdgcn_s_sleep(2);
        }
    } else {
        GET_CALL();
        const int lk = (wid - 1 + bid) % 7, src = (rank + 1 + lk) & (NRANK - 1);
        const gchar* rimg = (const gchar*)p.ws[src] + bufoff;
        gchar* limg = myws + bufoff + (size_t)src * KT_STRIDE;
        gu32* lflag = (gu32*)(myws + OFF_INFLAG + src * 4096);
        u32x4 in[6]; u32x2 in6 = {0u, 0u}, side = {0u, 0u};
#pragma unroll
        for (int b = 0; b < 6; ++b) in[b] = (u32x4){0u, 0u, 0u, 0u};
        const unsigned soff8 = (unsigned)lane * 8u;
        LAS unsigned char* sc = lds + wid * 1024;
#define PI_OF(S_, R_) (rimg + OFF_PI + 3072 + (size_t)((R_) < 64 ? (S_) * 64 + (R_) : 512 + (rank * 8 + (S_)) * 64 + (R_) - 64) * PI_ITEM)
#define RAW_OF(S_, R_) (rimg + ITEM_OFF(R_) + ((R_) < 64 ? (size_t)((S_) * 8 + src) * KT_STRIDE : 2 * MAT_BYTES + (size_t)(rank * 8 + (S_)) * KT_STRIDE))
#define DST_OF(S_, R_) (limg + ITEM_OFF(R_) + ((R_) < 64 ? 0 : MAT_BYTES) + (size_t)(S_) * 8 * KT_STRIDE)
#define PULL(k, BASE, META) do { \
        if ((k) == 0) asm volatile("s_nop 4\n\tglobal_load_dwordx2 %0, %1, %2 offset:3584 sc0 sc1" : "+v"(side) : "v"((soff8 >> 6) <= ((META) >> 5) ? soff8 : 0u), "s"(BASE) : "memory"); \
        if ((k) < 6) asm volatile("s_nop 4\n\tglobal_load_dwordx4 %0, %1, %2 offset:%3 sc0 sc1" : "+v"(in[(k) < 6 ? (k) : 0]) : "v"(soff0), "s"(BASE), "i"((k) * 1024 - 3072) : "memory"); \
        else asm volatile("s_nop 4\n\tglobal_load_dwordx2 %0, %1, %2 offset:3072 sc0 sc1" : "+v"(in6) : "v"(soff8), "s"(BASE) : "memory"); } while (0)
#define TAKE(DST, META, N_) do { unsigned e_; while (((e_ = __builtin_amdgcn_readfirstlane(__hip_atomic_load((LAS unsigned*)lctl + 28 + wid * 4 + ((N_) & 3), __ATOMIC_ACQUIRE, __HIP_MEMORY_SCOPE_WORKGROUP))) >> 20) != (unsigned)(N_)) __builtin_amdgcn_s_sleep(1); \
        lctl[7 + wid] = (unsigned)(N_); asm volatile("" ::: "memory"); DST = (int)(e_ & 0x7ffu); META = (e_ >> 12) & 255u; } while (0)
        int cur, s, r, pflag = -1; unsigned meta;
        TAKE(cur, meta, 1);
        s = cur >> 7; r = cur & 127;
        const gchar* sb = PI_OF(s, r);
#pragma unroll
        for (int k = 0; k < PW; ++k) PULL(k, sb, meta);
        for (int i = 0; cur < 1024; ++i) {
            int nxt = 1024, sn = 0, rn = 0; const gchar* sbn = sb; unsigned metan = 0;
            gchar* db = DST_OF(s, r);
            const bool isx = r < 64;
            const gchar* rraw = RAW_OF(s, r);
            asm volatile("" : "+s"(rraw), "+s"(db));
            const unsigned BB = ((meta & 31u) - 1u) * 0x04000400u;
            unsigned s0 = 0, s1 = 0, s2 = 0, dn[3] = {0u, 0u, 0u}; bool ovf = false;
            auto step = [&](auto KC) __attribute__((always_inline)) {
                constexpr int k = decltype(KC)::value;
                (void)&rraw; (void)&db; (void)&sb; (void)&soff0; (void)&soff8; (void)&in; (void)&in6; (void)&side;
                if (k + PW == 7) {
                    TAKE(nxt, metan, i + 2);
                    sn = (nxt >> 7) & 7; rn = nxt & 127;
                    sbn = nxt < 1024 ? PI_OF(sn, rn) : (const gchar*)myws + bufoff + OFF_PI + 3072;
                    if (nxt >= 1024 && lane == 0) __hip_atomic_fetch_add((gu32*)(myws + OFF_PDONE + lk * 512), 1u, __ATOMIC_RELAXED, __HIP_MEMORY_SCOPE_AGENT);
                }
                if (k < PW && i == 0) asm volatile("s_waitcnt vmcnt(%0)" :: "i"(PW - 1) : "memory");
                if (k < 6) asm volatile("s_waitcnt vmcnt(%2)" : "+v"(in[k < 6 ? k : 0]), "+v"(side) : "i"(pull_younger(k)) : "memory");
                else asm volatile("s_waitcnt vmcnt(%1)" : "+v"(in6) : "i"(pull_younger(k)) : "memory");
                if (k == PW && pflag >= 0 && lane == 0) __hip_atomic_store(lflag + pflag, call, __ATOMIC_RELAXED, __HIP_MEMORY_SCOPE_SYSTEM);
                if (k + PW < 7) PULL((k + PW) % 7, sb, meta); else PULL((k + PW) % 7, sbn, metan);
                __builtin_amdgcn_sched_barrier(0);
                if (k == 0) {
                    *(volatile LAS u32x2*)(sc + lane * 8) = side;
                    const unsigned lb = *(volatile LAS unsigned char*)(sc + lane);
                    ovf = lb == 255u;
                    const unsigned len = ovf ? 0u : lb, o = min(wave_scan(len) - len, 3583u);
                    const volatile LAS unsigned* w = (const volatile LAS unsigned*)(sc + 64) + (o >> 5);
                    const unsigned w0 = w[0], w1 = w[1], w2 = w[2], w3 = w[3];
                    s0 = __builtin_amdgcn_alignbit(w1, w0, o) | 1u; s1 = __builtin_amdgcn_alignbit(w2, w1, o); s2 = __builtin_amdgcn_alignbit(w3, w2, o);
                    asm volatile("" : "+v"(s0), "+v"(s1), "+v"(s2));
                }
                if (k == 1 || k == 3 || k == 5 || k == 6) {
                    constexpr int g = k >> 1;
#define PIN(n) ((n) < 24 ? in[((n) < 24 ? (n) : 0) >> 2][(n) & 3] : in6[(n) & 1])
                    unsigned H[8];
#pragma unroll
                    for (int e = 0; e < ((g & 1) ? 5 : 8); ++e) {
                        const int ci = (g & 1) * 8 + e; const unsigned pw = PIN(13 * (g >> 1) + ci);
                        H[e] = pw & 0x8fff8fffu;
                        if (ci < 12) { const unsigned pc = (pw >> (12 - 3 * (ci & 3))) & (0x00070007u << (3 * (ci & 3))); dn[ci >> 2] = (ci & 3) ? dn[ci >> 2] | pc : pc; }
                        else { H[5] = dn[0] | ((pw << 3) & 0x80008000u); H[6] = dn[1] | ((pw << 2) & 0x80008000u); H[7] = dn[2] | ((pw << 1) & 0x80008000u); }
                    }
                    unpack8<g>(H, BB, s0, s1, s2);
                    u32x4 v0 = {H[0], H[1], H[2], H[3]}, v1 = {H[4], H[5], H[6], H[7]};
                    const unsigned o0 = soff0 + (isx ? g * 2048 : (g & 1) * 16384 + (g >> 1) * 1024), o1 = o0 + (isx ? 1024 : 2048);
                    if (ovf) asm volatile("s_nop 4\n\tglobal_load_dwordx4 %0, %2, %4 sc0 sc1\n\tglobal_load_dwordx4 %1, %3, %4 sc0 sc1\n\ts_waitcnt vmcnt(0)"
                                          : "+v"(v0), "+v"(v1) : "v"(o0), "v"(o1), "s"(rraw) : "memory");
                    STORE16(o0, v0, db, 63); STORE16(o1, v1, db, 63);
                }
            };
            step(IC<0>{}); step(IC<1>{}); step(IC<2>{}); step(IC<3>{}); step(IC<4>{}); step(IC<5>{}); step(IC<6>{});
            pflag = cur; cur = nxt; s = sn; r = rn; sb = sbn; meta = metan;
        }
        asm volatile("s_waitcnt vmcnt(0)" : "+v"(in[0]), "+v"(in[1]), "+v"(in[2]), "+v"(in[3]), "+v"(side) :: "memory");
        if (lane == 0) __hip_atomic_store(lflag + pflag, call, __ATOMIC_RELAXED, __HIP_MEMORY_SCOPE_SYSTEM);
    }

    const int wr = wid >> 2, wc = wid & 3;
    const gchar* img = myws + bufoff;
    const unsigned voff = (unsigned)tid * 16u;
    const unsigned ldsw = (unsigned)wid * 1024u;
    const int aoff = wr * 8192 + lane * 16, boff = wc * 4096 + lane * 16;
    constexpr int HTB = 16384;
#define SA(b, h) (((b) * 2 + (h)) * HTB)
#define SB(b, h) ((4 + (b) * 2 + (h)) * HTB)
#define STAGE(bufo, gbase) do { _Pragma("unroll") for (int _i = 0; _i < 2; ++_i) \
        __builtin_amdgcn_global_load_lds((const GAS unsigned*)((gbase) + voff + _i * 8192), (LAS unsigned*)(lds + (bufo) + ldsw + _i * 8192), 16, 0, 17); } while (0)
#define LDA(dst, b, h) do { _Pragma("unroll") for (int m = 0; m < 4; ++m) _Pragma("unroll") for (int k = 0; k < 2; ++k) \
        dst[m][k] = *(const LAS half8*)(lds + SA(b, h) + aoff + m * 2048 + k * 1024); } while (0)
#define LDB(dst, b, h) do { _Pragma("unroll") for (int n = 0; n < 2; ++n) _Pragma("unroll") for (int k = 0; k < 2; ++k) \
        dst[n][k] = *(const LAS half8*)(lds + SB(b, h) + boff + n * 2048 + k * 1024); } while (0)
#define MMA(ai, bj, At_, Bt_) do { __builtin_amdgcn_s_setprio(1); \
        _Pragma("unroll") for (int m = 0; m < 4; ++m) _Pragma("unroll") for (int n = 0; n < 2; ++n) _Pragma("unroll") for (int k = 0; k < 2; ++k) \
            acc[ai][bj][m][n] = __builtin_amdgcn_mfma_f32_16x16x32_f16(At_[m][k], Bt_[n][k], acc[ai][bj][m][n], 0, 0, 0); \
        __builtin_amdgcn_s_setprio(0); } while (0)
#define WAIT_V(n) asm volatile("s_waitcnt vmcnt(" #n ")" ::: "memory")
#define WAIT_L(n) asm volatile("s_waitcnt lgkmcnt(" #n ")" ::: "memory")
#define BAR __builtin_amdgcn_s_barrier()
#define SCHED __builtin_amdgcn_sched_barrier(0)
    bool allin = false;
#define WAIT_SEG(c) do { if (wid == 0 && !allin) { int lz_ = lane; asm volatile("" : "+v"(lz_)); const int wh_ = lz_ >> 3; \
        const int it_ = wh_ < 4 && ((c) == 8 || (lz_ & 7) != dest) ? 4 * mt + wh_ : 64 + 4 * nt + (wh_ & 3); \
        gu32* fa_ = (gu32*)(myws + OFF_INFLAG + ((c) == 8 ? rank * 4096 + (lz_ & 7) * 512 : (lz_ & 7) * 4096 + (c) * 512) + it_ * 4); \
        bool now_ = false; \
        if ((c) < 8 && (dd == 0 || (c) > (grp == 0 ? 8 : grp == 1 ? 8 - LAG : 0))) { unsigned v_[9]; \
            v_[8] = __hip_atomic_load((gu32*)(myws + OFF_CDONE), __ATOMIC_RELAXED, __HIP_MEMORY_SCOPE_SYSTEM); \
            _Pragma("unroll") for (int k_ = 0; k_ < 8; ++k_) v_[k_] = __hip_atomic_load(fa_ + ((c) + k_ < 8 ? k_ : 0) * 128, __ATOMIC_RELAXED, __HIP_MEMORY_SCOPE_SYSTEM); \
            asm volatile("" : "+v"(v_[0]), "+v"(v_[1]), "+v"(v_[2]), "+v"(v_[3]), "+v"(v_[4]), "+v"(v_[5]), "+v"(v_[6]), "+v"(v_[7]), "+v"(v_[8]) :: "memory"); \
            now_ = __all(v_[0] >= call); \
            allin = now_ && __all(min(min(min(v_[1], v_[2]), min(v_[3], v_[4])), min(min(v_[5], v_[6]), v_[7])) >= call && (int)(v_[8] - call * (112u * CW)) >= 0); } \
        if (!now_) { if ((c) == (grp == 0 ? 7 : grp == 1 ? 8 - LAG : 0)) poll_ge<16>(fa_, call, tmo, call); else poll_ge<64>(fa_, call, tmo, call); } \
        if ((c) == 6 && !allin) poll_ge<64>((gu32*)(myws + OFF_CDONE), call * (112u * CW), tmo, call);     \
        } } while (0)

    f32x4 acc[2][2][4][2];
#pragma unroll
    for (int a = 0; a < 2; ++a)
#pragma unroll
        for (int b = 0; b < 2; ++b)
#pragma unroll
            for (int m = 0; m < 4; ++m)
#pragma unroll
                for (int n = 0; n < 2; ++n) acc[a][b][m][n] = (f32x4){0.f, 0.f, 0.f, 0.f};
    half8 At[4][2], B0[2][2], B1[2][2];

#define A_OF(t) (img + ((unsigned)__builtin_amdgcn_readlane(tabA, (t) & 63) & ~15u))
#define B_OF(t) (img + (unsigned)__builtin_amdgcn_readlane(tabB, (t) & 63))
#define GEMM_LOOP(T0, T1, HOOK) do { \
    const gchar* a3 = A_OF((T0) + 1); \
    for (int t = (T0); t < (T1); t += 2) { \
        const gchar* a1 = a3; \
        LDB(B0, 0, 0); SCHED; LDA(At, 0, 0); STAGE(SA(1, 1), a1 + HTB); SCHED; \
        const gchar* a2 = A_OF(t + 2); const gchar* b2 = B_OF(t + 2); const gchar* b3 = B_OF(t + 3); a3 = A_OF(t + 3); \
        WAIT_L(8); BAR; WAIT_L(0); MMA(0, 0, At, B0); \
        HOOK; \
        asm volatile("" ::: "memory"); BAR; SCHED; \
        LDB(B1, 0, 1); STAGE(SB(0, 0), b2); \
        BAR; WAIT_L(0); MMA(0, 1, At, B1); BAR; \
        LDA(At, 0, 1); STAGE(SA(0, 0), a2); \
        BAR; WAIT_L(0); MMA(1, 0, At, B0); BAR; SCHED; \
        STAGE(SB(0, 1), b2 + HTB); \
        WAIT_V(6); BAR; MMA(1, 1, At, B1); BAR; \
        LDB(B0, 1, 0); SCHED; LDA(At, 1, 0); STAGE(SA(0, 1), a2 + HTB); \
        WAIT_L(8); BAR; WAIT_L(0); MMA(0, 0, At, B0); BAR; SCHED; \
        LDB(B1, 1, 1); STAGE(SB(1, 0), b3); \
        BAR; WAIT_L(0); MMA(0, 1, At, B1); BAR; \
        LDA(At, 1, 1); STAGE(SA(1, 0), a3); \
        BAR; WAIT_L(0); MMA(1, 0, At, B0); BAR; SCHED; \
        STAGE(SB(1, 1), b3 + HTB); \
        WAIT_V(6); BAR; MMA(1, 1, At, B1); BAR; \
    } \
    if (wr == 0) BAR;                                                   \
    } while (0)

    const int src = (rank - dd) & (NRANK - 1);
    const bool early = dd != 0 && grp == 3;
#define DIAG() do { WAIT_V(0); BAR; if (wr == 1) BAR; asm volatile("" ::: "memory"); GEMM_LOOP(56, 64, ); } while (0)

    int tabA, tabB;
    {
        const int t0 = grp != 0 ? 0 : OWNA, u = lane < t0 ? lane : lane - 8, oo = 8 - dd;
        const int st = dd == 0 ? lane / 7 : u / 6, jj = dd == 0 ? lane % 7 : u % 6, o = jj + 1 + (jj + 1 >= oo ? 1 : 0);
        const bool own = dd != 0 && lane >= t0 && lane < t0 + 8;
        const int kt = own ? (lane - t0) * 8 + rank : (st & 7) * 8 + ((dest + o) & 7), q = lane & 7;
        tabA = lane < 56 ? (kt * (int)KT_STRIDE + mt * (int)PANEL) | (own ? 8 : st & 7) : ((q * 8 + rank) * (int)KT_STRIDE + (2 * rank + (mt & 1)) * (int)PANEL) | 7;
        tabB = nt * (int)PANEL + (lane < 56 ? (int)MAT_BYTES + kt * (int)KT_STRIDE
                                  : dd == 0 ? (int)MAT_BYTES + (q * 8 + rank) * (int)KT_STRIDE : 2 * (int)MAT_BYTES + (src * 8 + q) * (int)KT_STRIDE);
    }
    int have = __builtin_amdgcn_readlane(tabA, 0) & 15;
    BAR;
    WAIT_SEG(have);
    asm volatile("" ::: "memory"); BAR; asm volatile("" ::: "memory");
    { const gchar* a0 = A_OF(0); const gchar* b0 = B_OF(0); const gchar* a1 = A_OF(1); const gchar* b1 = B_OF(1);
      STAGE(SB(0, 0), b0); STAGE(SA(0, 0), a0); STAGE(SB(0, 1), b0 + HTB); STAGE(SA(0, 1), a0 + HTB);
      if (wr == 1) BAR;
      WAIT_V(4); BAR;
      STAGE(SB(1, 0), b1); STAGE(SA(1, 0), a1); STAGE(SB(1, 1), b1 + HTB);
      WAIT_V(6); BAR; }
    GEMM_LOOP(0, 56, { const int seg_ = __builtin_amdgcn_readlane(tabA, t + 3) & 15;
        if (seg_ != have) { have = seg_; WAIT_SEG(have); } });

    int tz = threadIdx.x; asm volatile("" : "+v"(tz));
    const int fr = tz & 15, fq = (tz >> 4) & 3;
    const unsigned lo = (unsigned)(fq * 4) * (NCOLS * 4) + (unsigned)fr * 16u;
    const unsigned soff = (unsigned)(tz & 63) * 16u;
    const size_t torg = ((size_t)((mt & 1) * 256 + wr * 64) * NCOLS + nt * 256 + wc * 64) * 4;
    const unsigned qnan = 0x7fc00000u;
    LAS unsigned char* sc = lds + 131072 + 256 + wid * 1088;
#define ROWOFF(c, g, e) ((size_t)(((c) >> 1) * 128 + (((c) & 1) * 2 + ((g) >> 1)) * 16 + ((g) & 1) * 2 + ((e) >> 2)) * (NCOLS * 4))
#define ACC(c, g, e) acc[(c) >> 1][((e) >> 1) & 1][((c) & 1) * 2 + ((g) >> 1)][(e) & 1][((g) & 1) * 2 + ((e) >> 2)]
    if (dd != 0) {
    if (__builtin_amdgcn_readfirstlane(__hip_atomic_load(tmo, __ATOMIC_RELAXED, __HIP_MEMORY_SCOPE_AGENT)) == call) {
#pragma unroll
        for (int i = 0; i < 32; ++i) {
            acc[i >> 4][(i >> 3) & 1][(i >> 1) & 3][i & 1] = __builtin_bit_cast(f32x4, (u32x4){qnan, qnan, qnan, qnan});
            asm volatile("" : "+v"(acc[i >> 4][(i >> 3) & 1][(i >> 1) & 3][i & 1]));
        }
    }
    unsigned u[4][4][8];
#pragma unroll
    for (int c = 0; c < 4; ++c)
#pragma unroll
        for (int g = 0; g < 4; ++g)
#pragma unroll
            for (int e = 0; e < 8; ++e) { const float v = ACC(c, g, e); u[c][g][e] = __builtin_bit_cast(unsigned, v); }
    for (int i = 0; i < XSTAG * wid; ++i) __builtin_amdgcn_s_sleep(1);
    unsigned meta;
    {
        unsigned se = 0;
#define U32(n) u[c][(n) >> 3][(n) & 7]
#pragma unroll
        for (int c = 0; c < 4; ++c)
#pragma unroll
            for (int g = 0; g < 4; ++g) {
#pragma unroll
                for (int e = 0; e < 8; ++e) u[c][g][e] = __builtin_amdgcn_alignbit(u[c][g][e], u[c][g][e], 31);
#pragma unroll
                for (int e = 0; e < 8; e += 2) se += __builtin_amdgcn_perm(u[c][g][e + 1], u[c][g][e], 0x0c070c03u);
                asm volatile("" : "+v"(u[c][g][0]), "+v"(u[c][g][1]), "+v"(u[c][g][2]), "+v"(u[c][g][3]), "+v"(u[c][g][4]), "+v"(u[c][g][5]), "+v"(u[c][g][6]), "+v"(u[c][g][7]));
            }
        const unsigned S = __builtin_amdgcn_readlane(wave_scan((se & 0xffffu) + (se >> 16)), 63);
        const unsigned b1 = min((S + 492u) >> 13, 253u), B = (b1 + 0xff00u - 5u) * 0x10001u;
        *(LAS u32x4*)(sc + soff) = (u32x4){0u, 0u, 0u, 0u};
        if (soff < 64u) *(LAS u32x4*)(sc + 1024 + soff) = (u32x4){0u, 0u, 0u, 0u};
        gchar* pk = myws + OFF_PK + (size_t)(bid * 8 + wid) * PK_WAVE;
        u64 st[4]; unsigned ln[4], worst = 0, up = 0;
#pragma unroll
        for (int c = 0; c < 4; ++c) {
            u32x2 cw = {0u, 0u};
            st[c] = 1; ln[c] = 0;
#pragma unroll
            for (int q = 7; q >= 0; --q) {
                unsigned W = 1, M[2];
#pragma unroll
                for (int h = 1; h >= 0; --h) {
                    const int n = 4 * q + 2 * h;
                    const unsigned E = __builtin_amdgcn_perm(U32(n + 1), U32(n), 0x0d070d03u), dq = pk_subs(E, B), ix = pk_min(dq, 0x00070007u) | 0x0c000c00u;
                    const unsigned a3 = pk_min(pk_subs(dq, 0x00070007u), 0x00030003u) * 3u;
                    up |= a3;
                    M[h] = __builtin_amdgcn_perm(0x00000001u, 0x02040507u, ix) + pk_subs(B, E) + a3;
                    cw[q >> 2] = (cw[q >> 2] << 2) | pk_subs(__builtin_amdgcn_perm(0x03020100u, 0u, ix), a3);
                    W = (W << ((M[h] >> 16) & 31u)) | 1u;
                    W = (W << (M[h] & 31u)) | 1u;
                }
                const unsigned ms = M[0] + M[1], sm = (ms & 0xffffu) + (ms >> 16);
                worst = max(worst, sm); ln[c] += sm;
                st[c] = (st[c] << (sm & 31u)) | W;
                asm volatile("" : "+v"(st[c]), "+v"(cw));
            }
#pragma unroll
            for (int i = 0; i < 6; ++i) {
#define CARR(n) __builtin_amdgcn_perm(U32(24 + (n) / 3), U32(n), 0x04020100u + (((n) % 3) << 24))
                const u32x4 v = {CARR(4 * i), CARR(4 * i + 1), CARR(4 * i + 2), CARR(4 * i + 3)};
                STORE16(soff, v, pk + c * PK_CHUNK + i * 1024, 63);
            }
            asm volatile("s_nop 4\n\tglobal_store_dwordx2 %0, %1, %2 sc0 sc1" :: "v"(soff >> 1), "v"(cw), "s"(pk + c * PK_CHUNK + 6144) : "memory");
        }
        bool ovf = max(max(ln[0], ln[1]), max(ln[2], ln[3])) > 63u || worst > 31u || (up & 0x00080008u) != 0u;
        const unsigned len = ovf ? 0u : ln[0] + ln[1] + ln[2] + ln[3], end = wave_scan(len);
        ovf |= end > 7679u;
        asm volatile("" ::: "memory");
        sc[soff >> 4] = (unsigned char)(ovf ? 255u : len);
        if (!ovf) {
            unsigned o = end - len;
#pragma unroll
            for (int c = 0; c < 4; ++c) {
                LAS unsigned* w = (LAS unsigned*)(sc + 64) + (o >> 5);
                const u64 a = st[c] << (o & 31u), b = (st[c] >> 32) << (o & 31u);
                __hip_atomic_fetch_or(w, (unsigned)a, __ATOMIC_RELAXED, __HIP_MEMORY_SCOPE_WAVEFRONT);
                __hip_atomic_fetch_or(w + 1, (unsigned)(a >> 32), __ATOMIC_RELAXED, __HIP_MEMORY_SCOPE_WAVEFRONT);
                __hip_atomic_fetch_or(w + 2, (unsigned)(b >> 32), __ATOMIC_RELAXED, __HIP_MEMORY_SCOPE_WAVEFRONT);
                o += ln[c];
            }
        }
        asm volatile("s_waitcnt lgkmcnt(0)" ::: "memory");
        const u32x4 sd = *(volatile LAS u32x4*)(sc + soff);
        STORE16(soff, sd, pk + 4 * PK_CHUNK, 63);
        meta = b1 | (((min((unsigned)__builtin_amdgcn_readlane(end, 63), 7679u) + 512u) >> 9) << 8);
        if (ovf) {
            gchar* rbase = myws + OFF_RAW + (size_t)(bid * 8 + wid) * RAW_WAVE;
#pragma unroll
            for (int c = 0; c < 4; ++c)
#pragma unroll
                for (int i = 0; i < 8; ++i) {
                    const u32x4 v = {U32(4 * i), U32(4 * i + 1), U32(4 * i + 2), U32(4 * i + 3)};
                    STORE16(soff, v, rbase + (c * 8 + i) * 1024, 63);
                }
        }
        asm volatile("s_waitcnt vmcnt(0)" ::: "memory");
        if ((tz & 63) == 0)
            __hip_atomic_store((gu32*)((gchar*)p.ws[dest] + OFF_OUTFLAG + (bid * 8 + wid) * 64), (call << 12) | meta, __ATOMIC_RELAXED, __HIP_MEMORY_SCOPE_SYSTEM);
    }
#pragma unroll
    for (int i = 0; i < 32; ++i) acc[i >> 4][(i >> 3) & 1][(i >> 1) & 3][i & 1] = (f32x4){0.f, 0.f, 0.f, 0.f};
    }
    if (early) DIAG();
    gchar* obase = (gchar*)p.out + torg + (size_t)src * NLOC * 4;
    if (dd != 0) {
        const unsigned f = __builtin_amdgcn_readfirstlane(poll_ge((gu32*)(myws + OFF_OUTFLAG + (bid * 8 + wid) * 64), call << 12, tmo, call));
        const bool bad = __builtin_amdgcn_readfirstlane(__hip_atomic_load(tmo, __ATOMIC_RELAXED, __HIP_MEMORY_SCOPE_AGENT)) == call;
        const gchar* pk = (const gchar*)p.ws[bad ? rank : src] + OFF_PK + (size_t)(bid * 8 + wid) * PK_WAVE;
        const gchar* rbase = (const gchar*)p.ws[bad ? rank : src] + OFF_RAW + (size_t)(bid * 8 + wid) * RAW_WAVE;
        u32x4 side;
        asm volatile("s_nop 4\n\tglobal_load_dwordx4 %0, %1, %2 sc0 sc1\n\ts_waitcnt vmcnt(0)" : "=&v"(side) : "v"((soff >> 6) <= ((f >> 8) & 15u) ? soff : 0u), "s"(pk + 4 * PK_CHUNK) : "memory");
        *(volatile LAS u32x4*)(sc + soff) = side;
        const unsigned lb = *(volatile LAS unsigned char*)(sc + (soff >> 4));
        const bool ovf = lb == 255u;
        unsigned ptr = wave_scan(ovf ? 0u : lb) - (ovf ? 0u : lb);
        const unsigned BB = ((f & 255u) - 1u) * 0x10001u;
        const unsigned poison = bad ? qnan : 0u;
        if (wc == 0) {
            if (grp == 0) poll_ge<16>((gu32*)(myws + OFF_PDONE + (7 - dd) * 512), (call - 1u) * NPROD + (unsigned)(xcd * 2 + wr) * NPROD / 16u + 1u, tmo, call);
            else poll_ge<8>((gu32*)(myws + OFF_GATE + ((bid - 8) * 2 + wr) * 128), call, tmo, call);
        }
#pragma unroll
        for (int c = 0; c < 4; ++c) {
            while (__builtin_amdgcn_readfirstlane(lctl[1 + wr]) + 1u < (unsigned)(c * 4 + wc)) __builtin_amdgcn_s_sleep(32);
            while (__builtin_amdgcn_readfirstlane(lctl[1 + wr]) < (unsigned)(c * 4 + wc)) __builtin_amdgcn_s_sleep(2);
            const int lag = (int)__builtin_amdgcn_readfirstlane(lctl[3 + wr]);
            u32x4 in[6]; u32x2 cv;
            const gchar* pc = pk + c * PK_CHUNK + 3072;
            asm volatile("s_nop 4\n\tglobal_load_dwordx2 %0, %1, %2 offset:3072 sc0 sc1 nt" : "=v"(cv) : "v"(soff >> 1), "s"(pc) : "memory");
#pragma unroll
            for (int i = 0; i < 6; ++i) {
                asm volatile("s_waitcnt vmcnt(%0)" :: "i"(EW + 1) : "memory");
                if (lag <= TH2) asm volatile("s_waitcnt vmcnt(%0)" :: "i"(EW) : "memory");
                if (lag <= TH) asm volatile("s_waitcnt vmcnt(%0)" :: "i"(EW - 1) : "memory");
                if (lag < -TH) asm volatile("s_waitcnt vmcnt(%0)" :: "i"(EW - 2) : "memory");
                asm volatile("s_nop 4\n\tglobal_load_dwordx4 %0, %1, %2 offset:%3 sc0 sc1 nt" : "=v"(in[i]) : "v"(soff), "s"(pc), "i"(i * 1024 - 3072) : "memory");
            }
            asm volatile("s_waitcnt vmcnt(%1)" : "+v"(cv) : "i"(EW) : "memory");
            lctl[1 + wr] = (unsigned)(c * 4 + wc) + 1u;
            if (c == 3 && (tz & 255) == 192) __hip_atomic_store((gu32*)(myws + OFF_GATE + (bid * 2 + wr) * 128), call, __ATOMIC_RELAXED, __HIP_MEMORY_SCOPE_AGENT);
            unsigned tot = 0;
            if ((tz & 63) == 0) asm volatile("s_nop 4\n\tglobal_atomic_add %0, %1, %2, %3 sc0" : "+v"(tot) : "v"(0u), "v"(1u), "s"(myws + OFF_PROG + (7 - dd) * 512) : "memory");
            __builtin_amdgcn_sched_barrier(0);
            unsigned ex[16], s0, s1;
            {
                ptr = min(ptr, 7679u);
                const volatile LAS unsigned* w = (const volatile LAS unsigned*)(sc + 64) + (ptr >> 5);
                const unsigned w0 = w[0], w1 = w[1], w2 = w[2];
                s0 = __builtin_amdgcn_alignbit(w1, w0, ptr) | 1u; s1 = __builtin_amdgcn_alignbit(w2, w1, ptr);
            }
#pragma unroll
            for (int q = 0; q < 8; ++q) {
                unsigned t = s0, M[2];
#pragma unroll
                for (int h = 0; h < 2; ++h) {
                    const unsigned C = (cv[q >> 2] >> (2 * ((2 * q + h) & 7))) & 0x00030003u;
                    const unsigned m0 = (unsigned)__builtin_ctz(t - ((C & 0xffffu) == 0u ? 1u : 0u));
                    t >>= m0 & 31u;
                    const unsigned m1 = (unsigned)__builtin_ctz(t - (C < 0x10000u ? 1u : 0u));
                    if (h == 0) t >>= m1 & 31u;
                    M[h] = m0 | (m1 << 16);
                    ex[2 * q + h] = __builtin_amdgcn_perm(0x030b0202u, 0x07010100u, pk_min(M[h], 0x00070007u)) - M[h] + C + BB;
                }
                const unsigned ms = M[0] + M[1], sm = (ms + (ms >> 16)) & 31u;
                s0 = __builtin_amdgcn_alignbit(s1, s0, sm); s1 >>= sm; ptr += sm;
                asm volatile("" : "+v"(ex[2 * q]), "+v"(ex[2 * q + 1]), "+v"(s0));
            }
            asm volatile("s_waitcnt vmcnt(0)" : "+v"(in[0]), "+v"(in[1]), "+v"(in[2]), "+v"(in[3]), "+v"(in[4]), "+v"(in[5]), "+v"(tot), "+v"(ptr) :: "memory");
            if ((tz & 63) == 0) lctl[3 + wr] = tot - (call - 1u) * 1024u + 8u - 16u * (unsigned)(grp * 16 + c * 4 + wc + 1);
#pragma unroll
            for (int g = 0; g < 4; ++g) {
                u32x4 q[2];
#pragma unroll
                for (int e = 0; e < 8; ++e) {
                    const int n = g * 8 + e;
#define CIN(n) in[(n) >> 2][(n) & 3]
                    const unsigned lo24 = n < 24 ? CIN(n < 24 ? n : 0)
                        : __builtin_amdgcn_perm(CIN(n < 24 ? 0 : 3 * (n - 24) + 2), __builtin_amdgcn_perm(CIN(n < 24 ? 0 : 3 * (n - 24) + 1), CIN(n < 24 ? 0 : 3 * (n - 24)), 0x00000703u), 0x00070100u);
                    q[e >> 2][e & 3] = __builtin_amdgcn_perm(ex[n >> 1], lo24, (n & 1) ? 0x06020100u : 0x04020100u);
                }
                if (ovf) asm volatile("s_nop 4\n\tglobal_load_dwordx4 %0, %2, %3 sc0 sc1\n\tglobal_load_dwordx4 %1, %2, %3 offset:1024 sc0 sc1\n\ts_waitcnt vmcnt(0)"
                                      : "+v"(q[0]), "+v"(q[1]) : "v"(soff + (c * 4 + g) * 2048u), "s"(rbase) : "memory");
                unsigned v[8];
#pragma unroll
                for (int e = 0; e < 8; ++e) v[e] = __builtin_amdgcn_alignbit(q[e >> 2][e & 3], q[e >> 2][e & 3], 1) | poison;
#pragma unroll
                for (int e = 0; e < 8; ++e) ACC(c, g, e) += __builtin_bit_cast(float, v[e]);
                if (early) {
                    const f32x4 w0 = {ACC(c, g, 0), ACC(c, g, 1), ACC(c, g, 2), ACC(c, g, 3)}, w1 = {ACC(c, g, 4), ACC(c, g, 5), ACC(c, g, 6), ACC(c, g, 7)};
                    STOREO(lo, w0, obase + ROWOFF(c, g, 0)); STOREO(lo, w1, obase + ROWOFF(c, g, 4));
                }
            }
        }
    }
    if (!early) {
        DIAG();
        const bool bad = __builtin_amdgcn_readfirstlane(__hip_atomic_load(tmo, __ATOMIC_RELAXED, __HIP_MEMORY_SCOPE_AGENT)) == call;
#pragma unroll
        for (int c = 0; c < 4; ++c)
#pragma unroll
            for (int g = 0; g < 4; ++g)
#pragma unroll
                for (int h = 0; h < 2; ++h) {
                    f32x4 v = {ACC(c, g, 4 * h), ACC(c, g, 4 * h + 1), ACC(c, g, 4 * h + 2), ACC(c, g, 4 * h + 3)};
                    if (bad) v = __builtin_bit_cast(f32x4, (u32x4){qnan, qnan, qnan, qnan});
                    STOREO(lo, v, obase + ROWOFF(c, g, 4 * h));
                }
    }
}

extern "C" void kernel_launch(int rank, int nranks, ncclComm_t comm,
                              hipStream_t stream,
                              void* const* d_in, const int* in_sizes, int n_in,
                              void* d_out, int out_size,
                              const int* in_dtypes, int out_dtype,
                              void* const* d_out_all,
                              void* const* d_ws_all, size_t ws_bytes) {
    if (nranks != NRANK || (unsigned)rank >= (unsigned)NRANK || ws_bytes < WS_NEED || n_in < 2 ||
        in_sizes[0] != 4096 * KSH || in_sizes[1] != KSH * NCOLS || out_size != 512 * NCOLS) {
        fprintf(stderr, "unexpected problem shape\n");
        return;
    }
    Params p;
    p.x = (const float*)d_in[0]; p.w = (const float*)d_in[1];
    p.out = (float*)d_out;
    for (int r = 0; r < NRANK; ++r) p.ws[r] = (char*)d_ws_all[r];
    p.rank = rank; p.pad = 0;
    gemm_rs_kernel<<<GRID, 512, 0, stream>>>(p);
    HIP_CHECK(hipGetLastError());
}
```

```cpp
#include <hip/hip_runtime.h>
#include <rccl/rccl.h>
#include <cstdio>

#define HIP_CHECK(expr) do { hipError_t _e = (expr); if (_e != hipSuccess) { \
    fprintf(stderr, "HIP error %s at %s:%d\n", hipGetErrorString(_e), __FILE__, __LINE__); \
    return; } } while (0)

#define LAS __attribute__((address_space(3)))
#define GAS __attribute__((address_space(1)))
typedef _Float16 half8 __attribute__((ext_vector_type(8)));
typedef _Float16 half2v __attribute__((ext_vector_type(2)));
typedef float f32x2 __attribute__((ext_vector_type(2)));
typedef float f32x4 __attribute__((ext_vector_type(4)));
typedef unsigned u32x4 __attribute__((ext_vector_type(4)));
typedef unsigned u32x2 __attribute__((ext_vector_type(2)));
typedef unsigned long long u64;
template <int N> struct IC { static constexpr int value = N; };
typedef GAS unsigned gu32;
typedef GAS char gchar;

constexpr int NRANK = 8;
constexpr int KSH = 512;
constexpr int NCOLS = 32768, NLOC = 4096;
constexpr int GRID = 256, NPROD = 32;
constexpr size_t CTRL_BYTES = 1u << 20;
constexpr size_t PANEL = 32768;
constexpr size_t KT_STRIDE = 16 * PANEL;
constexpr size_t MAT_BYTES = 64 * KT_STRIDE;
constexpr size_t PI_ITEM = 26 * 256 + 512;
constexpr size_t OFF_PI = 3 * MAT_BYTES;
constexpr size_t BUF_BYTES = 3 * MAT_BYTES + (33u << 20);
constexpr size_t PK_CHUNK = 26 * 256;
constexpr size_t PK_WAVE = 4 * PK_CHUNK + 1024;
constexpr size_t RAW_WAVE = 64 * 512;
constexpr size_t OFF_PK = CTRL_BYTES + 2 * BUF_BYTES;
constexpr size_t OFF_RAW = OFF_PK + (size_t)GRID * 8 * PK_WAVE;
constexpr size_t WS_NEED = OFF_RAW + (size_t)GRID * 8 * RAW_WAVE;
constexpr int OFF_TMO = 0, OFF_CDONE = 28672 + 768;
constexpr int OFF_QUEUE = 20480, OFF_PDONE = 24576 + 256, OFF_PROG = 24576 + 128;
constexpr int OFF_GATE = 425984;
constexpr int OFF_INFLAG = 32768;
constexpr int OFF_CFLAG = 65536;
constexpr int OFF_OUTFLAG = 131072;
constexpr int OFF_TICKET = 393216;
constexpr unsigned long long DEADLINE_TICKS = 100000000ull;
constexpr int NPUSH = 40;
constexpr int UW = 12;
constexpr int CW = 3;
constexpr int PW = 2;
constexpr int LAG = 5;
constexpr int OWNA = 18;
constexpr int XSTAG = 20;
constexpr int EW = 3;
constexpr int TH = 12, TH2 = 40;
constexpr int NITEM = 960;
constexpr unsigned QUEUE_PER_CALL = NITEM - NPUSH - 2 * NPROD + NPROD;

struct Params {
    const float* x; const float* w;
    float* out;
    char* ws[NRANK];
    int rank; int pad;
};


template <int SLEEP = 8>
__device__ __forceinline__ unsigned poll_ge(gu32* addr, unsigned need, gu32* tmo, unsigned call) {
    const unsigned long long t0 = __builtin_amdgcn_s_memrealtime();
    unsigned v;
    for (unsigned n = 1;; ++n) {
        v = __hip_atomic_load(addr, __ATOMIC_RELAXED, __HIP_MEMORY_SCOPE_SYSTEM);
        if (__all((int)(v - need) >= 0)) break;
        if ((n & 15u) == 0u) {
            const unsigned tv = __builtin_amdgcn_readfirstlane(__hip_atomic_load(tmo, __ATOMIC_RELAXED, __HIP_MEMORY_SCOPE_AGENT));
            if (tv == call || __builtin_amdgcn_s_memrealtime() - t0 > DEADLINE_TICKS) {
                __hip_atomic_store(tmo, call, __ATOMIC_RELAXED, __HIP_MEMORY_SCOPE_AGENT);
                break;
            }
        }
        __builtin_amdgcn_s_sleep(SLEEP);
    }
    asm volatile("" ::: "memory");
    return v;
}

__device__ __forceinline__ unsigned pk16(float a, float b) {
    f32x2 v = {a, b};
    return __builtin_bit_cast(unsigned, __builtin_convertvector(v, half2v));
}

typedef unsigned short u16x2 __attribute__((ext_vector_type(2)));
__device__ __forceinline__ unsigned pk_max(unsigned a, unsigned b) { return __builtin_bit_cast(unsigned, __builtin_elementwise_max(__builtin_bit_cast(u16x2, a), __builtin_bit_cast(u16x2, b))); }
__device__ __forceinline__ unsigned wave_scan(unsigned x) {
    x += (unsigned)__builtin_amdgcn_update_dpp(0, (int)x, 0x111, 0xf, 0xf, false);
    x += (unsigned)__builtin_amdgcn_update_dpp(0, (int)x, 0x112, 0xf, 0xf, false);
    x += (unsigned)__builtin_amdgcn_update_dpp(0, (int)x, 0x114, 0xf, 0xf, false);
    x += (unsigned)__builtin_amdgcn_update_dpp(0, (int)x, 0x118, 0xf, 0xf, false);
    x += (unsigned)__builtin_amdgcn_update_dpp(0, (int)x, 0x142, 0xa, 0xf, false);
    x += (unsigned)__builtin_amdgcn_update_dpp(0, (int)x, 0x143, 0xc, 0xf, false);
    return x;
}

__device__ __forceinline__ unsigned pk_min(unsigned a, unsigned b) { return __builtin_bit_cast(unsigned, __builtin_elementwise_min(__builtin_bit_cast(u16x2, a), __builtin_bit_cast(u16x2, b))); }
__device__ __forceinline__ unsigned pk_subs(unsigned a, unsigned b) { return __builtin_bit_cast(unsigned, __builtin_elementwise_sub_sat(__builtin_bit_cast(u16x2, a), __builtin_bit_cast(u16x2, b))); }

__device__ __forceinline__ unsigned pack_item(unsigned (&D)[32], unsigned (&P)[26], u32x2& sd, LAS unsigned char* sc, int lane) {
    unsigned se = 0;
#pragma unroll
    for (int i = 0; i < 32; ++i) se += (D[i] >> 10) & 0x001f001fu;
    const unsigned S = __builtin_amdgcn_readlane(wave_scan((se & 0xffffu) + (se >> 16)), 63);
    const unsigned b1 = min((S + 246u) >> 12, 29u), B = (b1 + 27u) * 0x10001u;
    *(LAS u32x2*)(sc + lane * 8) = (u32x2){0u, 0u};
    if (lane < 8) *(LAS u32x2*)(sc + 512 + lane * 8) = (u32x2){0u, 0u};
    u64 lo = 1; unsigned hi = 0, len = 0, worst = 0, up = 0;
    bool ovf = false;
#pragma unroll
    for (int q = 15; q >= 0; --q) {
        unsigned W = 1, M[2];
#pragma unroll
        for (int h = 1; h >= 0; --h) {
            const int i = 2 * q + h;
            const unsigned E = ((D[i] >> 10) & 0x001f001fu) | 0x00200020u, dd = pk_subs(E, B), ix = pk_min(dd, 0x00070007u) | 0x0c000c00u;
            const unsigned a3 = pk_subs(dd, 0x00070007u) * 3u;
            up = pk_max(up, a3);
            M[h] = __builtin_amdgcn_perm(0x00000001u, 0x02040507u, ix) + pk_subs(B, E) + a3;
            D[i] = (D[i] & 0x83ff83ffu) | (pk_subs(__builtin_amdgcn_perm(0x03020100u, 0u, ix), a3) << 10);
            W = (W << ((M[h] >> 16) & 31u)) | 1u;
            W = (W << (M[h] & 31u)) | 1u;
        }
        const unsigned ms = M[0] + M[1], sm = (ms & 0xffffu) + (ms >> 16);
        worst = max(worst, sm); len += sm;
        if (q < 8) hi = (unsigned)(((((u64)hi << 32) | (lo >> 32)) << (sm & 31u)) >> 32);
        lo = (lo << (sm & 31u)) | W;
        if (q == 8) ovf = len > 63u;
        asm volatile("" : "+v"(lo), "+v"(hi));
    }
    ovf |= len > 95u || worst > 31u || pk_subs(up, 0x00060006u) != 0u;
    const unsigned end = wave_scan(ovf ? 0u : len), o = end - (ovf ? 0u : len);
    ovf |= end > 3583u;
    asm volatile("" ::: "memory");
    sc[lane] = (unsigned char)(ovf ? 255u : len);
    if (!ovf) {
        LAS unsigned* w = (LAS unsigned*)(sc + 64) + (o >> 5);
        const u64 a = lo << (o & 31u), b = (((u64)hi << 32) | (lo >> 32)) << (o & 31u), c = (u64)hi << (o & 31u);
        __hip_atomic_fetch_or(w, (unsigned)a, __ATOMIC_RELAXED, __HIP_MEMORY_SCOPE_WAVEFRONT);
        __hip_atomic_fetch_or(w + 1, (unsigned)(a >> 32), __ATOMIC_RELAXED, __HIP_MEMORY_SCOPE_WAVEFRONT);
        __hip_atomic_fetch_or(w + 2, (unsigned)(b >> 32), __ATOMIC_RELAXED, __HIP_MEMORY_SCOPE_WAVEFRONT);
        __hip_atomic_fetch_or(w + 3, (unsigned)(c >> 32), __ATOMIC_RELAXED, __HIP_MEMORY_SCOPE_WAVEFRONT);
    }
#pragma unroll
    for (int g = 0; g < 2; ++g) {
#pragma unroll
        for (int i = 0; i < 12; ++i) P[13 * g + i] = D[16 * g + i] | ((D[16 * g + 13 + i / 4] << (12 - 3 * (i & 3))) & 0x70007000u);
        P[13 * g + 12] = D[16 * g + 12] | ((D[16 * g + 13] >> 3) & 0x10001000u) | ((D[16 * g + 14] >> 2) & 0x20002000u) | ((D[16 * g + 15] >> 1) & 0x40004000u);
    }
    asm volatile("s_waitcnt lgkmcnt(0)" ::: "memory");
    sd = *(volatile LAS u32x2*)(sc + lane * 8);
    const unsigned tot = min((unsigned)__builtin_amdgcn_readlane(end, 63), 3583u);
    return b1 | (((tot + 512u) >> 9) << 5);
}
template <int Q>
__device__ __forceinline__ void unpack8(unsigned (&H)[8], unsigned BB, unsigned& s0, unsigned& s1, unsigned& s2) {
#pragma unroll
    for (int q = 0; q < 4; ++q) {
        unsigned t = s0, M[2];
#pragma unroll
        for (int h = 0; h < 2; ++h) {
            const unsigned T = H[2 * q + h];
            const half2v f = __builtin_bit_cast(half2v, T);
            const unsigned m0 = (unsigned)__builtin_ctz(t - (__builtin_amdgcn_classh(f[0], 0xf0) ? 1u : 0u));
            t >>= m0 & 31u;
            const unsigned m1 = (unsigned)__builtin_ctz(t - (__builtin_amdgcn_classh(f[1], 0xf0) ? 1u : 0u));
            if (h == 0) t >>= m1 & 31u;
            M[h] = m0 | (m1 << 16);
            H[2 * q + h] = ((__builtin_amdgcn_perm(0x030b0202u, 0x07010100u, pk_min(M[h], 0x00070007u)) - M[h]) << 10) + (T + BB);
        }
        const unsigned ms = M[0] + M[1], sm = ms + (ms >> 16);
        s0 = __builtin_amdgcn_alignbit(s1, s0, sm);
        if (Q < 2) { s1 = __builtin_amdgcn_alignbit(s2, s1, sm); s2 >>= sm & 31u; } else s1 >>= sm & 31u;
        asm volatile("" : "+v"(H[2 * q]), "+v"(H[2 * q + 1]));
    }
}
constexpr int pull_younger(int k) {
    int n = 0;
    for (int j = k - PW; j < k; ++j) {
        const int jj = (j + 7) % 7;
        if (j > k - PW) n += (j + PW) % 7 == 0 ? 2 : 1;
        if (jj == 1 || jj == 3 || jj == 5 || jj == 6) n += 2;
    }
    return n;
}

__device__ __forceinline__ void item_decode(int it, int src, int& s, int& r) {
    int e;
    constexpr int n1 = 72 * LAG, n2 = n1 + 88 * (8 - LAG);
    if (it < n1) { s = it / 72; e = it - 72 * s; }
    else if (it < n2) { s = (it - n1) / 88; e = it - n1 - 88 * s; s += e < 72 ? LAG : 0; }
    else { const int v = it - 576; s = (v >> 4) & 7; e = 72 + (v >> 7) * 16 + (v & 15); }
    r = e < 56 ? (8 * src + 8 + e) & 63 : e + 8;
}

#define UNIT_LOAD(B, S_, R_, H_) do { const bool isx_ = (R_) < 64; \
        const gchar* b_ = isx_ ? sX + (size_t)((R_) * 64 + (H_) * 32) * (KSH * 4) + (size_t)(S_) * 256 \
                               : sW + (size_t)((S_) * 64 + (H_) * 32) * (NCOLS * 4) + (size_t)((R_) - 64) * 256; \
        const unsigned vo_ = isx_ ? xoff : woff; \
        const gchar* b1_ = b_ + (isx_ ? 16 : 1 * NCOLS * 4);          const gchar* b2_ = b_ + (isx_ ? 128 : 2 * NCOLS * 4); \
        const gchar* b3_ = b_ + (isx_ ? 144 : 3 * NCOLS * 4);         const gchar* b4_ = b_ + (isx_ ? 32768 : 4 * NCOLS * 4); \
        const gchar* b5_ = b_ + (isx_ ? 32768 + 16 : 5 * NCOLS * 4);  const gchar* b6_ = b_ + (isx_ ? 32768 + 128 : 6 * NCOLS * 4); \
        const gchar* b7_ = b_ + (isx_ ? 32768 + 144 : 7 * NCOLS * 4); \
        asm volatile("s_nop 4\n\tglobal_load_dwordx4 %0, %8, %9 nt\n\tglobal_load_dwordx4 %1, %8, %10 nt\n\t" \
                     "global_load_dwordx4 %2, %8, %11 nt\n\tglobal_load_dwordx4 %3, %8, %12 nt\n\t" \
                     "global_load_dwordx4 %4, %8, %13 nt\n\tglobal_load_dwordx4 %5, %8, %14 nt\n\t" \
                     "global_load_dwordx4 %6, %8, %15 nt\n\tglobal_load_dwordx4 %7, %8, %16 nt" \
                     : "+v"(B[0]), "+v"(B[1]), "+v"(B[2]), "+v"(B[3]), "+v"(B[4]), "+v"(B[5]), "+v"(B[6]), "+v"(B[7]) \
                     : "v"(vo_), "s"(b_), "s"(b1_), "s"(b2_), "s"(b3_), "s"(b4_), "s"(b5_), "s"(b6_), "s"(b7_) : "memory"); } while (0)
#define UNIT_WAIT(B, N) asm volatile("s_waitcnt vmcnt(%8)" : "+v"(B[0]), "+v"(B[1]), "+v"(B[2]), "+v"(B[3]), \
                                     "+v"(B[4]), "+v"(B[5]), "+v"(B[6]), "+v"(B[7]) : "i"(N) : "memory")
#define UNIT_CVT(O, B, R_) do { if ((R_) < 64) { \
            _Pragma("unroll") for (int b = 0; b < 4; ++b) { \
                const f32x4 lo_ = __builtin_bit_cast(f32x4, B[2 * b]), hi_ = __builtin_bit_cast(f32x4, B[2 * b + 1]); \
                O[b] = (u32x4){pk16(lo_[0], lo_[1]), pk16(lo_[2], lo_[3]), pk16(hi_[0], hi_[1]), pk16(hi_[2], hi_[3])}; } \
        } else { \
            _Pragma("unroll") for (int j = 0; j < 4; ++j) \
                O[j] = (u32x4){pk16(__builtin_bit_cast(f32x4, B[0])[j], __builtin_bit_cast(f32x4, B[1])[j]), \
                               pk16(__builtin_bit_cast(f32x4, B[2])[j], __builtin_bit_cast(f32x4, B[3])[j]), \
                               pk16(__builtin_bit_cast(f32x4, B[4])[j], __builtin_bit_cast(f32x4, B[5])[j]), \
                               pk16(__builtin_bit_cast(f32x4, B[6])[j], __builtin_bit_cast(f32x4, B[7])[j])}; } \
        asm volatile("" : "+v"(O[0]), "+v"(O[1]), "+v"(O[2]), "+v"(O[3])); } while (0)
#define ITEM_OFF(R_) ((R_) < 64 ? (size_t)(R_) * 8192 : (size_t)(((R_) - 64) >> 2) * PANEL + (size_t)((R_) & 3) * 4096)
#define UNIT_DST(BASE, R_, H_) const bool isxd_ = (R_) < 64; \
        gchar* t0_ = (BASE) + (isxd_ ? (H_) * 4096 : (H_) * 1024); \
        gchar* t1_ = t0_ + (isxd_ ? 1024 : 2048); gchar* t2_ = t0_ + (isxd_ ? 2048 : 16384); gchar* t3_ = t0_ + (isxd_ ? 3072 : 16384 + 2048)
#define STOREO(VOFF, DATA, BASE) asm volatile("s_nop 4\n\tglobal_store_dwordx4 %0, %1, %2 sc0 sc1 nt\n\ts_nop 1" :: "v"(VOFF), "v"(DATA), "s"(BASE) : "memory")
#define STORE16(VOFF, DATA, BASE, N) asm volatile("s_nop 3\n\ts_waitcnt vmcnt(%3)\n\tglobal_store_dwordx4 %0, %1, %2 sc0 sc1\n\ts_nop 1" \
                                                  :: "v"(VOFF), "v"(DATA), "s"(BASE), "i"(N) : "memory")

__global__ __launch_bounds__(512, 2) void gemm_rs_kernel(const Params p) {
    __shared__ __attribute__((aligned(16))) unsigned char shm[131072 + 256 + 8 * 1088];
    LAS unsigned char* lds = (LAS unsigned char*)shm;
    volatile LAS unsigned* lctl = (volatile LAS unsigned*)(lds + 131072);
    const int tid = threadIdx.x, wid = __builtin_amdgcn_readfirstlane(tid >> 6), lane = tid & 63;
    const int rank = p.rank & (NRANK - 1), bid = blockIdx.x;
    gchar* myws = (gchar*)p.ws[rank];
    gu32* tmo = (gu32*)(myws + OFF_TMO);

    if (tid == 0) {
        lctl[0] = __hip_atomic_fetch_add((gu32*)(myws + OFF_TICKET + bid * 128), 1u, __ATOMIC_RELAXED, __HIP_MEMORY_SCOPE_AGENT) + 1u;
    } else if (tid < 64) lctl[tid] = 0u;
    unsigned call; size_t bufoff;
#define GET_CALL() do { asm volatile("s_waitcnt lgkmcnt(0)" ::: "memory"); __builtin_amdgcn_s_barrier(); asm volatile("" ::: "memory"); \
        call = __builtin_amdgcn_readfirstlane(lctl[0]); bufoff = CTRL_BYTES + (size_t)(call & 1u) * BUF_BYTES; } while (0)

    const int xcd = bid & 7, idx = bid >> 3, dd = idx >> 2;
    const int dest = (rank + dd) & (NRANK - 1);
    const int mt = 2 * dest + (xcd & 1), grp = idx & 3, nt = 4 * grp + (xcd >> 1);
    const int hidx = (dd - 1) * 16 + (bid & 15);

    const unsigned soff0 = (unsigned)lane * 16u;
    const gchar* sX = (const gchar*)p.x;
    const unsigned xoff = (unsigned)(lane & 15) * (KSH * 4) + (unsigned)(lane >> 4) * 32u;
    const unsigned woff = (unsigned)(lane >> 4) * 8u * (NCOLS * 4) + (unsigned)(lane & 15) * 16u;
    if (bid >= NPROD && nt >= 8 && wid >= 1 && wid <= CW) {
        u32x4 buf0[8], buf1[8], o[4], o1[4];
#pragma unroll
        for (int b = 0; b < 8; ++b) { buf0[b] = (u32x4){0u, 0u, 0u, 0u}; buf1[b] = (u32x4){0u, 0u, 0u, 0u}; }
        bool first = true;
        static_assert(CW == 3, "3 x 336 runs of four");
        const int me = (wid - 1) * 112 + hidx;
        u32x4 fv = {0u, 0u, 0u, 0u};
        for (int n = 0; n < 14; ++n) {
            int s, e, sub = 0;
            if (n < 2) { const int job = NPUSH + me + 336 * n; if (job >= 576) continue; s = job / 192; e = job - 192 * s; }
            else {
                const int q = 335 - me + 336 * ((n - 2) >> 2); int v; sub = (n - 2) & 3;
                if (q < 240) { s = q / 48; v = q - 48 * s; s += 3; } else { s = ((q - 240) >> 5) & 7; v = 48 + ((q - 240) >> 8) * 32 + ((q - 240) & 31); }
                e = v < 16 ? 4 * v + sub : 64 + (((v - 16) >> 3) * 4 + sub) * 8 + (v & 7);
            }
            const int d = e < 64 ? rank : (rank + 1 + ((e - 64) & 7)) & (NRANK - 1);
            const int r = e < 64 ? (8 * rank + 8 + e) & 63 : 64 + ((e - 64) >> 3);
            const gchar* sW = (const gchar*)p.w + (size_t)d * NLOC * 4;
            UNIT_LOAD(buf0, s, r, 0); UNIT_LOAD(buf1, s, r, 1);
            if (first) { GET_CALL(); first = false; }
            gchar* tb = myws + bufoff + ITEM_OFF(r) + (e < 64 ? (size_t)(s * 8 + rank) * KT_STRIDE
                        : d == rank ? MAT_BYTES + (size_t)(s * 8 + rank) * KT_STRIDE : 2 * MAT_BYTES + (size_t)(d * 8 + s) * KT_STRIDE);
            UNIT_WAIT(buf0, 8); UNIT_CVT(o, buf0, r);
            { UNIT_DST(tb, r, 0); STORE16(soff0, o[0], t0_, 63); STORE16(soff0, o[1], t1_, 63); STORE16(soff0, o[2], t2_, 63); STORE16(soff0, o[3], t3_, 63); }
            UNIT_WAIT(buf1, 4); UNIT_CVT(o1, buf1, r);
            { UNIT_DST(tb, r, 1); STORE16(soff0, o1[0], t0_, 63); STORE16(soff0, o1[1], t1_, 63); STORE16(soff0, o1[2], t2_, 63); STORE16(soff0, o1[3], t3_, 63); }
            unsigned meta = 0;
            if (e < 56 || d != rank) {
                unsigned D[32], P[26]; u32x2 sd;
#pragma unroll
                for (int i = 0; i < 16; ++i) { D[i] = o[i >> 2][i & 3]; D[16 + i] = o1[i >> 2][i & 3]; }
                meta = pack_item(D, P, sd, lds + wid * 1024, lane);
                gchar* pi = myws + bufoff + OFF_PI + (size_t)(e < 64 ? s * 64 + r : 512 + (d * 8 + s) * 64 + r - 64) * PI_ITEM;
#pragma unroll
                for (int i = 0; i < 6; ++i) { const u32x4 v = {P[4 * i], P[4 * i + 1], P[4 * i + 2], P[4 * i + 3]}; STORE16(soff0, v, pi + i * 1024, 63); }
                const u32x2 v2 = {P[24], P[25]};
                asm volatile("s_nop 4\n\tglobal_store_dwordx2 %0, %1, %3 sc0 sc1\n\tglobal_store_dwordx2 %0, %2, %3 offset:512 sc0 sc1"
                             :: "v"(soff0 >> 1), "v"(v2), "v"(sd), "s"(pi + 6144) : "memory");
            }
            fv = (u32x4){fv[1], fv[2], fv[3], (call << 8) | meta};
            if (n >= 2 && sub != 3) continue;
            asm volatile("s_waitcnt vmcnt(0)" ::: "memory");
            const unsigned fo = (unsigned)(rank * 4096 + (s * 128 + r - sub) * 4);
#pragma unroll
            for (int k = 0; k < NRANK; ++k)
                if ((e < 56 || k == d) && lane == 0) {
                    gchar* fb = (gchar*)p.ws[k] + (k == rank ? OFF_INFLAG : OFF_CFLAG + (call & 1u) * 32768u);
                    if (n < 2) __hip_atomic_store((gu32*)(fb + fo), k == rank ? call : fv[3], __ATOMIC_RELAXED, __HIP_MEMORY_SCOPE_SYSTEM);
                    else { const u32x4 fw = k == rank ? (u32x4){call, call, call, call} : fv; STORE16(fo, fw, fb, 63); }
                }
        }
        if (lane == 0) __hip_atomic_fetch_add((gu32*)(myws + OFF_CDONE), 1u, __ATOMIC_RELAXED, __HIP_MEMORY_SCOPE_AGENT);
    } else if (bid >= NPROD && nt < 8 && wid == 1 && hidx < NPUSH) {
        u32x4 buf0[8], buf1[8], o0[4], o1[4];
#pragma unroll
        for (int b = 0; b < 8; ++b) { buf0[b] = (u32x4){0u, 0u, 0u, 0u}; buf1[b] = (u32x4){0u, 0u, 0u, 0u}; }
        const gchar* sW = sX;
        const int r = (8 * rank + 8 + hidx) & 63;
        UNIT_LOAD(buf0, 0, r, 0); UNIT_LOAD(buf1, 0, r, 1);
        GET_CALL();
        UNIT_WAIT(buf0, 8); UNIT_CVT(o0, buf0, 0);
        UNIT_WAIT(buf1, 0); UNIT_CVT(o1, buf1, 0);
        const size_t ioff = bufoff + (size_t)rank * KT_STRIDE + (size_t)r * 8192;
#pragma unroll
        for (int b = 0; b < 8; ++b)
            for (int k = 1; k <= NRANK; ++k) {
                gchar* t = (gchar*)p.ws[(rank + hidx + k) & (NRANK - 1)] + ioff + b * 1024;
                STORE16(soff0, b < 4 ? o0[b & 3] : o1[b & 3], t, UW - 1);
            }
        asm volatile("s_waitcnt vmcnt(0)" ::: "memory");
#pragma unroll
        for (int k = 0; k < NRANK; ++k)
            if (lane == 0) __hip_atomic_store((gu32*)((gchar*)p.ws[k] + OFF_INFLAG + rank * 4096 + r * 4), call, __ATOMIC_RELAXED, __HIP_MEMORY_SCOPE_SYSTEM);
    } else if (bid >= NPROD) {
        GET_CALL();
    } else if (wid == 0) {
        GET_CALL();
        const int j = lane & 7, lk = (j + bid) % 7, psrc = (rank + 1 + lk) & (NRANK - 1);
        const unsigned long long t0 = __builtin_amdgcn_s_memrealtime();
        unsigned n = 0; int item = 0; bool have = false, fin = lane >= 7, late = false;
        for (unsigned it = 1; !__all(fin); ++it) {
            if (!fin && !have && n <= lctl[8 + j]) {
                unsigned t = NPUSH + bid + n * NPROD;
                if (n >= 2) t = __hip_atomic_fetch_add((gu32*)(myws + OFF_QUEUE + lk * 512), 1u, __ATOMIC_RELAXED, __HIP_MEMORY_SCOPE_AGENT)
                                - (call - 1u) * QUEUE_PER_CALL + NPUSH + 2u * NPROD;
                int s, r; item_decode(t < (unsigned)NITEM ? (int)t : 0, psrc, s, r);
                item = t < (unsigned)NITEM ? s * 128 + r : 1024; have = true;
            }
            if (have) {
                bool ready = item >= 1024 || late; unsigned cf = 0;
                if (!ready) { cf = __hip_atomic_load((gu32*)(myws + OFF_CFLAG + (call & 1u) * 32768u + psrc * 4096 + item * 4), __ATOMIC_RELAXED, __HIP_MEMORY_SCOPE_SYSTEM); ready = (int)(cf - (call << 8)) >= 0; }
                if (ready) { ++n; __hip_atomic_store((LAS unsigned*)lctl + 32 + j * 4 + (n & 3u), (n << 20) | ((cf & 255u) << 12) | (unsigned)item, __ATOMIC_RELEASE, __HIP_MEMORY_SCOPE_WORKGROUP); have = false; fin = item >= 1024; }
            }
            if ((it & 63u) == 0u && !late) {
                late = __builtin_amdgcn_readfirstlane(__hip_atomic_load(tmo, __ATOMIC_RELAXED, __HIP_MEMORY_SCOPE_AGENT)) == call
                       || __builtin_amdgcn_s_memrealtime() - t0 > DEADLINE_TICKS;
                if (late) __hip_atomic_store(tmo, call, __ATOMIC_RELAXED, __HIP_MEMORY_SCOPE_AGENT);
            }
            __builtin_amdgcn_s_sleep(2);
        }
    } else {
        GET_CALL();
        const int lk = (wid - 1 + bid) % 7, src = (rank + 1 + lk) & (NRANK - 1);
        const gchar* rimg = (const gchar*)p.ws[src] + bufoff;
        gchar* limg = myws + bufoff + (size_t)src * KT_STRIDE;
        gu32* lflag = (gu32*)(myws + OFF_INFLAG + src * 4096);
        u32x4 in[6]; u32x2 in6 = {0u, 0u}, side = {0u, 0u};
#pragma unroll
        for (int b = 0; b < 6; ++b) in[b] = (u32x4){0u, 0u, 0u, 0u};
        const unsigned soff8 = (unsigned)lane * 8u;
        LAS unsigned char* sc = lds + wid * 1024;
#define PI_OF(S_, R_) (rimg + OFF_PI + 3072 + (size_t)((R_) < 64 ? (S_) * 64 + (R_) : 512 + (rank * 8 + (S_)) * 64 + (R_) - 64) * PI_ITEM)
#define RAW_OF(S_, R_) (rimg + ITEM_OFF(R_) + ((R_) < 64 ? (size_t)((S_) * 8 + src) * KT_STRIDE : 2 * MAT_BYTES + (size_t)(rank * 8 + (S_)) * KT_STRIDE))
#define DST_OF(S_, R_) (limg + ITEM_OFF(R_) + ((R_) < 64 ? 0 : MAT_BYTES) + (size_t)(S_) * 8 * KT_STRIDE)
#define PULL(k, BASE, META) do { \
        if ((k) == 0) asm volatile("s_nop 4\n\tglobal_load_dwordx2 %0, %1, %2 offset:3584 sc0 sc1" : "+v"(side) : "v"((soff8 >> 6) <= ((META) >> 5) ? soff8 : 0u), "s"(BASE) : "memory"); \
        if ((k) < 6) asm volatile("s_nop 4\n\tglobal_load_dwordx4 %0, %1, %2 offset:%3 sc0 sc1" : "+v"(in[(k) < 6 ? (k) : 0]) : "v"(soff0), "s"(BASE), "i"((k) * 1024 - 3072) : "memory"); \
        else asm volatile("s_nop 4\n\tglobal_load_dwordx2 %0, %1, %2 offset:3072 sc0 sc1" : "+v"(in6) : "v"(soff8), "s"(BASE) : "memory"); } while (0)
#define TAKE(DST, META, N_) do { unsigned e_; while (((e_ = __builtin_amdgcn_readfirstlane(__hip_atomic_load((LAS unsigned*)lctl + 28 + wid * 4 + ((N_) & 3), __ATOMIC_ACQUIRE, __HIP_MEMORY_SCOPE_WORKGROUP))) >> 20) != (unsigned)(N_)) __builtin_amdgcn_s_sleep(1); \
        lctl[7 + wid] = (unsigned)(N_); asm volatile("" ::: "memory"); DST = (int)(e_ & 0x7ffu); META = (e_ >> 12) & 255u; } while (0)
        int cur, s, r, pflag = -1; unsigned meta;
        TAKE(cur, meta, 1);
        s = cur >> 7; r = cur & 127;
        const gchar* sb = PI_OF(s, r);
#pragma unroll
        for (int k = 0; k < PW; ++k) PULL(k, sb, meta);
        for (int i = 0; cur < 1024; ++i) {
            int nxt = 1024, sn = 0, rn = 0; const gchar* sbn = sb; unsigned metan = 0;
            gchar* db = DST_OF(s, r);
            const bool isx = r < 64;
            const gchar* rraw = RAW_OF(s, r);
            asm volatile("" : "+s"(rraw), "+s"(db));
            const unsigned BB = ((meta & 31u) - 1u) * 0x04000400u;
            unsigned s0 = 0, s1 = 0, s2 = 0, dn[3] = {0u, 0u, 0u}; bool ovf = false;
            auto step = [&](auto KC) __attribute__((always_inline)) {
                constexpr int k = decltype(KC)::value;
                (void)&rraw; (void)&db; (void)&sb; (void)&soff0; (void)&soff8; (void)&in; (void)&in6; (void)&side;
                if (k + PW == 7) {
                    TAKE(nxt, metan, i + 2);
                    sn = (nxt >> 7) & 7; rn = nxt & 127;
                    sbn = nxt < 1024 ? PI_OF(sn, rn) : (const gchar*)myws + bufoff + OFF_PI + 3072;
                    if (nxt >= 1024 && lane == 0) __hip_atomic_fetch_add((gu32*)(myws + OFF_PDONE + lk * 512), 1u, __ATOMIC_RELAXED, __HIP_MEMORY_SCOPE_AGENT);
                }
                if (k < PW && i == 0) asm volatile("s_waitcnt vmcnt(%0)" :: "i"(PW - 1) : "memory");
                if (k < 6) asm volatile("s_waitcnt vmcnt(%2)" : "+v"(in[k < 6 ? k : 0]), "+v"(side) : "i"(pull_younger(k)) : "memory");
                else asm volatile("s_waitcnt vmcnt(%1)" : "+v"(in6) : "i"(pull_younger(k)) : "memory");
                if (k == PW && pflag >= 0 && lane == 0) __hip_atomic_store(lflag + pflag, call, __ATOMIC_RELAXED, __HIP_MEMORY_SCOPE_SYSTEM);
                if (k + PW < 7) PULL((k + PW) % 7, sb, meta); else PULL((k + PW) % 7, sbn, metan);
                __builtin_amdgcn_sched_barrier(0);
                if (k == 0) {
                    *(volatile LAS u32x2*)(sc + lane * 8) = side;
                    const unsigned lb = *(volatile LAS unsigned char*)(sc + lane);
                    ovf = lb == 255u;
                    const unsigned len = ovf ? 0u : lb, o = min(wave_scan(len) - len, 3583u);
                    const volatile LAS unsigned* w = (const volatile LAS unsigned*)(sc + 64) + (o >> 5);
                    const unsigned w0 = w[0], w1 = w[1], w2 = w[2], w3 = w[3];
                    s0 = __builtin_amdgcn_alignbit(w1, w0, o) | 1u; s1 = __builtin_amdgcn_alignbit(w2, w1, o); s2 = __builtin_amdgcn_alignbit(w3, w2, o);
                    asm volatile("" : "+v"(s0), "+v"(s1), "+v"(s2));
                }
                if (k == 1 || k == 3 || k == 5 || k == 6) {
                    constexpr int g = k >> 1;
#define PIN(n) ((n) < 24 ? in[((n) < 24 ? (n) : 0) >> 2][(n) & 3] : in6[(n) & 1])
                    unsigned H[8];
#pragma unroll
                    for (int e = 0; e < ((g & 1) ? 5 : 8); ++e) {
                        const int ci = (g & 1) * 8 + e; const unsigned pw = PIN(13 * (g >> 1) + ci);
                        H[e] = pw & 0x8fff8fffu;
                        if (ci < 12) { const unsigned pc = (pw >> (12 - 3 * (ci & 3))) & (0x00070007u << (3 * (ci & 3))); dn[ci >> 2] = (ci & 3) ? dn[ci >> 2] | pc : pc; }
                        else { H[5] = dn[0] | ((pw << 3) & 0x80008000u); H[6] = dn[1] | ((pw << 2) & 0x80008000u); H[7] = dn[2] | ((pw << 1) & 0x80008000u); }
                    }
                    unpack8<g>(H, BB, s0, s1, s2);
                    u32x4 v0 = {H[0], H[1], H[2], H[3]}, v1 = {H[4], H[5], H[6], H[7]};
                    const unsigned o0 = soff0 + (isx ? g * 2048 : (g & 1) * 16384 + (g >> 1) * 1024), o1 = o0 + (isx ? 1024 : 2048);
                    if (ovf) asm volatile("s_nop 4\n\tglobal_load_dwordx4 %0, %2, %4 sc0 sc1\n\tglobal_load_dwordx4 %1, %3, %4 sc0 sc1\n\ts_waitcnt vmcnt(0)"
                                          : "+v"(v0), "+v"(v1) : "v"(o0), "v"(o1), "s"(rraw) : "memory");
                    STORE16(o0, v0, db, 63); STORE16(o1, v1, db, 63);
                }
            };
            step(IC<0>{}); step(IC<1>{}); step(IC<2>{}); step(IC<3>{}); step(IC<4>{}); step(IC<5>{}); step(IC<6>{});
            pflag = cur; cur = nxt; s = sn; r = rn; sb = sbn; meta = metan;
        }
        asm volatile("s_waitcnt vmcnt(0)" : "+v"(in[0]), "+v"(in[1]), "+v"(in[2]), "+v"(in[3]), "+v"(side) :: "memory");
        if (lane == 0) __hip_atomic_store(lflag + pflag, call, __ATOMIC_RELAXED, __HIP_MEMORY_SCOPE_SYSTEM);
    }

    const int wr = wid >> 2, wc = wid & 3;
    const gchar* img = myws + bufoff;
    const unsigned voff = (unsigned)tid * 16u;
    const unsigned ldsw = (unsigned)wid * 1024u;
    const int aoff = wr * 8192 + lane * 16, boff = wc * 4096 + lane * 16;
    constexpr int HTB = 16384;
#define SA(b, h) (((b) * 2 + (h)) * HTB)
#define SB(b, h) ((4 + (b) * 2 + (h)) * HTB)
#define STAGE(bufo, gbase) do { _Pragma("unroll") for (int _i = 0; _i < 2; ++_i) \
        __builtin_amdgcn_global_load_lds((const GAS unsigned*)((gbase) + voff + _i * 8192), (LAS unsigned*)(lds + (bufo) + ldsw + _i * 8192), 16, 0, 17); } while (0)
#define LDA(dst, b, h) do { _Pragma("unroll") for (int m = 0; m < 4; ++m) _Pragma("unroll") for (int k = 0; k < 2; ++k) \
        dst[m][k] = *(const LAS half8*)(lds + SA(b, h) + aoff + m * 2048 + k * 1024); } while (0)
#define LDB(dst, b, h) do { _Pragma("unroll") for (int n = 0; n < 2; ++n) _Pragma("unroll") for (int k = 0; k < 2; ++k) \
        dst[n][k] = *(const LAS half8*)(lds + SB(b, h) + boff + n * 2048 + k * 1024); } while (0)
#define MMA(ai, bj, At_, Bt_) do { __builtin_amdgcn_s_setprio(1); \
        _Pragma("unroll") for (int m = 0; m < 4; ++m) _Pragma("unroll") for (int n = 0; n < 2; ++n) _Pragma("unroll") for (int k = 0; k < 2; ++k) \
            acc[ai][bj][m][n] = __builtin_amdgcn_mfma_f32_16x16x32_f16(At_[m][k], Bt_[n][k], acc[ai][bj][m][n], 0, 0, 0); \
        __builtin_amdgcn_s_setprio(0); } while (0)
#define WAIT_V(n) asm volatile("s_waitcnt vmcnt(" #n ")" ::: "memory")
#define WAIT_L(n) asm volatile("s_waitcnt lgkmcnt(" #n ")" ::: "memory")
#define BAR __builtin_amdgcn_s_barrier()
#define SCHED __builtin_amdgcn_sched_barrier(0)
    bool allin = false;
#define WAIT_SEG(c) do { if (wid == 0 && !allin) { int lz_ = lane; asm volatile("" : "+v"(lz_)); const int wh_ = lz_ >> 3; \
        const int it_ = wh_ < 4 && ((c) == 8 || (lz_ & 7) != dest) ? 4 * mt + wh_ : 64 + 4 * nt + (wh_ & 3); \
        gu32* fa_ = (gu32*)(myws + OFF_INFLAG + ((c) == 8 ? rank * 4096 + (lz_ & 7) * 512 : (lz_ & 7) * 4096 + (c) * 512) + it_ * 4); \
        bool now_ = false; \
        if ((c) < 8 && (dd == 0 || (c) > (grp == 0 ? 8 : grp == 1 ? 8 - LAG : 0))) { unsigned v_[9]; \
            v_[8] = __hip_atomic_load((gu32*)(myws + OFF_CDONE), __ATOMIC_RELAXED, __HIP_MEMORY_SCOPE_SYSTEM); \
            _Pragma("unroll") for (int k_ = 0; k_ < 8; ++k_) v_[k_] = __hip_atomic_load(fa_ + ((c) + k_ < 8 ? k_ : 0) * 128, __ATOMIC_RELAXED, __HIP_MEMORY_SCOPE_SYSTEM); \
            asm volatile("" : "+v"(v_[0]), "+v"(v_[1]), "+v"(v_[2]), "+v"(v_[3]), "+v"(v_[4]), "+v"(v_[5]), "+v"(v_[6]), "+v"(v_[7]), "+v"(v_[8]) :: "memory"); \
            now_ = __all(v_[0] >= call); \
            allin = now_ && __all(min(min(min(v_[1], v_[2]), min(v_[3], v_[4])), min(min(v_[5], v_[6]), v_[7])) >= call && (int)(v_[8] - call * (112u * CW)) >= 0); } \
        if (!now_) { if ((c) == (grp == 0 ? 7 : grp == 1 ? 8 - LAG : 0)) poll_ge<16>(fa_, call, tmo, call); else poll_ge<64>(fa_, call, tmo, call); } \
        if ((c) == 6 && !allin) poll_ge<64>((gu32*)(myws + OFF_CDONE), call * (112u * CW), tmo, call);     \
        } } while (0)

    f32x4 acc[2][2][4][2];
#pragma unroll
    for (int a = 0; a < 2; ++a)
#pragma unroll
        for (int b = 0; b < 2; ++b)
#pragma unroll
            for (int m = 0; m < 4; ++m)
#pragma unroll
                for (int n = 0; n < 2; ++n) acc[a][b][m][n] = (f32x4){0.f, 0.f, 0.f, 0.f};
    half8 At[4][2], B0[2][2], B1[2][2];

#define A_OF(t) (img + ((unsigned)__builtin_amdgcn_readlane(tabA, (t) & 63) & ~15u))
#define B_OF(t) (img + (unsigned)__builtin_amdgcn_readlane(tabB, (t) & 63))
#define GEMM_LOOP(T0, T1, HOOK) do { \
    const gchar* a3 = A_OF((T0) + 1); \
    for (int t = (T0); t < (T1); t += 2) { \
        const gchar* a1 = a3; \
        LDB(B0, 0, 0); SCHED; LDA(At, 0, 0); STAGE(SA(1, 1), a1 + HTB); SCHED; \
        const gchar* a2 = A_OF(t + 2); const gchar* b2 = B_OF(t + 2); const gchar* b3 = B_OF(t + 3); a3 = A_OF(t + 3); \
        WAIT_L(8); BAR; WAIT_L(0); MMA(0, 0, At, B0); \
        HOOK; \
        asm volatile("" ::: "memory"); BAR; SCHED; \
        LDB(B1, 0, 1); STAGE(SB(0, 0), b2); \
        BAR; WAIT_L(0); MMA(0, 1, At, B1); BAR; \
        LDA(At, 0, 1); STAGE(SA(0, 0), a2); \
        BAR; WAIT_L(0); MMA(1, 0, At, B0); BAR; SCHED; \
        STAGE(SB(0, 1), b2 + HTB); \
        WAIT_V(6); BAR; MMA(1, 1, At, B1); BAR; \
        LDB(B0, 1, 0); SCHED; LDA(At, 1, 0); STAGE(SA(0, 1), a2 + HTB); \
        WAIT_L(8); BAR; WAIT_L(0); MMA(0, 0, At, B0); BAR; SCHED; \
        LDB(B1, 1, 1); STAGE(SB(1, 0), b3); \
        BAR; WAIT_L(0); MMA(0, 1, At, B1); BAR; \
        LDA(At, 1, 1); STAGE(SA(1, 0), a3); \
        BAR; WAIT_L(0); MMA(1, 0, At, B0); BAR; SCHED; \
        STAGE(SB(1, 1), b3 + HTB); \
        WAIT_V(6); BAR; MMA(1, 1, At, B1); BAR; \
    } \
    if (wr == 0) BAR;                                                   \
    } while (0)

    const int src = (rank - dd) & (NRANK - 1);
    const bool early = dd != 0 && grp == 3;
#define DIAG() do { WAIT_V(0); BAR; if (wr == 1) BAR; asm volatile("" ::: "memory"); GEMM_LOOP(56, 64, ); } while (0)

    int tabA, tabB;
    {
        const int t0 = grp != 0 ? 0 : OWNA, u = lane < t0 ? lane : lane - 8, oo = 8 - dd;
        const int st = dd == 0 ? lane / 7 : u / 6, jj = dd == 0 ? lane % 7 : u % 6, o = jj + 1 + (jj + 1 >= oo ? 1 : 0);
        const bool own = dd != 0 && lane >= t0 && lane < t0 + 8;
        const int kt = own ? (lane - t0) * 8 + rank : (st & 7) * 8 + ((dest + o) & 7), q = lane & 7;
        tabA = lane < 56 ? (kt * (int)KT_STRIDE + mt * (int)PANEL) | (own ? 8 : st & 7) : ((q * 8 + rank) * (int)KT_STRIDE + (2 * rank + (mt & 1)) * (int)PANEL) | 7;
        tabB = nt * (int)PANEL + (lane < 56 ? (int)MAT_BYTES + kt * (int)KT_STRIDE
                                  : dd == 0 ? (int)MAT_BYTES + (q * 8 + rank) * (int)KT_STRIDE : 2 * (int)MAT_BYTES + (src * 8 + q) * (int)KT_STRIDE);
    }
    int have = __builtin_amdgcn_readlane(tabA, 0) & 15;
    BAR;
    WAIT_SEG(have);
    asm volatile("" ::: "memory"); BAR; asm volatile("" ::: "memory");
    { const gchar* a0 = A_OF(0); const gchar* b0 = B_OF(0); const gchar* a1 = A_OF(1); const gchar* b1 = B_OF(1);
      STAGE(SB(0, 0), b0); STAGE(SA(0, 0), a0); STAGE(SB(0, 1), b0 + HTB); STAGE(SA(0, 1), a0 + HTB);
      if (wr == 1) BAR;
      WAIT_V(4); BAR;
      STAGE(SB(1, 0), b1); STAGE(SA(1, 0), a1); STAGE(SB(1, 1), b1 + HTB);
      WAIT_V(6); BAR; }
    GEMM_LOOP(0, 56, { const int seg_ = __builtin_amdgcn_readlane(tabA, t + 3) & 15;
        if (seg_ != have) { have = seg_; WAIT_SEG(have); } });

    int tz = threadIdx.x; asm volatile("" : "+v"(tz));
    const int fr = tz & 15, fq = (tz >> 4) & 3;
    const unsigned lo = (unsigned)(fq * 4) * (NCOLS * 4) + (unsigned)fr * 16u;
    const unsigned soff = (unsigned)(tz & 63) * 16u;
    const size_t torg = ((size_t)((mt & 1) * 256 + wr * 64) * NCOLS + nt * 256 + wc * 64) * 4;
    const unsigned qnan = 0x7fc00000u;
    LAS unsigned char* sc = lds + 131072 + 256 + wid * 1088;
#define ROWOFF(c, g, e) ((size_t)(((c) >> 1) * 128 + (((c) & 1) * 2 + ((g) >> 1)) * 16 + ((g) & 1) * 2 + ((e) >> 2)) * (NCOLS * 4))
#define ACC(c, g, e) acc[(c) >> 1][((e) >> 1) & 1][((c) & 1) * 2 + ((g) >> 1)][(e) & 1][((g) & 1) * 2 + ((e) >> 2)]
    if (dd != 0) {
    if (__builtin_amdgcn_readfirstlane(__hip_atomic_load(tmo, __ATOMIC_RELAXED, __HIP_MEMORY_SCOPE_AGENT)) == call) {
#pragma unroll
        for (int i = 0; i < 32; ++i) {
            acc[i >> 4][(i >> 3) & 1][(i >> 1) & 3][i & 1] = __builtin_bit_cast(f32x4, (u32x4){qnan, qnan, qnan, qnan});
            asm volatile("" : "+v"(acc[i >> 4][(i >> 3) & 1][(i >> 1) & 3][i & 1]));
        }
    }
    unsigned u[4][4][8];
#pragma unroll
    for (int c = 0; c < 4; ++c)
#pragma unroll
        for (int g = 0; g < 4; ++g)
#pragma unroll
            for (int e = 0; e < 8; ++e) { const float v = ACC(c, g, e); u[c][g][e] = __builtin_bit_cast(unsigned, v); }
    for (int i = 0; i < XSTAG * wid; ++i) __builtin_amdgcn_s_sleep(1);
    unsigned meta;
    {
        unsigned se = 0;
#define U32(n) u[c][(n) >> 3][(n) & 7]
#pragma unroll
        for (int c = 0; c < 4; ++c)
#pragma unroll
            for (int g = 0; g < 4; ++g) {
#pragma unroll
                for (int e = 0; e < 8; ++e) u[c][g][e] = __builtin_amdgcn_alignbit(u[c][g][e], u[c][g][e], 31);
#pragma unroll
                for (int e = 0; e < 8; e += 2) se += __builtin_amdgcn_perm(u[c][g][e + 1], u[c][g][e], 0x0c070c03u);
                asm volatile("" : "+v"(u[c][g][0]), "+v"(u[c][g][1]), "+v"(u[c][g][2]), "+v"(u[c][g][3]), "+v"(u[c][g][4]), "+v"(u[c][g][5]), "+v"(u[c][g][6]), "+v"(u[c][g][7]));
            }
        const unsigned S = __builtin_amdgcn_readlane(wave_scan((se & 0xffffu) + (se >> 16)), 63);
        const unsigned b1 = min((S + 492u) >> 13, 253u), B = (b1 + 0xff00u - 5u) * 0x10001u;
        *(LAS u32x4*)(sc + soff) = (u32x4){0u, 0u, 0u, 0u};
        if (soff < 64u) *(LAS u32x4*)(sc + 1024 + soff) = (u32x4){0u, 0u, 0u, 0u};
        gchar* pk = myws + OFF_PK + (size_t)(bid * 8 + wid) * PK_WAVE;
        u64 st[4]; unsigned ln[4], worst = 0, up = 0;
#pragma unroll
        for (int c = 0; c < 4; ++c) {
            u32x2 cw = {0u, 0u};
            st[c] = 1; ln[c] = 0;
#pragma unroll
            for (int q = 7; q >= 0; --q) {
                unsigned W = 1, M[2];
#pragma unroll
                for (int h = 1; h >= 0; --h) {
                    const int n = 4 * q + 2 * h;
                    const unsigned E = __builtin_amdgcn_perm(U32(n + 1), U32(n), 0x0d070d03u), dq = pk_subs(E, B), ix = pk_min(dq, 0x00070007u) | 0x0c000c00u;
                    const unsigned a3 = pk_min(pk_subs(dq, 0x00070007u), 0x00030003u) * 3u;
                    up |= a3;
                    M[h] = __builtin_amdgcn_perm(0x00000001u, 0x02040507u, ix) + pk_subs(B, E) + a3;
                    cw[q >> 2] = (cw[q >> 2] << 2) | pk_subs(__builtin_amdgcn_perm(0x03020100u, 0u, ix), a3);
                    W = (W << ((M[h] >> 16) & 31u)) | 1u;
                    W = (W << (M[h] & 31u)) | 1u;
                }
                const unsigned ms = M[0] + M[1], sm = (ms & 0xffffu) + (ms >> 16);
                worst = max(worst, sm); ln[c] += sm;
                st[c] = (st[c] << (sm & 31u)) | W;
                asm volatile("" : "+v"(st[c]), "+v"(cw));
            }
#pragma unroll
            for (int i = 0; i < 6; ++i) {
#define CARR(n) __builtin_amdgcn_perm(U32(24 + (n) / 3), U32(n), 0x04020100u + (((n) % 3) << 24))
                const u32x4 v = {CARR(4 * i), CARR(4 * i + 1), CARR(4 * i + 2), CARR(4 * i + 3)};
                STORE16(soff, v, pk + c * PK_CHUNK + i * 1024, 63);
            }
            asm volatile("s_nop 4\n\tglobal_store_dwordx2 %0, %1, %2 sc0 sc1" :: "v"(soff >> 1), "v"(cw), "s"(pk + c * PK_CHUNK + 6144) : "memory");
        }
        bool ovf = max(max(ln[0], ln[1]), max(ln[2], ln[3])) > 63u || worst > 31u || (up & 0x00080008u) != 0u;
        const unsigned len = ovf ? 0u : ln[0] + ln[1] + ln[2] + ln[3], end = wave_scan(len);
        ovf |= end > 7679u;
        asm volatile("" ::: "memory");
        sc[soff >> 4] = (unsigned char)(ovf ? 255u : len);
        if (!ovf) {
            unsigned o = end - len;
#pragma unroll
            for (int c = 0; c < 4; ++c) {
                LAS unsigned* w = (LAS unsigned*)(sc + 64) + (o >> 5);
                const u64 a = st[c] << (o & 31u), b = (st[c] >> 32) << (o & 31u);
                __hip_atomic_fetch_or(w, (unsigned)a, __ATOMIC_RELAXED, __HIP_MEMORY_SCOPE_WAVEFRONT);
                __hip_atomic_fetch_or(w + 1, (unsigned)(a >> 32), __ATOMIC_RELAXED, __HIP_MEMORY_SCOPE_WAVEFRONT);
                __hip_atomic_fetch_or(w + 2, (unsigned)(b >> 32), __ATOMIC_RELAXED, __HIP_MEMORY_SCOPE_WAVEFRONT);
                o += ln[c];
            }
        }
        asm volatile("s_waitcnt lgkmcnt(0)" ::: "memory");
        const u32x4 sd = *(volatile LAS u32x4*)(sc + soff);
        STORE16(soff, sd, pk + 4 * PK_CHUNK, 63);
        meta = b1 | (((min((unsigned)__builtin_amdgcn_readlane(end, 63), 7679u) + 512u) >> 9) << 8);
        if (ovf) {
            gchar* rbase = myws + OFF_RAW + (size_t)(bid * 8 + wid) * RAW_WAVE;
#pragma unroll
            for (int c = 0; c < 4; ++c)
#pragma unroll
                for (int i = 0; i < 8; ++i) {
                    const u32x4 v = {U32(4 * i), U32(4 * i + 1), U32(4 * i + 2), U32(4 * i + 3)};
                    STORE16(soff, v, rbase + (c * 8 + i) * 1024, 63);
                }
        }
        asm volatile("s_waitcnt vmcnt(0)" ::: "memory");
        lctl[16 + wid] = (call << 12) | meta;
        unsigned old = 0;
        if ((tz & 63) == 0) old = __hip_atomic_fetch_add((LAS unsigned*)lctl + 5, 1u, __ATOMIC_ACQ_REL, __HIP_MEMORY_SCOPE_WORKGROUP);
        if (__builtin_amdgcn_readfirstlane(old) == 7 && (tz & 63) < 2) {
            const u32x4 fw = *(volatile LAS u32x4*)(lctl + 16 + (tz & 1) * 4);
            STORE16(soff, fw, (gchar*)p.ws[dest] + OFF_OUTFLAG + bid * 128, 63);
        }
    }
#pragma unroll
    for (int i = 0; i < 32; ++i) acc[i >> 4][(i >> 3) & 1][(i >> 1) & 3][i & 1] = (f32x4){0.f, 0.f, 0.f, 0.f};
    }
    if (early) DIAG();
    gchar* obase = (gchar*)p.out + torg + (size_t)src * NLOC * 4;
    if (dd != 0) {
        const unsigned f = __builtin_amdgcn_readfirstlane(poll_ge<32>((gu32*)(myws + OFF_OUTFLAG + bid * 128 + wid * 4), call << 12, tmo, call));
        const bool bad = __builtin_amdgcn_readfirstlane(__hip_atomic_load(tmo, __ATOMIC_RELAXED, __HIP_MEMORY_SCOPE_AGENT)) == call;
        const gchar* pk = (const gchar*)p.ws[bad ? rank : src] + OFF_PK + (size_t)(bid * 8 + wid) * PK_WAVE;
        const gchar* rbase = (const gchar*)p.ws[bad ? rank : src] + OFF_RAW + (size_t)(bid * 8 + wid) * RAW_WAVE;
        u32x4 side;
        asm volatile("s_nop 4\n\tglobal_load_dwordx4 %0, %1, %2 sc0 sc1\n\ts_waitcnt vmcnt(0)" : "=&v"(side) : "v"((soff >> 6) <= ((f >> 8) & 15u) ? soff : 0u), "s"(pk + 4 * PK_CHUNK) : "memory");
        *(volatile LAS u32x4*)(sc + soff) = side;
        const unsigned lb = *(volatile LAS unsigned char*)(sc + (soff >> 4));
        const bool ovf = lb == 255u;
        unsigned ptr = wave_scan(ovf ? 0u : lb) - (ovf ? 0u : lb);
        const unsigned BB = ((f & 255u) - 1u) * 0x10001u;
        const unsigned poison = bad ? qnan : 0u;
        if (wc == 0) {
            if (grp == 0) poll_ge<16>((gu32*)(myws + OFF_PDONE + (7 - dd) * 512), (call - 1u) * NPROD + (unsigned)(xcd * 2 + wr) * NPROD / 16u + 1u, tmo, call);
            else poll_ge<8>((gu32*)(myws + OFF_GATE + ((bid - 8) * 2 + wr) * 128), call, tmo, call);
        }
#pragma unroll
        for (int c = 0; c < 4; ++c) {
            while (__builtin_amdgcn_readfirstlane(lctl[1 + wr]) + 1u < (unsigned)(c * 4 + wc)) __builtin_amdgcn_s_sleep(32);
            while (__builtin_amdgcn_readfirstlane(lctl[1 + wr]) < (unsigned)(c * 4 + wc)) __builtin_amdgcn_s_sleep(2);
            const int lag = (int)__builtin_amdgcn_readfirstlane(lctl[3 + wr]);
            u32x4 in[6]; u32x2 cv;
            const gchar* pc = pk + c * PK_CHUNK + 3072;
            asm volatile("s_nop 4\n\tglobal_load_dwordx2 %0, %1, %2 offset:3072 sc0 sc1 nt" : "=v"(cv) : "v"(soff >> 1), "s"(pc) : "memory");
#pragma unroll
            for (int i = 0; i < 6; ++i) {
                asm volatile("s_waitcnt vmcnt(%0)" :: "i"(EW + 1) : "memory");
                if (lag <= TH2) asm volatile("s_waitcnt vmcnt(%0)" :: "i"(EW) : "memory");
                if (lag <= TH) asm volatile("s_waitcnt vmcnt(%0)" :: "i"(EW - 1) : "memory");
                if (lag < -TH) asm volatile("s_waitcnt vmcnt(%0)" :: "i"(EW - 2) : "memory");
                asm volatile("s_nop 4\n\tglobal_load_dwordx4 %0, %1, %2 offset:%3 sc0 sc1 nt" : "=v"(in[i]) : "v"(soff), "s"(pc), "i"(i * 1024 - 3072) : "memory");
            }
            asm volatile("s_waitcnt vmcnt(%1)" : "+v"(cv) : "i"(EW) : "memory");
            lctl[1 + wr] = (unsigned)(c * 4 + wc) + 1u;
            if (c == 3 && (tz & 255) == 192) __hip_atomic_store((gu32*)(myws + OFF_GATE + (bid * 2 + wr) * 128), call, __ATOMIC_RELAXED, __HIP_MEMORY_SCOPE_AGENT);
            unsigned tot = 0;
            if ((tz & 63) == 0) asm volatile("s_nop 4\n\tglobal_atomic_add %0, %1, %2, %3 sc0" : "+v"(tot) : "v"(0u), "v"(1u), "s"(myws + OFF_PROG + (7 - dd) * 512) : "memory");
            __builtin_amdgcn_sched_barrier(0);
            unsigned ex[16], s0, s1;
            {
                ptr = min(ptr, 7679u);
                const volatile LAS unsigned* w = (const volatile LAS unsigned*)(sc + 64) + (ptr >> 5);
                const unsigned w0 = w[0], w1 = w[1], w2 = w[2];
                s0 = __builtin_amdgcn_alignbit(w1, w0, ptr) | 1u; s1 = __builtin_amdgcn_alignbit(w2, w1, ptr);
            }
#pragma unroll
            for (int q = 0; q < 8; ++q) {
                unsigned t = s0, M[2];
#pragma unroll
                for (int h = 0; h < 2; ++h) {
                    const unsigned C = (cv[q >> 2] >> (2 * ((2 * q + h) & 7))) & 0x00030003u;
                    const unsigned m0 = (unsigned)__builtin_ctz(t - ((C & 0xffffu) == 0u ? 1u : 0u));
                    t >>= m0 & 31u;
                    const unsigned m1 = (unsigned)__builtin_ctz(t - (C < 0x10000u ? 1u : 0u));
                    if (h == 0) t >>= m1 & 31u;
                    M[h] = m0 | (m1 << 16);
                    ex[2 * q + h] = __builtin_amdgcn_perm(0x030b0202u, 0x07010100u, pk_min(M[h], 0x00070007u)) - M[h] + C + BB;
                }
                const unsigned ms = M[0] + M[1], sm = (ms + (ms >> 16)) & 31u;
                s0 = __builtin_amdgcn_alignbit(s1, s0, sm); s1 >>= sm; ptr += sm;
                asm volatile("" : "+v"(ex[2 * q]), "+v"(ex[2 * q + 1]), "+v"(s0));
            }
            asm volatile("s_waitcnt vmcnt(0)" : "+v"(in[0]), "+v"(in[1]), "+v"(in[2]), "+v"(in[3]), "+v"(in[4]), "+v"(in[5]), "+v"(tot), "+v"(ptr) :: "memory");
            if ((tz & 63) == 0) lctl[3 + wr] = tot - (call - 1u) * 1024u + 8u - 16u * (unsigned)(grp * 16 + c * 4 + wc + 1);
#pragma unroll
            for (int g = 0; g < 4; ++g) {
                u32x4 q[2];
#pragma unroll
                for (int e = 0; e < 8; ++e) {
                    const int n = g * 8 + e;
#define CIN(n) in[(n) >> 2][(n) & 3]
                    const unsigned lo24 = n < 24 ? CIN(n < 24 ? n : 0)
                        : __builtin_amdgcn_perm(CIN(n < 24 ? 0 : 3 * (n - 24) + 2), __builtin_amdgcn_perm(CIN(n < 24 ? 0 : 3 * (n - 24) + 1), CIN(n < 24 ? 0 : 3 * (n - 24)), 0x00000703u), 0x00070100u);
                    q[e >> 2][e & 3] = __builtin_amdgcn_perm(ex[n >> 1], lo24, (n & 1) ? 0x06020100u : 0x04020100u);
                }
                if (ovf) asm volatile("s_nop 4\n\tglobal_load_dwordx4 %0, %2, %3 sc0 sc1\n\tglobal_load_dwordx4 %1, %2, %3 offset:1024 sc0 sc1\n\ts_waitcnt vmcnt(0)"
                                      : "+v"(q[0]), "+v"(q[1]) : "v"(soff + (c * 4 + g) * 2048u), "s"(rbase) : "memory");
                unsigned v[8];
#pragma unroll
                for (int e = 0; e < 8; ++e) v[e] = __builtin_amdgcn_alignbit(q[e >> 2][e & 3], q[e >> 2][e & 3], 1) | poison;
#pragma unroll
                for (int e = 0; e < 8; ++e) ACC(c, g, e) += __builtin_bit_cast(float, v[e]);
                if (early) {
                    const f32x4 w0 = {ACC(c, g, 0), ACC(c, g, 1), ACC(c, g, 2), ACC(c, g, 3)}, w1 = {ACC(c, g, 4), ACC(c, g, 5), ACC(c, g, 6), ACC(c, g, 7)};
                    STOREO(lo, w0, obase + ROWOFF(c, g, 0)); STOREO(lo, w1, obase + ROWOFF(c, g, 4));
                }
            }
        }
    }
    if (!early) {
        DIAG();
        const bool bad = __builtin_amdgcn_readfirstlane(__hip_atomic_load(tmo, __ATOMIC_RELAXED, __HIP_MEMORY_SCOPE_AGENT)) == call;
#pragma unroll
        for (int c = 0; c < 4; ++c)
#pragma unroll
            for (int g = 0; g < 4; ++g)
#pragma unroll
                for (int h = 0; h < 2; ++h) {
                    f32x4 v = {ACC(c, g, 4 * h), ACC(c, g, 4 * h + 1), ACC(c, g, 4 * h + 2), ACC(c, g, 4 * h + 3)};
                    if (bad) v = __builtin_bit_cast(f32x4, (u32x4){qnan, qnan, qnan, qnan});
                    STOREO(lo, v, obase + ROWOFF(c, g, 4 * h));
                }
    }
}

extern "C" void kernel_launch(int rank, int nranks, ncclComm_t comm,
                              hipStream_t stream,
                              void* const* d_in, const int* in_sizes, int n_in,
                              void* d_out, int out_size,
                              const int* in_dtypes, int out_dtype,
                              void* const* d_out_all,
                              void* const* d_ws_all, size_t ws_bytes) {
    if (nranks != NRANK || (unsigned)rank >= (unsigned)NRANK || ws_bytes < WS_NEED || n_in < 2 ||
        in_sizes[0] != 4096 * KSH || in_sizes[1] != KSH * NCOLS || out_size != 512 * NCOLS) {
        fprintf(stderr, "unexpected problem shape\n");
        return;
    }
    Params p;
    p.x = (const float*)d_in[0]; p.w = (const float*)d_in[1];
    p.out = (float*)d_out;
    for (int r = 0; r < NRANK; ++r) p.ws[r] = (char*)d_ws_all[r];
    p.rank = rank; p.pad = 0;
    gemm_rs_kernel<<<GRID, 512, 0, stream>>>(p);
    HIP_CHECK(hipGetLastError());
}
```

```cpp
#include <hip/hip_runtime.h>
#include <rccl/rccl.h>
#include <cstdio>

#define HIP_CHECK(expr) do { hipError_t _e = (expr); if (_e != hipSuccess) { \
    fprintf(stderr, "HIP error %s at %s:%d\n", hipGetErrorString(_e), __FILE__, __LINE__); \
    return; } } while (0)

#define LAS __attribute__((address_space(3)))
#define GAS __attribute__((address_space(1)))
typedef _Float16 half8 __attribute__((ext_vector_type(8)));
typedef _Float16 half2v __attribute__((ext_vector_type(2)));
typedef float f32x2 __attribute__((ext_vector_type(2)));
typedef float f32x4 __attribute__((ext_vector_type(4)));
typedef unsigned u32x4 __attribute__((ext_vector_type(4)));
typedef unsigned u32x2 __attribute__((ext_vector_type(2)));
typedef unsigned long long u64;
template <int N> struct IC { static constexpr int value = N; };
typedef GAS unsigned gu32;
typedef GAS char gchar;

constexpr int NRANK = 8;
constexpr int KSH = 512;
constexpr int NCOLS = 32768, NLOC = 4096;
constexpr int GRID = 256, NPROD = 32;
constexpr size_t CTRL_BYTES = 1u << 20;
constexpr size_t PANEL = 32768;
constexpr size_t KT_STRIDE = 16 * PANEL;
constexpr size_t MAT_BYTES = 64 * KT_STRIDE;
constexpr size_t PI_ITEM = 26 * 256 + 512;
constexpr size_t OFF_PI = 3 * MAT_BYTES;
constexpr size_t BUF_BYTES = 3 * MAT_BYTES + (33u << 20);
constexpr size_t PK_CHUNK = 26 * 256;
constexpr size_t PK_WAVE = 4 * PK_CHUNK + 1024;
constexpr size_t RAW_WAVE = 64 * 512;
constexpr size_t OFF_PK = CTRL_BYTES + 2 * BUF_BYTES;
constexpr size_t OFF_RAW = OFF_PK + (size_t)GRID * 8 * PK_WAVE;
constexpr size_t WS_NEED = OFF_RAW + (size_t)GRID * 8 * RAW_WAVE;
constexpr int OFF_TMO = 0, OFF_CDONE = 28672 + 768;
constexpr int OFF_QUEUE = 20480, OFF_PDONE = 24576 + 256, OFF_PROG = 24576 + 128;
constexpr int OFF_GATE = 425984;
constexpr int OFF_INFLAG = 32768;
constexpr int OFF_CFLAG = 65536;
constexpr int OFF_OUTFLAG = 131072;
constexpr int OFF_TICKET = 393216;
constexpr unsigned long long DEADLINE_TICKS = 100000000ull;
constexpr int NPUSH = 40;
constexpr int UW = 12;
constexpr int CW = 3;
constexpr int PW = 2;
constexpr int LAG = 4;
constexpr int OWNA = 18;
constexpr int EW = 3;
constexpr int TH = 12, TH2 = 40;
static_assert(OFF_PI + (size_t)(512 + 4096) * PI_ITEM <= BUF_BYTES && OFF_GATE + 512 * 128 <= (int)CTRL_BYTES && NPUSH <= 56, "layout");
constexpr int NITEM = 960;
constexpr unsigned QUEUE_PER_CALL = NITEM - NPUSH - 2 * NPROD + NPROD;

struct Params {
    const float* x; const float* w;
    float* out;
    char* ws[NRANK];
    int rank; int pad;
};

template <int SLEEP = 8>
__device__ __forceinline__ unsigned poll_ge(gu32* addr, unsigned need, gu32* tmo, unsigned call) {
    const unsigned long long t0 = __builtin_amdgcn_s_memrealtime();
    unsigned v;
    for (unsigned n = 1;; ++n) {
        v = __hip_atomic_load(addr, __ATOMIC_RELAXED, __HIP_MEMORY_SCOPE_SYSTEM);
        if (__all((int)(v - need) >= 0)) break;
        if ((n & 15u) == 0u) {
            const unsigned tv = __builtin_amdgcn_readfirstlane(__hip_atomic_load(tmo, __ATOMIC_RELAXED, __HIP_MEMORY_SCOPE_AGENT));
            if (tv == call || __builtin_amdgcn_s_memrealtime() - t0 > DEADLINE_TICKS) {
                __hip_atomic_store(tmo, call, __ATOMIC_RELAXED, __HIP_MEMORY_SCOPE_AGENT);
                break;
            }
        }
        __builtin_amdgcn_s_sleep(SLEEP);
    }
    asm volatile("" ::: "memory");
    return v;
}

__device__ __forceinline__ unsigned pk16(float a, float b) {
    f32x2 v = {a, b};
    return __builtin_bit_cast(unsigned, __builtin_convertvector(v, half2v));
}

typedef unsigned short u16x2 __attribute__((ext_vector_type(2)));
__device__ __forceinline__ unsigned pk_max(unsigned a, unsigned b) { return __builtin_bit_cast(unsigned, __builtin_elementwise_max(__builtin_bit_cast(u16x2, a), __builtin_bit_cast(u16x2, b))); }
__device__ __forceinline__ unsigned wave_scan(unsigned x) {
    x += (unsigned)__builtin_amdgcn_update_dpp(0, (int)x, 0x111, 0xf, 0xf, false);
    x += (unsigned)__builtin_amdgcn_update_dpp(0, (int)x, 0x112, 0xf, 0xf, false);
    x += (unsigned)__builtin_amdgcn_update_dpp(0, (int)x, 0x114, 0xf, 0xf, false);
    x += (unsigned)__builtin_amdgcn_update_dpp(0, (int)x, 0x118, 0xf, 0xf, false);
    x += (unsigned)__builtin_amdgcn_update_dpp(0, (int)x, 0x142, 0xa, 0xf, false);
    x += (unsigned)__builtin_amdgcn_update_dpp(0, (int)x, 0x143, 0xc, 0xf, false);
    return x;
}

__device__ __forceinline__ unsigned pk_min(unsigned a, unsigned b) { return __builtin_bit_cast(unsigned, __builtin_elementwise_min(__builtin_bit_cast(u16x2, a), __builtin_bit_cast(u16x2, b))); }
__device__ __forceinline__ unsigned pk_subs(unsigned a, unsigned b) { return __builtin_bit_cast(unsigned, __builtin_elementwise_sub_sat(__builtin_bit_cast(u16x2, a), __builtin_bit_cast(u16x2, b))); }

__device__ __forceinline__ unsigned pack_item(unsigned (&D)[32], unsigned (&P)[26], u32x2& sd, LAS unsigned char* sc, int lane) {
    unsigned se = 0;
#pragma unroll
    for (int i = 0; i < 32; ++i) se += (D[i] >> 10) & 0x001f001fu;
    const unsigned S = __builtin_amdgcn_readlane(wave_scan((se & 0xffffu) + (se >> 16)), 63);
    const unsigned b1 = min((S + 246u) >> 12, 29u), B = (b1 + 27u) * 0x10001u;
    *(LAS u32x2*)(sc + lane * 8) = (u32x2){0u, 0u};
    if (lane < 8) *(LAS u32x2*)(sc + 512 + lane * 8) = (u32x2){0u, 0u};
    u64 lo = 1; unsigned hi = 0, len = 0, worst = 0, up = 0;
    bool ovf = false;
#pragma unroll
    for (int q = 15; q >= 0; --q) {
        unsigned W = 1, M[2];
#pragma unroll
        for (int h = 1; h >= 0; --h) {
            const int i = 2 * q + h;
            const unsigned E = ((D[i] >> 10) & 0x001f001fu) | 0x00200020u, dd = pk_subs(E, B), ix = pk_min(dd, 0x00070007u) | 0x0c000c00u;
            const unsigned a3 = pk_subs(dd, 0x00070007u) * 3u;
            up = pk_max(up, a3);
            M[h] = __builtin_amdgcn_perm(0x00000001u, 0x02040507u, ix) + pk_subs(B, E) + a3;
            D[i] = (D[i] & 0x83ff83ffu) | (pk_subs(__builtin_amdgcn_perm(0x03020100u, 0u, ix), a3) << 10);
            W = (W << ((M[h] >> 16) & 31u)) | 1u;
            W = (W << (M[h] & 31u)) | 1u;
        }
        const unsigned ms = M[0] + M[1], sm = (ms & 0xffffu) + (ms >> 16);
        worst = max(worst, sm); len += sm;
        if (q < 8) hi = (unsigned)(((((u64)hi << 32) | (lo >> 32)) << (sm & 31u)) >> 32);
        lo = (lo << (sm & 31u)) | W;
        if (q == 8) ovf = len > 63u;
        asm volatile("" : "+v"(lo), "+v"(hi));
    }
    ovf |= len > 95u || worst > 31u || pk_subs(up, 0x00060006u) != 0u;
    const unsigned end = wave_scan(ovf ? 0u : len), o = end - (ovf ? 0u : len);
    ovf |= end > 3583u;
    asm volatile("" ::: "memory");
    sc[lane] = (unsigned char)(ovf ? 255u : len);
    if (!ovf) {
        LAS unsigned* w = (LAS unsigned*)(sc + 64) + (o >> 5);
        const u64 a = lo << (o & 31u), b = (((u64)hi << 32) | (lo >> 32)) << (o & 31u), c = (u64)hi << (o & 31u);
        __hip_atomic_fetch_or(w, (unsigned)a, __ATOMIC_RELAXED, __HIP_MEMORY_SCOPE_WAVEFRONT);
        __hip_atomic_fetch_or(w + 1, (unsigned)(a >> 32), __ATOMIC_RELAXED, __HIP_MEMORY_SCOPE_WAVEFRONT);
        __hip_atomic_fetch_or(w + 2, (unsigned)(b >> 32), __ATOMIC_RELAXED, __HIP_MEMORY_SCOPE_WAVEFRONT);
        __hip_atomic_fetch_or(w + 3, (unsigned)(c >> 32), __ATOMIC_RELAXED, __HIP_MEMORY_SCOPE_WAVEFRONT);
    }
#pragma unroll
    for (int g = 0; g < 2; ++g) {
#pragma unroll
        for (int i = 0; i < 12; ++i) P[13 * g + i] = D[16 * g + i] | ((D[16 * g + 13 + i / 4] << (12 - 3 * (i & 3))) & 0x70007000u);
        P[13 * g + 12] = D[16 * g + 12] | ((D[16 * g + 13] >> 3) & 0x10001000u) | ((D[16 * g + 14] >> 2) & 0x20002000u) | ((D[16 * g + 15] >> 1) & 0x40004000u);
    }
    asm volatile("s_waitcnt lgkmcnt(0)" ::: "memory");
    sd = *(volatile LAS u32x2*)(sc + lane * 8);
    const unsigned tot = min((unsigned)__builtin_amdgcn_readlane(end, 63), 3583u);
    return __builtin_popcountll(__ballot(ovf)) > 2 ? 31u : b1 | (((tot + 512u) >> 9) << 5);
}
template <int Q>
__device__ __forceinline__ void unpack8(unsigned (&H)[8], unsigned BB, unsigned& s0, unsigned& s1, unsigned& s2) {
#pragma unroll
    for (int q = 0; q < 4; ++q) {
        unsigned t = s0, M[2];
#pragma unroll
        for (int h = 0; h < 2; ++h) {
            const unsigned T = H[2 * q + h];
            const half2v f = __builtin_bit_cast(half2v, T);
            const unsigned m0 = (unsigned)__builtin_ctz(t - (__builtin_amdgcn_classh(f[0], 0xf0) ? 1u : 0u));
            t >>= m0 & 31u;
            const unsigned m1 = (unsigned)__builtin_ctz(t - (__builtin_amdgcn_classh(f[1], 0xf0) ? 1u : 0u));
            if (h == 0) t >>= m1 & 31u;
            M[h] = m0 | (m1 << 16);
            H[2 * q + h] = ((__builtin_amdgcn_perm(0x030b0202u, 0x07010100u, pk_min(M[h], 0x00070007u)) - M[h]) << 10) + (T + BB);
        }
        const unsigned ms = M[0] + M[1], sm = ms + (ms >> 16);
        s0 = __builtin_amdgcn_alignbit(s1, s0, sm);
        if (Q < 2) { s1 = __builtin_amdgcn_alignbit(s2, s1, sm); s2 >>= sm & 31u; } else s1 >>= sm & 31u;
        asm volatile("" : "+v"(H[2 * q]), "+v"(H[2 * q + 1]));
    }
}
constexpr int pull_younger(int k) {
    int n = 0;
    for (int j = k - PW; j < k; ++j) {
        const int jj = (j + 7) % 7;
        if (j > k - PW) n += (j + PW) % 7 == 0 ? 2 : 1;
        if (jj == 1 || jj == 3 || jj == 5 || jj == 6) n += 2;
    }
    return n;
}

__device__ __forceinline__ void item_decode(int it, int src, int& s, int& r) {
    int e;
    constexpr int n1 = 72 * LAG, n2 = n1 + 88 * (8 - LAG);
    if (it < n1) { s = it / 72; e = it - 72 * s; }
    else if (it < n2) { s = (it - n1) / 88; e = it - n1 - 88 * s; s += e < 72 ? LAG : 0; }
    else { const int v = it - 576; s = (v >> 4) & 7; e = 72 + (v >> 7) * 16 + (v & 15); }
    r = e < 56 ? (8 * src + 8 + e) & 63 : e + 8;
}

#define UNIT_LOAD(B, S_, R_, H_) do { const bool isx_ = (R_) < 64; \
        const gchar* b_ = isx_ ? sX + (size_t)((R_) * 64 + (H_) * 32) * (KSH * 4) + (size_t)(S_) * 256 \
                               : sW + (size_t)((S_) * 64 + (H_) * 32) * (NCOLS * 4) + (size_t)((R_) - 64) * 256; \
        const unsigned vo_ = isx_ ? xoff : woff; \
        const gchar* b1_ = b_ + (isx_ ? 16 : 1 * NCOLS * 4);          const gchar* b2_ = b_ + (isx_ ? 128 : 2 * NCOLS * 4); \
        const gchar* b3_ = b_ + (isx_ ? 144 : 3 * NCOLS * 4);         const gchar* b4_ = b_ + (isx_ ? 32768 : 4 * NCOLS * 4); \
        const gchar* b5_ = b_ + (isx_ ? 32768 + 16 : 5 * NCOLS * 4);  const gchar* b6_ = b_ + (isx_ ? 32768 + 128 : 6 * NCOLS * 4); \
        const gchar* b7_ = b_ + (isx_ ? 32768 + 144 : 7 * NCOLS * 4); \
        asm volatile("s_nop 4\n\tglobal_load_dwordx4 %0, %8, %9 nt\n\tglobal_load_dwordx4 %1, %8, %10 nt\n\t" \
                     "global_load_dwordx4 %2, %8, %11 nt\n\tglobal_load_dwordx4 %3, %8, %12 nt\n\t" \
                     "global_load_dwordx4 %4, %8, %13 nt\n\tglobal_load_dwordx4 %5, %8, %14 nt\n\t" \
                     "global_load_dwordx4 %6, %8, %15 nt\n\tglobal_load_dwordx4 %7, %8, %16 nt" \
                     : "+v"(B[0]), "+v"(B[1]), "+v"(B[2]), "+v"(B[3]), "+v"(B[4]), "+v"(B[5]), "+v"(B[6]), "+v"(B[7]) \
                     : "v"(vo_), "s"(b_), "s"(b1_), "s"(b2_), "s"(b3_), "s"(b4_), "s"(b5_), "s"(b6_), "s"(b7_) : "memory"); } while (0)
#define UNIT_WAIT(B, N) asm volatile("s_waitcnt vmcnt(%8)" : "+v"(B[0]), "+v"(B[1]), "+v"(B[2]), "+v"(B[3]), \
                                     "+v"(B[4]), "+v"(B[5]), "+v"(B[6]), "+v"(B[7]) : "i"(N) : "memory")
#define UNIT_CVT(O, B, R_) do { if ((R_) < 64) { \
            _Pragma("unroll") for (int b = 0; b < 4; ++b) { \
                const f32x4 lo_ = __builtin_bit_cast(f32x4, B[2 * b]), hi_ = __builtin_bit_cast(f32x4, B[2 * b + 1]); \
                O[b] = (u32x4){pk16(lo_[0], lo_[1]), pk16(lo_[2], lo_[3]), pk16(hi_[0], hi_[1]), pk16(hi_[2], hi_[3])}; } \
        } else { \
            _Pragma("unroll") for (int j = 0; j < 4; ++j) \
                O[j] = (u32x4){pk16(__builtin_bit_cast(f32x4, B[0])[j], __builtin_bit_cast(f32x4, B[1])[j]), \
                               pk16(__builtin_bit_cast(f32x4, B[2])[j], __builtin_bit_cast(f32x4, B[3])[j]), \
                               pk16(__builtin_bit_cast(f32x4, B[4])[j], __builtin_bit_cast(f32x4, B[5])[j]), \
                               pk16(__builtin_bit_cast(f32x4, B[6])[j], __builtin_bit_cast(f32x4, B[7])[j])}; } \
        asm volatile("" : "+v"(O[0]), "+v"(O[1]), "+v"(O[2]), "+v"(O[3])); } while (0)
#define ITEM_OFF(R_) ((R_) < 64 ? (size_t)(R_) * 8192 : (size_t)(((R_) - 64) >> 2) * PANEL + (size_t)((R_) & 3) * 4096)
#define UNIT_DST(BASE, R_, H_) const bool isxd_ = (R_) < 64; \
        gchar* t0_ = (BASE) + (isxd_ ? (H_) * 4096 : (H_) * 1024); \
        gchar* t1_ = t0_ + (isxd_ ? 1024 : 2048); gchar* t2_ = t0_ + (isxd_ ? 2048 : 16384); gchar* t3_ = t0_ + (isxd_ ? 3072 : 16384 + 2048)
#define STOREO(VOFF, DATA, BASE) asm volatile("s_nop 4\n\tglobal_store_dwordx4 %0, %1, %2 sc0 sc1 nt\n\ts_nop 1" :: "v"(VOFF), "v"(DATA), "s"(BASE) : "memory")
#define STORE16(VOFF, DATA, BASE, N) asm volatile("s_nop 3\n\ts_waitcnt vmcnt(%3)\n\tglobal_store_dwordx4 %0, %1, %2 sc0 sc1\n\ts_nop 1" \
                                                  :: "v"(VOFF), "v"(DATA), "s"(BASE), "i"(N) : "memory")

__global__ __launch_bounds__(512, 2) void gemm_rs_kernel(const Params p) {
    __shared__ __attribute__((aligned(16))) unsigned char shm[131072 + 256 + 8 * 1088];
    LAS unsigned char* lds = (LAS unsigned char*)shm;
    volatile LAS unsigned* lctl = (volatile LAS unsigned*)(lds + 131072);
    const int tid = threadIdx.x, wid = __builtin_amdgcn_readfirstlane(tid >> 6), lane = tid & 63;
    const int rank = p.rank & (NRANK - 1), bid = blockIdx.x;
    gchar* myws = (gchar*)p.ws[rank];
    gu32* tmo = (gu32*)(myws + OFF_TMO);

    if (tid == 0) {
        lctl[0] = __hip_atomic_fetch_add((gu32*)(myws + OFF_TICKET + bid * 128), 1u, __ATOMIC_RELAXED, __HIP_MEMORY_SCOPE_AGENT) + 1u;
    } else if (tid < 64) lctl[tid] = 0u;
    unsigned call; size_t bufoff;
#define GET_CALL() do { asm volatile("s_waitcnt lgkmcnt(0)" ::: "memory"); __builtin_amdgcn_s_barrier(); asm volatile("" ::: "memory"); \
        call = __builtin_amdgcn_readfirstlane(lctl[0]); bufoff = CTRL_BYTES + (size_t)(call & 1u) * BUF_BYTES; } while (0)

    const int xcd = bid & 7, idx = bid >> 3;
    int dd = 0, par = xcd & 1, grp = idx & 3, nt = 4 * grp + (xcd >> 1), hidx = 0;
    if (bid >= NPROD) {
        const int hi = idx >= 18, jj = idx - 4 - 14 * hi, a = xcd >> 1, n0 = a < 2 ? 8 : 6, odd = jj >= n0, k = odd ? jj - n0 : jj;
        const int mi = (odd ? 3 * a + (a == 3) : 4 * a - (a == 3)) + (k >> 1);
        grp = 2 * hi + odd; nt = 4 * grp + 2 * (xcd & 1) + (k & 1); par = mi >= 7; dd = 1 + mi - 7 * par;
        hidx = jj * 8 + xcd;
    }
    const int dest = (rank + dd) & (NRANK - 1), mt = 2 * dest + par;

    const unsigned soff0 = (unsigned)lane * 16u;
    const gchar* sX = (const gchar*)p.x;
    const unsigned xoff = (unsigned)(lane & 15) * (KSH * 4) + (unsigned)(lane >> 4) * 32u;
    const unsigned woff = (unsigned)(lane >> 4) * 8u * (NCOLS * 4) + (unsigned)(lane & 15) * 16u;
    if (bid >= NPROD && nt >= 8 && wid >= 1 && wid <= CW) {
        u32x4 buf0[8], buf1[8], o[4], o1[4];
#pragma unroll
        for (int b = 0; b < 8; ++b) { buf0[b] = (u32x4){0u, 0u, 0u, 0u}; buf1[b] = (u32x4){0u, 0u, 0u, 0u}; }
        bool first = true;
        static_assert(CW == 3, "3 x 336 runs of four");
        const int me = (wid - 1) * 112 + hidx;
        u32x4 fv = {0u, 0u, 0u, 0u};
        for (int n = 0; n < 14; ++n) {
            int s, e, sub = 0;
            if (n < 2) { const int job = NPUSH + me + 336 * n; if (job >= 576) continue; s = job / 192; e = job - 192 * s; }
            else {
                const int q = 335 - me + 336 * ((n - 2) >> 2); int v; sub = (n - 2) & 3;
                if (q < 240) { s = q / 48; v = q - 48 * s; s += 3; } else { s = ((q - 240) >> 5) & 7; v = 48 + ((q - 240) >> 8) * 32 + ((q - 240) & 31); }
                e = v < 16 ? 4 * v + sub : 64 + (((v - 16) >> 3) * 4 + sub) * 8 + (v & 7);
            }
            const int d = e < 64 ? rank : (rank + 1 + ((e - 64) & 7)) & (NRANK - 1);
            const int r = e < 64 ? (8 * rank + 8 + e) & 63 : 64 + ((e - 64) >> 3);
            const gchar* sW = (const gchar*)p.w + (size_t)d * NLOC * 4;
            UNIT_LOAD(buf0, s, r, 0); UNIT_LOAD(buf1, s, r, 1);
            if (first) { GET_CALL(); first = false; }
            gchar* tb = myws + bufoff + ITEM_OFF(r) + (e < 64 ? (size_t)(s * 8 + rank) * KT_STRIDE
                        : d == rank ? MAT_BYTES + (size_t)(s * 8 + rank) * KT_STRIDE : 2 * MAT_BYTES + (size_t)(d * 8 + s) * KT_STRIDE);
            UNIT_WAIT(buf0, 8); UNIT_CVT(o, buf0, r);
            { UNIT_DST(tb, r, 0); STORE16(soff0, o[0], t0_, 63); STORE16(soff0, o[1], t1_, 63); STORE16(soff0, o[2], t2_, 63); STORE16(soff0, o[3], t3_, 63); }
            UNIT_WAIT(buf1, 4); UNIT_CVT(o1, buf1, r);
            { UNIT_DST(tb, r, 1); STORE16(soff0, o1[0], t0_, 63); STORE16(soff0, o1[1], t1_, 63); STORE16(soff0, o1[2], t2_, 63); STORE16(soff0, o1[3], t3_, 63); }
            unsigned meta = 0;
            if (e < 56 || d != rank) {
                unsigned D[32], P[26]; u32x2 sd;
#pragma unroll
                for (int i = 0; i < 16; ++i) { D[i] = o[i >> 2][i & 3]; D[16 + i] = o1[i >> 2][i & 3]; }
                meta = pack_item(D, P, sd, lds + wid * 1024, lane);
                gchar* pi = myws + bufoff + OFF_PI + (size_t)(e < 64 ? s * 64 + r : 512 + (d * 8 + s) * 64 + r - 64) * PI_ITEM;
#pragma unroll
                for (int i = 0; i < 6; ++i) { const u32x4 v = {P[4 * i], P[4 * i + 1], P[4 * i + 2], P[4 * i + 3]}; STORE16(soff0, v, pi + i * 1024, 63); }
                const u32x2 v2 = {P[24], P[25]};
                asm volatile("s_nop 4\n\tglobal_store_dwordx2 %0, %1, %3 sc0 sc1\n\tglobal_store_dwordx2 %0, %2, %3 offset:512 sc0 sc1"
                             :: "v"(soff0 >> 1), "v"(v2), "v"(sd), "s"(pi + 6144) : "memory");
            }
            fv = (u32x4){fv[1], fv[2], fv[3], (call << 8) | meta};
            if (n >= 2 && sub != 3) continue;
            asm volatile("s_waitcnt vmcnt(0)" ::: "memory");
            const unsigned fo = (unsigned)(rank * 4096 + (s * 128 + r - sub) * 4);
#pragma unroll
            for (int k = 0; k < NRANK; ++k)
                if ((e < 56 || k == d) && lane == 0) {
                    gchar* fb = (gchar*)p.ws[k] + (k == rank ? OFF_INFLAG : OFF_CFLAG + (call & 1u) * 32768u);
                    if (n < 2) __hip_atomic_store((gu32*)(fb + fo), k == rank ? call : fv[3], __ATOMIC_RELAXED, __HIP_MEMORY_SCOPE_SYSTEM);
                    else { const u32x4 fw = k == rank ? (u32x4){call, call, call, call} : fv; STORE16(fo, fw, fb, 63); }
                }
        }
        if (lane == 0) __hip_atomic_fetch_add((gu32*)(myws + OFF_CDONE), 1u, __ATOMIC_RELAXED, __HIP_MEMORY_SCOPE_AGENT);
    } else if (bid >= NPROD && nt < 8 && wid == 1 && hidx < NPUSH) {
        u32x4 buf0[8], buf1[8], o0[4], o1[4];
#pragma unroll
        for (int b = 0; b < 8; ++b) { buf0[b] = (u32x4){0u, 0u, 0u, 0u}; buf1[b] = (u32x4){0u, 0u, 0u, 0u}; }
        const gchar* sW = sX;
        const int r = (8 * rank + 8 + hidx) & 63;
        UNIT_LOAD(buf0, 0, r, 0); UNIT_LOAD(buf1, 0, r, 1);
        GET_CALL();
        UNIT_WAIT(buf0, 8); UNIT_CVT(o0, buf0, 0);
        UNIT_WAIT(buf1, 0); UNIT_CVT(o1, buf1, 0);
        const size_t ioff = bufoff + (size_t)rank * KT_STRIDE + (size_t)r * 8192;
#pragma unroll
        for (int b = 0; b < 8; ++b)
            for (int k = 1; k <= NRANK; ++k) {
                gchar* t = (gchar*)p.ws[(rank + hidx + k) & (NRANK - 1)] + ioff + b * 1024;
                STORE16(soff0, b < 4 ? o0[b & 3] : o1[b & 3], t, UW - 1);
            }
        asm volatile("s_waitcnt vmcnt(0)" ::: "memory");
#pragma unroll
        for (int k = 0; k < NRANK; ++k)
            if (lane == 0) __hip_atomic_store((gu32*)((gchar*)p.ws[k] + OFF_INFLAG + rank * 4096 + r * 4), call, __ATOMIC_RELAXED, __HIP_MEMORY_SCOPE_SYSTEM);
    } else if (bid >= NPROD) {
        GET_CALL();
    } else if (wid == 0) {
        GET_CALL();
        const int j = lane & 7, lk = (j + bid) % 7, psrc = (rank + 1 + lk) & (NRANK - 1);
        const unsigned long long t0 = __builtin_amdgcn_s_memrealtime();
        while (__builtin_amdgcn_s_memrealtime() - t0 < 600) __builtin_amdgcn_s_sleep(4);
        unsigned n = 0; int item = 0; bool have = false, fin = lane >= 7, late = false;
        for (unsigned it = 1; !__all(fin); ++it) {
            if (!fin && !have && n <= lctl[8 + j]) {
                unsigned t = NPUSH + bid + n * NPROD;
                if (n >= 2) t = __hip_atomic_fetch_add((gu32*)(myws + OFF_QUEUE + lk * 512), 1u, __ATOMIC_RELAXED, __HIP_MEMORY_SCOPE_AGENT)
                                - (call - 1u) * QUEUE_PER_CALL + NPUSH + 2u * NPROD;
                int s, r; item_decode(t < (unsigned)NITEM ? (int)t : 0, psrc, s, r);
                item = t < (unsigned)NITEM ? s * 128 + r : 1024; have = true;
            }
            if (have) {
                bool ready = item >= 1024 || late; unsigned cf = 0;
                if (!ready) { cf = __hip_atomic_load((gu32*)(myws + OFF_CFLAG + (call & 1u) * 32768u + psrc * 4096 + item * 4), __ATOMIC_RELAXED, __HIP_MEMORY_SCOPE_SYSTEM); ready = (int)(cf - (call << 8)) >= 0; }
                if (ready) { ++n; __hip_atomic_store((LAS unsigned*)lctl + 32 + j * 4 + (n & 3u), (n << 20) | ((cf & 255u) << 12) | (unsigned)item, __ATOMIC_RELEASE, __HIP_MEMORY_SCOPE_WORKGROUP); have = false; fin = item >= 1024; }
            }
            if ((it & 63u) == 0u && !late) {
                late = __builtin_amdgcn_readfirstlane(__hip_atomic_load(tmo, __ATOMIC_RELAXED, __HIP_MEMORY_SCOPE_AGENT)) == call
                       || __builtin_amdgcn_s_memrealtime() - t0 > DEADLINE_TICKS;
                if (late) __hip_atomic_store(tmo, call, __ATOMIC_RELAXED, __HIP_MEMORY_SCOPE_AGENT);
            }
            __builtin_amdgcn_s_sleep(2);
        }
    } else {
        GET_CALL();
        const int lk = (wid - 1 + bid) % 7, src = (rank + 1 + lk) & (NRANK - 1);
        const gchar* rimg = (const gchar*)p.ws[src] + bufoff;
        gchar* limg = myws + bufoff + (size_t)src * KT_STRIDE;
        gu32* lflag = (gu32*)(myws + OFF_INFLAG + src * 4096);
        u32x4 in[6]; u32x2 in6 = {0u, 0u}, side = {0u, 0u};
#pragma unroll
        for (int b = 0; b < 6; ++b) in[b] = (u32x4){0u, 0u, 0u, 0u};
        const unsigned soff8 = (unsigned)lane * 8u;
        LAS unsigned char* sc = lds + wid * 1024;
#define PI_OF(S_, R_) (rimg + OFF_PI + 3072 + (size_t)((R_) < 64 ? (S_) * 64 + (R_) : 512 + (rank * 8 + (S_)) * 64 + (R_) - 64) * PI_ITEM)
#define RAW_OF(S_, R_) (rimg + ITEM_OFF(R_) + ((R_) < 64 ? (size_t)((S_) * 8 + src) * KT_STRIDE : 2 * MAT_BYTES + (size_t)(rank * 8 + (S_)) * KT_STRIDE))
#define DST_OF(S_, R_) (limg + ITEM_OFF(R_) + ((R_) < 64 ? 0 : MAT_BYTES) + (size_t)(S_) * 8 * KT_STRIDE)
#define PULL(k, BASE, META) do { \
        if ((k) == 0) asm volatile("s_nop 4\n\tglobal_load_dwordx2 %0, %1, %2 offset:3584 sc0 sc1" : "+v"(side) : "v"((soff8 >> 6) <= ((META) >> 5) ? soff8 : 0u), "s"(BASE) : "memory"); \
        if ((k) < 6) asm volatile("s_nop 4\n\tglobal_load_dwordx4 %0, %1, %2 offset:%3 sc0 sc1" : "+v"(in[(k) < 6 ? (k) : 0]) : "v"(soff0), "s"(BASE), "i"((k) * 1024 - 3072) : "memory"); \
        else asm volatile("s_nop 4\n\tglobal_load_dwordx2 %0, %1, %2 offset:3072 sc0 sc1" : "+v"(in6) : "v"(soff8), "s"(BASE) : "memory"); } while (0)
#define TAKE(DST, META, N_) do { unsigned e_; while (((e_ = __builtin_amdgcn_readfirstlane(__hip_atomic_load((LAS unsigned*)lctl + 28 + wid * 4 + ((N_) & 3), __ATOMIC_ACQUIRE, __HIP_MEMORY_SCOPE_WORKGROUP))) >> 20) != (unsigned)(N_)) __builtin_amdgcn_s_sleep(1); \
        lctl[7 + wid] = (unsigned)(N_); asm volatile("" ::: "memory"); DST = (int)(e_ & 0x7ffu); META = (e_ >> 12) & 255u; } while (0)
        int cur, s, r, pflag = -1; unsigned meta;
        TAKE(cur, meta, 1);
        s = cur >> 7; r = cur & 127;
        const gchar* idle = (const gchar*)myws + bufoff + OFF_PI + 3072;
        const gchar* sb = meta == 31u ? idle : PI_OF(s, r);
#pragma unroll
        for (int k = 0; k < PW; ++k) PULL(k, sb, meta);
        for (int i = 0; cur < 1024; ++i) {
            int nxt = 1024, sn = 0, rn = 0; const gchar* sbn = sb; unsigned metan = 0;
            gchar* db = DST_OF(s, r);
            const bool isx = r < 64;
            const gchar* rraw = RAW_OF(s, r);
            asm volatile("" : "+s"(rraw), "+s"(db));
            const unsigned BB = ((meta & 31u) - 1u) * 0x04000400u;
            unsigned s0 = 0, s1 = 0, s2 = 0, dn[3] = {0u, 0u, 0u}; bool ovf = false;
            auto step = [&](auto KC) __attribute__((always_inline)) {
                constexpr int k = decltype(KC)::value;
                (void)&rraw; (void)&db; (void)&sb; (void)&soff0; (void)&soff8; (void)&in; (void)&in6; (void)&side;
                if (k + PW == 7) {
                    TAKE(nxt, metan, i + 2);
                    sn = (nxt >> 7) & 7; rn = nxt & 127;
                    sbn = nxt < 1024 && metan != 31u ? PI_OF(sn, rn) : idle;
                    if (nxt >= 1024 && lane == 0) __hip_atomic_fetch_add((gu32*)(myws + OFF_PDONE + lk * 512), 1u, __ATOMIC_RELAXED, __HIP_MEMORY_SCOPE_AGENT);
                }
                if (k < PW && i == 0) asm volatile("s_waitcnt vmcnt(%0)" :: "i"(PW - 1) : "memory");
                if (k < 6) asm volatile("s_waitcnt vmcnt(%2)" : "+v"(in[k < 6 ? k : 0]), "+v"(side) : "i"(pull_younger(k)) : "memory");
                else asm volatile("s_waitcnt vmcnt(%1)" : "+v"(in6) : "i"(pull_younger(k)) : "memory");
                if (k == PW && pflag >= 0 && lane == 0) __hip_atomic_store(lflag + pflag, call, __ATOMIC_RELAXED, __HIP_MEMORY_SCOPE_SYSTEM);
                if (k + PW < 7) PULL((k + PW) % 7, sb, meta); else PULL((k + PW) % 7, sbn, metan);
                __builtin_amdgcn_sched_barrier(0);
                if (k == 0) {
                    *(volatile LAS u32x2*)(sc + lane * 8) = side;
                    const unsigned lb = *(volatile LAS unsigned char*)(sc + lane);
                    ovf = lb == 255u || meta == 31u;
                    const unsigned len = ovf ? 0u : lb, o = min(wave_scan(len) - len, 3583u);
                    const volatile LAS unsigned* w = (const volatile LAS unsigned*)(sc + 64) + (o >> 5);
                    const unsigned w0 = w[0], w1 = w[1], w2 = w[2], w3 = w[3];
                    s0 = __builtin_amdgcn_alignbit(w1, w0, o) | 1u; s1 = __builtin_amdgcn_alignbit(w2, w1, o); s2 = __builtin_amdgcn_alignbit(w3, w2, o);
                    asm volatile("" : "+v"(s0), "+v"(s1), "+v"(s2));
                }
                if (k == 1 || k == 3 || k == 5 || k == 6) {
                    constexpr int g = k >> 1;
#define PIN(n) ((n) < 24 ? in[((n) < 24 ? (n) : 0) >> 2][(n) & 3] : in6[(n) & 1])
                    unsigned H[8];
#pragma unroll
                    for (int e = 0; e < ((g & 1) ? 5 : 8); ++e) {
                        const int ci = (g & 1) * 8 + e; const unsigned pw = PIN(13 * (g >> 1) + ci);
                        H[e] = pw & 0x8fff8fffu;
                        if (ci < 12) { const unsigned pc = (pw >> (12 - 3 * (ci & 3))) & (0x00070007u << (3 * (ci & 3))); dn[ci >> 2] = (ci & 3) ? dn[ci >> 2] | pc : pc; }
                        else { H[5] = dn[0] | ((pw << 3) & 0x80008000u); H[6] = dn[1] | ((pw << 2) & 0x80008000u); H[7] = dn[2] | ((pw << 1) & 0x80008000u); }
                    }
                    unpack8<g>(H, BB, s0, s1, s2);
                    u32x4 v0 = {H[0], H[1], H[2], H[3]}, v1 = {H[4], H[5], H[6], H[7]};
                    const unsigned o0 = soff0 + (isx ? g * 2048 : (g & 1) * 16384 + (g >> 1) * 1024), o1 = o0 + (isx ? 1024 : 2048);
                    if (ovf) asm volatile("s_nop 4\n\tglobal_load_dwordx4 %0, %2, %4 sc0 sc1\n\tglobal_load_dwordx4 %1, %3, %4 sc0 sc1\n\ts_waitcnt vmcnt(0)"
                                          : "+v"(v0), "+v"(v1) : "v"(o0), "v"(o1), "s"(rraw) : "memory");
                    STORE16(o0, v0, db, 63); STORE16(o1, v1, db, 63);
                }
            };
            step(IC<0>{}); step(IC<1>{}); step(IC<2>{}); step(IC<3>{}); step(IC<4>{}); step(IC<5>{}); step(IC<6>{});
            pflag = cur; cur = nxt; s = sn; r = rn; sb = sbn; meta = metan;
        }
        asm volatile("s_waitcnt vmcnt(0)" : "+v"(in[0]), "+v"(in[1]), "+v"(in[2]), "+v"(in[3]), "+v"(side) :: "memory");
        if (lane == 0) __hip_atomic_store(lflag + pflag, call, __ATOMIC_RELAXED, __HIP_MEMORY_SCOPE_SYSTEM);
    }

    const int wr = wid >> 2, wc = wid & 3;
    const gchar* img = myws + bufoff;
    const unsigned voff = (unsigned)tid * 16u;
    const unsigned ldsw = (unsigned)wid * 1024u;
    const int aoff = wr * 8192 + lane * 16, boff = wc * 4096 + lane * 16;
    constexpr int HTB = 16384;
#define SA(b, h) (((b) * 2 + (h)) * HTB)
#define SB(b, h) ((4 + (b) * 2 + (h)) * HTB)
#define STAGE(bufo, gbase) do { _Pragma("unroll") for (int _i = 0; _i < 2; ++_i) \
        __builtin_amdgcn_global_load_lds((const GAS unsigned*)((gbase) + voff + _i * 8192), (LAS unsigned*)(lds + (bufo) + ldsw + _i * 8192), 16, 0, 17); } while (0)
#define LDA(dst, b, h) do { _Pragma("unroll") for (int m = 0; m < 4; ++m) _Pragma("unroll") for (int k = 0; k < 2; ++k) \
        dst[m][k] = *(const LAS half8*)(lds + SA(b, h) + aoff + m * 2048 + k * 1024); } while (0)
#define LDB(dst, b, h) do { _Pragma("unroll") for (int n = 0; n < 2; ++n) _Pragma("unroll") for (int k = 0; k < 2; ++k) \
        dst[n][k] = *(const LAS half8*)(lds + SB(b, h) + boff + n * 2048 + k * 1024); } while (0)
#define MMA(ai, bj, At_, Bt_) do { __builtin_amdgcn_s_setprio(1); \
        _Pragma("unroll") for (int m = 0; m < 4; ++m) _Pragma("unroll") for (int n = 0; n < 2; ++n) _Pragma("unroll") for (int k = 0; k < 2; ++k) \
            acc[ai][bj][m][n] = __builtin_amdgcn_mfma_f32_16x16x32_f16(At_[m][k], Bt_[n][k], acc[ai][bj][m][n], 0, 0, 0); \
        __builtin_amdgcn_s_setprio(0); } while (0)
#define WAIT_V(n) asm volatile("s_waitcnt vmcnt(" #n ")" ::: "memory")
#define WAIT_L(n) asm volatile("s_waitcnt lgkmcnt(" #n ")" ::: "memory")
#define BAR __builtin_amdgcn_s_barrier()
#define SCHED __builtin_amdgcn_sched_barrier(0)
    bool allin = false;
#define WAIT_SEG(c) do { if (wid == 0 && !allin) { int lz_ = lane; asm volatile("" : "+v"(lz_)); const int wh_ = lz_ >> 3; \
        const int it_ = wh_ < 4 && ((c) == 8 || (lz_ & 7) != dest) ? 4 * mt + wh_ : 64 + 4 * nt + (wh_ & 3); \
        gu32* fa_ = (gu32*)(myws + OFF_INFLAG + ((c) == 8 ? rank * 4096 + (lz_ & 7) * 512 : (lz_ & 7) * 4096 + (c) * 512) + it_ * 4); \
        bool now_ = false; \
        if ((c) < 8 && (dd == 0 || (c) > (grp == 0 ? 8 : grp == 1 ? 8 - LAG : 0))) { unsigned v_[9]; \
            v_[8] = __hip_atomic_load((gu32*)(myws + OFF_CDONE), __ATOMIC_RELAXED, __HIP_MEMORY_SCOPE_SYSTEM); \
            _Pragma("unroll") for (int k_ = 0; k_ < 8; ++k_) v_[k_] = __hip_atomic_load(fa_ + ((c) + k_ < 8 ? k_ : 0) * 128, __ATOMIC_RELAXED, __HIP_MEMORY_SCOPE_SYSTEM); \
            asm volatile("" : "+v"(v_[0]), "+v"(v_[1]), "+v"(v_[2]), "+v"(v_[3]), "+v"(v_[4]), "+v"(v_[5]), "+v"(v_[6]), "+v"(v_[7]), "+v"(v_[8]) :: "memory"); \
            now_ = __all(v_[0] >= call); \
            allin = now_ && __all(min(min(min(v_[1], v_[2]), min(v_[3], v_[4])), min(min(v_[5], v_[6]), v_[7])) >= call && (int)(v_[8] - call * (112u * CW)) >= 0); } \
        if (!now_) { if ((c) == (grp == 0 ? 7 : grp == 1 ? 8 - LAG : 0)) poll_ge<16>(fa_, call, tmo, call); else poll_ge<64>(fa_, call, tmo, call); } \
        if ((c) == 6 && !allin) poll_ge<64>((gu32*)(myws + OFF_CDONE), call * (112u * CW), tmo, call);     \
        } } while (0)

    f32x4 acc[2][2][4][2];
#pragma unroll
    for (int a = 0; a < 2; ++a)
#pragma unroll
        for (int b = 0; b < 2; ++b)
#pragma unroll
            for (int m = 0; m < 4; ++m)
#pragma unroll
                for (int n = 0; n < 2; ++n) acc[a][b][m][n] = (f32x4){0.f, 0.f, 0.f, 0.f};
    half8 At[4][2], B0[2][2], B1[2][2];

#define A_OF(t) (img + ((unsigned)__builtin_amdgcn_readlane(tabA, (t) & 63) & ~15u))
#define B_OF(t) (img + (unsigned)__builtin_amdgcn_readlane(tabB, (t) & 63))
#define GEMM_LOOP(T0, T1, HOOK) do { \
    const gchar* a3 = A_OF((T0) + 1); \
    for (int t = (T0); t < (T1); t += 2) { \
        const gchar* a1 = a3; \
        LDB(B0, 0, 0); SCHED; LDA(At, 0, 0); STAGE(SA(1, 1), a1 + HTB); SCHED; \
        const gchar* a2 = A_OF(t + 2); const gchar* b2 = B_OF(t + 2); const gchar* b3 = B_OF(t + 3); a3 = A_OF(t + 3); \
        WAIT_L(8); BAR; WAIT_L(0); MMA(0, 0, At, B0); \
        HOOK; \
        asm volatile("" ::: "memory"); BAR; SCHED; \
        LDB(B1, 0, 1); STAGE(SB(0, 0), b2); \
        BAR; WAIT_L(0); MMA(0, 1, At, B1); BAR; \
        LDA(At, 0, 1); STAGE(SA(0, 0), a2); \
        BAR; WAIT_L(0); MMA(1, 0, At, B0); BAR; SCHED; \
        STAGE(SB(0, 1), b2 + HTB); \
        WAIT_V(6); BAR; MMA(1, 1, At, B1); BAR; \
        LDB(B0, 1, 0); SCHED; LDA(At, 1, 0); STAGE(SA(0, 1), a2 + HTB); \
        WAIT_L(8); BAR; WAIT_L(0); MMA(0, 0, At, B0); BAR; SCHED; \
        LDB(B1, 1, 1); STAGE(SB(1, 0), b3); \
        BAR; WAIT_L(0); MMA(0, 1, At, B1); BAR; \
        LDA(At, 1, 1); STAGE(SA(1, 0), a3); \
        BAR; WAIT_L(0); MMA(1, 0, At, B0); BAR; SCHED; \
        STAGE(SB(1, 1), b3 + HTB); \
        WAIT_V(6); BAR; MMA(1, 1, At, B1); BAR; \
    } \
    if (wr == 0) BAR;                                                   \
    } while (0)

    const int src = (rank - dd) & (NRANK - 1);
    const bool early = dd != 0 && grp == 3;
#define DIAG() do { WAIT_V(0); BAR; if (wr == 1) BAR; asm volatile("" ::: "memory"); GEMM_LOOP(56, 64, ); } while (0)

    int tabA, tabB;
    {
        const int t0 = grp != 0 ? 0 : OWNA, u = lane < t0 ? lane : lane - 8, oo = 8 - dd;
        const int st = dd == 0 ? lane / 7 : u / 6, jj = dd == 0 ? lane % 7 : u % 6, o = jj + 1 + (jj + 1 >= oo ? 1 : 0);
        const bool own = dd != 0 && lane >= t0 && lane < t0 + 8;
        const int kt = own ? (lane - t0) * 8 + rank : (st & 7) * 8 + ((dest + o) & 7), q = lane & 7;
        tabA = lane < 56 ? (kt * (int)KT_STRIDE + mt * (int)PANEL) | (own ? 8 : st & 7) : ((q * 8 + rank) * (int)KT_STRIDE + (2 * rank + (mt & 1)) * (int)PANEL) | 7;
        tabB = nt * (int)PANEL + (lane < 56 ? (int)MAT_BYTES + kt * (int)KT_STRIDE
                                  : dd == 0 ? (int)MAT_BYTES + (q * 8 + rank) * (int)KT_STRIDE : 2 * (int)MAT_BYTES + (src * 8 + q) * (int)KT_STRIDE);
    }
    int have = __builtin_amdgcn_readlane(tabA, 0) & 15;
    BAR;
    WAIT_SEG(have);
    asm volatile("" ::: "memory"); BAR; asm volatile("" ::: "memory");
    { const gchar* a0 = A_OF(0); const gchar* b0 = B_OF(0); const gchar* a1 = A_OF(1); const gchar* b1 = B_OF(1);
      STAGE(SB(0, 0), b0); STAGE(SA(0, 0), a0); STAGE(SB(0, 1), b0 + HTB); STAGE(SA(0, 1), a0 + HTB);
      if (wr == 1) BAR;
      WAIT_V(4); BAR;
      STAGE(SB(1, 0), b1); STAGE(SA(1, 0), a1); STAGE(SB(1, 1), b1 + HTB);
      WAIT_V(6); BAR; }
    GEMM_LOOP(0, 56, { const int seg_ = __builtin_amdgcn_readlane(tabA, t + 3) & 15;
        if (seg_ != have) { have = seg_; WAIT_SEG(have); } });

    int tz = threadIdx.x; asm volatile("" : "+v"(tz));
    const int fr = tz & 15, fq = (tz >> 4) & 3;
    const unsigned lo = (unsigned)(fq * 4) * (NCOLS * 4) + (unsigned)fr * 16u;
    const unsigned soff = (unsigned)(tz & 63) * 16u;
    const size_t torg = ((size_t)((mt & 1) * 256 + wr * 64) * NCOLS + nt * 256 + wc * 64) * 4;
    const unsigned qnan = 0x7fc00000u;
    LAS unsigned char* sc = lds + 131072 + 256 + wid * 1088;
#define ROWOFF(c, g, e) ((size_t)(((c) >> 1) * 128 + (((c) & 1) * 2 + ((g) >> 1)) * 16 + ((g) & 1) * 2 + ((e) >> 2)) * (NCOLS * 4))
#define ACC(c, g, e) acc[(c) >> 1][((e) >> 1) & 1][((c) & 1) * 2 + ((g) >> 1)][(e) & 1][((g) & 1) * 2 + ((e) >> 2)]
    if (dd != 0) {
    if (__builtin_amdgcn_readfirstlane(__hip_atomic_load(tmo, __ATOMIC_RELAXED, __HIP_MEMORY_SCOPE_AGENT)) == call) {
#pragma unroll
        for (int i = 0; i < 32; ++i) {
            acc[i >> 4][(i >> 3) & 1][(i >> 1) & 3][i & 1] = __builtin_bit_cast(f32x4, (u32x4){qnan, qnan, qnan, qnan});
            asm volatile("" : "+v"(acc[i >> 4][(i >> 3) & 1][(i >> 1) & 3][i & 1]));
        }
    }
    unsigned u[4][4][8];
#pragma unroll
    for (int c = 0; c < 4; ++c)
#pragma unroll
        for (int g = 0; g < 4; ++g)
#pragma unroll
            for (int e = 0; e < 8; ++e) { const float v = ACC(c, g, e); u[c][g][e] = __builtin_bit_cast(unsigned, v); }
    unsigned meta;
    {
        unsigned se = 0;
#define U32(n) u[c][(n) >> 3][(n) & 7]
#pragma unroll
        for (int c = 0; c < 4; ++c)
#pragma unroll
            for (int g = 0; g < 4; ++g) {
#pragma unroll
                for (int e = 0; e < 8; ++e) u[c][g][e] = __builtin_amdgcn_alignbit(u[c][g][e], u[c][g][e], 31);
#pragma unroll
                for (int e = 0; e < 8; e += 2) se += __builtin_amdgcn_perm(u[c][g][e + 1], u[c][g][e], 0x0c070c03u);
                asm volatile("" : "+v"(u[c][g][0]), "+v"(u[c][g][1]), "+v"(u[c][g][2]), "+v"(u[c][g][3]), "+v"(u[c][g][4]), "+v"(u[c][g][5]), "+v"(u[c][g][6]), "+v"(u[c][g][7]));
            }
        const unsigned S = __builtin_amdgcn_readlane(wave_scan((se & 0xffffu) + (se >> 16)), 63);
        const unsigned b1 = min((S + 492u) >> 13, 253u), B = (b1 + 0xff00u - 5u) * 0x10001u;
        *(LAS u32x4*)(sc + soff) = (u32x4){0u, 0u, 0u, 0u};
        if (soff < 64u) *(LAS u32x4*)(sc + 1024 + soff) = (u32x4){0u, 0u, 0u, 0u};
        gchar* pk = myws + OFF_PK + (size_t)(bid * 8 + wid) * PK_WAVE;
        u64 st[4]; unsigned ln[4], worst = 0, up = 0;
#pragma unroll
        for (int c = 0; c < 4; ++c) {
            u32x2 cw = {0u, 0u};
            st[c] = 1; ln[c] = 0;
#pragma unroll
            for (int q = 7; q >= 0; --q) {
                unsigned W = 1, M[2];
#pragma unroll
                for (int h = 1; h >= 0; --h) {
                    const int n = 4 * q + 2 * h;
                    const unsigned E = __builtin_amdgcn_perm(U32(n + 1), U32(n), 0x0d070d03u), dq = pk_subs(E, B), ix = pk_min(dq, 0x00070007u) | 0x0c000c00u;
                    const unsigned a3 = pk_min(pk_subs(dq, 0x00070007u), 0x00030003u) * 3u;
                    up |= a3;
                    M[h] = __builtin_amdgcn_perm(0x00000001u, 0x02040507u, ix) + pk_subs(B, E) + a3;
                    cw[q >> 2] = (cw[q >> 2] << 2) | pk_subs(__builtin_amdgcn_perm(0x03020100u, 0u, ix), a3);
                    W = (W << ((M[h] >> 16) & 31u)) | 1u;
                    W = (W << (M[h] & 31u)) | 1u;
                }
                const unsigned ms = M[0] + M[1], sm = (ms & 0xffffu) + (ms >> 16);
                worst = max(worst, sm); ln[c] += sm;
                st[c] = (st[c] << (sm & 31u)) | W;
                asm volatile("" : "+v"(st[c]), "+v"(cw));
            }
#pragma unroll
            for (int i = 0; i < 6; ++i) {
#define CARR(n) __builtin_amdgcn_perm(U32(24 + (n) / 3), U32(n), 0x04020100u + (((n) % 3) << 24))
                const u32x4 v = {CARR(4 * i), CARR(4 * i + 1), CARR(4 * i + 2), CARR(4 * i + 3)};
                STORE16(soff, v, pk + c * PK_CHUNK + i * 1024, 63);
            }
            asm volatile("s_nop 4\n\tglobal_store_dwordx2 %0, %1, %2 sc0 sc1" :: "v"(soff >> 1), "v"(cw), "s"(pk + c * PK_CHUNK + 6144) : "memory");
        }
        bool ovf = max(max(ln[0], ln[1]), max(ln[2], ln[3])) > 63u || worst > 31u || (up & 0x00080008u) != 0u;
        const unsigned len = ovf ? 0u : ln[0] + ln[1] + ln[2] + ln[3], end = wave_scan(len);
        ovf |= end > 7679u;
        asm volatile("" ::: "memory");
        sc[soff >> 4] = (unsigned char)(ovf ? 255u : len);
        if (!ovf) {
            unsigned o = end - len;
#pragma unroll
            for (int c = 0; c < 4; ++c) {
                LAS unsigned* w = (LAS unsigned*)(sc + 64) + (o >> 5);
                const u64 a = st[c] << (o & 31u), b = (st[c] >> 32) << (o & 31u);
                __hip_atomic_fetch_or(w, (unsigned)a, __ATOMIC_RELAXED, __HIP_MEMORY_SCOPE_WAVEFRONT);
                __hip_atomic_fetch_or(w + 1, (unsigned)(a >> 32), __ATOMIC_RELAXED, __HIP_MEMORY_SCOPE_WAVEFRONT);
                __hip_atomic_fetch_or(w + 2, (unsigned)(b >> 32), __ATOMIC_RELAXED, __HIP_MEMORY_SCOPE_WAVEFRONT);
                o += ln[c];
            }
        }
        asm volatile("s_waitcnt lgkmcnt(0)" ::: "memory");
        const u32x4 sd = *(volatile LAS u32x4*)(sc + soff);
        STORE16(soff, sd, pk + 4 * PK_CHUNK, 63);
        meta = b1 | (((min((unsigned)__builtin_amdgcn_readlane(end, 63), 7679u) + 512u) >> 9) << 8);
        if (__builtin_popcountll(__ballot(ovf)) > 2) { meta = 255u; ovf = true; }
        if (ovf) {
            gchar* rbase = myws + OFF_RAW + (size_t)(bid * 8 + wid) * RAW_WAVE;
#pragma unroll
            for (int c = 0; c < 4; ++c)
#pragma unroll
                for (int i = 0; i < 8; ++i) {
                    const u32x4 v = {U32(4 * i), U32(4 * i + 1), U32(4 * i + 2), U32(4 * i + 3)};
                    STORE16(soff, v, rbase + (c * 8 + i) * 1024, 63);
                }
        }
        asm volatile("s_waitcnt vmcnt(0)" ::: "memory");
        lctl[16 + wid] = (call << 12) | meta;
        unsigned old = 0;
        if ((tz & 63) == 0) old = __hip_atomic_fetch_add((LAS unsigned*)lctl + 5, 1u, __ATOMIC_ACQ_REL, __HIP_MEMORY_SCOPE_WORKGROUP);
        if (__builtin_amdgcn_readfirstlane(old) == 7 && (tz & 63) < 2) {
            const u32x4 fw = *(volatile LAS u32x4*)(lctl + 16 + (tz & 1) * 4);
            STORE16(soff, fw, (gchar*)p.ws[dest] + OFF_OUTFLAG + bid * 128, 63);
        }
    }
#pragma unroll
    for (int i = 0; i < 32; ++i) acc[i >> 4][(i >> 3) & 1][(i >> 1) & 3][i & 1] = (f32x4){0.f, 0.f, 0.f, 0.f};
    }
    if (early) DIAG();
    gchar* obase = (gchar*)p.out + torg + (size_t)src * NLOC * 4;
    if (dd != 0) {
        const unsigned f = __builtin_amdgcn_readfirstlane(poll_ge<32>((gu32*)(myws + OFF_OUTFLAG + bid * 128 + wid * 4), call << 12, tmo, call));
        const bool bad = __builtin_amdgcn_readfirstlane(__hip_atomic_load(tmo, __ATOMIC_RELAXED, __HIP_MEMORY_SCOPE_AGENT)) == call;
        const bool whole = (f & 255u) == 255u;
        const gchar* pk = (const gchar*)p.ws[bad || whole ? rank : src] + OFF_PK + (size_t)(bid * 8 + wid) * PK_WAVE;
        const gchar* rbase = (const gchar*)p.ws[bad ? rank : src] + OFF_RAW + (size_t)(bid * 8 + wid) * RAW_WAVE;
        u32x4 side;
        asm volatile("s_nop 4\n\tglobal_load_dwordx4 %0, %1, %2 sc0 sc1\n\ts_waitcnt vmcnt(0)" : "=&v"(side) : "v"((soff >> 6) <= ((f >> 8) & 15u) ? soff : 0u), "s"(pk + 4 * PK_CHUNK) : "memory");
        *(volatile LAS u32x4*)(sc + soff) = side;
        const unsigned lb = *(volatile LAS unsigned char*)(sc + (soff >> 4));
        const bool ovf = lb == 255u || whole;
        unsigned ptr = wave_scan(ovf ? 0u : lb) - (ovf ? 0u : lb);
        const unsigned BB = ((f & 255u) - 1u) * 0x10001u;
        const unsigned poison = bad ? qnan : 0u;
        if (wc == 0) {
            if (grp == 0) poll_ge<16>((gu32*)(myws + OFF_PDONE + (7 - dd) * 512), (call - 1u) * NPROD + (unsigned)((par * 4 + nt) * 2 + wr) * 2u + 1u, tmo, call);
            else poll_ge<8>((gu32*)(myws + OFF_GATE + ((mt * 16 + nt - 4) * 2 + wr) * 128), call, tmo, call);
        }
#pragma unroll
        for (int c = 0; c < 4; ++c) {
            while (__builtin_amdgcn_readfirstlane(lctl[1 + wr]) + 1u < (unsigned)(c * 4 + wc)) __builtin_amdgcn_s_sleep(32);
            while (__builtin_amdgcn_readfirstlane(lctl[1 + wr]) < (unsigned)(c * 4 + wc)) __builtin_amdgcn_s_sleep(2);
            const int lag = (int)__builtin_amdgcn_readfirstlane(lctl[3 + wr]);
            u32x4 in[6]; u32x2 cv;
            const gchar* pc = pk + c * PK_CHUNK + 3072;
            asm volatile("s_nop 4\n\tglobal_load_dwordx2 %0, %1, %2 offset:3072 sc0 sc1 nt" : "=v"(cv) : "v"(soff >> 1), "s"(pc) : "memory");
#pragma unroll
            for (int i = 0; i < 6; ++i) {
                asm volatile("s_waitcnt vmcnt(%0)" :: "i"(EW + 1) : "memory");
                if (lag <= TH2) asm volatile("s_waitcnt vmcnt(%0)" :: "i"(EW) : "memory");
                if (lag <= TH) asm volatile("s_waitcnt vmcnt(%0)" :: "i"(EW - 1) : "memory");
                if (lag < -TH) asm volatile("s_waitcnt vmcnt(%0)" :: "i"(EW - 2) : "memory");
                asm volatile("s_nop 4\n\tglobal_load_dwordx4 %0, %1, %2 offset:%3 sc0 sc1 nt" : "=v"(in[i]) : "v"(soff), "s"(pc), "i"(i * 1024 - 3072) : "memory");
            }
            asm volatile("s_waitcnt vmcnt(%1)" : "+v"(cv) : "i"(EW) : "memory");
#define PASS_ON() do { lctl[1 + wr] = (unsigned)(c * 4 + wc) + 1u; \
                if (c == 3 && (tz & 255) == 192) __hip_atomic_store((gu32*)(myws + OFF_GATE + ((mt * 16 + nt) * 2 + wr) * 128), call, __ATOMIC_RELAXED, __HIP_MEMORY_SCOPE_AGENT); } while (0)
            if (!whole) PASS_ON();
            unsigned tot = 0;
            if ((tz & 63) == 0) asm volatile("s_nop 4\n\tglobal_atomic_add %0, %1, %2, %3 sc0" : "+v"(tot) : "v"(0u), "v"(1u), "s"(myws + OFF_PROG + (7 - dd) * 512) : "memory");
            __builtin_amdgcn_sched_barrier(0);
            unsigned ex[16], s0, s1;
            {
                ptr = min(ptr, 7679u);
                const volatile LAS unsigned* w = (const volatile LAS unsigned*)(sc + 64) + (ptr >> 5);
                const unsigned w0 = w[0], w1 = w[1], w2 = w[2];
                s0 = __builtin_amdgcn_alignbit(w1, w0, ptr) | 1u; s1 = __builtin_amdgcn_alignbit(w2, w1, ptr);
            }
#pragma unroll
            for (int q = 0; q < 8; ++q) {
                unsigned t = s0, M[2];
#pragma unroll
                for (int h = 0; h < 2; ++h) {
                    const unsigned C = (cv[q >> 2] >> (2 * ((2 * q + h) & 7))) & 0x00030003u;
                    const unsigned m0 = (unsigned)__builtin_ctz(t - ((C & 0xffffu) == 0u ? 1u : 0u));
                    t >>= m0 & 31u;
                    const unsigned m1 = (unsigned)__builtin_ctz(t - (C < 0x10000u ? 1u : 0u));
                    if (h == 0) t >>= m1 & 31u;
                    M[h] = m0 | (m1 << 16);
                    ex[2 * q + h] = __builtin_amdgcn_perm(0x030b0202u, 0x07010100u, pk_min(M[h], 0x00070007u)) - M[h] + C + BB;
                }
                const unsigned ms = M[0] + M[1], sm = (ms + (ms >> 16)) & 31u;
                s0 = __builtin_amdgcn_alignbit(s1, s0, sm); s1 >>= sm; ptr += sm;
                asm volatile("" : "+v"(ex[2 * q]), "+v"(ex[2 * q + 1]), "+v"(s0));
            }
            asm volatile("s_waitcnt vmcnt(0)" : "+v"(in[0]), "+v"(in[1]), "+v"(in[2]), "+v"(in[3]), "+v"(in[4]), "+v"(in[5]), "+v"(tot), "+v"(ptr) :: "memory");
            if ((tz & 63) == 0) lctl[3 + wr] = tot - (call - 1u) * 1024u + 8u - 16u * (unsigned)(grp * 16 + c * 4 + wc + 1);
#pragma unroll
            for (int g = 0; g < 4; ++g) {
                u32x4 q[2];
#pragma unroll
                for (int e = 0; e < 8; ++e) {
                    const int n = g * 8 + e;
#define CIN(n) in[(n) >> 2][(n) & 3]
                    const unsigned lo24 = n < 24 ? CIN(n < 24 ? n : 0)
                        : __builtin_amdgcn_perm(CIN(n < 24 ? 0 : 3 * (n - 24) + 2), __builtin_amdgcn_perm(CIN(n < 24 ? 0 : 3 * (n - 24) + 1), CIN(n < 24 ? 0 : 3 * (n - 24)), 0x00000703u), 0x00070100u);
                    q[e >> 2][e & 3] = __builtin_amdgcn_perm(ex[n >> 1], lo24, (n & 1) ? 0x06020100u : 0x04020100u);
                }
                if (ovf) asm volatile("s_nop 4\n\tglobal_load_dwordx4 %0, %2, %3 sc0 sc1\n\tglobal_load_dwordx4 %1, %2, %3 offset:1024 sc0 sc1\n\ts_waitcnt vmcnt(0)"
                                      : "+v"(q[0]), "+v"(q[1]) : "v"(soff + (c * 4 + g) * 2048u), "s"(rbase) : "memory");
                if (whole && g == 1) PASS_ON();
                unsigned v[8];
#pragma unroll
                for (int e = 0; e < 8; ++e) v[e] = __builtin_amdgcn_alignbit(q[e >> 2][e & 3], q[e >> 2][e & 3], 1) | poison;
#pragma unroll
                for (int e = 0; e < 8; ++e) ACC(c, g, e) += __builtin_bit_cast(float, v[e]);
                if (early) {
                    const f32x4 w0 = {ACC(c, g, 0), ACC(c, g, 1), ACC(c, g, 2), ACC(c, g, 3)}, w1 = {ACC(c, g, 4), ACC(c, g, 5), ACC(c, g, 6), ACC(c, g, 7)};
                    STOREO(lo, w0, obase + ROWOFF(c, g, 0)); STOREO(lo, w1, obase + ROWOFF(c, g, 4));
                }
            }
        }
    }
    if (!early) {
        DIAG();
        const bool bad = __builtin_amdgcn_readfirstlane(__hip_atomic_load(tmo, __ATOMIC_RELAXED, __HIP_MEMORY_SCOPE_AGENT)) == call;
#pragma unroll
        for (int c = 0; c < 4; ++c)
#pragma unroll
            for (int g = 0; g < 4; ++g)
#pragma unroll
                for (int h = 0; h < 2; ++h) {
                    f32x4 v = {ACC(c, g, 4 * h), ACC(c, g, 4 * h + 1), ACC(c, g, 4 * h + 2), ACC(c, g, 4 * h + 3)};
                    if (bad) v = __builtin_bit_cast(f32x4, (u32x4){qnan, qnan, qnan, qnan});
                    STOREO(lo, v, obase + ROWOFF(c, g, 4 * h));
                }
    }
}

extern "C" void kernel_launch(int rank, int nranks, ncclComm_t comm,
                              hipStream_t stream,
                              void* const* d_in, const int* in_sizes, int n_in,
                              void* d_out, int out_size,
                              const int* in_dtypes, int out_dtype,
                              void* const* d_out_all,
                              void* const* d_ws_all, size_t ws_bytes) {
    if (nranks != NRANK || (unsigned)rank >= (unsigned)NRANK || ws_bytes < WS_NEED || n_in < 2 ||
        in_sizes[0] != 4096 * KSH || in_sizes[1] != KSH * NCOLS || out_size != 512 * NCOLS) {
        fprintf(stderr, "unexpected problem shape\n");
        return;
    }
    Params p;
    p.x = (const float*)d_in[0]; p.w = (const float*)d_in[1];
    p.out = (float*)d_out;
    for (int r = 0; r < NRANK; ++r) p.ws[r] = (char*)d_ws_all[r];
    p.rank = rank; p.pad = 0;
    gemm_rs_kernel<<<GRID, 512, 0, stream>>>(p);
    HIP_CHECK(hipGetLastError());
}
```
